```python
import numpy as np

kernel_name = "gpu_a2a_gemm_m4096_k4096_n16384_bf16_gelu"


ABSMAX_RTOL = 0.008

COMPUTE_DTYPE = "bfloat16"
EPILOGUE = "gelu"
IN_QUANT = None
OUT_QUANT = None
IN_DTYPES = {"x": "bfloat16", "w_mat": "bfloat16"}
WORLD_SIZE = 8

def _to_bf16(x):
    u = np.ascontiguousarray(x, dtype=np.float32).view(np.uint32)
    return ((u + 0x7FFF + ((u >> 16) & 1)) & 0xFFFF0000).view(np.float32)

def setup_inputs():

    x = _to_bf16(np.random.randn(4096, 4096).astype(np.float32))
    w_mat = _to_bf16((np.random.randn(4096, 16384) / np.sqrt(4096)).astype(np.float32))
    return {"x": x, "w_mat": w_mat}

def _epilogue(y):
    c = 0.7978845608028654
    return (0.5 * y * (1.0 + np.tanh(c * (y + 0.044715 * y ** 3)))).astype(np.float32)

def reference(x, w_mat):
    y = (x @ w_mat).astype(np.float32)
    return _epilogue(y)
```

```cpp
#include <hip/hip_runtime.h>
#include <rccl/rccl.h>
#include <cstdio>

#define HIP_CHECK(expr) do { hipError_t _e = (expr); if (_e != hipSuccess) { \
    fprintf(stderr, "HIP error %s at %s:%d\n", hipGetErrorString(_e), __FILE__, __LINE__); \
    return; } } while (0)

struct Params { const char* x; const char* w; int rank; int pad; char* out; char* ws[8]; };
constexpr int LDS_BYTES = 163840;

__global__ __launch_bounds__(256, 1) void a2a_gemm_gelu(Params p) {
  extern __shared__ __attribute__((aligned(16))) char smem[];
  const int bid = blockIdx.x, xcd = bid & 7, idx = bid >> 3;
  const int wave = __builtin_amdgcn_readfirstlane(threadIdx.x >> 6);
  int l = threadIdx.x & 63;
  const int rank = p.rank & 7;
  const int ntile = xcd * 16 + (idx >> 1), mhalf = idx & 1, n0 = ntile * 128;
  const unsigned lds0 = (unsigned)(size_t)smem;
  const unsigned OOB = 0x80000000u;
  unsigned sv[4], woff[4];
  { const int g = l >> 4;
    const unsigned srow = (rank * 512 + mhalf * 256 + wave * 64 + (l >> 3)) * 1024;
    sv[0] = srow + (((l & 7) ^ g) * 16); sv[1] = srow + (((l & 7) ^ (4 + g)) * 16) + 7168; sv[2] = sv[0] + 14336; sv[3] = sv[1] + 14336;
    for (int i = 0; i < 4; ++i) { int s = ((g & 1) << 2) | ((i >> 1) & 1);
      woff[i] = ((wave * 4 + i) * 4 + ((l >> 3) & 3)) * 32768 + n0 * 2 + (l >> 5) * 128 + (((l & 7) ^ s) * 16) - i * 1024; } }
  unsigned ssb = lds0 + wave * 4096, rot = (ntile + 4 * mhalf) & 7, locw = rank << 24;
  const char* xp = p.x; const char* wp = p.w;
  asm volatile(
    "s_mov_b32 s46, 0x400000\n"
    "s_mov_b32 s47, 0x00020000\n"
    "s_mov_b32 s38, 0x8000000\n"
    "s_mov_b32 s39, 0x00020000\n"
    "s_mov_b32 s67, s85\n"
    "s_add_u32 s66, s86, 0\n"
    "s_and_b32 s66, s66, 7\n"
    "s_lshl_b32 s84, s66, 7\n"
    "s_add_u32 m0, s85, 65536\n"
    "s_add_u32 s84, s84, 0\n"
    "buffer_load_dwordx4 v32, s[44:47], s84 offen offset:0 lds\n"
    "buffer_load_dwordx4 v33, s[44:47], s84 offen offset:1024 lds\n"
    "buffer_load_dwordx4 v34, s[44:47], s84 offen offset:2048 lds\n"
    "buffer_load_dwordx4 v35, s[44:47], s84 offen offset:3072 lds\n"
    "s_add_u32 s66, s86, 0\n"
    "s_and_b32 s66, s66, 7\n"
    "s_lshl_b32 s84, s66, 7\n"
    "s_add_u32 m0, s85, 81920\n"
    "s_add_u32 s84, s84, 32768\n"
    "buffer_load_dwordx4 v32, s[44:47], s84 offen offset:0 lds\n"
    "buffer_load_dwordx4 v33, s[44:47], s84 offen offset:1024 lds\n"
    "buffer_load_dwordx4 v34, s[44:47], s84 offen offset:2048 lds\n"
    "buffer_load_dwordx4 v35, s[44:47], s84 offen offset:3072 lds\n"
    "s_add_u32 s66, s86, 0\n"
    "s_and_b32 s66, s66, 7\n"
    "s_lshl_b32 s64, s66, 21\n"
    "s_add_u32 m0, s67, 0\n"
    "s_add_u32 s64, s64, s87\n"
    "buffer_load_dwordx4 v56, s[36:39], s64 offen offset:0 lds\n"
    "buffer_load_dwordx4 v57, s[36:39], s64 offen offset:1024 lds\n"
    "buffer_load_dwordx4 v58, s[36:39], s64 offen offset:2048 lds\n"
    "buffer_load_dwordx4 v59, s[36:39], s64 offen offset:3072 lds\n"
    : "+{s[44:45]}"(xp), "+{s[36:37]}"(wp), "+{v32}"(sv[0]), "+{v56}"(woff[0]), "+{v33}"(sv[1]), "+{v57}"(woff[1]), "+{v34}"(sv[2]), "+{v58}"(woff[2]), "+{v35}"(sv[3]), "+{v59}"(woff[3]), "+{s86}"(rot), "+{s87}"(locw), "+{s85}"(ssb), "+v"(l)
    :
    : "s46", "s47", "s38", "s39", "s67", "s66", "s84", "s64", "memory", "scc");
  const int r16 = l & 15, g = l >> 4, q4 = (l >> 2) & 3, p4 = l & 3;
  unsigned at0 = (mhalf * 16 + wave * 4) * 2048 + l * 16;
  unsigned mv0 = ssb + 65536 + r16 * 128 + ((g ^ (r16 >> 1)) * 16), mv1 = ssb + 65536 + r16 * 128 + (((4 + g) ^ (r16 >> 1)) * 16);
  unsigned bl[4], xs[4];
  for (int c = 0; c < 4; ++c) { int cp = c >> 1, f = c & 1;
    bl[c] = lds0 + 2048 * g + 128 * q4 + 64 * (cp ^ (q4 >> 1)) + 16 * (p4 ^ (g & 1)) + 8 * ((p4 & 1) ^ f); }
  const int li = l;
  unsigned tab = (unsigned)((((rank + 1 + li % 7) & 7) * 8 + ((li / 7) & 7)) << 16);
  unsigned ooff = (mhalf * 256 + wave * 64 + r16) * 32768 + n0 * 2 + g * 16;
  char* wsl = p.ws[0];
  const bool pusher = bid < 224;
  const int dp = pusher ? 1 + (bid >> 5) : 0, slot = bid & 31, pj = slot >> 2, pq = slot & 3;
  const unsigned peer = (rank + dp) & 7;
  char* wsp = p.ws[dp];
  const unsigned xrow = (peer * 512 + pq * 128 + wave * 32 + (l >> 3)) * 1024 + pj * 128;
  xs[0] = xrow + (((l & 7) ^ g) * 16); xs[1] = xrow + (((l & 7) ^ (4 + g)) * 16) + 7168; xs[2] = xs[0] + 14336; xs[3] = xs[1] + 14336;
  unsigned so = (rank * 8 + pj) * 65536 + (pq * 8 + wave * 2) * 2048 + l * 16;
  unsigned foff = (l < 8 && wave == 0) ? ((l * 2 + (pq >> 1)) * 8 + rank) * 128 + pj * 4 : OOB;
  unsigned next0 = pusher ? (unsigned)(slot * 5 / 4) << 16 : 0x7fffffffu, pend0 = pusher ? 0 : 2;
  unsigned poff = (l < 56 && wave == 0) ? 0x4000 + ((xcd * 2 + mhalf) * 8 + ((rank + 1 + li % 7) & 7)) * 128 + (li / 7) * 4 : OOB;
  unsigned cr = lds0 + 147456, cw = cr + wave * 64;
  unsigned cntoff = (bid * 4 + wave) * 4, wv = wave, mh = mhalf ? 4 : 99;
  char* outp = p.out;
  asm volatile(
    "s_mov_b32 s46, 0x400000\n"
    "s_mov_b32 s47, 0x00020000\n"
    "s_mov_b32 s38, 0x8000000\n"
    "s_mov_b32 s39, 0x00020000\n"
    "s_mov_b32 s67, s85\n"
    "s_mov_b32 s58, 0x10000\n"
    "s_mov_b32 s59, 0x00020000\n"
    "buffer_load_dword v42, v43, s[56:59], 0 offen sc1\n"
    "s_add_u32 m0, s85, 147456\n"
    "s_nop 0\n"
    "buffer_load_dwordx4 v96, s[44:47], 0 offen offset:0 lds\n"
    "buffer_load_dwordx4 v97, s[44:47], 0 offen offset:1024 lds\n"
    "buffer_load_dwordx4 v98, s[44:47], 0 offen offset:2048 lds\n"
    "buffer_load_dwordx4 v99, s[44:47], 0 offen offset:3072 lds\n"
    "s_add_u32 s66, s86, 1\n"
    "s_and_b32 s66, s66, 7\n"
    "s_lshl_b32 s84, s66, 7\n"
    "s_add_u32 m0, s85, 98304\n"
    "s_add_u32 s84, s84, 0\n"
    "buffer_load_dwordx4 v32, s[44:47], s84 offen offset:0 lds\n"
    "buffer_load_dwordx4 v33, s[44:47], s84 offen offset:1024 lds\n"
    "buffer_load_dwordx4 v34, s[44:47], s84 offen offset:2048 lds\n"
    "buffer_load_dwordx4 v35, s[44:47], s84 offen offset:3072 lds\n"
    "s_add_u32 s66, s86, 1\n"
    "s_and_b32 s66, s66, 7\n"
    "s_lshl_b32 s84, s66, 7\n"
    "s_add_u32 m0, s85, 114688\n"
    "s_add_u32 s84, s84, 32768\n"
    "buffer_load_dwordx4 v32, s[44:47], s84 offen offset:0 lds\n"
    "buffer_load_dwordx4 v33, s[44:47], s84 offen offset:1024 lds\n"
    "buffer_load_dwordx4 v34, s[44:47], s84 offen offset:2048 lds\n"
    "buffer_load_dwordx4 v35, s[44:47], s84 offen offset:3072 lds\n"
    "s_add_u32 s66, s86, 1\n"
    "s_and_b32 s66, s66, 7\n"
    "s_lshl_b32 s64, s66, 21\n"
    "s_add_u32 m0, s67, 16384\n"
    "s_add_u32 s64, s64, s87\n"
    "buffer_load_dwordx4 v56, s[36:39], s64 offen offset:0 lds\n"
    "buffer_load_dwordx4 v57, s[36:39], s64 offen offset:1024 lds\n"
    "buffer_load_dwordx4 v58, s[36:39], s64 offen offset:2048 lds\n"
    "buffer_load_dwordx4 v59, s[36:39], s64 offen offset:3072 lds\n"
    "s_memrealtime s[78:79]\n"
    "s_mov_b64 s[16:17], 0\n"
    "s_mov_b64 s[18:19], 0\n"
    "s_add_u32 s40, s56, 0x100000\n"
    "s_addc_u32 s41, s57, 0\n"
    "s_mov_b32 s42, 0x400000\n"
    "s_mov_b32 s43, 0x00020000\n"
    "s_add_u32 s48, s52, 0x100000\n"
    "s_addc_u32 s49, s53, 0\n"
    "s_mov_b32 s50, 0x400000\n"
    "s_mov_b32 s51, 0x00020000\n"
    "s_add_u32 s52, s52, 0x4000\n"
    "s_addc_u32 s53, s53, 0\n"
    "s_mov_b32 s54, 0x4000\n"
    "s_mov_b32 s55, 0x00020000\n"
    "s_mov_b32 s62, 0x1000000\n"
    "s_mov_b32 s63, 0x00020000\n"
    "s_mov_b64 s[20:21], s[56:57]\n"
    "s_mov_b32 s22, 0\n"
    "s_mov_b32 s23, 0x00020000\n"
    "v_add_u32 v55, 0x1000, v54\n"
    "v_add_u32 v30, 0x10000, v28\n"
    "v_add_u32 v31, 0x10000, v29\n"
    "v_add_u32 v64, 0x10000, v60\n"
    "v_add_u32 v68, 0x20000, v60\n"
    "v_add_u32 v65, 0x10000, v61\n"
    "v_add_u32 v69, 0x20000, v61\n"
    "v_add_u32 v66, 0x10000, v62\n"
    "v_add_u32 v70, 0x20000, v62\n"
    "v_add_u32 v67, 0x10000, v63\n"
    "v_add_u32 v71, 0x20000, v63\n"
    "s_mov_b32 s88, 0\n"
    "s_mov_b32 s89, 524288\n"
    "s_mov_b32 s90, 1048576\n"
    "s_mov_b32 s91, 1572864\n"
    "s_mov_b64 s[72:73], 0\n"
    "s_mov_b64 s[80:81], 0\n"
    "s_mov_b64 s[82:83], 0\n"
    "v_mov_b32 v24, 1\n"
    "v_mov_b32 v47, 0\n"
    "v_mov_b32 v48, 0\n"
    "s_mov_b32 s92, 0xffff0000\n"
    "s_mov_b32 s93, 0xffff0000\n"
    "v_mov_b32 v45, 0xc0135761\n"
    "v_mov_b32 v44, 0xbdd2d3e8\n"
    "s_waitcnt lgkmcnt(0)\n"
    "s_branch 91f\n"
    "90:\n"
    "s_add_u32 s28, s28, 8\n"
    "s_addc_u32 s29, s29, 0\n"
    "s_cmp_eq_u32 s71, 0\n"
    "s_cbranch_scc1 86f\n"
    "s_cmp_eq_u32 s71, 3\n"
    "s_cbranch_scc1 85f\n"
    "s_sub_u32 s24, s25, s27\n"
    "s_sext_i32_i16 s24, s24\n"
    "88:\n"
    "s_getreg_b32 s95, hwreg(HW_REG_IB_STS)\n"
    "s_bfe_u32 s30, s95, 0x20016\n"
    "s_and_b32 s95, s95, 15\n"
    "s_lshl4_add_u32 s95, s30, s95\n"
    "s_cmp_le_i32 s95, s24\n"
    "s_cbranch_scc0 88b\n"
    "s_mov_b32 s71, 3\n"
    "s_mov_b32 s24, 0\n"
    "s_setpc_b64 s[28:29]\n"
    "85:\n"
    "buffer_atomic_add v24, v50, s[52:55], 0 offen sc1\n"
    "s_mov_b32 s71, 2\n"
    "s_mov_b32 s24, 0x7fffffff\n"
    "s_setpc_b64 s[28:29]\n"
    "86:\n"
    "buffer_store_dwordx4 v[72:75], v51, s[48:51], 0 offen offset:0 sc0 sc1\n"
    "buffer_store_dwordx4 v[76:79], v51, s[48:51], 0 offen offset:1024 sc0 sc1\n"
    "buffer_store_dwordx4 v[80:83], v51, s[48:51], 0 offen offset:2048 sc0 sc1\n"
    "buffer_store_dwordx4 v[84:87], v51, s[48:51], 0 offen offset:3072 sc0 sc1\n"
    "s_mov_b32 s71, 1\n"
    "s_mov_b32 s27, s25\n"
    "s_add_u32 s24, s25, 196608\n"
    "s_and_b32 s24, s24, 0xfffe0000\n"
    "s_setpc_b64 s[28:29]\n"
    "91:\n"
    "v_mov_b32 v128, 0\n"
    "v_mov_b32 v129, 0\n"
    "v_mov_b32 v130, 0\n"
    "v_mov_b32 v131, 0\n"
    "v_mov_b32 v132, 0\n"
    "v_mov_b32 v133, 0\n"
    "v_mov_b32 v134, 0\n"
    "v_mov_b32 v135, 0\n"
    "v_mov_b32 v136, 0\n"
    "v_mov_b32 v137, 0\n"
    "v_mov_b32 v138, 0\n"
    "v_mov_b32 v139, 0\n"
    "v_mov_b32 v140, 0\n"
    "v_mov_b32 v141, 0\n"
    "v_mov_b32 v142, 0\n"
    "v_mov_b32 v143, 0\n"
    "v_mov_b32 v144, 0\n"
    "v_mov_b32 v145, 0\n"
    "v_mov_b32 v146, 0\n"
    "v_mov_b32 v147, 0\n"
    "v_mov_b32 v148, 0\n"
    "v_mov_b32 v149, 0\n"
    "v_mov_b32 v150, 0\n"
    "v_mov_b32 v151, 0\n"
    "v_mov_b32 v152, 0\n"
    "v_mov_b32 v153, 0\n"
    "v_mov_b32 v154, 0\n"
    "v_mov_b32 v155, 0\n"
    "v_mov_b32 v156, 0\n"
    "v_mov_b32 v157, 0\n"
    "v_mov_b32 v158, 0\n"
    "v_mov_b32 v159, 0\n"
    "v_mov_b32 v160, 0\n"
    "v_mov_b32 v161, 0\n"
    "v_mov_b32 v162, 0\n"
    "v_mov_b32 v163, 0\n"
    "v_mov_b32 v164, 0\n"
    "v_mov_b32 v165, 0\n"
    "v_mov_b32 v166, 0\n"
    "v_mov_b32 v167, 0\n"
    "v_mov_b32 v168, 0\n"
    "v_mov_b32 v169, 0\n"
    "v_mov_b32 v170, 0\n"
    "v_mov_b32 v171, 0\n"
    "v_mov_b32 v172, 0\n"
    "v_mov_b32 v173, 0\n"
    "v_mov_b32 v174, 0\n"
    "v_mov_b32 v175, 0\n"
    "v_mov_b32 v176, 0\n"
    "v_mov_b32 v177, 0\n"
    "v_mov_b32 v178, 0\n"
    "v_mov_b32 v179, 0\n"
    "v_mov_b32 v180, 0\n"
    "v_mov_b32 v181, 0\n"
    "v_mov_b32 v182, 0\n"
    "v_mov_b32 v183, 0\n"
    "v_mov_b32 v184, 0\n"
    "v_mov_b32 v185, 0\n"
    "v_mov_b32 v186, 0\n"
    "v_mov_b32 v187, 0\n"
    "v_mov_b32 v188, 0\n"
    "v_mov_b32 v189, 0\n"
    "v_mov_b32 v190, 0\n"
    "v_mov_b32 v191, 0\n"
    "v_mov_b32 v192, 0\n"
    "v_mov_b32 v193, 0\n"
    "v_mov_b32 v194, 0\n"
    "v_mov_b32 v195, 0\n"
    "v_mov_b32 v196, 0\n"
    "v_mov_b32 v197, 0\n"
    "v_mov_b32 v198, 0\n"
    "v_mov_b32 v199, 0\n"
    "v_mov_b32 v200, 0\n"
    "v_mov_b32 v201, 0\n"
    "v_mov_b32 v202, 0\n"
    "v_mov_b32 v203, 0\n"
    "v_mov_b32 v204, 0\n"
    "v_mov_b32 v205, 0\n"
    "v_mov_b32 v206, 0\n"
    "v_mov_b32 v207, 0\n"
    "v_mov_b32 v208, 0\n"
    "v_mov_b32 v209, 0\n"
    "v_mov_b32 v210, 0\n"
    "v_mov_b32 v211, 0\n"
    "v_mov_b32 v212, 0\n"
    "v_mov_b32 v213, 0\n"
    "v_mov_b32 v214, 0\n"
    "v_mov_b32 v215, 0\n"
    "v_mov_b32 v216, 0\n"
    "v_mov_b32 v217, 0\n"
    "v_mov_b32 v218, 0\n"
    "v_mov_b32 v219, 0\n"
    "v_mov_b32 v220, 0\n"
    "v_mov_b32 v221, 0\n"
    "v_mov_b32 v222, 0\n"
    "v_mov_b32 v223, 0\n"
    "v_mov_b32 v224, 0\n"
    "v_mov_b32 v225, 0\n"
    "v_mov_b32 v226, 0\n"
    "v_mov_b32 v227, 0\n"
    "v_mov_b32 v228, 0\n"
    "v_mov_b32 v229, 0\n"
    "v_mov_b32 v230, 0\n"
    "v_mov_b32 v231, 0\n"
    "v_mov_b32 v232, 0\n"
    "v_mov_b32 v233, 0\n"
    "v_mov_b32 v234, 0\n"
    "v_mov_b32 v235, 0\n"
    "v_mov_b32 v236, 0\n"
    "v_mov_b32 v237, 0\n"
    "v_mov_b32 v238, 0\n"
    "v_mov_b32 v239, 0\n"
    "v_mov_b32 v240, 0\n"
    "v_mov_b32 v241, 0\n"
    "v_mov_b32 v242, 0\n"
    "v_mov_b32 v243, 0\n"
    "v_mov_b32 v244, 0\n"
    "v_mov_b32 v245, 0\n"
    "v_mov_b32 v246, 0\n"
    "v_mov_b32 v247, 0\n"
    "v_mov_b32 v248, 0\n"
    "v_mov_b32 v249, 0\n"
    "v_mov_b32 v250, 0\n"
    "v_mov_b32 v251, 0\n"
    "v_mov_b32 v252, 0\n"
    "v_mov_b32 v253, 0\n"
    "v_mov_b32 v254, 0\n"
    "v_mov_b32 v255, 0\n"
    "s_waitcnt vmcnt(12)\n"
    "v_add_u32 v42, 1, v42\n"
    "s_nop 0\n"
    "v_readfirstlane_b32 s94, v42\n"
    "s_nop 1\n"
    "s_and_b32 s30, s94, 1\n"
    "s_lshl_b32 s30, s30, 22\n"
    "s_add_u32 s48, s48, s30\n"
    "s_addc_u32 s49, s49, 0\n"
    "ds_read_b128 v[72:75], v30 offset:16384\n"
    "ds_read_b128 v[76:79], v31 offset:16384\n"
    "ds_read_b128 v[80:83], v30 offset:18432\n"
    "ds_read_b128 v[84:87], v31 offset:18432\n"
    "s_add_u32 s66, s86, 2\n"
    "s_and_b32 s66, s66, 7\n"
    "s_lshl_b32 s84, s66, 7\n"
    "s_add_u32 m0, s85, 131072\n"
    "s_add_u32 s84, s84, 0\n"
    "buffer_load_dwordx4 v32, s[44:47], s84 offen offset:0 lds\n"
    "buffer_load_dwordx4 v33, s[44:47], s84 offen offset:1024 lds\n"
    "buffer_load_dwordx4 v34, s[44:47], s84 offen offset:2048 lds\n"
    "buffer_load_dwordx4 v35, s[44:47], s84 offen offset:3072 lds\n"
    "s_add_u32 s66, s86, 2\n"
    "s_and_b32 s66, s66, 7\n"
    "s_lshl_b32 s64, s66, 21\n"
    "s_add_u32 m0, s67, 32768\n"
    "s_add_u32 s64, s64, s87\n"
    "buffer_load_dwordx4 v56, s[36:39], s64 offen offset:0 lds\n"
    "buffer_load_dwordx4 v57, s[36:39], s64 offen offset:1024 lds\n"
    "buffer_load_dwordx4 v58, s[36:39], s64 offen offset:2048 lds\n"
    "buffer_load_dwordx4 v59, s[36:39], s64 offen offset:3072 lds\n"
    "buffer_store_dword v42, v43, s[56:59], 0 offen\n"
    "s_lshl_b32 s70, s94, 1\n"
    "s_add_u32 s40, s40, s30\n"
    "s_addc_u32 s41, s41, 0\n"
    "s_waitcnt vmcnt(34)\n"
    "ds_read_b128 a[0:3], v28 offset:0\n"
    "ds_read_b128 a[4:7], v29 offset:0\n"
    "ds_read_b128 a[8:11], v28 offset:2048\n"
    "ds_read_b128 a[12:15], v29 offset:2048\n"
    "s_waitcnt vmcnt(30)\n"
    "ds_read_b128 a[16:19], v28 offset:16384\n"
    "ds_read_b128 a[20:23], v29 offset:16384\n"
    "ds_read_b128 a[24:27], v28 offset:18432\n"
    "ds_read_b128 a[28:31], v29 offset:18432\n"
    "s_waitcnt vmcnt(26)\n"
    "s_barrier\n"
    "ds_read_b64_tr_b16 v[96:97], v60 offset:0\n"
    "ds_read_b64_tr_b16 v[98:99], v60 offset:1024\n"
    "ds_read_b64_tr_b16 v[100:101], v60 offset:8192\n"
    "ds_read_b64_tr_b16 v[102:103], v60 offset:9216\n"
    "ds_read_b64_tr_b16 v[104:105], v61 offset:0\n"
    "ds_read_b64_tr_b16 v[106:107], v61 offset:1024\n"
    "ds_read_b64_tr_b16 v[108:109], v61 offset:8192\n"
    "ds_read_b64_tr_b16 v[110:111], v61 offset:9216\n"
    "ds_read_b64_tr_b16 v[112:113], v62 offset:0\n"
    "ds_read_b64_tr_b16 v[114:115], v62 offset:1024\n"
    "ds_read_b64_tr_b16 v[116:117], v62 offset:8192\n"
    "ds_read_b64_tr_b16 v[118:119], v62 offset:9216\n"
    "s_waitcnt lgkmcnt(12)\n"
    "s_add_u32 s66, s86, 2\n"
    "s_and_b32 s66, s66, 7\n"
    "s_lshl_b32 s84, s66, 7\n"
    "s_add_u32 m0, s85, 65536\n"
    "s_add_u32 s84, s84, 32768\n"
    "buffer_load_dwordx4 v32, s[44:47], s84 offen offset:0 lds\n"
    "buffer_load_dwordx4 v33, s[44:47], s84 offen offset:1024 lds\n"
    "buffer_load_dwordx4 v34, s[44:47], s84 offen offset:2048 lds\n"
    "buffer_load_dwordx4 v35, s[44:47], s84 offen offset:3072 lds\n"
    "s_add_u32 s66, s86, 3\n"
    "s_and_b32 s66, s66, 7\n"
    "s_lshl_b32 s84, s66, 7\n"
    "s_add_u32 m0, s85, 81920\n"
    "s_add_u32 s84, s84, 0\n"
    "buffer_load_dwordx4 v32, s[44:47], s84 offen offset:0 lds\n"
    "buffer_load_dwordx4 v33, s[44:47], s84 offen offset:1024 lds\n"
    "buffer_load_dwordx4 v34, s[44:47], s84 offen offset:2048 lds\n"
    "buffer_load_dwordx4 v35, s[44:47], s84 offen offset:3072 lds\n"
    "s_mov_b32 s25, 46\n"
    "s_getpc_b64 s[28:29]\n"
    "s_cmp_ge_i32 s25, s24\n"
    "s_cbranch_scc1 90b\n"
    "s_waitcnt lgkmcnt(8)\n"
    "v_mfma_f32_16x16x32_bf16 v[128:131], v[96:99], a[0:3], v[128:131]\n"
    "ds_read_b64_tr_b16 v[120:121], v63 offset:0\n"
    "v_mfma_f32_16x16x32_bf16 v[128:131], v[100:103], a[4:7], v[128:131]\n"
    "v_mfma_f32_16x16x32_bf16 v[160:163], v[100:103], a[12:15], v[160:163]\n"
    "ds_read_b64_tr_b16 v[122:123], v63 offset:1024\n"
    "v_mfma_f32_16x16x32_bf16 v[160:163], v[96:99], a[8:11], v[160:163]\n"
    "v_mfma_f32_16x16x32_bf16 v[192:195], v[96:99], a[16:19], v[192:195]\n"
    "ds_read_b64_tr_b16 v[124:125], v63 offset:8192\n"
    "s_waitcnt vmcnt(25)\n"
    "ds_read_b128 a[32:35], v28 offset:32768\n"
    "v_mfma_f32_16x16x32_bf16 v[192:195], v[100:103], a[20:23], v[192:195]\n"
    "ds_read_b128 a[36:39], v29 offset:32768\n"
    "ds_read_b128 a[40:43], v28 offset:34816\n"
    "ds_read_b128 a[44:47], v29 offset:34816\n"
    "v_mfma_f32_16x16x32_bf16 v[224:227], v[100:103], a[28:31], v[224:227]\n"
    "ds_read_b64_tr_b16 v[126:127], v63 offset:9216\n"
    "s_waitcnt vmcnt(21)\n"
    "ds_read_b128 a[48:51], v28 offset:49152\n"
    "v_mfma_f32_16x16x32_bf16 v[224:227], v[96:99], a[24:27], v[224:227]\n"
    "ds_read_b128 a[52:55], v29 offset:49152\n"
    "ds_read_b128 a[56:59], v28 offset:51200\n"
    "ds_read_b128 a[60:63], v29 offset:51200\n"
    "s_waitcnt lgkmcnt(15)\n"
    "v_mfma_f32_16x16x32_bf16 v[228:231], v[104:107], a[24:27], v[228:231]\n"
    "ds_read_b64_tr_b16 v[96:97], v60 offset:512\n"
    "s_add_u32 s66, s86, 3\n"
    "s_and_b32 s66, s66, 7\n"
    "v_mfma_f32_16x16x32_bf16 v[228:231], v[108:111], a[28:31], v[228:231]\n"
    "s_mov_b32 s25, 65582\n"
    "s_getpc_b64 s[28:29]\n"
    "s_cmp_ge_i32 s25, s24\n"
    "s_cbranch_scc1 90b\n"
    "v_mfma_f32_16x16x32_bf16 v[196:199], v[108:111], a[20:23], v[196:199]\n"
    "ds_read_b64_tr_b16 v[98:99], v60 offset:1536\n"
    "s_lshl_b32 s64, s66, 21\n"
    "s_add_u32 m0, s67, 49152\n"
    "v_mfma_f32_16x16x32_bf16 v[196:199], v[104:107], a[16:19], v[196:199]\n"
    "s_add_u32 s64, s64, s87\n"
    "v_mfma_f32_16x16x32_bf16 v[164:167], v[104:107], a[8:11], v[164:167]\n"
    "ds_read_b64_tr_b16 v[100:101], v60 offset:8704\n"
    "v_mfma_f32_16x16x32_bf16 v[164:167], v[108:111], a[12:15], v[164:167]\n"
    "buffer_load_dwordx4 v56, s[36:39], s64 offen offset:0 lds\n"
    "v_mfma_f32_16x16x32_bf16 v[132:135], v[108:111], a[4:7], v[132:135]\n"
    "ds_read_b64_tr_b16 v[102:103], v60 offset:9728\n"
    "v_mfma_f32_16x16x32_bf16 v[132:135], v[104:107], a[0:3], v[132:135]\n"
    "buffer_load_dwordx4 v57, s[36:39], s64 offen offset:1024 lds\n"
    "s_waitcnt lgkmcnt(15)\n"
    "v_mfma_f32_16x16x32_bf16 v[136:139], v[112:115], a[0:3], v[136:139]\n"
    "ds_read_b64_tr_b16 v[104:105], v61 offset:512\n"
    "v_mfma_f32_16x16x32_bf16 v[136:139], v[116:119], a[4:7], v[136:139]\n"
    "v_mfma_f32_16x16x32_bf16 v[168:171], v[116:119], a[12:15], v[168:171]\n"
    "ds_read_b64_tr_b16 v[106:107], v61 offset:1536\n"
    "v_mfma_f32_16x16x32_bf16 v[168:171], v[112:115], a[8:11], v[168:171]\n"
    "v_mfma_f32_16x16x32_bf16 v[200:203], v[112:115], a[16:19], v[200:203]\n"
    "ds_read_b64_tr_b16 v[108:109], v61 offset:8704\n"
    "v_mfma_f32_16x16x32_bf16 v[200:203], v[116:119], a[20:23], v[200:203]\n"
    "buffer_load_dwordx4 v58, s[36:39], s64 offen offset:2048 lds\n"
    "v_mfma_f32_16x16x32_bf16 v[232:235], v[116:119], a[28:31], v[232:235]\n"
    "ds_read_b64_tr_b16 v[110:111], v61 offset:9728\n"
    "v_mfma_f32_16x16x32_bf16 v[232:235], v[112:115], a[24:27], v[232:235]\n"
    "s_waitcnt lgkmcnt(12)\n"
    "v_mfma_f32_16x16x32_bf16 v[236:239], v[120:123], a[24:27], v[236:239]\n"
    "ds_read_b64_tr_b16 v[112:113], v62 offset:512\n"
    "v_mfma_f32_16x16x32_bf16 v[236:239], v[124:127], a[28:31], v[236:239]\n"
    "v_mfma_f32_16x16x32_bf16 v[204:207], v[124:127], a[20:23], v[204:207]\n"
    "ds_read_b64_tr_b16 v[114:115], v62 offset:1536\n"
    "v_mfma_f32_16x16x32_bf16 v[204:207], v[120:123], a[16:19], v[204:207]\n"
    "buffer_load_dwordx4 v59, s[36:39], s64 offen offset:3072 lds\n"
    "v_mfma_f32_16x16x32_bf16 v[172:175], v[120:123], a[8:11], v[172:175]\n"
    "ds_read_b64_tr_b16 v[116:117], v62 offset:8704\n"
    "v_mfma_f32_16x16x32_bf16 v[172:175], v[124:127], a[12:15], v[172:175]\n"
    "v_mfma_f32_16x16x32_bf16 v[140:143], v[124:127], a[4:7], v[140:143]\n"
    "ds_read_b64_tr_b16 v[118:119], v62 offset:9728\n"
    "v_mfma_f32_16x16x32_bf16 v[140:143], v[120:123], a[0:3], v[140:143]\n"
    "s_waitcnt lgkmcnt(8)\n"
    "v_mfma_f32_16x16x32_bf16 v[144:147], v[96:99], a[0:3], v[144:147]\n"
    "ds_read_b64_tr_b16 v[120:121], v63 offset:512\n"
    "s_add_u32 s66, s86, 3\n"
    "s_and_b32 s66, s66, 7\n"
    "v_mfma_f32_16x16x32_bf16 v[144:147], v[100:103], a[4:7], v[144:147]\n"
    "s_lshl_b32 s84, s66, 7\n"
    "s_add_u32 m0, s85, 98304\n"
    "s_add_u32 s84, s84, 32768\n"
    "v_mfma_f32_16x16x32_bf16 v[176:179], v[100:103], a[12:15], v[176:179]\n"
    "ds_read_b64_tr_b16 v[122:123], v63 offset:1536\n"
    "buffer_load_dwordx4 v32, s[44:47], s84 offen offset:0 lds\n"
    "v_mfma_f32_16x16x32_bf16 v[176:179], v[96:99], a[8:11], v[176:179]\n"
    "buffer_load_dwordx4 v33, s[44:47], s84 offen offset:1024 lds\n"
    "v_mfma_f32_16x16x32_bf16 v[208:211], v[96:99], a[16:19], v[208:211]\n"
    "ds_read_b64_tr_b16 v[124:125], v63 offset:8704\n"
    "buffer_load_dwordx4 v34, s[44:47], s84 offen offset:2048 lds\n"
    "v_mfma_f32_16x16x32_bf16 v[208:211], v[100:103], a[20:23], v[208:211]\n"
    "s_mov_b32 s25, 131125\n"
    "s_getpc_b64 s[28:29]\n"
    "s_cmp_ge_i32 s25, s24\n"
    "s_cbranch_scc1 90b\n"
    "v_mfma_f32_16x16x32_bf16 v[240:243], v[100:103], a[28:31], v[240:243]\n"
    "ds_read_b64_tr_b16 v[126:127], v63 offset:9728\n"
    "v_mfma_f32_16x16x32_bf16 v[240:243], v[96:99], a[24:27], v[240:243]\n"
    "buffer_load_dwordx4 v35, s[44:47], s84 offen offset:3072 lds\n"
    "s_waitcnt lgkmcnt(8)\n"
    "s_waitcnt vmcnt(25)\n"
    "s_barrier\n"
    "v_mfma_f32_16x16x32_bf16 v[244:247], v[104:107], a[24:27], v[244:247]\n"
    "ds_read_b64_tr_b16 v[96:97], v60 offset:16384\n"
    "s_add_u32 s66, s86, 4\n"
    "s_and_b32 s66, s66, 7\n"
    "v_mfma_f32_16x16x32_bf16 v[244:247], v[108:111], a[28:31], v[244:247]\n"
    "s_lshl_b32 s84, s66, 7\n"
    "s_add_u32 m0, s85, 114688\n"
    "s_add_u32 s84, s84, 0\n"
    "v_mfma_f32_16x16x32_bf16 v[212:215], v[108:111], a[20:23], v[212:215]\n"
    "ds_read_b64_tr_b16 v[98:99], v60 offset:17408\n"
    "v_mfma_f32_16x16x32_bf16 v[212:215], v[104:107], a[16:19], v[212:215]\n"
    "buffer_load_dwordx4 v32, s[44:47], s84 offen offset:0 lds\n"
    "v_mfma_f32_16x16x32_bf16 v[180:183], v[104:107], a[8:11], v[180:183]\n"
    "ds_read_b64_tr_b16 v[100:101], v60 offset:24576\n"
    "v_mfma_f32_16x16x32_bf16 v[180:183], v[108:111], a[12:15], v[180:183]\n"
    "v_mfma_f32_16x16x32_bf16 v[148:151], v[108:111], a[4:7], v[148:151]\n"
    "ds_read_b64_tr_b16 v[102:103], v60 offset:25600\n"
    "v_mfma_f32_16x16x32_bf16 v[148:151], v[104:107], a[0:3], v[148:151]\n"
    "s_waitcnt lgkmcnt(8)\n"
    "v_mfma_f32_16x16x32_bf16 v[152:155], v[112:115], a[0:3], v[152:155]\n"
    "ds_read_b64_tr_b16 v[104:105], v61 offset:16384\n"
    "v_mfma_f32_16x16x32_bf16 v[152:155], v[116:119], a[4:7], v[152:155]\n"
    "buffer_load_dwordx4 v33, s[44:47], s84 offen offset:1024 lds\n"
    "v_mfma_f32_16x16x32_bf16 v[184:187], v[116:119], a[12:15], v[184:187]\n"
    "ds_read_b64_tr_b16 v[106:107], v61 offset:17408\n"
    "v_mfma_f32_16x16x32_bf16 v[184:187], v[112:115], a[8:11], v[184:187]\n"
    "v_mfma_f32_16x16x32_bf16 v[216:219], v[112:115], a[16:19], v[216:219]\n"
    "ds_read_b64_tr_b16 v[108:109], v61 offset:24576\n"
    "v_mfma_f32_16x16x32_bf16 v[216:219], v[116:119], a[20:23], v[216:219]\n"
    "v_mfma_f32_16x16x32_bf16 v[248:251], v[116:119], a[28:31], v[248:251]\n"
    "ds_read_b64_tr_b16 v[110:111], v61 offset:25600\n"
    "v_mfma_f32_16x16x32_bf16 v[248:251], v[112:115], a[24:27], v[248:251]\n"
    "buffer_load_dwordx4 v34, s[44:47], s84 offen offset:2048 lds\n"
    "s_waitcnt lgkmcnt(8)\n"
    "v_mfma_f32_16x16x32_bf16 v[252:255], v[120:123], a[24:27], v[252:255]\n"
    "ds_read_b64_tr_b16 v[112:113], v62 offset:16384\n"
    "v_mfma_f32_16x16x32_bf16 v[252:255], v[124:127], a[28:31], v[252:255]\n"
    "v_mfma_f32_16x16x32_bf16 v[220:223], v[124:127], a[20:23], v[220:223]\n"
    "ds_read_b64_tr_b16 v[114:115], v62 offset:17408\n"
    "v_mfma_f32_16x16x32_bf16 v[220:223], v[120:123], a[16:19], v[220:223]\n"
    "v_mfma_f32_16x16x32_bf16 v[188:191], v[120:123], a[8:11], v[188:191]\n"
    "ds_read_b64_tr_b16 v[116:117], v62 offset:24576\n"
    "v_mfma_f32_16x16x32_bf16 v[188:191], v[124:127], a[12:15], v[188:191]\n"
    "buffer_load_dwordx4 v35, s[44:47], s84 offen offset:3072 lds\n"
    "v_mfma_f32_16x16x32_bf16 v[156:159], v[124:127], a[4:7], v[156:159]\n"
    "ds_read_b64_tr_b16 v[118:119], v62 offset:25600\n"
    "v_mfma_f32_16x16x32_bf16 v[156:159], v[120:123], a[0:3], v[156:159]\n"
    "s_waitcnt lgkmcnt(8)\n"
    "v_mfma_f32_16x16x32_bf16 v[128:131], v[96:99], a[32:35], v[128:131]\n"
    "ds_read_b64_tr_b16 v[120:121], v63 offset:16384\n"
    "v_mfma_f32_16x16x32_bf16 v[128:131], v[100:103], a[36:39], v[128:131]\n"
    "v_mfma_f32_16x16x32_bf16 v[160:163], v[100:103], a[44:47], v[160:163]\n"
    "ds_read_b64_tr_b16 v[122:123], v63 offset:17408\n"
    "v_mfma_f32_16x16x32_bf16 v[160:163], v[96:99], a[40:43], v[160:163]\n"
    "v_mfma_f32_16x16x32_bf16 v[192:195], v[96:99], a[48:51], v[192:195]\n"
    "ds_read_b64_tr_b16 v[124:125], v63 offset:24576\n"
    "s_waitcnt vmcnt(25)\n"
    "ds_read_b128 a[64:67], v30 offset:0\n"
    "v_mfma_f32_16x16x32_bf16 v[192:195], v[100:103], a[52:55], v[192:195]\n"
    "ds_read_b128 a[68:71], v31 offset:0\n"
    "ds_read_b128 a[72:75], v30 offset:2048\n"
    "ds_read_b128 a[76:79], v31 offset:2048\n"
    "v_mfma_f32_16x16x32_bf16 v[224:227], v[100:103], a[60:63], v[224:227]\n"
    "ds_read_b64_tr_b16 v[126:127], v63 offset:25600\n"
    "s_waitcnt vmcnt(16)\n"
    "ds_read_b128 a[80:83], v28 offset:0\n"
    "v_mfma_f32_16x16x32_bf16 v[224:227], v[96:99], a[56:59], v[224:227]\n"
    "ds_read_b128 a[84:87], v29 offset:0\n"
    "ds_read_b128 a[88:91], v28 offset:2048\n"
    "ds_read_b128 a[92:95], v29 offset:2048\n"
    "s_waitcnt lgkmcnt(15)\n"
    "v_mfma_f32_16x16x32_bf16 v[228:231], v[104:107], a[56:59], v[228:231]\n"
    "ds_read_b64_tr_b16 v[96:97], v60 offset:16896\n"
    "v_mfma_f32_16x16x32_bf16 v[228:231], v[108:111], a[60:63], v[228:231]\n"
    "s_mov_b32 s25, 196666\n"
    "s_getpc_b64 s[28:29]\n"
    "s_cmp_ge_i32 s25, s24\n"
    "s_cbranch_scc1 90b\n"
    "v_mfma_f32_16x16x32_bf16 v[196:199], v[108:111], a[52:55], v[196:199]\n"
    "ds_read_b64_tr_b16 v[98:99], v60 offset:17920\n"
    "v_mfma_f32_16x16x32_bf16 v[196:199], v[104:107], a[48:51], v[196:199]\n"
    "v_mfma_f32_16x16x32_bf16 v[164:167], v[104:107], a[40:43], v[164:167]\n"
    "ds_read_b64_tr_b16 v[100:101], v60 offset:25088\n"
    "v_mfma_f32_16x16x32_bf16 v[164:167], v[108:111], a[44:47], v[164:167]\n"
    "v_mfma_f32_16x16x32_bf16 v[132:135], v[108:111], a[36:39], v[132:135]\n"
    "ds_read_b64_tr_b16 v[102:103], v60 offset:26112\n"
    "v_mfma_f32_16x16x32_bf16 v[132:135], v[104:107], a[32:35], v[132:135]\n"
    "s_waitcnt lgkmcnt(15)\n"
    "v_mfma_f32_16x16x32_bf16 v[136:139], v[112:115], a[32:35], v[136:139]\n"
    "ds_read_b64_tr_b16 v[104:105], v61 offset:16896\n"
    "v_mfma_f32_16x16x32_bf16 v[136:139], v[116:119], a[36:39], v[136:139]\n"
    "v_mfma_f32_16x16x32_bf16 v[168:171], v[116:119], a[44:47], v[168:171]\n"
    "ds_read_b64_tr_b16 v[106:107], v61 offset:17920\n"
    "v_mfma_f32_16x16x32_bf16 v[168:171], v[112:115], a[40:43], v[168:171]\n"
    "v_mfma_f32_16x16x32_bf16 v[200:203], v[112:115], a[48:51], v[200:203]\n"
    "ds_read_b64_tr_b16 v[108:109], v61 offset:25088\n"
    "v_mfma_f32_16x16x32_bf16 v[200:203], v[116:119], a[52:55], v[200:203]\n"
    "v_mfma_f32_16x16x32_bf16 v[232:235], v[116:119], a[60:63], v[232:235]\n"
    "ds_read_b64_tr_b16 v[110:111], v61 offset:26112\n"
    "v_mfma_f32_16x16x32_bf16 v[232:235], v[112:115], a[56:59], v[232:235]\n"
    "s_waitcnt lgkmcnt(12)\n"
    "v_mfma_f32_16x16x32_bf16 v[236:239], v[120:123], a[56:59], v[236:239]\n"
    "ds_read_b64_tr_b16 v[112:113], v62 offset:16896\n"
    "v_mfma_f32_16x16x32_bf16 v[236:239], v[124:127], a[60:63], v[236:239]\n"
    "v_mfma_f32_16x16x32_bf16 v[204:207], v[124:127], a[52:55], v[204:207]\n"
    "ds_read_b64_tr_b16 v[114:115], v62 offset:17920\n"
    "v_mfma_f32_16x16x32_bf16 v[204:207], v[120:123], a[48:51], v[204:207]\n"
    "v_mfma_f32_16x16x32_bf16 v[172:175], v[120:123], a[40:43], v[172:175]\n"
    "ds_read_b64_tr_b16 v[116:117], v62 offset:25088\n"
    "v_mfma_f32_16x16x32_bf16 v[172:175], v[124:127], a[44:47], v[172:175]\n"
    "v_mfma_f32_16x16x32_bf16 v[140:143], v[124:127], a[36:39], v[140:143]\n"
    "ds_read_b64_tr_b16 v[118:119], v62 offset:26112\n"
    "v_mfma_f32_16x16x32_bf16 v[140:143], v[120:123], a[32:35], v[140:143]\n"
    "s_waitcnt lgkmcnt(8)\n"
    "v_mfma_f32_16x16x32_bf16 v[144:147], v[96:99], a[32:35], v[144:147]\n"
    "ds_read_b64_tr_b16 v[120:121], v63 offset:16896\n"
    "s_add_u32 s66, s86, 4\n"
    "s_and_b32 s66, s66, 7\n"
    "v_mfma_f32_16x16x32_bf16 v[144:147], v[100:103], a[36:39], v[144:147]\n"
    "s_lshl_b32 s84, s66, 7\n"
    "s_add_u32 m0, s85, 131072\n"
    "s_add_u32 s84, s84, 32768\n"
    "v_mfma_f32_16x16x32_bf16 v[176:179], v[100:103], a[44:47], v[176:179]\n"
    "ds_read_b64_tr_b16 v[122:123], v63 offset:17920\n"
    "buffer_load_dwordx4 v32, s[44:47], s84 offen offset:0 lds\n"
    "v_mfma_f32_16x16x32_bf16 v[176:179], v[96:99], a[40:43], v[176:179]\n"
    "buffer_load_dwordx4 v33, s[44:47], s84 offen offset:1024 lds\n"
    "v_mfma_f32_16x16x32_bf16 v[208:211], v[96:99], a[48:51], v[208:211]\n"
    "ds_read_b64_tr_b16 v[124:125], v63 offset:25088\n"
    "buffer_load_dwordx4 v34, s[44:47], s84 offen offset:2048 lds\n"
    "v_mfma_f32_16x16x32_bf16 v[208:211], v[100:103], a[52:55], v[208:211]\n"
    "s_mov_b32 s25, 262205\n"
    "s_getpc_b64 s[28:29]\n"
    "s_cmp_ge_i32 s25, s24\n"
    "s_cbranch_scc1 90b\n"
    "v_mfma_f32_16x16x32_bf16 v[240:243], v[100:103], a[60:63], v[240:243]\n"
    "ds_read_b64_tr_b16 v[126:127], v63 offset:26112\n"
    "v_mfma_f32_16x16x32_bf16 v[240:243], v[96:99], a[56:59], v[240:243]\n"
    "buffer_load_dwordx4 v35, s[44:47], s84 offen offset:3072 lds\n"
    "s_waitcnt lgkmcnt(8)\n"
    "s_waitcnt vmcnt(25)\n"
    "s_barrier\n"
    "v_mfma_f32_16x16x32_bf16 v[244:247], v[104:107], a[56:59], v[244:247]\n"
    "ds_read_b64_tr_b16 v[96:97], v60 offset:32768\n"
    "s_add_u32 s66, s86, 4\n"
    "s_and_b32 s66, s66, 7\n"
    "v_mfma_f32_16x16x32_bf16 v[244:247], v[108:111], a[60:63], v[244:247]\n"
    "s_lshl_b32 s64, s66, 21\n"
    "s_add_u32 m0, s67, 0\n"
    "s_add_u32 s64, s64, s87\n"
    "v_mfma_f32_16x16x32_bf16 v[212:215], v[108:111], a[52:55], v[212:215]\n"
    "ds_read_b64_tr_b16 v[98:99], v60 offset:33792\n"
    "v_mfma_f32_16x16x32_bf16 v[212:215], v[104:107], a[48:51], v[212:215]\n"
    "buffer_load_dwordx4 v56, s[36:39], s64 offen offset:0 lds\n"
    "v_mfma_f32_16x16x32_bf16 v[180:183], v[104:107], a[40:43], v[180:183]\n"
    "ds_read_b64_tr_b16 v[100:101], v60 offset:40960\n"
    "v_mfma_f32_16x16x32_bf16 v[180:183], v[108:111], a[44:47], v[180:183]\n"
    "buffer_load_dwordx4 v57, s[36:39], s64 offen offset:1024 lds\n"
    "v_mfma_f32_16x16x32_bf16 v[148:151], v[108:111], a[36:39], v[148:151]\n"
    "ds_read_b64_tr_b16 v[102:103], v60 offset:41984\n"
    "v_mfma_f32_16x16x32_bf16 v[148:151], v[104:107], a[32:35], v[148:151]\n"
    "s_waitcnt lgkmcnt(8)\n"
    "v_mfma_f32_16x16x32_bf16 v[152:155], v[112:115], a[32:35], v[152:155]\n"
    "ds_read_b64_tr_b16 v[104:105], v61 offset:32768\n"
    "v_mfma_f32_16x16x32_bf16 v[152:155], v[116:119], a[36:39], v[152:155]\n"
    "buffer_load_dwordx4 v58, s[36:39], s64 offen offset:2048 lds\n"
    "v_mfma_f32_16x16x32_bf16 v[184:187], v[116:119], a[44:47], v[184:187]\n"
    "ds_read_b64_tr_b16 v[106:107], v61 offset:33792\n"
    "v_mfma_f32_16x16x32_bf16 v[184:187], v[112:115], a[40:43], v[184:187]\n"
    "buffer_load_dwordx4 v59, s[36:39], s64 offen offset:3072 lds\n"
    "v_mfma_f32_16x16x32_bf16 v[216:219], v[112:115], a[48:51], v[216:219]\n"
    "ds_read_b64_tr_b16 v[108:109], v61 offset:40960\n"
    "s_add_u32 s66, s86, 5\n"
    "s_and_b32 s66, s66, 7\n"
    "v_mfma_f32_16x16x32_bf16 v[216:219], v[116:119], a[52:55], v[216:219]\n"
    "s_lshl_b32 s84, s66, 7\n"
    "s_add_u32 m0, s85, 65536\n"
    "s_add_u32 s84, s84, 0\n"
    "v_mfma_f32_16x16x32_bf16 v[248:251], v[116:119], a[60:63], v[248:251]\n"
    "ds_read_b64_tr_b16 v[110:111], v61 offset:41984\n"
    "v_mfma_f32_16x16x32_bf16 v[248:251], v[112:115], a[56:59], v[248:251]\n"
    "buffer_load_dwordx4 v32, s[44:47], s84 offen offset:0 lds\n"
    "s_waitcnt lgkmcnt(8)\n"
    "v_mfma_f32_16x16x32_bf16 v[252:255], v[120:123], a[56:59], v[252:255]\n"
    "ds_read_b64_tr_b16 v[112:113], v62 offset:32768\n"
    "v_mfma_f32_16x16x32_bf16 v[252:255], v[124:127], a[60:63], v[252:255]\n"
    "buffer_load_dwordx4 v33, s[44:47], s84 offen offset:1024 lds\n"
    "v_mfma_f32_16x16x32_bf16 v[220:223], v[124:127], a[52:55], v[220:223]\n"
    "ds_read_b64_tr_b16 v[114:115], v62 offset:33792\n"
    "v_mfma_f32_16x16x32_bf16 v[220:223], v[120:123], a[48:51], v[220:223]\n"
    "buffer_load_dwordx4 v34, s[44:47], s84 offen offset:2048 lds\n"
    "v_mfma_f32_16x16x32_bf16 v[188:191], v[120:123], a[40:43], v[188:191]\n"
    "ds_read_b64_tr_b16 v[116:117], v62 offset:40960\n"
    "v_mfma_f32_16x16x32_bf16 v[188:191], v[124:127], a[44:47], v[188:191]\n"
    "v_mfma_f32_16x16x32_bf16 v[156:159], v[124:127], a[36:39], v[156:159]\n"
    "ds_read_b64_tr_b16 v[118:119], v62 offset:41984\n"
    "v_mfma_f32_16x16x32_bf16 v[156:159], v[120:123], a[32:35], v[156:159]\n"
    "buffer_load_dwordx4 v35, s[44:47], s84 offen offset:3072 lds\n"
    "s_waitcnt lgkmcnt(8)\n"
    "v_mfma_f32_16x16x32_bf16 v[128:131], v[96:99], a[64:67], v[128:131]\n"
    "ds_read_b64_tr_b16 v[120:121], v63 offset:32768\n"
    "v_mfma_f32_16x16x32_bf16 v[128:131], v[100:103], a[68:71], v[128:131]\n"
    "v_mfma_f32_16x16x32_bf16 v[160:163], v[100:103], a[76:79], v[160:163]\n"
    "ds_read_b64_tr_b16 v[122:123], v63 offset:33792\n"
    "v_mfma_f32_16x16x32_bf16 v[160:163], v[96:99], a[72:75], v[160:163]\n"
    "v_mfma_f32_16x16x32_bf16 v[192:195], v[96:99], a[80:83], v[192:195]\n"
    "ds_read_b64_tr_b16 v[124:125], v63 offset:40960\n"
    "s_waitcnt vmcnt(24)\n"
    "ds_read_b128 a[96:99], v28 offset:16384\n"
    "v_mfma_f32_16x16x32_bf16 v[192:195], v[100:103], a[84:87], v[192:195]\n"
    "ds_read_b128 a[100:103], v29 offset:16384\n"
    "ds_read_b128 a[104:107], v28 offset:18432\n"
    "ds_read_b128 a[108:111], v29 offset:18432\n"
    "v_mfma_f32_16x16x32_bf16 v[224:227], v[100:103], a[92:95], v[224:227]\n"
    "ds_read_b64_tr_b16 v[126:127], v63 offset:41984\n"
    "s_waitcnt vmcnt(16)\n"
    "ds_read_b128 a[112:115], v28 offset:32768\n"
    "v_mfma_f32_16x16x32_bf16 v[224:227], v[96:99], a[88:91], v[224:227]\n"
    "ds_read_b128 a[116:119], v29 offset:32768\n"
    "ds_read_b128 a[120:123], v28 offset:34816\n"
    "ds_read_b128 a[124:127], v29 offset:34816\n"
    "s_waitcnt lgkmcnt(15)\n"
    "v_mfma_f32_16x16x32_bf16 v[228:231], v[104:107], a[88:91], v[228:231]\n"
    "ds_read_b64_tr_b16 v[96:97], v60 offset:33280\n"
    "v_mfma_f32_16x16x32_bf16 v[228:231], v[108:111], a[92:95], v[228:231]\n"
    "s_mov_b32 s25, 327750\n"
    "s_getpc_b64 s[28:29]\n"
    "s_cmp_ge_i32 s25, s24\n"
    "s_cbranch_scc1 90b\n"
    "v_mfma_f32_16x16x32_bf16 v[196:199], v[108:111], a[84:87], v[196:199]\n"
    "ds_read_b64_tr_b16 v[98:99], v60 offset:34304\n"
    "v_mfma_f32_16x16x32_bf16 v[196:199], v[104:107], a[80:83], v[196:199]\n"
    "v_mfma_f32_16x16x32_bf16 v[164:167], v[104:107], a[72:75], v[164:167]\n"
    "ds_read_b64_tr_b16 v[100:101], v60 offset:41472\n"
    "v_mfma_f32_16x16x32_bf16 v[164:167], v[108:111], a[76:79], v[164:167]\n"
    "v_mfma_f32_16x16x32_bf16 v[132:135], v[108:111], a[68:71], v[132:135]\n"
    "ds_read_b64_tr_b16 v[102:103], v60 offset:42496\n"
    "v_mfma_f32_16x16x32_bf16 v[132:135], v[104:107], a[64:67], v[132:135]\n"
    "s_waitcnt lgkmcnt(15)\n"
    "v_mfma_f32_16x16x32_bf16 v[136:139], v[112:115], a[64:67], v[136:139]\n"
    "ds_read_b64_tr_b16 v[104:105], v61 offset:33280\n"
    "v_mfma_f32_16x16x32_bf16 v[136:139], v[116:119], a[68:71], v[136:139]\n"
    "v_mfma_f32_16x16x32_bf16 v[168:171], v[116:119], a[76:79], v[168:171]\n"
    "ds_read_b64_tr_b16 v[106:107], v61 offset:34304\n"
    "v_mfma_f32_16x16x32_bf16 v[168:171], v[112:115], a[72:75], v[168:171]\n"
    "v_mfma_f32_16x16x32_bf16 v[200:203], v[112:115], a[80:83], v[200:203]\n"
    "ds_read_b64_tr_b16 v[108:109], v61 offset:41472\n"
    "v_mfma_f32_16x16x32_bf16 v[200:203], v[116:119], a[84:87], v[200:203]\n"
    "v_mfma_f32_16x16x32_bf16 v[232:235], v[116:119], a[92:95], v[232:235]\n"
    "ds_read_b64_tr_b16 v[110:111], v61 offset:42496\n"
    "v_mfma_f32_16x16x32_bf16 v[232:235], v[112:115], a[88:91], v[232:235]\n"
    "s_waitcnt lgkmcnt(12)\n"
    "v_mfma_f32_16x16x32_bf16 v[236:239], v[120:123], a[88:91], v[236:239]\n"
    "ds_read_b64_tr_b16 v[112:113], v62 offset:33280\n"
    "v_mfma_f32_16x16x32_bf16 v[236:239], v[124:127], a[92:95], v[236:239]\n"
    "v_mfma_f32_16x16x32_bf16 v[204:207], v[124:127], a[84:87], v[204:207]\n"
    "ds_read_b64_tr_b16 v[114:115], v62 offset:34304\n"
    "v_mfma_f32_16x16x32_bf16 v[204:207], v[120:123], a[80:83], v[204:207]\n"
    "v_mfma_f32_16x16x32_bf16 v[172:175], v[120:123], a[72:75], v[172:175]\n"
    "ds_read_b64_tr_b16 v[116:117], v62 offset:41472\n"
    "v_mfma_f32_16x16x32_bf16 v[172:175], v[124:127], a[76:79], v[172:175]\n"
    "v_mfma_f32_16x16x32_bf16 v[140:143], v[124:127], a[68:71], v[140:143]\n"
    "ds_read_b64_tr_b16 v[118:119], v62 offset:42496\n"
    "v_mfma_f32_16x16x32_bf16 v[140:143], v[120:123], a[64:67], v[140:143]\n"
    "s_waitcnt lgkmcnt(8)\n"
    "v_mfma_f32_16x16x32_bf16 v[144:147], v[96:99], a[64:67], v[144:147]\n"
    "ds_read_b64_tr_b16 v[120:121], v63 offset:33280\n"
    "s_add_u32 s66, s86, 5\n"
    "s_and_b32 s66, s66, 7\n"
    "v_mfma_f32_16x16x32_bf16 v[144:147], v[100:103], a[68:71], v[144:147]\n"
    "s_lshl_b32 s84, s66, 7\n"
    "s_add_u32 m0, s85, 81920\n"
    "s_add_u32 s84, s84, 32768\n"
    "v_mfma_f32_16x16x32_bf16 v[176:179], v[100:103], a[76:79], v[176:179]\n"
    "ds_read_b64_tr_b16 v[122:123], v63 offset:34304\n"
    "buffer_load_dwordx4 v32, s[44:47], s84 offen offset:0 lds\n"
    "v_mfma_f32_16x16x32_bf16 v[176:179], v[96:99], a[72:75], v[176:179]\n"
    "buffer_load_dwordx4 v33, s[44:47], s84 offen offset:1024 lds\n"
    "v_mfma_f32_16x16x32_bf16 v[208:211], v[96:99], a[80:83], v[208:211]\n"
    "ds_read_b64_tr_b16 v[124:125], v63 offset:41472\n"
    "buffer_load_dwordx4 v34, s[44:47], s84 offen offset:2048 lds\n"
    "v_mfma_f32_16x16x32_bf16 v[208:211], v[100:103], a[84:87], v[208:211]\n"
    "s_mov_b32 s25, 393289\n"
    "s_getpc_b64 s[28:29]\n"
    "s_cmp_ge_i32 s25, s24\n"
    "s_cbranch_scc1 90b\n"
    "v_mfma_f32_16x16x32_bf16 v[240:243], v[100:103], a[92:95], v[240:243]\n"
    "ds_read_b64_tr_b16 v[126:127], v63 offset:42496\n"
    "v_mfma_f32_16x16x32_bf16 v[240:243], v[96:99], a[88:91], v[240:243]\n"
    "buffer_load_dwordx4 v35, s[44:47], s84 offen offset:3072 lds\n"
    "s_waitcnt lgkmcnt(8)\n"
    "s_waitcnt vmcnt(24)\n"
    "s_barrier\n"
    "v_mfma_f32_16x16x32_bf16 v[244:247], v[104:107], a[88:91], v[244:247]\n"
    "ds_read_b64_tr_b16 v[96:97], v60 offset:49152\n"
    "s_add_u32 s66, s86, 5\n"
    "s_and_b32 s66, s66, 7\n"
    "v_mfma_f32_16x16x32_bf16 v[244:247], v[108:111], a[92:95], v[244:247]\n"
    "s_lshl_b32 s64, s66, 21\n"
    "s_add_u32 m0, s67, 16384\n"
    "s_add_u32 s64, s64, s87\n"
    "v_mfma_f32_16x16x32_bf16 v[212:215], v[108:111], a[84:87], v[212:215]\n"
    "ds_read_b64_tr_b16 v[98:99], v60 offset:50176\n"
    "v_mfma_f32_16x16x32_bf16 v[212:215], v[104:107], a[80:83], v[212:215]\n"
    "buffer_load_dwordx4 v56, s[36:39], s64 offen offset:0 lds\n"
    "v_mfma_f32_16x16x32_bf16 v[180:183], v[104:107], a[72:75], v[180:183]\n"
    "ds_read_b64_tr_b16 v[100:101], v60 offset:57344\n"
    "v_mfma_f32_16x16x32_bf16 v[180:183], v[108:111], a[76:79], v[180:183]\n"
    "buffer_load_dwordx4 v57, s[36:39], s64 offen offset:1024 lds\n"
    "v_mfma_f32_16x16x32_bf16 v[148:151], v[108:111], a[68:71], v[148:151]\n"
    "ds_read_b64_tr_b16 v[102:103], v60 offset:58368\n"
    "v_mfma_f32_16x16x32_bf16 v[148:151], v[104:107], a[64:67], v[148:151]\n"
    "s_waitcnt lgkmcnt(8)\n"
    "v_mfma_f32_16x16x32_bf16 v[152:155], v[112:115], a[64:67], v[152:155]\n"
    "ds_read_b64_tr_b16 v[104:105], v61 offset:49152\n"
    "v_mfma_f32_16x16x32_bf16 v[152:155], v[116:119], a[68:71], v[152:155]\n"
    "buffer_load_dwordx4 v58, s[36:39], s64 offen offset:2048 lds\n"
    "v_mfma_f32_16x16x32_bf16 v[184:187], v[116:119], a[76:79], v[184:187]\n"
    "ds_read_b64_tr_b16 v[106:107], v61 offset:50176\n"
    "v_mfma_f32_16x16x32_bf16 v[184:187], v[112:115], a[72:75], v[184:187]\n"
    "buffer_load_dwordx4 v59, s[36:39], s64 offen offset:3072 lds\n"
    "v_mfma_f32_16x16x32_bf16 v[216:219], v[112:115], a[80:83], v[216:219]\n"
    "ds_read_b64_tr_b16 v[108:109], v61 offset:57344\n"
    "s_add_u32 s66, s86, 6\n"
    "s_and_b32 s66, s66, 7\n"
    "v_mfma_f32_16x16x32_bf16 v[216:219], v[116:119], a[84:87], v[216:219]\n"
    "s_lshl_b32 s84, s66, 7\n"
    "s_add_u32 m0, s85, 98304\n"
    "s_add_u32 s84, s84, 0\n"
    "v_mfma_f32_16x16x32_bf16 v[248:251], v[116:119], a[92:95], v[248:251]\n"
    "ds_read_b64_tr_b16 v[110:111], v61 offset:58368\n"
    "v_mfma_f32_16x16x32_bf16 v[248:251], v[112:115], a[88:91], v[248:251]\n"
    "buffer_load_dwordx4 v32, s[44:47], s84 offen offset:0 lds\n"
    "s_waitcnt lgkmcnt(8)\n"
    "v_mfma_f32_16x16x32_bf16 v[252:255], v[120:123], a[88:91], v[252:255]\n"
    "ds_read_b64_tr_b16 v[112:113], v62 offset:49152\n"
    "v_mfma_f32_16x16x32_bf16 v[252:255], v[124:127], a[92:95], v[252:255]\n"
    "buffer_load_dwordx4 v33, s[44:47], s84 offen offset:1024 lds\n"
    "v_mfma_f32_16x16x32_bf16 v[220:223], v[124:127], a[84:87], v[220:223]\n"
    "ds_read_b64_tr_b16 v[114:115], v62 offset:50176\n"
    "v_mfma_f32_16x16x32_bf16 v[220:223], v[120:123], a[80:83], v[220:223]\n"
    "buffer_load_dwordx4 v34, s[44:47], s84 offen offset:2048 lds\n"
    "v_mfma_f32_16x16x32_bf16 v[188:191], v[120:123], a[72:75], v[188:191]\n"
    "ds_read_b64_tr_b16 v[116:117], v62 offset:57344\n"
    "v_mfma_f32_16x16x32_bf16 v[188:191], v[124:127], a[76:79], v[188:191]\n"
    "v_mfma_f32_16x16x32_bf16 v[156:159], v[124:127], a[68:71], v[156:159]\n"
    "ds_read_b64_tr_b16 v[118:119], v62 offset:58368\n"
    "v_mfma_f32_16x16x32_bf16 v[156:159], v[120:123], a[64:67], v[156:159]\n"
    "buffer_load_dwordx4 v35, s[44:47], s84 offen offset:3072 lds\n"
    "s_waitcnt lgkmcnt(8)\n"
    "v_mfma_f32_16x16x32_bf16 v[128:131], v[96:99], a[96:99], v[128:131]\n"
    "ds_read_b64_tr_b16 v[120:121], v63 offset:49152\n"
    "v_mfma_f32_16x16x32_bf16 v[128:131], v[100:103], a[100:103], v[128:131]\n"
    "v_mfma_f32_16x16x32_bf16 v[160:163], v[100:103], a[108:111], v[160:163]\n"
    "ds_read_b64_tr_b16 v[122:123], v63 offset:50176\n"
    "v_mfma_f32_16x16x32_bf16 v[160:163], v[96:99], a[104:107], v[160:163]\n"
    "v_mfma_f32_16x16x32_bf16 v[192:195], v[96:99], a[112:115], v[192:195]\n"
    "ds_read_b64_tr_b16 v[124:125], v63 offset:57344\n"
    "s_waitcnt vmcnt(24)\n"
    "ds_read_b128 a[128:131], v28 offset:49152\n"
    "v_mfma_f32_16x16x32_bf16 v[192:195], v[100:103], a[116:119], v[192:195]\n"
    "ds_read_b128 a[132:135], v29 offset:49152\n"
    "ds_read_b128 a[136:139], v28 offset:51200\n"
    "ds_read_b128 a[140:143], v29 offset:51200\n"
    "v_mfma_f32_16x16x32_bf16 v[224:227], v[100:103], a[124:127], v[224:227]\n"
    "ds_read_b64_tr_b16 v[126:127], v63 offset:58368\n"
    "s_waitcnt vmcnt(20)\n"
    "ds_read_b128 a[144:147], v30 offset:0\n"
    "v_mfma_f32_16x16x32_bf16 v[224:227], v[96:99], a[120:123], v[224:227]\n"
    "ds_read_b128 a[148:151], v31 offset:0\n"
    "ds_read_b128 a[152:155], v30 offset:2048\n"
    "ds_read_b128 a[156:159], v31 offset:2048\n"
    "s_waitcnt lgkmcnt(15)\n"
    "v_mfma_f32_16x16x32_bf16 v[228:231], v[104:107], a[120:123], v[228:231]\n"
    "ds_read_b64_tr_b16 v[96:97], v60 offset:49664\n"
    "v_mfma_f32_16x16x32_bf16 v[228:231], v[108:111], a[124:127], v[228:231]\n"
    "s_mov_b32 s25, 458834\n"
    "s_getpc_b64 s[28:29]\n"
    "s_cmp_ge_i32 s25, s24\n"
    "s_cbranch_scc1 90b\n"
    "v_mfma_f32_16x16x32_bf16 v[196:199], v[108:111], a[116:119], v[196:199]\n"
    "ds_read_b64_tr_b16 v[98:99], v60 offset:50688\n"
    "v_mfma_f32_16x16x32_bf16 v[196:199], v[104:107], a[112:115], v[196:199]\n"
    "v_mfma_f32_16x16x32_bf16 v[164:167], v[104:107], a[104:107], v[164:167]\n"
    "ds_read_b64_tr_b16 v[100:101], v60 offset:57856\n"
    "v_mfma_f32_16x16x32_bf16 v[164:167], v[108:111], a[108:111], v[164:167]\n"
    "v_mfma_f32_16x16x32_bf16 v[132:135], v[108:111], a[100:103], v[132:135]\n"
    "ds_read_b64_tr_b16 v[102:103], v60 offset:58880\n"
    "v_mfma_f32_16x16x32_bf16 v[132:135], v[104:107], a[96:99], v[132:135]\n"
    "s_waitcnt lgkmcnt(15)\n"
    "v_mfma_f32_16x16x32_bf16 v[136:139], v[112:115], a[96:99], v[136:139]\n"
    "ds_read_b64_tr_b16 v[104:105], v61 offset:49664\n"
    "v_mfma_f32_16x16x32_bf16 v[136:139], v[116:119], a[100:103], v[136:139]\n"
    "v_mfma_f32_16x16x32_bf16 v[168:171], v[116:119], a[108:111], v[168:171]\n"
    "ds_read_b64_tr_b16 v[106:107], v61 offset:50688\n"
    "v_mfma_f32_16x16x32_bf16 v[168:171], v[112:115], a[104:107], v[168:171]\n"
    "v_mfma_f32_16x16x32_bf16 v[200:203], v[112:115], a[112:115], v[200:203]\n"
    "ds_read_b64_tr_b16 v[108:109], v61 offset:57856\n"
    "v_mfma_f32_16x16x32_bf16 v[200:203], v[116:119], a[116:119], v[200:203]\n"
    "v_mfma_f32_16x16x32_bf16 v[232:235], v[116:119], a[124:127], v[232:235]\n"
    "ds_read_b64_tr_b16 v[110:111], v61 offset:58880\n"
    "v_mfma_f32_16x16x32_bf16 v[232:235], v[112:115], a[120:123], v[232:235]\n"
    "s_ff0_i32_b64 s30, s[82:83]\n"
    "s_bfm_b64 vcc, s30, 0\n"
    "s_andn2_b64 vcc, vcc, s[80:81]\n"
    "s_bcnt1_i32_b64 s30, vcc\n"
    "s_cmp_lt_u32 s30, 64\n"
    "s_cselect_b32 s22, 0x10000, 0\n"
    "s_waitcnt lgkmcnt(12)\n"
    "v_mfma_f32_16x16x32_bf16 v[236:239], v[120:123], a[120:123], v[236:239]\n"
    "ds_read_b64_tr_b16 v[112:113], v62 offset:49664\n"
    "v_mfma_f32_16x16x32_bf16 v[236:239], v[124:127], a[124:127], v[236:239]\n"
    "s_bcnt1_i32_b64 s30, s[82:83]\n"
    "s_cmp_ge_u32 s30, 56\n"
    "s_cselect_b32 s22, 0, s22\n"
    "v_mfma_f32_16x16x32_bf16 v[204:207], v[124:127], a[116:119], v[204:207]\n"
    "ds_read_b64_tr_b16 v[114:115], v62 offset:50688\n"
    "v_mfma_f32_16x16x32_bf16 v[204:207], v[120:123], a[112:115], v[204:207]\n"
    "buffer_load_dword v48, v49, s[20:23], 0 offen sc0 sc1\n"
    "v_mfma_f32_16x16x32_bf16 v[172:175], v[120:123], a[104:107], v[172:175]\n"
    "ds_read_b64_tr_b16 v[116:117], v62 offset:57856\n"
    "v_mfma_f32_16x16x32_bf16 v[172:175], v[124:127], a[108:111], v[172:175]\n"
    "v_mfma_f32_16x16x32_bf16 v[140:143], v[124:127], a[100:103], v[140:143]\n"
    "ds_read_b64_tr_b16 v[118:119], v62 offset:58880\n"
    "v_mfma_f32_16x16x32_bf16 v[140:143], v[120:123], a[96:99], v[140:143]\n"
    "s_waitcnt lgkmcnt(8)\n"
    "v_mfma_f32_16x16x32_bf16 v[144:147], v[96:99], a[96:99], v[144:147]\n"
    "ds_read_b64_tr_b16 v[120:121], v63 offset:49664\n"
    "s_add_u32 s66, s86, 6\n"
    "s_and_b32 s66, s66, 7\n"
    "v_mfma_f32_16x16x32_bf16 v[144:147], v[100:103], a[100:103], v[144:147]\n"
    "s_lshl_b32 s84, s66, 7\n"
    "s_add_u32 m0, s85, 114688\n"
    "s_add_u32 s84, s84, 32768\n"
    "v_mfma_f32_16x16x32_bf16 v[176:179], v[100:103], a[108:111], v[176:179]\n"
    "ds_read_b64_tr_b16 v[122:123], v63 offset:50688\n"
    "buffer_load_dwordx4 v32, s[44:47], s84 offen offset:0 lds\n"
    "v_mfma_f32_16x16x32_bf16 v[176:179], v[96:99], a[104:107], v[176:179]\n"
    "buffer_load_dwordx4 v33, s[44:47], s84 offen offset:1024 lds\n"
    "v_mfma_f32_16x16x32_bf16 v[208:211], v[96:99], a[112:115], v[208:211]\n"
    "ds_read_b64_tr_b16 v[124:125], v63 offset:57856\n"
    "buffer_load_dwordx4 v34, s[44:47], s84 offen offset:2048 lds\n"
    "v_mfma_f32_16x16x32_bf16 v[208:211], v[100:103], a[116:119], v[208:211]\n"
    "s_mov_b32 s25, 524374\n"
    "s_getpc_b64 s[28:29]\n"
    "s_cmp_ge_i32 s25, s24\n"
    "s_cbranch_scc1 90b\n"
    "v_mfma_f32_16x16x32_bf16 v[240:243], v[100:103], a[124:127], v[240:243]\n"
    "ds_read_b64_tr_b16 v[126:127], v63 offset:58880\n"
    "v_mfma_f32_16x16x32_bf16 v[240:243], v[96:99], a[120:123], v[240:243]\n"
    "buffer_load_dwordx4 v35, s[44:47], s84 offen offset:3072 lds\n"
    "s_waitcnt lgkmcnt(8)\n"
    "s_waitcnt vmcnt(21)\n"
    "s_barrier\n"
    "v_mfma_f32_16x16x32_bf16 v[244:247], v[104:107], a[120:123], v[244:247]\n"
    "ds_read_b64_tr_b16 v[96:97], v60 offset:0\n"
    "s_add_u32 s66, s86, 6\n"
    "s_and_b32 s66, s66, 7\n"
    "v_mfma_f32_16x16x32_bf16 v[244:247], v[108:111], a[124:127], v[244:247]\n"
    "s_lshl_b32 s64, s66, 21\n"
    "s_add_u32 m0, s67, 32768\n"
    "s_add_u32 s64, s64, s87\n"
    "v_mfma_f32_16x16x32_bf16 v[212:215], v[108:111], a[116:119], v[212:215]\n"
    "ds_read_b64_tr_b16 v[98:99], v60 offset:1024\n"
    "v_mfma_f32_16x16x32_bf16 v[212:215], v[104:107], a[112:115], v[212:215]\n"
    "buffer_load_dwordx4 v56, s[36:39], s64 offen offset:0 lds\n"
    "v_mfma_f32_16x16x32_bf16 v[180:183], v[104:107], a[104:107], v[180:183]\n"
    "ds_read_b64_tr_b16 v[100:101], v60 offset:8192\n"
    "v_mfma_f32_16x16x32_bf16 v[180:183], v[108:111], a[108:111], v[180:183]\n"
    "buffer_load_dwordx4 v57, s[36:39], s64 offen offset:1024 lds\n"
    "v_mfma_f32_16x16x32_bf16 v[148:151], v[108:111], a[100:103], v[148:151]\n"
    "ds_read_b64_tr_b16 v[102:103], v60 offset:9216\n"
    "v_mfma_f32_16x16x32_bf16 v[148:151], v[104:107], a[96:99], v[148:151]\n"
    "s_waitcnt lgkmcnt(8)\n"
    "v_mfma_f32_16x16x32_bf16 v[152:155], v[112:115], a[96:99], v[152:155]\n"
    "ds_read_b64_tr_b16 v[104:105], v61 offset:0\n"
    "v_mfma_f32_16x16x32_bf16 v[152:155], v[116:119], a[100:103], v[152:155]\n"
    "buffer_load_dwordx4 v58, s[36:39], s64 offen offset:2048 lds\n"
    "v_mfma_f32_16x16x32_bf16 v[184:187], v[116:119], a[108:111], v[184:187]\n"
    "ds_read_b64_tr_b16 v[106:107], v61 offset:1024\n"
    "v_mfma_f32_16x16x32_bf16 v[184:187], v[112:115], a[104:107], v[184:187]\n"
    "buffer_load_dwordx4 v59, s[36:39], s64 offen offset:3072 lds\n"
    "v_mfma_f32_16x16x32_bf16 v[216:219], v[112:115], a[112:115], v[216:219]\n"
    "ds_read_b64_tr_b16 v[108:109], v61 offset:8192\n"
    "s_add_u32 s66, s86, 7\n"
    "s_and_b32 s66, s66, 7\n"
    "v_mfma_f32_16x16x32_bf16 v[216:219], v[116:119], a[116:119], v[216:219]\n"
    "s_lshl_b32 s84, s66, 7\n"
    "s_add_u32 m0, s85, 131072\n"
    "s_add_u32 s84, s84, 0\n"
    "v_mfma_f32_16x16x32_bf16 v[248:251], v[116:119], a[124:127], v[248:251]\n"
    "ds_read_b64_tr_b16 v[110:111], v61 offset:9216\n"
    "v_mfma_f32_16x16x32_bf16 v[248:251], v[112:115], a[120:123], v[248:251]\n"
    "buffer_load_dwordx4 v32, s[44:47], s84 offen offset:0 lds\n"
    "s_waitcnt lgkmcnt(8)\n"
    "v_mfma_f32_16x16x32_bf16 v[252:255], v[120:123], a[120:123], v[252:255]\n"
    "ds_read_b64_tr_b16 v[112:113], v62 offset:0\n"
    "v_mfma_f32_16x16x32_bf16 v[252:255], v[124:127], a[124:127], v[252:255]\n"
    "buffer_load_dwordx4 v33, s[44:47], s84 offen offset:1024 lds\n"
    "v_mfma_f32_16x16x32_bf16 v[220:223], v[124:127], a[116:119], v[220:223]\n"
    "ds_read_b64_tr_b16 v[114:115], v62 offset:1024\n"
    "v_mfma_f32_16x16x32_bf16 v[220:223], v[120:123], a[112:115], v[220:223]\n"
    "buffer_load_dwordx4 v34, s[44:47], s84 offen offset:2048 lds\n"
    "v_mfma_f32_16x16x32_bf16 v[188:191], v[120:123], a[104:107], v[188:191]\n"
    "ds_read_b64_tr_b16 v[116:117], v62 offset:8192\n"
    "v_mfma_f32_16x16x32_bf16 v[188:191], v[124:127], a[108:111], v[188:191]\n"
    "v_mfma_f32_16x16x32_bf16 v[156:159], v[124:127], a[100:103], v[156:159]\n"
    "ds_read_b64_tr_b16 v[118:119], v62 offset:9216\n"
    "v_mfma_f32_16x16x32_bf16 v[156:159], v[120:123], a[96:99], v[156:159]\n"
    "buffer_load_dwordx4 v35, s[44:47], s84 offen offset:3072 lds\n"
    "s_waitcnt lgkmcnt(8)\n"
    "v_mfma_f32_16x16x32_bf16 v[128:131], v[96:99], a[128:131], v[128:131]\n"
    "ds_read_b64_tr_b16 v[120:121], v63 offset:0\n"
    "v_mfma_f32_16x16x32_bf16 v[128:131], v[100:103], a[132:135], v[128:131]\n"
    "v_mfma_f32_16x16x32_bf16 v[160:163], v[100:103], a[140:143], v[160:163]\n"
    "ds_read_b64_tr_b16 v[122:123], v63 offset:1024\n"
    "v_mfma_f32_16x16x32_bf16 v[160:163], v[96:99], a[136:139], v[160:163]\n"
    "v_mfma_f32_16x16x32_bf16 v[192:195], v[96:99], a[144:147], v[192:195]\n"
    "ds_read_b64_tr_b16 v[124:125], v63 offset:8192\n"
    "s_waitcnt vmcnt(25)\n"
    "ds_read_b128 a[160:163], v28 offset:0\n"
    "v_mfma_f32_16x16x32_bf16 v[192:195], v[100:103], a[148:151], v[192:195]\n"
    "ds_read_b128 a[164:167], v29 offset:0\n"
    "ds_read_b128 a[168:171], v28 offset:2048\n"
    "ds_read_b128 a[172:175], v29 offset:2048\n"
    "v_mfma_f32_16x16x32_bf16 v[224:227], v[100:103], a[156:159], v[224:227]\n"
    "ds_read_b64_tr_b16 v[126:127], v63 offset:9216\n"
    "s_waitcnt vmcnt(21)\n"
    "ds_read_b128 a[176:179], v28 offset:16384\n"
    "v_mfma_f32_16x16x32_bf16 v[224:227], v[96:99], a[152:155], v[224:227]\n"
    "ds_read_b128 a[180:183], v29 offset:16384\n"
    "ds_read_b128 a[184:187], v28 offset:18432\n"
    "ds_read_b128 a[188:191], v29 offset:18432\n"
    "s_waitcnt lgkmcnt(15)\n"
    "v_mfma_f32_16x16x32_bf16 v[228:231], v[104:107], a[152:155], v[228:231]\n"
    "ds_read_b64_tr_b16 v[96:97], v60 offset:512\n"
    "v_mfma_f32_16x16x32_bf16 v[228:231], v[108:111], a[156:159], v[228:231]\n"
    "s_mov_b32 s25, 589919\n"
    "s_getpc_b64 s[28:29]\n"
    "s_cmp_ge_i32 s25, s24\n"
    "s_cbranch_scc1 90b\n"
    "v_mfma_f32_16x16x32_bf16 v[196:199], v[108:111], a[148:151], v[196:199]\n"
    "ds_read_b64_tr_b16 v[98:99], v60 offset:1536\n"
    "v_mfma_f32_16x16x32_bf16 v[196:199], v[104:107], a[144:147], v[196:199]\n"
    "v_mfma_f32_16x16x32_bf16 v[164:167], v[104:107], a[136:139], v[164:167]\n"
    "ds_read_b64_tr_b16 v[100:101], v60 offset:8704\n"
    "v_mfma_f32_16x16x32_bf16 v[164:167], v[108:111], a[140:143], v[164:167]\n"
    "v_cmp_le_u32 vcc, s70, v47\n"
    "v_mfma_f32_16x16x32_bf16 v[132:135], v[108:111], a[132:135], v[132:135]\n"
    "ds_read_b64_tr_b16 v[102:103], v60 offset:9728\n"
    "v_mfma_f32_16x16x32_bf16 v[132:135], v[104:107], a[128:131], v[132:135]\n"
    "s_or_b64 s[82:83], s[82:83], vcc\n"
    "v_cmp_le_u32 vcc, s70, v48\n"
    "s_waitcnt lgkmcnt(15)\n"
    "v_mfma_f32_16x16x32_bf16 v[136:139], v[112:115], a[128:131], v[136:139]\n"
    "ds_read_b64_tr_b16 v[104:105], v61 offset:512\n"
    "v_mfma_f32_16x16x32_bf16 v[136:139], v[116:119], a[132:135], v[136:139]\n"
    "s_or_b64 s[82:83], s[82:83], vcc\n"
    "v_mfma_f32_16x16x32_bf16 v[168:171], v[116:119], a[140:143], v[168:171]\n"
    "ds_read_b64_tr_b16 v[106:107], v61 offset:1536\n"
    "v_mfma_f32_16x16x32_bf16 v[168:171], v[112:115], a[136:139], v[168:171]\n"
    "v_mfma_f32_16x16x32_bf16 v[200:203], v[112:115], a[144:147], v[200:203]\n"
    "ds_read_b64_tr_b16 v[108:109], v61 offset:8704\n"
    "v_mfma_f32_16x16x32_bf16 v[200:203], v[116:119], a[148:151], v[200:203]\n"
    "v_mov_b32 v38, s82\n"
    "v_mov_b32 v39, s83\n"
    "ds_write_b64 v41, v[38:39] offset:0\n"
    "v_mfma_f32_16x16x32_bf16 v[232:235], v[116:119], a[156:159], v[232:235]\n"
    "ds_read_b64_tr_b16 v[110:111], v61 offset:9728\n"
    "v_mfma_f32_16x16x32_bf16 v[232:235], v[112:115], a[152:155], v[232:235]\n"
    "s_ff0_i32_b64 s30, s[82:83]\n"
    "s_bfm_b64 vcc, s30, 0\n"
    "s_andn2_b64 vcc, vcc, s[80:81]\n"
    "s_bcnt1_i32_b64 s30, vcc\n"
    "s_cmp_lt_u32 s30, 64\n"
    "s_cselect_b32 s22, 0x10000, 0\n"
    "s_waitcnt lgkmcnt(13)\n"
    "v_mfma_f32_16x16x32_bf16 v[236:239], v[120:123], a[152:155], v[236:239]\n"
    "ds_read_b64_tr_b16 v[112:113], v62 offset:512\n"
    "v_mfma_f32_16x16x32_bf16 v[236:239], v[124:127], a[156:159], v[236:239]\n"
    "s_bcnt1_i32_b64 s30, s[82:83]\n"
    "s_cmp_ge_u32 s30, 56\n"
    "s_cselect_b32 s22, 0, s22\n"
    "v_mfma_f32_16x16x32_bf16 v[204:207], v[124:127], a[148:151], v[204:207]\n"
    "ds_read_b64_tr_b16 v[114:115], v62 offset:1536\n"
    "v_mfma_f32_16x16x32_bf16 v[204:207], v[120:123], a[144:147], v[204:207]\n"
    "buffer_load_dword v47, v49, s[20:23], 0 offen sc0 sc1\n"
    "v_mfma_f32_16x16x32_bf16 v[172:175], v[120:123], a[136:139], v[172:175]\n"
    "ds_read_b64_tr_b16 v[116:117], v62 offset:8704\n"
    "v_mfma_f32_16x16x32_bf16 v[172:175], v[124:127], a[140:143], v[172:175]\n"
    "v_mfma_f32_16x16x32_bf16 v[140:143], v[124:127], a[132:135], v[140:143]\n"
    "ds_read_b64_tr_b16 v[118:119], v62 offset:9728\n"
    "v_mfma_f32_16x16x32_bf16 v[140:143], v[120:123], a[128:131], v[140:143]\n"
    "s_waitcnt lgkmcnt(9)\n"
    "v_mfma_f32_16x16x32_bf16 v[144:147], v[96:99], a[128:131], v[144:147]\n"
    "ds_read_b64_tr_b16 v[120:121], v63 offset:512\n"
    "s_add_u32 s66, s86, 7\n"
    "s_and_b32 s66, s66, 7\n"
    "v_mfma_f32_16x16x32_bf16 v[144:147], v[100:103], a[132:135], v[144:147]\n"
    "s_lshl_b32 s84, s66, 7\n"
    "s_add_u32 m0, s85, 65536\n"
    "s_add_u32 s84, s84, 32768\n"
    "v_mfma_f32_16x16x32_bf16 v[176:179], v[100:103], a[140:143], v[176:179]\n"
    "ds_read_b64_tr_b16 v[122:123], v63 offset:1536\n"
    "buffer_load_dwordx4 v32, s[44:47], s84 offen offset:0 lds\n"
    "v_mfma_f32_16x16x32_bf16 v[176:179], v[96:99], a[136:139], v[176:179]\n"
    "buffer_load_dwordx4 v33, s[44:47], s84 offen offset:1024 lds\n"
    "v_mfma_f32_16x16x32_bf16 v[208:211], v[96:99], a[144:147], v[208:211]\n"
    "ds_read_b64_tr_b16 v[124:125], v63 offset:8704\n"
    "buffer_load_dwordx4 v34, s[44:47], s84 offen offset:2048 lds\n"
    "v_mfma_f32_16x16x32_bf16 v[208:211], v[100:103], a[148:151], v[208:211]\n"
    "s_mov_b32 s25, 655459\n"
    "s_getpc_b64 s[28:29]\n"
    "s_cmp_ge_i32 s25, s24\n"
    "s_cbranch_scc1 90b\n"
    "v_mfma_f32_16x16x32_bf16 v[240:243], v[100:103], a[156:159], v[240:243]\n"
    "ds_read_b64_tr_b16 v[126:127], v63 offset:9728\n"
    "v_mfma_f32_16x16x32_bf16 v[240:243], v[96:99], a[152:155], v[240:243]\n"
    "buffer_load_dwordx4 v35, s[44:47], s84 offen offset:3072 lds\n"
    "s_waitcnt lgkmcnt(8)\n"
    "s_waitcnt vmcnt(22)\n"
    "s_barrier\n"
    "v_mfma_f32_16x16x32_bf16 v[244:247], v[104:107], a[152:155], v[244:247]\n"
    "ds_read_b64_tr_b16 v[96:97], v60 offset:16384\n"
    "s_add_u32 s66, s86, 7\n"
    "s_and_b32 s66, s66, 7\n"
    "v_mfma_f32_16x16x32_bf16 v[244:247], v[108:111], a[156:159], v[244:247]\n"
    "ds_read_b64 v[36:37], v40 offset:0\n"
    "v_mfma_f32_16x16x32_bf16 v[212:215], v[108:111], a[148:151], v[212:215]\n"
    "ds_read_b64_tr_b16 v[98:99], v60 offset:17408\n"
    "s_lshl_b32 s64, s66, 21\n"
    "s_add_u32 m0, s67, 49152\n"
    "v_mfma_f32_16x16x32_bf16 v[212:215], v[104:107], a[144:147], v[212:215]\n"
    "s_add_u32 s64, s64, s87\n"
    "v_mfma_f32_16x16x32_bf16 v[180:183], v[104:107], a[136:139], v[180:183]\n"
    "ds_read_b64_tr_b16 v[100:101], v60 offset:24576\n"
    "v_mfma_f32_16x16x32_bf16 v[180:183], v[108:111], a[140:143], v[180:183]\n"
    "buffer_load_dwordx4 v56, s[36:39], s64 offen offset:0 lds\n"
    "v_mfma_f32_16x16x32_bf16 v[148:151], v[108:111], a[132:135], v[148:151]\n"
    "ds_read_b64_tr_b16 v[102:103], v60 offset:25600\n"
    "v_mfma_f32_16x16x32_bf16 v[148:151], v[104:107], a[128:131], v[148:151]\n"
    "s_waitcnt lgkmcnt(9)\n"
    "v_mfma_f32_16x16x32_bf16 v[152:155], v[112:115], a[128:131], v[152:155]\n"
    "ds_read_b64_tr_b16 v[104:105], v61 offset:16384\n"
    "v_mfma_f32_16x16x32_bf16 v[152:155], v[116:119], a[132:135], v[152:155]\n"
    "buffer_load_dwordx4 v57, s[36:39], s64 offen offset:1024 lds\n"
    "v_mfma_f32_16x16x32_bf16 v[184:187], v[116:119], a[140:143], v[184:187]\n"
    "ds_read_b64_tr_b16 v[106:107], v61 offset:17408\n"
    "v_mfma_f32_16x16x32_bf16 v[184:187], v[112:115], a[136:139], v[184:187]\n"
    "v_mfma_f32_16x16x32_bf16 v[216:219], v[112:115], a[144:147], v[216:219]\n"
    "ds_read_b64_tr_b16 v[108:109], v61 offset:24576\n"
    "v_mfma_f32_16x16x32_bf16 v[216:219], v[116:119], a[148:151], v[216:219]\n"
    "v_mfma_f32_16x16x32_bf16 v[248:251], v[116:119], a[156:159], v[248:251]\n"
    "ds_read_b64_tr_b16 v[110:111], v61 offset:25600\n"
    "v_mfma_f32_16x16x32_bf16 v[248:251], v[112:115], a[152:155], v[248:251]\n"
    "buffer_load_dwordx4 v58, s[36:39], s64 offen offset:2048 lds\n"
    "s_waitcnt lgkmcnt(9)\n"
    "v_mfma_f32_16x16x32_bf16 v[252:255], v[120:123], a[152:155], v[252:255]\n"
    "ds_read_b64_tr_b16 v[112:113], v62 offset:16384\n"
    "v_mfma_f32_16x16x32_bf16 v[252:255], v[124:127], a[156:159], v[252:255]\n"
    "v_mfma_f32_16x16x32_bf16 v[220:223], v[124:127], a[148:151], v[220:223]\n"
    "ds_read_b64_tr_b16 v[114:115], v62 offset:17408\n"
    "v_mfma_f32_16x16x32_bf16 v[220:223], v[120:123], a[144:147], v[220:223]\n"
    "v_mfma_f32_16x16x32_bf16 v[188:191], v[120:123], a[136:139], v[188:191]\n"
    "ds_read_b64_tr_b16 v[116:117], v62 offset:24576\n"
    "v_mfma_f32_16x16x32_bf16 v[188:191], v[124:127], a[140:143], v[188:191]\n"
    "buffer_load_dwordx4 v59, s[36:39], s64 offen offset:3072 lds\n"
    "v_mfma_f32_16x16x32_bf16 v[156:159], v[124:127], a[132:135], v[156:159]\n"
    "ds_read_b64_tr_b16 v[118:119], v62 offset:25600\n"
    "v_mfma_f32_16x16x32_bf16 v[156:159], v[120:123], a[128:131], v[156:159]\n"
    "s_waitcnt lgkmcnt(8)\n"
    "v_mfma_f32_16x16x32_bf16 v[128:131], v[96:99], a[160:163], v[128:131]\n"
    "ds_read_b64_tr_b16 v[120:121], v63 offset:16384\n"
    "v_mfma_f32_16x16x32_bf16 v[128:131], v[100:103], a[164:167], v[128:131]\n"
    "v_readfirstlane_b32 vcc_lo, v36\n"
    "v_readfirstlane_b32 vcc_hi, v37\n"
    "v_mfma_f32_16x16x32_bf16 v[160:163], v[100:103], a[172:175], v[160:163]\n"
    "ds_read_b64_tr_b16 v[122:123], v63 offset:17408\n"
    "v_mfma_f32_16x16x32_bf16 v[160:163], v[96:99], a[168:171], v[160:163]\n"
    "s_or_b64 s[72:73], s[72:73], vcc\n"
    "v_mfma_f32_16x16x32_bf16 v[192:195], v[96:99], a[176:179], v[192:195]\n"
    "ds_read_b64_tr_b16 v[124:125], v63 offset:24576\n"
    "s_waitcnt vmcnt(22)\n"
    "ds_read_b128 a[192:195], v28 offset:32768\n"
    "v_mfma_f32_16x16x32_bf16 v[192:195], v[100:103], a[180:183], v[192:195]\n"
    "ds_read_b128 a[196:199], v29 offset:32768\n"
    "ds_read_b128 a[200:203], v28 offset:34816\n"
    "ds_read_b128 a[204:207], v29 offset:34816\n"
    "v_mfma_f32_16x16x32_bf16 v[224:227], v[100:103], a[188:191], v[224:227]\n"
    "ds_read_b64_tr_b16 v[126:127], v63 offset:25600\n"
    "s_waitcnt vmcnt(17)\n"
    "ds_read_b128 a[208:211], v28 offset:49152\n"
    "v_mfma_f32_16x16x32_bf16 v[224:227], v[96:99], a[184:187], v[224:227]\n"
    "ds_read_b128 a[212:215], v29 offset:49152\n"
    "ds_read_b128 a[216:219], v28 offset:51200\n"
    "ds_read_b128 a[220:223], v29 offset:51200\n"
    "s_waitcnt lgkmcnt(15)\n"
    "v_mfma_f32_16x16x32_bf16 v[228:231], v[104:107], a[184:187], v[228:231]\n"
    "ds_read_b64_tr_b16 v[96:97], v60 offset:16896\n"
    "s_andn2_b64 s[76:77], s[72:73], s[80:81]\n"
    "s_cbranch_scc0 1011f\n"
    "1012:\n"
    "s_ff1_i32_b64 s69, s[76:77]\n"
    "v_readlane_b32 s65, v53, s69\n"
    "v_mfma_f32_16x16x32_bf16 v[228:231], v[108:111], a[188:191], v[228:231]\n"
    "s_mov_b32 s25, 721000\n"
    "s_getpc_b64 s[28:29]\n"
    "s_cmp_ge_i32 s25, s24\n"
    "s_cbranch_scc1 90b\n"
    "v_mfma_f32_16x16x32_bf16 v[196:199], v[108:111], a[180:183], v[196:199]\n"
    "ds_read_b64_tr_b16 v[98:99], v60 offset:17920\n"
    "s_bitset1_b64 s[80:81], s69\n"
    "s_nop 0\n"
    "v_mfma_f32_16x16x32_bf16 v[196:199], v[104:107], a[176:179], v[196:199]\n"
    "s_nop 0\n"
    "s_lshl_b32 s64, s65, 5\n"
    "v_mfma_f32_16x16x32_bf16 v[164:167], v[104:107], a[168:171], v[164:167]\n"
    "ds_read_b64_tr_b16 v[100:101], v60 offset:25088\n"
    "buffer_load_dwordx4 a[0:3], v54, s[40:43], s65 offen offset:0\n"
    "v_mfma_f32_16x16x32_bf16 v[164:167], v[108:111], a[172:175], v[164:167]\n"
    "v_cmp_le_u32 vcc, s70, v47\n"
    "v_mfma_f32_16x16x32_bf16 v[132:135], v[108:111], a[164:167], v[132:135]\n"
    "ds_read_b64_tr_b16 v[102:103], v60 offset:26112\n"
    "buffer_load_dwordx4 a[4:7], v54, s[40:43], s65 offen offset:1024\n"
    "v_mfma_f32_16x16x32_bf16 v[132:135], v[104:107], a[160:163], v[132:135]\n"
    "s_or_b64 s[82:83], s[82:83], vcc\n"
    "v_cmp_le_u32 vcc, s70, v48\n"
    "s_waitcnt lgkmcnt(15)\n"
    "v_mfma_f32_16x16x32_bf16 v[136:139], v[112:115], a[160:163], v[136:139]\n"
    "ds_read_b64_tr_b16 v[104:105], v61 offset:16896\n"
    "buffer_load_dwordx4 a[8:11], v54, s[40:43], s65 offen offset:2048\n"
    "v_mfma_f32_16x16x32_bf16 v[136:139], v[116:119], a[164:167], v[136:139]\n"
    "s_or_b64 s[82:83], s[82:83], vcc\n"
    "v_mfma_f32_16x16x32_bf16 v[168:171], v[116:119], a[172:175], v[168:171]\n"
    "ds_read_b64_tr_b16 v[106:107], v61 offset:17920\n"
    "v_mfma_f32_16x16x32_bf16 v[168:171], v[112:115], a[168:171], v[168:171]\n"
    "v_mfma_f32_16x16x32_bf16 v[200:203], v[112:115], a[176:179], v[200:203]\n"
    "ds_read_b64_tr_b16 v[108:109], v61 offset:25088\n"
    "buffer_load_dwordx4 a[12:15], v54, s[40:43], s65 offen offset:3072\n"
    "v_mfma_f32_16x16x32_bf16 v[200:203], v[116:119], a[180:183], v[200:203]\n"
    "v_mov_b32 v38, s82\n"
    "v_mov_b32 v39, s83\n"
    "ds_write_b64 v41, v[38:39] offset:8\n"
    "v_mfma_f32_16x16x32_bf16 v[232:235], v[116:119], a[188:191], v[232:235]\n"
    "ds_read_b64_tr_b16 v[110:111], v61 offset:26112\n"
    "v_mfma_f32_16x16x32_bf16 v[232:235], v[112:115], a[184:187], v[232:235]\n"
    "s_ff0_i32_b64 s30, s[82:83]\n"
    "s_bfm_b64 vcc, s30, 0\n"
    "s_andn2_b64 vcc, vcc, s[80:81]\n"
    "s_bcnt1_i32_b64 s30, vcc\n"
    "s_cmp_lt_u32 s30, 64\n"
    "s_cselect_b32 s22, 0x10000, 0\n"
    "s_waitcnt lgkmcnt(13)\n"
    "v_mfma_f32_16x16x32_bf16 v[236:239], v[120:123], a[184:187], v[236:239]\n"
    "ds_read_b64_tr_b16 v[112:113], v62 offset:16896\n"
    "v_mfma_f32_16x16x32_bf16 v[236:239], v[124:127], a[188:191], v[236:239]\n"
    "s_bcnt1_i32_b64 s30, s[82:83]\n"
    "s_cmp_ge_u32 s30, 56\n"
    "s_cselect_b32 s22, 0, s22\n"
    "v_mfma_f32_16x16x32_bf16 v[204:207], v[124:127], a[180:183], v[204:207]\n"
    "ds_read_b64_tr_b16 v[114:115], v62 offset:17920\n"
    "buffer_load_dwordx4 a[16:19], v55, s[40:43], s65 offen offset:0\n"
    "v_mfma_f32_16x16x32_bf16 v[204:207], v[120:123], a[176:179], v[204:207]\n"
    "buffer_load_dword v48, v49, s[20:23], 0 offen sc0 sc1\n"
    "v_mfma_f32_16x16x32_bf16 v[172:175], v[120:123], a[168:171], v[172:175]\n"
    "ds_read_b64_tr_b16 v[116:117], v62 offset:25088\n"
    "v_mfma_f32_16x16x32_bf16 v[172:175], v[124:127], a[172:175], v[172:175]\n"
    "v_mfma_f32_16x16x32_bf16 v[140:143], v[124:127], a[164:167], v[140:143]\n"
    "ds_read_b64_tr_b16 v[118:119], v62 offset:26112\n"
    "buffer_load_dwordx4 a[20:23], v55, s[40:43], s65 offen offset:1024\n"
    "v_mfma_f32_16x16x32_bf16 v[140:143], v[120:123], a[160:163], v[140:143]\n"
    "s_waitcnt lgkmcnt(9)\n"
    "v_mfma_f32_16x16x32_bf16 v[144:147], v[96:99], a[160:163], v[144:147]\n"
    "ds_read_b64_tr_b16 v[120:121], v63 offset:16896\n"
    "v_mfma_f32_16x16x32_bf16 v[144:147], v[100:103], a[164:167], v[144:147]\n"
    "buffer_load_dwordx4 a[24:27], v55, s[40:43], s65 offen offset:2048\n"
    "v_mfma_f32_16x16x32_bf16 v[176:179], v[100:103], a[172:175], v[176:179]\n"
    "ds_read_b64_tr_b16 v[122:123], v63 offset:17920\n"
    "v_mfma_f32_16x16x32_bf16 v[176:179], v[96:99], a[168:171], v[176:179]\n"
    "v_mfma_f32_16x16x32_bf16 v[208:211], v[96:99], a[176:179], v[208:211]\n"
    "ds_read_b64_tr_b16 v[124:125], v63 offset:25088\n"
    "buffer_load_dwordx4 a[28:31], v55, s[40:43], s65 offen offset:3072\n"
    "v_mfma_f32_16x16x32_bf16 v[208:211], v[100:103], a[180:183], v[208:211]\n"
    "s_mov_b32 s25, 786545\n"
    "s_getpc_b64 s[28:29]\n"
    "s_cmp_ge_i32 s25, s24\n"
    "s_cbranch_scc1 90b\n"
    "v_mfma_f32_16x16x32_bf16 v[240:243], v[100:103], a[188:191], v[240:243]\n"
    "ds_read_b64_tr_b16 v[126:127], v63 offset:26112\n"
    "v_mfma_f32_16x16x32_bf16 v[240:243], v[96:99], a[184:187], v[240:243]\n"
    "s_waitcnt lgkmcnt(8)\n"
    "s_waitcnt vmcnt(22)\n"
    "s_barrier\n"
    "v_mfma_f32_16x16x32_bf16 v[244:247], v[104:107], a[184:187], v[244:247]\n"
    "ds_read_b64_tr_b16 v[96:97], v60 offset:32768\n"
    "s_add_u32 m0, s67, 0\n"
    "s_nop 0\n"
    "v_mfma_f32_16x16x32_bf16 v[244:247], v[108:111], a[188:191], v[244:247]\n"
    "ds_read_b64 v[36:37], v40 offset:8\n"
    "v_mfma_f32_16x16x32_bf16 v[212:215], v[108:111], a[180:183], v[212:215]\n"
    "ds_read_b64_tr_b16 v[98:99], v60 offset:33792\n"
    "v_mfma_f32_16x16x32_bf16 v[212:215], v[104:107], a[176:179], v[212:215]\n"
    "buffer_load_dwordx4 v56, s[36:39], s64 offen offset:0 lds\n"
    "v_mfma_f32_16x16x32_bf16 v[180:183], v[104:107], a[168:171], v[180:183]\n"
    "ds_read_b64_tr_b16 v[100:101], v60 offset:40960\n"
    "v_mfma_f32_16x16x32_bf16 v[180:183], v[108:111], a[172:175], v[180:183]\n"
    "v_mfma_f32_16x16x32_bf16 v[148:151], v[108:111], a[164:167], v[148:151]\n"
    "ds_read_b64_tr_b16 v[102:103], v60 offset:41984\n"
    "v_mfma_f32_16x16x32_bf16 v[148:151], v[104:107], a[160:163], v[148:151]\n"
    "s_waitcnt lgkmcnt(9)\n"
    "v_mfma_f32_16x16x32_bf16 v[152:155], v[112:115], a[160:163], v[152:155]\n"
    "ds_read_b64_tr_b16 v[104:105], v61 offset:32768\n"
    "v_mfma_f32_16x16x32_bf16 v[152:155], v[116:119], a[164:167], v[152:155]\n"
    "buffer_load_dwordx4 v57, s[36:39], s64 offen offset:1024 lds\n"
    "v_mfma_f32_16x16x32_bf16 v[184:187], v[116:119], a[172:175], v[184:187]\n"
    "ds_read_b64_tr_b16 v[106:107], v61 offset:33792\n"
    "v_mfma_f32_16x16x32_bf16 v[184:187], v[112:115], a[168:171], v[184:187]\n"
    "v_mfma_f32_16x16x32_bf16 v[216:219], v[112:115], a[176:179], v[216:219]\n"
    "ds_read_b64_tr_b16 v[108:109], v61 offset:40960\n"
    "v_mfma_f32_16x16x32_bf16 v[216:219], v[116:119], a[180:183], v[216:219]\n"
    "v_mfma_f32_16x16x32_bf16 v[248:251], v[116:119], a[188:191], v[248:251]\n"
    "ds_read_b64_tr_b16 v[110:111], v61 offset:41984\n"
    "v_mfma_f32_16x16x32_bf16 v[248:251], v[112:115], a[184:187], v[248:251]\n"
    "buffer_load_dwordx4 v58, s[36:39], s64 offen offset:2048 lds\n"
    "s_waitcnt lgkmcnt(9)\n"
    "v_mfma_f32_16x16x32_bf16 v[252:255], v[120:123], a[184:187], v[252:255]\n"
    "ds_read_b64_tr_b16 v[112:113], v62 offset:32768\n"
    "v_mfma_f32_16x16x32_bf16 v[252:255], v[124:127], a[188:191], v[252:255]\n"
    "v_mfma_f32_16x16x32_bf16 v[220:223], v[124:127], a[180:183], v[220:223]\n"
    "ds_read_b64_tr_b16 v[114:115], v62 offset:33792\n"
    "v_mfma_f32_16x16x32_bf16 v[220:223], v[120:123], a[176:179], v[220:223]\n"
    "v_mfma_f32_16x16x32_bf16 v[188:191], v[120:123], a[168:171], v[188:191]\n"
    "ds_read_b64_tr_b16 v[116:117], v62 offset:40960\n"
    "v_mfma_f32_16x16x32_bf16 v[188:191], v[124:127], a[172:175], v[188:191]\n"
    "buffer_load_dwordx4 v59, s[36:39], s64 offen offset:3072 lds\n"
    "v_mfma_f32_16x16x32_bf16 v[156:159], v[124:127], a[164:167], v[156:159]\n"
    "ds_read_b64_tr_b16 v[118:119], v62 offset:41984\n"
    "v_mfma_f32_16x16x32_bf16 v[156:159], v[120:123], a[160:163], v[156:159]\n"
    "s_waitcnt lgkmcnt(8)\n"
    "v_mfma_f32_16x16x32_bf16 v[128:131], v[96:99], a[192:195], v[128:131]\n"
    "ds_read_b64_tr_b16 v[120:121], v63 offset:32768\n"
    "v_mfma_f32_16x16x32_bf16 v[128:131], v[100:103], a[196:199], v[128:131]\n"
    "v_readfirstlane_b32 vcc_lo, v36\n"
    "v_readfirstlane_b32 vcc_hi, v37\n"
    "v_mfma_f32_16x16x32_bf16 v[160:163], v[100:103], a[204:207], v[160:163]\n"
    "ds_read_b64_tr_b16 v[122:123], v63 offset:33792\n"
    "v_mfma_f32_16x16x32_bf16 v[160:163], v[96:99], a[200:203], v[160:163]\n"
    "s_or_b64 s[72:73], s[72:73], vcc\n"
    "v_mfma_f32_16x16x32_bf16 v[192:195], v[96:99], a[208:211], v[192:195]\n"
    "ds_read_b64_tr_b16 v[124:125], v63 offset:40960\n"
    "s_waitcnt vmcnt(22)\n"
    "ds_read_b128 a[224:227], v30 offset:0\n"
    "v_mfma_f32_16x16x32_bf16 v[192:195], v[100:103], a[212:215], v[192:195]\n"
    "ds_read_b128 a[228:231], v31 offset:0\n"
    "ds_read_b128 a[232:235], v30 offset:2048\n"
    "ds_read_b128 a[236:239], v31 offset:2048\n"
    "v_mfma_f32_16x16x32_bf16 v[224:227], v[100:103], a[220:223], v[224:227]\n"
    "ds_read_b64_tr_b16 v[126:127], v63 offset:41984\n"
    "s_waitcnt vmcnt(17)\n"
    "ds_read_b128 a[240:243], v28 offset:0\n"
    "v_mfma_f32_16x16x32_bf16 v[224:227], v[96:99], a[216:219], v[224:227]\n"
    "ds_read_b128 a[244:247], v29 offset:0\n"
    "ds_read_b128 a[248:251], v28 offset:2048\n"
    "ds_read_b128 a[252:255], v29 offset:2048\n"
    "s_waitcnt lgkmcnt(15)\n"
    "v_mfma_f32_16x16x32_bf16 v[228:231], v[104:107], a[216:219], v[228:231]\n"
    "ds_read_b64_tr_b16 v[96:97], v60 offset:33280\n"
    "s_andn2_b64 s[76:77], s[72:73], s[80:81]\n"
    "s_cbranch_scc0 1021f\n"
    "1022:\n"
    "s_ff1_i32_b64 s69, s[76:77]\n"
    "v_readlane_b32 s65, v53, s69\n"
    "v_mfma_f32_16x16x32_bf16 v[228:231], v[108:111], a[220:223], v[228:231]\n"
    "s_mov_b32 s25, 852085\n"
    "s_getpc_b64 s[28:29]\n"
    "s_cmp_ge_i32 s25, s24\n"
    "s_cbranch_scc1 90b\n"
    "v_mfma_f32_16x16x32_bf16 v[196:199], v[108:111], a[212:215], v[196:199]\n"
    "ds_read_b64_tr_b16 v[98:99], v60 offset:34304\n"
    "s_bitset1_b64 s[80:81], s69\n"
    "s_nop 0\n"
    "v_mfma_f32_16x16x32_bf16 v[196:199], v[104:107], a[208:211], v[196:199]\n"
    "s_nop 0\n"
    "s_lshl_b32 s64, s65, 5\n"
    "v_mfma_f32_16x16x32_bf16 v[164:167], v[104:107], a[200:203], v[164:167]\n"
    "ds_read_b64_tr_b16 v[100:101], v60 offset:41472\n"
    "buffer_load_dwordx4 a[32:35], v54, s[40:43], s65 offen offset:0\n"
    "v_mfma_f32_16x16x32_bf16 v[164:167], v[108:111], a[204:207], v[164:167]\n"
    "v_cmp_le_u32 vcc, s70, v47\n"
    "v_mfma_f32_16x16x32_bf16 v[132:135], v[108:111], a[196:199], v[132:135]\n"
    "ds_read_b64_tr_b16 v[102:103], v60 offset:42496\n"
    "buffer_load_dwordx4 a[36:39], v54, s[40:43], s65 offen offset:1024\n"
    "v_mfma_f32_16x16x32_bf16 v[132:135], v[104:107], a[192:195], v[132:135]\n"
    "s_or_b64 s[82:83], s[82:83], vcc\n"
    "v_cmp_le_u32 vcc, s70, v48\n"
    "s_waitcnt lgkmcnt(15)\n"
    "v_mfma_f32_16x16x32_bf16 v[136:139], v[112:115], a[192:195], v[136:139]\n"
    "ds_read_b64_tr_b16 v[104:105], v61 offset:33280\n"
    "buffer_load_dwordx4 a[40:43], v54, s[40:43], s65 offen offset:2048\n"
    "v_mfma_f32_16x16x32_bf16 v[136:139], v[116:119], a[196:199], v[136:139]\n"
    "s_or_b64 s[82:83], s[82:83], vcc\n"
    "v_mfma_f32_16x16x32_bf16 v[168:171], v[116:119], a[204:207], v[168:171]\n"
    "ds_read_b64_tr_b16 v[106:107], v61 offset:34304\n"
    "v_mfma_f32_16x16x32_bf16 v[168:171], v[112:115], a[200:203], v[168:171]\n"
    "v_mfma_f32_16x16x32_bf16 v[200:203], v[112:115], a[208:211], v[200:203]\n"
    "ds_read_b64_tr_b16 v[108:109], v61 offset:41472\n"
    "buffer_load_dwordx4 a[44:47], v54, s[40:43], s65 offen offset:3072\n"
    "v_mfma_f32_16x16x32_bf16 v[200:203], v[116:119], a[212:215], v[200:203]\n"
    "v_mov_b32 v38, s82\n"
    "v_mov_b32 v39, s83\n"
    "ds_write_b64 v41, v[38:39] offset:0\n"
    "v_mfma_f32_16x16x32_bf16 v[232:235], v[116:119], a[220:223], v[232:235]\n"
    "ds_read_b64_tr_b16 v[110:111], v61 offset:42496\n"
    "v_mfma_f32_16x16x32_bf16 v[232:235], v[112:115], a[216:219], v[232:235]\n"
    "s_ff0_i32_b64 s30, s[82:83]\n"
    "s_bfm_b64 vcc, s30, 0\n"
    "s_andn2_b64 vcc, vcc, s[80:81]\n"
    "s_bcnt1_i32_b64 s30, vcc\n"
    "s_cmp_lt_u32 s30, 64\n"
    "s_cselect_b32 s22, 0x10000, 0\n"
    "s_waitcnt lgkmcnt(13)\n"
    "v_mfma_f32_16x16x32_bf16 v[236:239], v[120:123], a[216:219], v[236:239]\n"
    "ds_read_b64_tr_b16 v[112:113], v62 offset:33280\n"
    "v_mfma_f32_16x16x32_bf16 v[236:239], v[124:127], a[220:223], v[236:239]\n"
    "s_bcnt1_i32_b64 s30, s[82:83]\n"
    "s_cmp_ge_u32 s30, 56\n"
    "s_cselect_b32 s22, 0, s22\n"
    "v_mfma_f32_16x16x32_bf16 v[204:207], v[124:127], a[212:215], v[204:207]\n"
    "ds_read_b64_tr_b16 v[114:115], v62 offset:34304\n"
    "buffer_load_dwordx4 a[48:51], v55, s[40:43], s65 offen offset:0\n"
    "v_mfma_f32_16x16x32_bf16 v[204:207], v[120:123], a[208:211], v[204:207]\n"
    "buffer_load_dword v47, v49, s[20:23], 0 offen sc0 sc1\n"
    "v_mfma_f32_16x16x32_bf16 v[172:175], v[120:123], a[200:203], v[172:175]\n"
    "ds_read_b64_tr_b16 v[116:117], v62 offset:41472\n"
    "v_mfma_f32_16x16x32_bf16 v[172:175], v[124:127], a[204:207], v[172:175]\n"
    "v_mfma_f32_16x16x32_bf16 v[140:143], v[124:127], a[196:199], v[140:143]\n"
    "ds_read_b64_tr_b16 v[118:119], v62 offset:42496\n"
    "buffer_load_dwordx4 a[52:55], v55, s[40:43], s65 offen offset:1024\n"
    "v_mfma_f32_16x16x32_bf16 v[140:143], v[120:123], a[192:195], v[140:143]\n"
    "s_waitcnt lgkmcnt(9)\n"
    "v_mfma_f32_16x16x32_bf16 v[144:147], v[96:99], a[192:195], v[144:147]\n"
    "ds_read_b64_tr_b16 v[120:121], v63 offset:33280\n"
    "v_mfma_f32_16x16x32_bf16 v[144:147], v[100:103], a[196:199], v[144:147]\n"
    "buffer_load_dwordx4 a[56:59], v55, s[40:43], s65 offen offset:2048\n"
    "v_mfma_f32_16x16x32_bf16 v[176:179], v[100:103], a[204:207], v[176:179]\n"
    "ds_read_b64_tr_b16 v[122:123], v63 offset:34304\n"
    "v_mfma_f32_16x16x32_bf16 v[176:179], v[96:99], a[200:203], v[176:179]\n"
    "v_mfma_f32_16x16x32_bf16 v[208:211], v[96:99], a[208:211], v[208:211]\n"
    "ds_read_b64_tr_b16 v[124:125], v63 offset:41472\n"
    "buffer_load_dwordx4 a[60:63], v55, s[40:43], s65 offen offset:3072\n"
    "v_mfma_f32_16x16x32_bf16 v[208:211], v[100:103], a[212:215], v[208:211]\n"
    "s_mov_b32 s25, 917630\n"
    "s_getpc_b64 s[28:29]\n"
    "s_cmp_ge_i32 s25, s24\n"
    "s_cbranch_scc1 90b\n"
    "v_mfma_f32_16x16x32_bf16 v[240:243], v[100:103], a[220:223], v[240:243]\n"
    "ds_read_b64_tr_b16 v[126:127], v63 offset:42496\n"
    "v_mfma_f32_16x16x32_bf16 v[240:243], v[96:99], a[216:219], v[240:243]\n"
    "s_waitcnt lgkmcnt(8)\n"
    "s_waitcnt vmcnt(22)\n"
    "s_barrier\n"
    "v_mfma_f32_16x16x32_bf16 v[244:247], v[104:107], a[216:219], v[244:247]\n"
    "ds_read_b64_tr_b16 v[96:97], v60 offset:49152\n"
    "s_add_u32 m0, s67, 16384\n"
    "s_nop 0\n"
    "v_mfma_f32_16x16x32_bf16 v[244:247], v[108:111], a[220:223], v[244:247]\n"
    "ds_read_b64 v[36:37], v40 offset:0\n"
    "v_mfma_f32_16x16x32_bf16 v[212:215], v[108:111], a[212:215], v[212:215]\n"
    "ds_read_b64_tr_b16 v[98:99], v60 offset:50176\n"
    "v_mfma_f32_16x16x32_bf16 v[212:215], v[104:107], a[208:211], v[212:215]\n"
    "buffer_load_dwordx4 v56, s[36:39], s64 offen offset:0 lds\n"
    "v_mfma_f32_16x16x32_bf16 v[180:183], v[104:107], a[200:203], v[180:183]\n"
    "ds_read_b64_tr_b16 v[100:101], v60 offset:57344\n"
    "v_mfma_f32_16x16x32_bf16 v[180:183], v[108:111], a[204:207], v[180:183]\n"
    "v_mfma_f32_16x16x32_bf16 v[148:151], v[108:111], a[196:199], v[148:151]\n"
    "ds_read_b64_tr_b16 v[102:103], v60 offset:58368\n"
    "v_mfma_f32_16x16x32_bf16 v[148:151], v[104:107], a[192:195], v[148:151]\n"
    "s_waitcnt lgkmcnt(9)\n"
    "v_mfma_f32_16x16x32_bf16 v[152:155], v[112:115], a[192:195], v[152:155]\n"
    "ds_read_b64_tr_b16 v[104:105], v61 offset:49152\n"
    "v_mfma_f32_16x16x32_bf16 v[152:155], v[116:119], a[196:199], v[152:155]\n"
    "buffer_load_dwordx4 v57, s[36:39], s64 offen offset:1024 lds\n"
    "v_mfma_f32_16x16x32_bf16 v[184:187], v[116:119], a[204:207], v[184:187]\n"
    "ds_read_b64_tr_b16 v[106:107], v61 offset:50176\n"
    "v_mfma_f32_16x16x32_bf16 v[184:187], v[112:115], a[200:203], v[184:187]\n"
    "v_mfma_f32_16x16x32_bf16 v[216:219], v[112:115], a[208:211], v[216:219]\n"
    "ds_read_b64_tr_b16 v[108:109], v61 offset:57344\n"
    "v_mfma_f32_16x16x32_bf16 v[216:219], v[116:119], a[212:215], v[216:219]\n"
    "v_mfma_f32_16x16x32_bf16 v[248:251], v[116:119], a[220:223], v[248:251]\n"
    "ds_read_b64_tr_b16 v[110:111], v61 offset:58368\n"
    "v_mfma_f32_16x16x32_bf16 v[248:251], v[112:115], a[216:219], v[248:251]\n"
    "buffer_load_dwordx4 v58, s[36:39], s64 offen offset:2048 lds\n"
    "s_waitcnt lgkmcnt(9)\n"
    "v_mfma_f32_16x16x32_bf16 v[252:255], v[120:123], a[216:219], v[252:255]\n"
    "ds_read_b64_tr_b16 v[112:113], v62 offset:49152\n"
    "v_mfma_f32_16x16x32_bf16 v[252:255], v[124:127], a[220:223], v[252:255]\n"
    "v_mfma_f32_16x16x32_bf16 v[220:223], v[124:127], a[212:215], v[220:223]\n"
    "ds_read_b64_tr_b16 v[114:115], v62 offset:50176\n"
    "v_mfma_f32_16x16x32_bf16 v[220:223], v[120:123], a[208:211], v[220:223]\n"
    "v_mfma_f32_16x16x32_bf16 v[188:191], v[120:123], a[200:203], v[188:191]\n"
    "ds_read_b64_tr_b16 v[116:117], v62 offset:57344\n"
    "v_mfma_f32_16x16x32_bf16 v[188:191], v[124:127], a[204:207], v[188:191]\n"
    "buffer_load_dwordx4 v59, s[36:39], s64 offen offset:3072 lds\n"
    "v_mfma_f32_16x16x32_bf16 v[156:159], v[124:127], a[196:199], v[156:159]\n"
    "ds_read_b64_tr_b16 v[118:119], v62 offset:58368\n"
    "v_mfma_f32_16x16x32_bf16 v[156:159], v[120:123], a[192:195], v[156:159]\n"
    "s_waitcnt lgkmcnt(8)\n"
    "v_mfma_f32_16x16x32_bf16 v[128:131], v[96:99], a[224:227], v[128:131]\n"
    "ds_read_b64_tr_b16 v[120:121], v63 offset:49152\n"
    "v_mfma_f32_16x16x32_bf16 v[128:131], v[100:103], a[228:231], v[128:131]\n"
    "v_readfirstlane_b32 vcc_lo, v36\n"
    "v_readfirstlane_b32 vcc_hi, v37\n"
    "v_mfma_f32_16x16x32_bf16 v[160:163], v[100:103], a[236:239], v[160:163]\n"
    "ds_read_b64_tr_b16 v[122:123], v63 offset:50176\n"
    "v_mfma_f32_16x16x32_bf16 v[160:163], v[96:99], a[232:235], v[160:163]\n"
    "s_or_b64 s[72:73], s[72:73], vcc\n"
    "v_mfma_f32_16x16x32_bf16 v[192:195], v[96:99], a[240:243], v[192:195]\n"
    "ds_read_b64_tr_b16 v[124:125], v63 offset:57344\n"
    "s_andn2_b64 s[76:77], s[72:73], s[80:81]\n"
    "s_cbranch_scc0 1031f\n"
    "1032:\n"
    "s_ff1_i32_b64 s69, s[76:77]\n"
    "v_readlane_b32 s65, v53, s69\n"
    "v_mfma_f32_16x16x32_bf16 v[192:195], v[100:103], a[244:247], v[192:195]\n"
    "s_bitset1_b64 s[80:81], s69\n"
    "s_nop 0\n"
    "s_nop 0\n"
    "v_mfma_f32_16x16x32_bf16 v[224:227], v[100:103], a[252:255], v[224:227]\n"
    "ds_read_b64_tr_b16 v[126:127], v63 offset:58368\n"
    "s_lshl_b32 s64, s65, 5\n"
    "v_mfma_f32_16x16x32_bf16 v[224:227], v[96:99], a[248:251], v[224:227]\n"
    "buffer_load_dwordx4 a[64:67], v54, s[40:43], s65 offen offset:0\n"
    "s_waitcnt lgkmcnt(8)\n"
    "v_mfma_f32_16x16x32_bf16 v[228:231], v[104:107], a[248:251], v[228:231]\n"
    "ds_read_b64_tr_b16 v[96:97], v60 offset:49664\n"
    "v_mfma_f32_16x16x32_bf16 v[228:231], v[108:111], a[252:255], v[228:231]\n"
    "s_mov_b32 s25, 983171\n"
    "s_getpc_b64 s[28:29]\n"
    "s_cmp_ge_i32 s25, s24\n"
    "s_cbranch_scc1 90b\n"
    "v_mfma_f32_16x16x32_bf16 v[196:199], v[108:111], a[244:247], v[196:199]\n"
    "ds_read_b64_tr_b16 v[98:99], v60 offset:50688\n"
    "v_mfma_f32_16x16x32_bf16 v[196:199], v[104:107], a[240:243], v[196:199]\n"
    "buffer_load_dwordx4 a[68:71], v54, s[40:43], s65 offen offset:1024\n"
    "v_mfma_f32_16x16x32_bf16 v[164:167], v[104:107], a[232:235], v[164:167]\n"
    "ds_read_b64_tr_b16 v[100:101], v60 offset:57856\n"
    "v_mfma_f32_16x16x32_bf16 v[164:167], v[108:111], a[236:239], v[164:167]\n"
    "v_cmp_le_u32 vcc, s70, v47\n"
    "v_mfma_f32_16x16x32_bf16 v[132:135], v[108:111], a[228:231], v[132:135]\n"
    "ds_read_b64_tr_b16 v[102:103], v60 offset:58880\n"
    "v_mfma_f32_16x16x32_bf16 v[132:135], v[104:107], a[224:227], v[132:135]\n"
    "s_or_b64 s[82:83], s[82:83], vcc\n"
    "v_cmp_le_u32 vcc, s70, v48\n"
    "s_waitcnt lgkmcnt(8)\n"
    "v_mfma_f32_16x16x32_bf16 v[136:139], v[112:115], a[224:227], v[136:139]\n"
    "ds_read_b64_tr_b16 v[104:105], v61 offset:49664\n"
    "buffer_load_dwordx4 a[72:75], v54, s[40:43], s65 offen offset:2048\n"
    "v_mfma_f32_16x16x32_bf16 v[136:139], v[116:119], a[228:231], v[136:139]\n"
    "s_or_b64 s[82:83], s[82:83], vcc\n"
    "v_mfma_f32_16x16x32_bf16 v[168:171], v[116:119], a[236:239], v[168:171]\n"
    "ds_read_b64_tr_b16 v[106:107], v61 offset:50688\n"
    "v_mfma_f32_16x16x32_bf16 v[168:171], v[112:115], a[232:235], v[168:171]\n"
    "v_mfma_f32_16x16x32_bf16 v[200:203], v[112:115], a[240:243], v[200:203]\n"
    "ds_read_b64_tr_b16 v[108:109], v61 offset:57856\n"
    "buffer_load_dwordx4 a[76:79], v54, s[40:43], s65 offen offset:3072\n"
    "v_mfma_f32_16x16x32_bf16 v[200:203], v[116:119], a[244:247], v[200:203]\n"
    "v_mov_b32 v38, s82\n"
    "v_mov_b32 v39, s83\n"
    "ds_write_b64 v41, v[38:39] offset:8\n"
    "v_mfma_f32_16x16x32_bf16 v[232:235], v[116:119], a[252:255], v[232:235]\n"
    "ds_read_b64_tr_b16 v[110:111], v61 offset:58880\n"
    "v_mfma_f32_16x16x32_bf16 v[232:235], v[112:115], a[248:251], v[232:235]\n"
    "s_ff0_i32_b64 s30, s[82:83]\n"
    "s_bfm_b64 vcc, s30, 0\n"
    "s_andn2_b64 vcc, vcc, s[80:81]\n"
    "s_bcnt1_i32_b64 s30, vcc\n"
    "s_cmp_lt_u32 s30, 64\n"
    "s_cselect_b32 s22, 0x10000, 0\n"
    "s_waitcnt lgkmcnt(9)\n"
    "v_mfma_f32_16x16x32_bf16 v[236:239], v[120:123], a[248:251], v[236:239]\n"
    "ds_read_b64_tr_b16 v[112:113], v62 offset:49664\n"
    "v_mfma_f32_16x16x32_bf16 v[236:239], v[124:127], a[252:255], v[236:239]\n"
    "s_bcnt1_i32_b64 s30, s[82:83]\n"
    "s_cmp_ge_u32 s30, 56\n"
    "s_cselect_b32 s22, 0, s22\n"
    "v_mfma_f32_16x16x32_bf16 v[204:207], v[124:127], a[244:247], v[204:207]\n"
    "ds_read_b64_tr_b16 v[114:115], v62 offset:50688\n"
    "buffer_load_dwordx4 a[80:83], v55, s[40:43], s65 offen offset:0\n"
    "v_mfma_f32_16x16x32_bf16 v[204:207], v[120:123], a[240:243], v[204:207]\n"
    "buffer_load_dword v48, v49, s[20:23], 0 offen sc0 sc1\n"
    "v_mfma_f32_16x16x32_bf16 v[172:175], v[120:123], a[232:235], v[172:175]\n"
    "ds_read_b64_tr_b16 v[116:117], v62 offset:57856\n"
    "v_mfma_f32_16x16x32_bf16 v[172:175], v[124:127], a[236:239], v[172:175]\n"
    "v_mfma_f32_16x16x32_bf16 v[140:143], v[124:127], a[228:231], v[140:143]\n"
    "ds_read_b64_tr_b16 v[118:119], v62 offset:58880\n"
    "buffer_load_dwordx4 a[84:87], v55, s[40:43], s65 offen offset:1024\n"
    "v_mfma_f32_16x16x32_bf16 v[140:143], v[120:123], a[224:227], v[140:143]\n"
    "s_waitcnt lgkmcnt(9)\n"
    "v_mfma_f32_16x16x32_bf16 v[144:147], v[96:99], a[224:227], v[144:147]\n"
    "ds_read_b64_tr_b16 v[120:121], v63 offset:49664\n"
    "v_mfma_f32_16x16x32_bf16 v[144:147], v[100:103], a[228:231], v[144:147]\n"
    "buffer_load_dwordx4 a[88:91], v55, s[40:43], s65 offen offset:2048\n"
    "v_mfma_f32_16x16x32_bf16 v[176:179], v[100:103], a[236:239], v[176:179]\n"
    "ds_read_b64_tr_b16 v[122:123], v63 offset:50688\n"
    "v_mfma_f32_16x16x32_bf16 v[176:179], v[96:99], a[232:235], v[176:179]\n"
    "v_mfma_f32_16x16x32_bf16 v[208:211], v[96:99], a[240:243], v[208:211]\n"
    "ds_read_b64_tr_b16 v[124:125], v63 offset:57856\n"
    "buffer_load_dwordx4 a[92:95], v55, s[40:43], s65 offen offset:3072\n"
    "v_mfma_f32_16x16x32_bf16 v[208:211], v[100:103], a[244:247], v[208:211]\n"
    "s_mov_b32 s25, 1048715\n"
    "s_getpc_b64 s[28:29]\n"
    "s_cmp_ge_i32 s25, s24\n"
    "s_cbranch_scc1 90b\n"
    "v_mfma_f32_16x16x32_bf16 v[240:243], v[100:103], a[252:255], v[240:243]\n"
    "ds_read_b64_tr_b16 v[126:127], v63 offset:58880\n"
    "v_mfma_f32_16x16x32_bf16 v[240:243], v[96:99], a[248:251], v[240:243]\n"
    "s_waitcnt lgkmcnt(8)\n"
    "s_waitcnt vmcnt(22)\n"
    "s_barrier\n"
    "v_mfma_f32_16x16x32_bf16 v[244:247], v[104:107], a[248:251], v[244:247]\n"
    "ds_read_b64_tr_b16 v[96:97], v60 offset:0\n"
    "s_add_u32 m0, s67, 32768\n"
    "s_nop 0\n"
    "v_mfma_f32_16x16x32_bf16 v[244:247], v[108:111], a[252:255], v[244:247]\n"
    "ds_read_b64 v[36:37], v40 offset:8\n"
    "v_mfma_f32_16x16x32_bf16 v[212:215], v[108:111], a[244:247], v[212:215]\n"
    "ds_read_b64_tr_b16 v[98:99], v60 offset:1024\n"
    "v_mfma_f32_16x16x32_bf16 v[212:215], v[104:107], a[240:243], v[212:215]\n"
    "buffer_load_dwordx4 v56, s[36:39], s64 offen offset:0 lds\n"
    "v_mfma_f32_16x16x32_bf16 v[180:183], v[104:107], a[232:235], v[180:183]\n"
    "ds_read_b64_tr_b16 v[100:101], v60 offset:8192\n"
    "v_mfma_f32_16x16x32_bf16 v[180:183], v[108:111], a[236:239], v[180:183]\n"
    "v_mfma_f32_16x16x32_bf16 v[148:151], v[108:111], a[228:231], v[148:151]\n"
    "ds_read_b64_tr_b16 v[102:103], v60 offset:9216\n"
    "v_mfma_f32_16x16x32_bf16 v[148:151], v[104:107], a[224:227], v[148:151]\n"
    "s_waitcnt lgkmcnt(9)\n"
    "v_mfma_f32_16x16x32_bf16 v[152:155], v[112:115], a[224:227], v[152:155]\n"
    "ds_read_b64_tr_b16 v[104:105], v61 offset:0\n"
    "v_mfma_f32_16x16x32_bf16 v[152:155], v[116:119], a[228:231], v[152:155]\n"
    "buffer_load_dwordx4 v57, s[36:39], s64 offen offset:1024 lds\n"
    "v_mfma_f32_16x16x32_bf16 v[184:187], v[116:119], a[236:239], v[184:187]\n"
    "ds_read_b64_tr_b16 v[106:107], v61 offset:1024\n"
    "v_mfma_f32_16x16x32_bf16 v[184:187], v[112:115], a[232:235], v[184:187]\n"
    "v_mfma_f32_16x16x32_bf16 v[216:219], v[112:115], a[240:243], v[216:219]\n"
    "ds_read_b64_tr_b16 v[108:109], v61 offset:8192\n"
    "v_mfma_f32_16x16x32_bf16 v[216:219], v[116:119], a[244:247], v[216:219]\n"
    "v_mfma_f32_16x16x32_bf16 v[248:251], v[116:119], a[252:255], v[248:251]\n"
    "ds_read_b64_tr_b16 v[110:111], v61 offset:9216\n"
    "v_mfma_f32_16x16x32_bf16 v[248:251], v[112:115], a[248:251], v[248:251]\n"
    "buffer_load_dwordx4 v58, s[36:39], s64 offen offset:2048 lds\n"
    "s_waitcnt lgkmcnt(9)\n"
    "v_mfma_f32_16x16x32_bf16 v[252:255], v[120:123], a[248:251], v[252:255]\n"
    "ds_read_b64_tr_b16 v[112:113], v62 offset:0\n"
    "v_mfma_f32_16x16x32_bf16 v[252:255], v[124:127], a[252:255], v[252:255]\n"
    "v_mfma_f32_16x16x32_bf16 v[220:223], v[124:127], a[244:247], v[220:223]\n"
    "ds_read_b64_tr_b16 v[114:115], v62 offset:1024\n"
    "v_mfma_f32_16x16x32_bf16 v[220:223], v[120:123], a[240:243], v[220:223]\n"
    "v_mfma_f32_16x16x32_bf16 v[188:191], v[120:123], a[232:235], v[188:191]\n"
    "ds_read_b64_tr_b16 v[116:117], v62 offset:8192\n"
    "v_mfma_f32_16x16x32_bf16 v[188:191], v[124:127], a[236:239], v[188:191]\n"
    "buffer_load_dwordx4 v59, s[36:39], s64 offen offset:3072 lds\n"
    "v_mfma_f32_16x16x32_bf16 v[156:159], v[124:127], a[228:231], v[156:159]\n"
    "ds_read_b64_tr_b16 v[118:119], v62 offset:9216\n"
    "v_mfma_f32_16x16x32_bf16 v[156:159], v[120:123], a[224:227], v[156:159]\n"
    "s_waitcnt lgkmcnt(8)\n"
    "v_mfma_f32_16x16x32_bf16 v[128:131], v[96:99], a[0:3], v[128:131]\n"
    "ds_read_b64_tr_b16 v[120:121], v63 offset:0\n"
    "v_mfma_f32_16x16x32_bf16 v[128:131], v[100:103], a[4:7], v[128:131]\n"
    "v_readfirstlane_b32 vcc_lo, v36\n"
    "v_readfirstlane_b32 vcc_hi, v37\n"
    "v_mfma_f32_16x16x32_bf16 v[160:163], v[100:103], a[12:15], v[160:163]\n"
    "ds_read_b64_tr_b16 v[122:123], v63 offset:1024\n"
    "v_mfma_f32_16x16x32_bf16 v[160:163], v[96:99], a[8:11], v[160:163]\n"
    "s_or_b64 s[72:73], s[72:73], vcc\n"
    "v_mfma_f32_16x16x32_bf16 v[192:195], v[96:99], a[16:19], v[192:195]\n"
    "ds_read_b64_tr_b16 v[124:125], v63 offset:8192\n"
    "s_andn2_b64 s[76:77], s[72:73], s[80:81]\n"
    "s_cbranch_scc0 1041f\n"
    "1042:\n"
    "s_ff1_i32_b64 s69, s[76:77]\n"
    "v_readlane_b32 s65, v53, s69\n"
    "v_mfma_f32_16x16x32_bf16 v[192:195], v[100:103], a[20:23], v[192:195]\n"
    "s_bitset1_b64 s[80:81], s69\n"
    "s_nop 0\n"
    "s_nop 0\n"
    "v_mfma_f32_16x16x32_bf16 v[224:227], v[100:103], a[28:31], v[224:227]\n"
    "ds_read_b64_tr_b16 v[126:127], v63 offset:9216\n"
    "s_lshl_b32 s64, s65, 5\n"
    "v_mfma_f32_16x16x32_bf16 v[224:227], v[96:99], a[24:27], v[224:227]\n"
    "buffer_load_dwordx4 a[96:99], v54, s[40:43], s65 offen offset:0\n"
    "s_waitcnt lgkmcnt(8)\n"
    "v_mfma_f32_16x16x32_bf16 v[228:231], v[104:107], a[24:27], v[228:231]\n"
    "ds_read_b64_tr_b16 v[96:97], v60 offset:512\n"
    "v_mfma_f32_16x16x32_bf16 v[228:231], v[108:111], a[28:31], v[228:231]\n"
    "s_mov_b32 s25, 1114256\n"
    "s_getpc_b64 s[28:29]\n"
    "s_cmp_ge_i32 s25, s24\n"
    "s_cbranch_scc1 90b\n"
    "v_mfma_f32_16x16x32_bf16 v[196:199], v[108:111], a[20:23], v[196:199]\n"
    "ds_read_b64_tr_b16 v[98:99], v60 offset:1536\n"
    "v_mfma_f32_16x16x32_bf16 v[196:199], v[104:107], a[16:19], v[196:199]\n"
    "buffer_load_dwordx4 a[100:103], v54, s[40:43], s65 offen offset:1024\n"
    "v_mfma_f32_16x16x32_bf16 v[164:167], v[104:107], a[8:11], v[164:167]\n"
    "ds_read_b64_tr_b16 v[100:101], v60 offset:8704\n"
    "v_mfma_f32_16x16x32_bf16 v[164:167], v[108:111], a[12:15], v[164:167]\n"
    "v_cmp_le_u32 vcc, s70, v47\n"
    "v_mfma_f32_16x16x32_bf16 v[132:135], v[108:111], a[4:7], v[132:135]\n"
    "ds_read_b64_tr_b16 v[102:103], v60 offset:9728\n"
    "v_mfma_f32_16x16x32_bf16 v[132:135], v[104:107], a[0:3], v[132:135]\n"
    "s_or_b64 s[82:83], s[82:83], vcc\n"
    "v_cmp_le_u32 vcc, s70, v48\n"
    "s_waitcnt lgkmcnt(8)\n"
    "v_mfma_f32_16x16x32_bf16 v[136:139], v[112:115], a[0:3], v[136:139]\n"
    "ds_read_b64_tr_b16 v[104:105], v61 offset:512\n"
    "buffer_load_dwordx4 a[104:107], v54, s[40:43], s65 offen offset:2048\n"
    "v_mfma_f32_16x16x32_bf16 v[136:139], v[116:119], a[4:7], v[136:139]\n"
    "s_or_b64 s[82:83], s[82:83], vcc\n"
    "v_mfma_f32_16x16x32_bf16 v[168:171], v[116:119], a[12:15], v[168:171]\n"
    "ds_read_b64_tr_b16 v[106:107], v61 offset:1536\n"
    "v_mfma_f32_16x16x32_bf16 v[168:171], v[112:115], a[8:11], v[168:171]\n"
    "v_mfma_f32_16x16x32_bf16 v[200:203], v[112:115], a[16:19], v[200:203]\n"
    "ds_read_b64_tr_b16 v[108:109], v61 offset:8704\n"
    "buffer_load_dwordx4 a[108:111], v54, s[40:43], s65 offen offset:3072\n"
    "v_mfma_f32_16x16x32_bf16 v[200:203], v[116:119], a[20:23], v[200:203]\n"
    "v_mov_b32 v38, s82\n"
    "v_mov_b32 v39, s83\n"
    "ds_write_b64 v41, v[38:39] offset:0\n"
    "v_mfma_f32_16x16x32_bf16 v[232:235], v[116:119], a[28:31], v[232:235]\n"
    "ds_read_b64_tr_b16 v[110:111], v61 offset:9728\n"
    "v_mfma_f32_16x16x32_bf16 v[232:235], v[112:115], a[24:27], v[232:235]\n"
    "s_ff0_i32_b64 s30, s[82:83]\n"
    "s_bfm_b64 vcc, s30, 0\n"
    "s_andn2_b64 vcc, vcc, s[80:81]\n"
    "s_bcnt1_i32_b64 s30, vcc\n"
    "s_cmp_lt_u32 s30, 64\n"
    "s_cselect_b32 s22, 0x10000, 0\n"
    "s_waitcnt lgkmcnt(9)\n"
    "v_mfma_f32_16x16x32_bf16 v[236:239], v[120:123], a[24:27], v[236:239]\n"
    "ds_read_b64_tr_b16 v[112:113], v62 offset:512\n"
    "v_mfma_f32_16x16x32_bf16 v[236:239], v[124:127], a[28:31], v[236:239]\n"
    "s_bcnt1_i32_b64 s30, s[82:83]\n"
    "s_cmp_ge_u32 s30, 56\n"
    "s_cselect_b32 s22, 0, s22\n"
    "v_mfma_f32_16x16x32_bf16 v[204:207], v[124:127], a[20:23], v[204:207]\n"
    "ds_read_b64_tr_b16 v[114:115], v62 offset:1536\n"
    "buffer_load_dwordx4 a[112:115], v55, s[40:43], s65 offen offset:0\n"
    "v_mfma_f32_16x16x32_bf16 v[204:207], v[120:123], a[16:19], v[204:207]\n"
    "buffer_load_dword v47, v49, s[20:23], 0 offen sc0 sc1\n"
    "v_mfma_f32_16x16x32_bf16 v[172:175], v[120:123], a[8:11], v[172:175]\n"
    "ds_read_b64_tr_b16 v[116:117], v62 offset:8704\n"
    "v_mfma_f32_16x16x32_bf16 v[172:175], v[124:127], a[12:15], v[172:175]\n"
    "v_mfma_f32_16x16x32_bf16 v[140:143], v[124:127], a[4:7], v[140:143]\n"
    "ds_read_b64_tr_b16 v[118:119], v62 offset:9728\n"
    "buffer_load_dwordx4 a[116:119], v55, s[40:43], s65 offen offset:1024\n"
    "v_mfma_f32_16x16x32_bf16 v[140:143], v[120:123], a[0:3], v[140:143]\n"
    "s_waitcnt lgkmcnt(9)\n"
    "v_mfma_f32_16x16x32_bf16 v[144:147], v[96:99], a[0:3], v[144:147]\n"
    "ds_read_b64_tr_b16 v[120:121], v63 offset:512\n"
    "v_mfma_f32_16x16x32_bf16 v[144:147], v[100:103], a[4:7], v[144:147]\n"
    "buffer_load_dwordx4 a[120:123], v55, s[40:43], s65 offen offset:2048\n"
    "v_mfma_f32_16x16x32_bf16 v[176:179], v[100:103], a[12:15], v[176:179]\n"
    "ds_read_b64_tr_b16 v[122:123], v63 offset:1536\n"
    "v_mfma_f32_16x16x32_bf16 v[176:179], v[96:99], a[8:11], v[176:179]\n"
    "v_mfma_f32_16x16x32_bf16 v[208:211], v[96:99], a[16:19], v[208:211]\n"
    "ds_read_b64_tr_b16 v[124:125], v63 offset:8704\n"
    "buffer_load_dwordx4 a[124:127], v55, s[40:43], s65 offen offset:3072\n"
    "v_mfma_f32_16x16x32_bf16 v[208:211], v[100:103], a[20:23], v[208:211]\n"
    "s_mov_b32 s25, 1179800\n"
    "s_getpc_b64 s[28:29]\n"
    "s_cmp_ge_i32 s25, s24\n"
    "s_cbranch_scc1 90b\n"
    "v_mfma_f32_16x16x32_bf16 v[240:243], v[100:103], a[28:31], v[240:243]\n"
    "ds_read_b64_tr_b16 v[126:127], v63 offset:9728\n"
    "v_mfma_f32_16x16x32_bf16 v[240:243], v[96:99], a[24:27], v[240:243]\n"
    "s_waitcnt lgkmcnt(8)\n"
    "s_waitcnt vmcnt(22)\n"
    "s_barrier\n"
    "v_mfma_f32_16x16x32_bf16 v[244:247], v[104:107], a[24:27], v[244:247]\n"
    "ds_read_b64_tr_b16 v[96:97], v60 offset:16384\n"
    "s_add_u32 m0, s67, 49152\n"
    "s_nop 0\n"
    "v_mfma_f32_16x16x32_bf16 v[244:247], v[108:111], a[28:31], v[244:247]\n"
    "ds_read_b64 v[36:37], v40 offset:0\n"
    "v_mfma_f32_16x16x32_bf16 v[212:215], v[108:111], a[20:23], v[212:215]\n"
    "ds_read_b64_tr_b16 v[98:99], v60 offset:17408\n"
    "v_mfma_f32_16x16x32_bf16 v[212:215], v[104:107], a[16:19], v[212:215]\n"
    "buffer_load_dwordx4 v56, s[36:39], s64 offen offset:0 lds\n"
    "v_mfma_f32_16x16x32_bf16 v[180:183], v[104:107], a[8:11], v[180:183]\n"
    "ds_read_b64_tr_b16 v[100:101], v60 offset:24576\n"
    "v_mfma_f32_16x16x32_bf16 v[180:183], v[108:111], a[12:15], v[180:183]\n"
    "v_mfma_f32_16x16x32_bf16 v[148:151], v[108:111], a[4:7], v[148:151]\n"
    "ds_read_b64_tr_b16 v[102:103], v60 offset:25600\n"
    "v_mfma_f32_16x16x32_bf16 v[148:151], v[104:107], a[0:3], v[148:151]\n"
    "s_waitcnt lgkmcnt(9)\n"
    "v_mfma_f32_16x16x32_bf16 v[152:155], v[112:115], a[0:3], v[152:155]\n"
    "ds_read_b64_tr_b16 v[104:105], v61 offset:16384\n"
    "v_mfma_f32_16x16x32_bf16 v[152:155], v[116:119], a[4:7], v[152:155]\n"
    "buffer_load_dwordx4 v57, s[36:39], s64 offen offset:1024 lds\n"
    "v_mfma_f32_16x16x32_bf16 v[184:187], v[116:119], a[12:15], v[184:187]\n"
    "ds_read_b64_tr_b16 v[106:107], v61 offset:17408\n"
    "v_mfma_f32_16x16x32_bf16 v[184:187], v[112:115], a[8:11], v[184:187]\n"
    "v_mfma_f32_16x16x32_bf16 v[216:219], v[112:115], a[16:19], v[216:219]\n"
    "ds_read_b64_tr_b16 v[108:109], v61 offset:24576\n"
    "v_mfma_f32_16x16x32_bf16 v[216:219], v[116:119], a[20:23], v[216:219]\n"
    "v_mfma_f32_16x16x32_bf16 v[248:251], v[116:119], a[28:31], v[248:251]\n"
    "ds_read_b64_tr_b16 v[110:111], v61 offset:25600\n"
    "v_mfma_f32_16x16x32_bf16 v[248:251], v[112:115], a[24:27], v[248:251]\n"
    "buffer_load_dwordx4 v58, s[36:39], s64 offen offset:2048 lds\n"
    "s_waitcnt lgkmcnt(9)\n"
    "v_mfma_f32_16x16x32_bf16 v[252:255], v[120:123], a[24:27], v[252:255]\n"
    "ds_read_b64_tr_b16 v[112:113], v62 offset:16384\n"
    "v_mfma_f32_16x16x32_bf16 v[252:255], v[124:127], a[28:31], v[252:255]\n"
    "v_mfma_f32_16x16x32_bf16 v[220:223], v[124:127], a[20:23], v[220:223]\n"
    "ds_read_b64_tr_b16 v[114:115], v62 offset:17408\n"
    "v_mfma_f32_16x16x32_bf16 v[220:223], v[120:123], a[16:19], v[220:223]\n"
    "v_mfma_f32_16x16x32_bf16 v[188:191], v[120:123], a[8:11], v[188:191]\n"
    "ds_read_b64_tr_b16 v[116:117], v62 offset:24576\n"
    "v_mfma_f32_16x16x32_bf16 v[188:191], v[124:127], a[12:15], v[188:191]\n"
    "buffer_load_dwordx4 v59, s[36:39], s64 offen offset:3072 lds\n"
    "v_mfma_f32_16x16x32_bf16 v[156:159], v[124:127], a[4:7], v[156:159]\n"
    "ds_read_b64_tr_b16 v[118:119], v62 offset:25600\n"
    "v_mfma_f32_16x16x32_bf16 v[156:159], v[120:123], a[0:3], v[156:159]\n"
    "s_waitcnt lgkmcnt(8)\n"
    "v_mfma_f32_16x16x32_bf16 v[128:131], v[96:99], a[32:35], v[128:131]\n"
    "ds_read_b64_tr_b16 v[120:121], v63 offset:16384\n"
    "v_mfma_f32_16x16x32_bf16 v[128:131], v[100:103], a[36:39], v[128:131]\n"
    "v_readfirstlane_b32 vcc_lo, v36\n"
    "v_readfirstlane_b32 vcc_hi, v37\n"
    "v_mfma_f32_16x16x32_bf16 v[160:163], v[100:103], a[44:47], v[160:163]\n"
    "ds_read_b64_tr_b16 v[122:123], v63 offset:17408\n"
    "v_mfma_f32_16x16x32_bf16 v[160:163], v[96:99], a[40:43], v[160:163]\n"
    "s_or_b64 s[72:73], s[72:73], vcc\n"
    "v_mfma_f32_16x16x32_bf16 v[192:195], v[96:99], a[48:51], v[192:195]\n"
    "ds_read_b64_tr_b16 v[124:125], v63 offset:24576\n"
    "s_andn2_b64 s[76:77], s[72:73], s[80:81]\n"
    "s_cbranch_scc0 1051f\n"
    "1052:\n"
    "s_ff1_i32_b64 s69, s[76:77]\n"
    "v_mfma_f32_16x16x32_bf16 v[192:195], v[100:103], a[52:55], v[192:195]\n"
    "v_readlane_b32 s65, v53, s69\n"
    "s_bitset1_b64 s[80:81], s69\n"
    "v_mfma_f32_16x16x32_bf16 v[224:227], v[100:103], a[60:63], v[224:227]\n"
    "ds_read_b64_tr_b16 v[126:127], v63 offset:25600\n"
    "s_nop 0\n"
    "v_mfma_f32_16x16x32_bf16 v[224:227], v[96:99], a[56:59], v[224:227]\n"
    "s_nop 0\n"
    "s_lshl_b32 s64, s65, 5\n"
    "s_waitcnt lgkmcnt(8)\n"
    "v_mfma_f32_16x16x32_bf16 v[228:231], v[104:107], a[56:59], v[228:231]\n"
    "ds_read_b64_tr_b16 v[96:97], v60 offset:16896\n"
    "s_add_u32 m0, s67, 65536\n"
    "v_mfma_f32_16x16x32_bf16 v[228:231], v[108:111], a[60:63], v[228:231]\n"
    "s_mov_b32 s25, 1245340\n"
    "s_getpc_b64 s[28:29]\n"
    "s_cmp_ge_i32 s25, s24\n"
    "s_cbranch_scc1 90b\n"
    "v_mfma_f32_16x16x32_bf16 v[196:199], v[108:111], a[52:55], v[196:199]\n"
    "ds_read_b64_tr_b16 v[98:99], v60 offset:17920\n"
    "s_nop 0\n"
    "v_mfma_f32_16x16x32_bf16 v[196:199], v[104:107], a[48:51], v[196:199]\n"
    "buffer_load_dwordx4 v56, s[36:39], s64 offen offset:0 lds\n"
    "v_mfma_f32_16x16x32_bf16 v[164:167], v[104:107], a[40:43], v[164:167]\n"
    "ds_read_b64_tr_b16 v[100:101], v60 offset:25088\n"
    "v_mfma_f32_16x16x32_bf16 v[164:167], v[108:111], a[44:47], v[164:167]\n"
    "v_cmp_le_u32 vcc, s70, v47\n"
    "v_mfma_f32_16x16x32_bf16 v[132:135], v[108:111], a[36:39], v[132:135]\n"
    "ds_read_b64_tr_b16 v[102:103], v60 offset:26112\n"
    "buffer_load_dwordx4 v57, s[36:39], s64 offen offset:1024 lds\n"
    "v_mfma_f32_16x16x32_bf16 v[132:135], v[104:107], a[32:35], v[132:135]\n"
    "s_or_b64 s[82:83], s[82:83], vcc\n"
    "v_cmp_le_u32 vcc, s70, v48\n"
    "s_waitcnt lgkmcnt(8)\n"
    "v_mfma_f32_16x16x32_bf16 v[136:139], v[112:115], a[32:35], v[136:139]\n"
    "ds_read_b64_tr_b16 v[104:105], v61 offset:16896\n"
    "v_mfma_f32_16x16x32_bf16 v[136:139], v[116:119], a[36:39], v[136:139]\n"
    "s_or_b64 s[82:83], s[82:83], vcc\n"
    "v_mfma_f32_16x16x32_bf16 v[168:171], v[116:119], a[44:47], v[168:171]\n"
    "ds_read_b64_tr_b16 v[106:107], v61 offset:17920\n"
    "buffer_load_dwordx4 v58, s[36:39], s64 offen offset:2048 lds\n"
    "v_mfma_f32_16x16x32_bf16 v[168:171], v[112:115], a[40:43], v[168:171]\n"
    "buffer_load_dwordx4 v59, s[36:39], s64 offen offset:3072 lds\n"
    "v_mfma_f32_16x16x32_bf16 v[200:203], v[112:115], a[48:51], v[200:203]\n"
    "ds_read_b64_tr_b16 v[108:109], v61 offset:25088\n"
    "v_mfma_f32_16x16x32_bf16 v[200:203], v[116:119], a[52:55], v[200:203]\n"
    "v_mov_b32 v38, s82\n"
    "v_mov_b32 v39, s83\n"
    "ds_write_b64 v41, v[38:39] offset:8\n"
    "v_mfma_f32_16x16x32_bf16 v[232:235], v[116:119], a[60:63], v[232:235]\n"
    "ds_read_b64_tr_b16 v[110:111], v61 offset:26112\n"
    "v_mfma_f32_16x16x32_bf16 v[232:235], v[112:115], a[56:59], v[232:235]\n"
    "s_ff0_i32_b64 s30, s[82:83]\n"
    "s_bfm_b64 vcc, s30, 0\n"
    "s_andn2_b64 vcc, vcc, s[80:81]\n"
    "s_bcnt1_i32_b64 s30, vcc\n"
    "s_cmp_lt_u32 s30, 64\n"
    "s_cselect_b32 s22, 0x10000, 0\n"
    "s_waitcnt lgkmcnt(9)\n"
    "v_mfma_f32_16x16x32_bf16 v[236:239], v[120:123], a[56:59], v[236:239]\n"
    "ds_read_b64_tr_b16 v[112:113], v62 offset:16896\n"
    "buffer_load_dwordx4 a[128:131], v54, s[40:43], s65 offen offset:0\n"
    "v_mfma_f32_16x16x32_bf16 v[236:239], v[124:127], a[60:63], v[236:239]\n"
    "s_bcnt1_i32_b64 s30, s[82:83]\n"
    "s_cmp_ge_u32 s30, 56\n"
    "s_cselect_b32 s22, 0, s22\n"
    "v_mfma_f32_16x16x32_bf16 v[204:207], v[124:127], a[52:55], v[204:207]\n"
    "ds_read_b64_tr_b16 v[114:115], v62 offset:17920\n"
    "v_mfma_f32_16x16x32_bf16 v[204:207], v[120:123], a[48:51], v[204:207]\n"
    "buffer_load_dword v48, v49, s[20:23], 0 offen sc0 sc1\n"
    "v_mfma_f32_16x16x32_bf16 v[172:175], v[120:123], a[40:43], v[172:175]\n"
    "ds_read_b64_tr_b16 v[116:117], v62 offset:25088\n"
    "buffer_load_dwordx4 a[132:135], v54, s[40:43], s65 offen offset:1024\n"
    "v_mfma_f32_16x16x32_bf16 v[172:175], v[124:127], a[44:47], v[172:175]\n"
    "buffer_load_dwordx4 a[136:139], v54, s[40:43], s65 offen offset:2048\n"
    "v_mfma_f32_16x16x32_bf16 v[140:143], v[124:127], a[36:39], v[140:143]\n"
    "ds_read_b64_tr_b16 v[118:119], v62 offset:26112\n"
    "v_mfma_f32_16x16x32_bf16 v[140:143], v[120:123], a[32:35], v[140:143]\n"
    "s_waitcnt lgkmcnt(9)\n"
    "v_mfma_f32_16x16x32_bf16 v[144:147], v[96:99], a[32:35], v[144:147]\n"
    "ds_read_b64_tr_b16 v[120:121], v63 offset:16896\n"
    "v_mfma_f32_16x16x32_bf16 v[144:147], v[100:103], a[36:39], v[144:147]\n"
    "buffer_load_dwordx4 a[140:143], v54, s[40:43], s65 offen offset:3072\n"
    "v_mfma_f32_16x16x32_bf16 v[176:179], v[100:103], a[44:47], v[176:179]\n"
    "ds_read_b64_tr_b16 v[122:123], v63 offset:17920\n"
    "v_mfma_f32_16x16x32_bf16 v[176:179], v[96:99], a[40:43], v[176:179]\n"
    "v_mfma_f32_16x16x32_bf16 v[208:211], v[96:99], a[48:51], v[208:211]\n"
    "ds_read_b64_tr_b16 v[124:125], v63 offset:25088\n"
    "v_mfma_f32_16x16x32_bf16 v[208:211], v[100:103], a[52:55], v[208:211]\n"
    "s_mov_b32 s25, 1310885\n"
    "s_getpc_b64 s[28:29]\n"
    "s_cmp_ge_i32 s25, s24\n"
    "s_cbranch_scc1 90b\n"
    "v_mfma_f32_16x16x32_bf16 v[240:243], v[100:103], a[60:63], v[240:243]\n"
    "ds_read_b64_tr_b16 v[126:127], v63 offset:26112\n"
    "buffer_load_dwordx4 a[144:147], v55, s[40:43], s65 offen offset:0\n"
    "v_mfma_f32_16x16x32_bf16 v[240:243], v[96:99], a[56:59], v[240:243]\n"
    "buffer_load_dwordx4 a[148:151], v55, s[40:43], s65 offen offset:1024\n"
    "s_waitcnt lgkmcnt(8)\n"
    "s_waitcnt vmcnt(24)\n"
    "s_barrier\n"
    "v_mfma_f32_16x16x32_bf16 v[244:247], v[104:107], a[56:59], v[244:247]\n"
    "ds_read_b64_tr_b16 v[96:97], v60 offset:32768\n"
    "v_mfma_f32_16x16x32_bf16 v[244:247], v[108:111], a[60:63], v[244:247]\n"
    "ds_read_b64 v[36:37], v40 offset:8\n"
    "v_mfma_f32_16x16x32_bf16 v[212:215], v[108:111], a[52:55], v[212:215]\n"
    "ds_read_b64_tr_b16 v[98:99], v60 offset:33792\n"
    "v_mfma_f32_16x16x32_bf16 v[212:215], v[104:107], a[48:51], v[212:215]\n"
    "buffer_load_dwordx4 a[152:155], v55, s[40:43], s65 offen offset:2048\n"
    "v_mfma_f32_16x16x32_bf16 v[180:183], v[104:107], a[40:43], v[180:183]\n"
    "ds_read_b64_tr_b16 v[100:101], v60 offset:40960\n"
    "v_mfma_f32_16x16x32_bf16 v[180:183], v[108:111], a[44:47], v[180:183]\n"
    "v_mfma_f32_16x16x32_bf16 v[148:151], v[108:111], a[36:39], v[148:151]\n"
    "ds_read_b64_tr_b16 v[102:103], v60 offset:41984\n"
    "v_mfma_f32_16x16x32_bf16 v[148:151], v[104:107], a[32:35], v[148:151]\n"
    "buffer_load_dwordx4 a[156:159], v55, s[40:43], s65 offen offset:3072\n"
    "s_waitcnt lgkmcnt(9)\n"
    "v_mfma_f32_16x16x32_bf16 v[152:155], v[112:115], a[32:35], v[152:155]\n"
    "ds_read_b64_tr_b16 v[104:105], v61 offset:32768\n"
    "s_andn2_b64 s[76:77], s[72:73], s[80:81]\n"
    "s_cbranch_scc0 1061f\n"
    "1062:\n"
    "s_ff1_i32_b64 s69, s[76:77]\n"
    "v_mfma_f32_16x16x32_bf16 v[152:155], v[116:119], a[36:39], v[152:155]\n"
    "v_readlane_b32 s65, v53, s69\n"
    "s_bitset1_b64 s[80:81], s69\n"
    "v_mfma_f32_16x16x32_bf16 v[184:187], v[116:119], a[44:47], v[184:187]\n"
    "ds_read_b64_tr_b16 v[106:107], v61 offset:33792\n"
    "s_nop 0\n"
    "v_mfma_f32_16x16x32_bf16 v[184:187], v[112:115], a[40:43], v[184:187]\n"
    "s_nop 0\n"
    "s_lshl_b32 s64, s65, 5\n"
    "v_mfma_f32_16x16x32_bf16 v[216:219], v[112:115], a[48:51], v[216:219]\n"
    "ds_read_b64_tr_b16 v[108:109], v61 offset:40960\n"
    "s_add_u32 m0, s67, 81920\n"
    "v_mfma_f32_16x16x32_bf16 v[216:219], v[116:119], a[52:55], v[216:219]\n"
    "s_nop 0\n"
    "v_mfma_f32_16x16x32_bf16 v[248:251], v[116:119], a[60:63], v[248:251]\n"
    "ds_read_b64_tr_b16 v[110:111], v61 offset:41984\n"
    "v_mfma_f32_16x16x32_bf16 v[248:251], v[112:115], a[56:59], v[248:251]\n"
    "buffer_load_dwordx4 v56, s[36:39], s64 offen offset:0 lds\n"
    "s_waitcnt lgkmcnt(9)\n"
    "v_mfma_f32_16x16x32_bf16 v[252:255], v[120:123], a[56:59], v[252:255]\n"
    "ds_read_b64_tr_b16 v[112:113], v62 offset:32768\n"
    "v_mfma_f32_16x16x32_bf16 v[252:255], v[124:127], a[60:63], v[252:255]\n"
    "buffer_load_dwordx4 v57, s[36:39], s64 offen offset:1024 lds\n"
    "v_mfma_f32_16x16x32_bf16 v[220:223], v[124:127], a[52:55], v[220:223]\n"
    "ds_read_b64_tr_b16 v[114:115], v62 offset:33792\n"
    "v_mfma_f32_16x16x32_bf16 v[220:223], v[120:123], a[48:51], v[220:223]\n"
    "buffer_load_dwordx4 v58, s[36:39], s64 offen offset:2048 lds\n"
    "v_mfma_f32_16x16x32_bf16 v[188:191], v[120:123], a[40:43], v[188:191]\n"
    "ds_read_b64_tr_b16 v[116:117], v62 offset:40960\n"
    "v_mfma_f32_16x16x32_bf16 v[188:191], v[124:127], a[44:47], v[188:191]\n"
    "v_mfma_f32_16x16x32_bf16 v[156:159], v[124:127], a[36:39], v[156:159]\n"
    "ds_read_b64_tr_b16 v[118:119], v62 offset:41984\n"
    "v_mfma_f32_16x16x32_bf16 v[156:159], v[120:123], a[32:35], v[156:159]\n"
    "buffer_load_dwordx4 v59, s[36:39], s64 offen offset:3072 lds\n"
    "s_waitcnt lgkmcnt(8)\n"
    "v_mfma_f32_16x16x32_bf16 v[128:131], v[96:99], a[64:67], v[128:131]\n"
    "ds_read_b64_tr_b16 v[120:121], v63 offset:32768\n"
    "v_mfma_f32_16x16x32_bf16 v[128:131], v[100:103], a[68:71], v[128:131]\n"
    "v_readfirstlane_b32 vcc_lo, v36\n"
    "v_readfirstlane_b32 vcc_hi, v37\n"
    "v_mfma_f32_16x16x32_bf16 v[160:163], v[100:103], a[76:79], v[160:163]\n"
    "ds_read_b64_tr_b16 v[122:123], v63 offset:33792\n"
    "v_mfma_f32_16x16x32_bf16 v[160:163], v[96:99], a[72:75], v[160:163]\n"
    "s_or_b64 s[72:73], s[72:73], vcc\n"
    "v_mfma_f32_16x16x32_bf16 v[192:195], v[96:99], a[80:83], v[192:195]\n"
    "ds_read_b64_tr_b16 v[124:125], v63 offset:40960\n"
    "buffer_load_dwordx4 a[160:163], v54, s[40:43], s65 offen offset:0\n"
    "v_mfma_f32_16x16x32_bf16 v[192:195], v[100:103], a[84:87], v[192:195]\n"
    "v_mfma_f32_16x16x32_bf16 v[224:227], v[100:103], a[92:95], v[224:227]\n"
    "ds_read_b64_tr_b16 v[126:127], v63 offset:41984\n"
    "v_mfma_f32_16x16x32_bf16 v[224:227], v[96:99], a[88:91], v[224:227]\n"
    "buffer_load_dwordx4 a[164:167], v54, s[40:43], s65 offen offset:1024\n"
    "s_waitcnt lgkmcnt(8)\n"
    "v_mfma_f32_16x16x32_bf16 v[228:231], v[104:107], a[88:91], v[228:231]\n"
    "ds_read_b64_tr_b16 v[96:97], v60 offset:33280\n"
    "v_mfma_f32_16x16x32_bf16 v[228:231], v[108:111], a[92:95], v[228:231]\n"
    "s_mov_b32 s25, 1376431\n"
    "s_getpc_b64 s[28:29]\n"
    "s_cmp_ge_i32 s25, s24\n"
    "s_cbranch_scc1 90b\n"
    "v_mfma_f32_16x16x32_bf16 v[196:199], v[108:111], a[84:87], v[196:199]\n"
    "ds_read_b64_tr_b16 v[98:99], v60 offset:34304\n"
    "v_mfma_f32_16x16x32_bf16 v[196:199], v[104:107], a[80:83], v[196:199]\n"
    "buffer_load_dwordx4 a[168:171], v54, s[40:43], s65 offen offset:2048\n"
    "v_mfma_f32_16x16x32_bf16 v[164:167], v[104:107], a[72:75], v[164:167]\n"
    "ds_read_b64_tr_b16 v[100:101], v60 offset:41472\n"
    "v_mfma_f32_16x16x32_bf16 v[164:167], v[108:111], a[76:79], v[164:167]\n"
    "v_cmp_le_u32 vcc, s70, v47\n"
    "v_mfma_f32_16x16x32_bf16 v[132:135], v[108:111], a[68:71], v[132:135]\n"
    "ds_read_b64_tr_b16 v[102:103], v60 offset:42496\n"
    "buffer_load_dwordx4 a[172:175], v54, s[40:43], s65 offen offset:3072\n"
    "v_mfma_f32_16x16x32_bf16 v[132:135], v[104:107], a[64:67], v[132:135]\n"
    "s_or_b64 s[82:83], s[82:83], vcc\n"
    "v_cmp_le_u32 vcc, s70, v48\n"
    "s_waitcnt lgkmcnt(8)\n"
    "v_mfma_f32_16x16x32_bf16 v[136:139], v[112:115], a[64:67], v[136:139]\n"
    "ds_read_b64_tr_b16 v[104:105], v61 offset:33280\n"
    "v_mfma_f32_16x16x32_bf16 v[136:139], v[116:119], a[68:71], v[136:139]\n"
    "s_or_b64 s[82:83], s[82:83], vcc\n"
    "v_mfma_f32_16x16x32_bf16 v[168:171], v[116:119], a[76:79], v[168:171]\n"
    "ds_read_b64_tr_b16 v[106:107], v61 offset:34304\n"
    "buffer_load_dwordx4 a[176:179], v55, s[40:43], s65 offen offset:0\n"
    "v_mfma_f32_16x16x32_bf16 v[168:171], v[112:115], a[72:75], v[168:171]\n"
    "buffer_load_dwordx4 a[180:183], v55, s[40:43], s65 offen offset:1024\n"
    "v_mfma_f32_16x16x32_bf16 v[200:203], v[112:115], a[80:83], v[200:203]\n"
    "ds_read_b64_tr_b16 v[108:109], v61 offset:41472\n"
    "v_mfma_f32_16x16x32_bf16 v[200:203], v[116:119], a[84:87], v[200:203]\n"
    "v_mov_b32 v38, s82\n"
    "v_mov_b32 v39, s83\n"
    "ds_write_b64 v41, v[38:39] offset:0\n"
    "v_mfma_f32_16x16x32_bf16 v[232:235], v[116:119], a[92:95], v[232:235]\n"
    "ds_read_b64_tr_b16 v[110:111], v61 offset:42496\n"
    "v_mfma_f32_16x16x32_bf16 v[232:235], v[112:115], a[88:91], v[232:235]\n"
    "s_ff0_i32_b64 s30, s[82:83]\n"
    "s_bfm_b64 vcc, s30, 0\n"
    "s_andn2_b64 vcc, vcc, s[80:81]\n"
    "s_bcnt1_i32_b64 s30, vcc\n"
    "s_cmp_lt_u32 s30, 64\n"
    "s_cselect_b32 s22, 0x10000, 0\n"
    "s_waitcnt lgkmcnt(9)\n"
    "v_mfma_f32_16x16x32_bf16 v[236:239], v[120:123], a[88:91], v[236:239]\n"
    "ds_read_b64_tr_b16 v[112:113], v62 offset:33280\n"
    "v_mfma_f32_16x16x32_bf16 v[236:239], v[124:127], a[92:95], v[236:239]\n"
    "s_bcnt1_i32_b64 s30, s[82:83]\n"
    "s_cmp_ge_u32 s30, 56\n"
    "s_cselect_b32 s22, 0, s22\n"
    "v_mfma_f32_16x16x32_bf16 v[204:207], v[124:127], a[84:87], v[204:207]\n"
    "ds_read_b64_tr_b16 v[114:115], v62 offset:34304\n"
    "buffer_load_dwordx4 a[184:187], v55, s[40:43], s65 offen offset:2048\n"
    "v_mfma_f32_16x16x32_bf16 v[204:207], v[120:123], a[80:83], v[204:207]\n"
    "buffer_load_dword v47, v49, s[20:23], 0 offen sc0 sc1\n"
    "v_mfma_f32_16x16x32_bf16 v[172:175], v[120:123], a[72:75], v[172:175]\n"
    "ds_read_b64_tr_b16 v[116:117], v62 offset:41472\n"
    "buffer_load_dwordx4 a[188:191], v55, s[40:43], s65 offen offset:3072\n"
    "v_mfma_f32_16x16x32_bf16 v[172:175], v[124:127], a[76:79], v[172:175]\n"
    "s_andn2_b64 s[76:77], s[72:73], s[80:81]\n"
    "s_cbranch_scc0 1071f\n"
    "1072:\n"
    "s_ff1_i32_b64 s69, s[76:77]\n"
    "v_readlane_b32 s65, v53, s69\n"
    "v_mfma_f32_16x16x32_bf16 v[140:143], v[124:127], a[68:71], v[140:143]\n"
    "ds_read_b64_tr_b16 v[118:119], v62 offset:42496\n"
    "s_bitset1_b64 s[80:81], s69\n"
    "v_mfma_f32_16x16x32_bf16 v[140:143], v[120:123], a[64:67], v[140:143]\n"
    "s_nop 0\n"
    "s_nop 0\n"
    "s_waitcnt lgkmcnt(9)\n"
    "v_mfma_f32_16x16x32_bf16 v[144:147], v[96:99], a[64:67], v[144:147]\n"
    "ds_read_b64_tr_b16 v[120:121], v63 offset:33280\n"
    "s_lshl_b32 s64, s65, 5\n"
    "v_mfma_f32_16x16x32_bf16 v[144:147], v[100:103], a[68:71], v[144:147]\n"
    "s_add_u32 m0, s67, 98304\n"
    "s_nop 0\n"
    "v_mfma_f32_16x16x32_bf16 v[176:179], v[100:103], a[76:79], v[176:179]\n"
    "ds_read_b64_tr_b16 v[122:123], v63 offset:34304\n"
    "v_mfma_f32_16x16x32_bf16 v[176:179], v[96:99], a[72:75], v[176:179]\n"
    "buffer_load_dwordx4 v56, s[36:39], s64 offen offset:0 lds\n"
    "v_mfma_f32_16x16x32_bf16 v[208:211], v[96:99], a[80:83], v[208:211]\n"
    "ds_read_b64_tr_b16 v[124:125], v63 offset:41472\n"
    "v_mfma_f32_16x16x32_bf16 v[208:211], v[100:103], a[84:87], v[208:211]\n"
    "s_mov_b32 s25, 1441975\n"
    "s_getpc_b64 s[28:29]\n"
    "s_cmp_ge_i32 s25, s24\n"
    "s_cbranch_scc1 90b\n"
    "v_mfma_f32_16x16x32_bf16 v[240:243], v[100:103], a[92:95], v[240:243]\n"
    "ds_read_b64_tr_b16 v[126:127], v63 offset:42496\n"
    "buffer_load_dwordx4 v57, s[36:39], s64 offen offset:1024 lds\n"
    "v_mfma_f32_16x16x32_bf16 v[240:243], v[96:99], a[88:91], v[240:243]\n"
    "buffer_load_dwordx4 v58, s[36:39], s64 offen offset:2048 lds\n"
    "s_waitcnt lgkmcnt(8)\n"
    "s_waitcnt vmcnt(29)\n"
    "s_barrier\n"
    "v_mfma_f32_16x16x32_bf16 v[244:247], v[104:107], a[88:91], v[244:247]\n"
    "ds_read_b64_tr_b16 v[96:97], v60 offset:49152\n"
    "v_mfma_f32_16x16x32_bf16 v[244:247], v[108:111], a[92:95], v[244:247]\n"
    "ds_read_b64 v[36:37], v40 offset:0\n"
    "v_mfma_f32_16x16x32_bf16 v[212:215], v[108:111], a[84:87], v[212:215]\n"
    "ds_read_b64_tr_b16 v[98:99], v60 offset:50176\n"
    "v_mfma_f32_16x16x32_bf16 v[212:215], v[104:107], a[80:83], v[212:215]\n"
    "buffer_load_dwordx4 v59, s[36:39], s64 offen offset:3072 lds\n"
    "v_mfma_f32_16x16x32_bf16 v[180:183], v[104:107], a[72:75], v[180:183]\n"
    "ds_read_b64_tr_b16 v[100:101], v60 offset:57344\n"
    "v_mfma_f32_16x16x32_bf16 v[180:183], v[108:111], a[76:79], v[180:183]\n"
    "v_mfma_f32_16x16x32_bf16 v[148:151], v[108:111], a[68:71], v[148:151]\n"
    "ds_read_b64_tr_b16 v[102:103], v60 offset:58368\n"
    "v_mfma_f32_16x16x32_bf16 v[148:151], v[104:107], a[64:67], v[148:151]\n"
    "buffer_load_dwordx4 a[192:195], v54, s[40:43], s65 offen offset:0\n"
    "s_waitcnt lgkmcnt(9)\n"
    "v_mfma_f32_16x16x32_bf16 v[152:155], v[112:115], a[64:67], v[152:155]\n"
    "ds_read_b64_tr_b16 v[104:105], v61 offset:49152\n"
    "v_mfma_f32_16x16x32_bf16 v[152:155], v[116:119], a[68:71], v[152:155]\n"
    "v_mfma_f32_16x16x32_bf16 v[184:187], v[116:119], a[76:79], v[184:187]\n"
    "ds_read_b64_tr_b16 v[106:107], v61 offset:50176\n"
    "v_mfma_f32_16x16x32_bf16 v[184:187], v[112:115], a[72:75], v[184:187]\n"
    "buffer_load_dwordx4 a[196:199], v54, s[40:43], s65 offen offset:1024\n"
    "v_mfma_f32_16x16x32_bf16 v[216:219], v[112:115], a[80:83], v[216:219]\n"
    "ds_read_b64_tr_b16 v[108:109], v61 offset:57344\n"
    "v_mfma_f32_16x16x32_bf16 v[216:219], v[116:119], a[84:87], v[216:219]\n"
    "buffer_load_dwordx4 a[200:203], v54, s[40:43], s65 offen offset:2048\n"
    "v_mfma_f32_16x16x32_bf16 v[248:251], v[116:119], a[92:95], v[248:251]\n"
    "ds_read_b64_tr_b16 v[110:111], v61 offset:58368\n"
    "v_mfma_f32_16x16x32_bf16 v[248:251], v[112:115], a[88:91], v[248:251]\n"
    "s_waitcnt lgkmcnt(9)\n"
    "v_mfma_f32_16x16x32_bf16 v[252:255], v[120:123], a[88:91], v[252:255]\n"
    "ds_read_b64_tr_b16 v[112:113], v62 offset:49152\n"
    "v_mfma_f32_16x16x32_bf16 v[252:255], v[124:127], a[92:95], v[252:255]\n"
    "buffer_load_dwordx4 a[204:207], v54, s[40:43], s65 offen offset:3072\n"
    "v_mfma_f32_16x16x32_bf16 v[220:223], v[124:127], a[84:87], v[220:223]\n"
    "ds_read_b64_tr_b16 v[114:115], v62 offset:50176\n"
    "v_mfma_f32_16x16x32_bf16 v[220:223], v[120:123], a[80:83], v[220:223]\n"
    "v_mfma_f32_16x16x32_bf16 v[188:191], v[120:123], a[72:75], v[188:191]\n"
    "ds_read_b64_tr_b16 v[116:117], v62 offset:57344\n"
    "v_mfma_f32_16x16x32_bf16 v[188:191], v[124:127], a[76:79], v[188:191]\n"
    "buffer_load_dwordx4 a[208:211], v55, s[40:43], s65 offen offset:0\n"
    "v_mfma_f32_16x16x32_bf16 v[156:159], v[124:127], a[68:71], v[156:159]\n"
    "ds_read_b64_tr_b16 v[118:119], v62 offset:58368\n"
    "v_mfma_f32_16x16x32_bf16 v[156:159], v[120:123], a[64:67], v[156:159]\n"
    "buffer_load_dwordx4 a[212:215], v55, s[40:43], s65 offen offset:1024\n"
    "s_waitcnt lgkmcnt(8)\n"
    "v_mfma_f32_16x16x32_bf16 v[128:131], v[96:99], a[96:99], v[128:131]\n"
    "ds_read_b64_tr_b16 v[120:121], v63 offset:49152\n"
    "v_mfma_f32_16x16x32_bf16 v[128:131], v[100:103], a[100:103], v[128:131]\n"
    "v_readfirstlane_b32 vcc_lo, v36\n"
    "v_readfirstlane_b32 vcc_hi, v37\n"
    "v_mfma_f32_16x16x32_bf16 v[160:163], v[100:103], a[108:111], v[160:163]\n"
    "ds_read_b64_tr_b16 v[122:123], v63 offset:50176\n"
    "v_mfma_f32_16x16x32_bf16 v[160:163], v[96:99], a[104:107], v[160:163]\n"
    "s_or_b64 s[72:73], s[72:73], vcc\n"
    "v_mfma_f32_16x16x32_bf16 v[192:195], v[96:99], a[112:115], v[192:195]\n"
    "ds_read_b64_tr_b16 v[124:125], v63 offset:57344\n"
    "v_mfma_f32_16x16x32_bf16 v[192:195], v[100:103], a[116:119], v[192:195]\n"
    "buffer_load_dwordx4 a[216:219], v55, s[40:43], s65 offen offset:2048\n"
    "v_mfma_f32_16x16x32_bf16 v[224:227], v[100:103], a[124:127], v[224:227]\n"
    "ds_read_b64_tr_b16 v[126:127], v63 offset:58368\n"
    "v_mfma_f32_16x16x32_bf16 v[224:227], v[96:99], a[120:123], v[224:227]\n"
    "buffer_load_dwordx4 a[220:223], v55, s[40:43], s65 offen offset:3072\n"
    "s_waitcnt lgkmcnt(8)\n"
    "v_mfma_f32_16x16x32_bf16 v[228:231], v[104:107], a[120:123], v[228:231]\n"
    "ds_read_b64_tr_b16 v[96:97], v60 offset:49664\n"
    "s_andn2_b64 s[76:77], s[72:73], s[80:81]\n"
    "s_cbranch_scc0 1081f\n"
    "1082:\n"
    "s_ff1_i32_b64 s69, s[76:77]\n"
    "v_mfma_f32_16x16x32_bf16 v[228:231], v[108:111], a[124:127], v[228:231]\n"
    "s_mov_b32 s25, 1507522\n"
    "s_getpc_b64 s[28:29]\n"
    "s_cmp_ge_i32 s25, s24\n"
    "s_cbranch_scc1 90b\n"
    "v_mfma_f32_16x16x32_bf16 v[196:199], v[108:111], a[116:119], v[196:199]\n"
    "ds_read_b64_tr_b16 v[98:99], v60 offset:50688\n"
    "v_readlane_b32 s65, v53, s69\n"
    "v_mfma_f32_16x16x32_bf16 v[196:199], v[104:107], a[112:115], v[196:199]\n"
    "s_bitset1_b64 s[80:81], s69\n"
    "s_nop 0\n"
    "v_mfma_f32_16x16x32_bf16 v[164:167], v[104:107], a[104:107], v[164:167]\n"
    "ds_read_b64_tr_b16 v[100:101], v60 offset:57856\n"
    "s_nop 0\n"
    "v_mfma_f32_16x16x32_bf16 v[164:167], v[108:111], a[108:111], v[164:167]\n"
    "v_cmp_le_u32 vcc, s70, v47\n"
    "v_mfma_f32_16x16x32_bf16 v[132:135], v[108:111], a[100:103], v[132:135]\n"
    "ds_read_b64_tr_b16 v[102:103], v60 offset:58880\n"
    "s_lshl_b32 s64, s65, 5\n"
    "v_mfma_f32_16x16x32_bf16 v[132:135], v[104:107], a[96:99], v[132:135]\n"
    "s_or_b64 s[82:83], s[82:83], vcc\n"
    "v_cmp_le_u32 vcc, s70, v48\n"
    "s_waitcnt lgkmcnt(8)\n"
    "v_mfma_f32_16x16x32_bf16 v[136:139], v[112:115], a[96:99], v[136:139]\n"
    "ds_read_b64_tr_b16 v[104:105], v61 offset:49664\n"
    "s_add_u32 m0, s67, 114688\n"
    "v_mfma_f32_16x16x32_bf16 v[136:139], v[116:119], a[100:103], v[136:139]\n"
    "s_or_b64 s[82:83], s[82:83], vcc\n"
    "v_mfma_f32_16x16x32_bf16 v[168:171], v[116:119], a[108:111], v[168:171]\n"
    "ds_read_b64_tr_b16 v[106:107], v61 offset:50688\n"
    "s_nop 0\n"
    "v_mfma_f32_16x16x32_bf16 v[168:171], v[112:115], a[104:107], v[168:171]\n"
    "buffer_load_dwordx4 v56, s[36:39], s64 offen offset:0 lds\n"
    "v_mfma_f32_16x16x32_bf16 v[200:203], v[112:115], a[112:115], v[200:203]\n"
    "ds_read_b64_tr_b16 v[108:109], v61 offset:57856\n"
    "buffer_load_dwordx4 v57, s[36:39], s64 offen offset:1024 lds\n"
    "v_mfma_f32_16x16x32_bf16 v[200:203], v[116:119], a[116:119], v[200:203]\n"
    "v_mov_b32 v38, s82\n"
    "v_mov_b32 v39, s83\n"
    "ds_write_b64 v41, v[38:39] offset:8\n"
    "v_mfma_f32_16x16x32_bf16 v[232:235], v[116:119], a[124:127], v[232:235]\n"
    "ds_read_b64_tr_b16 v[110:111], v61 offset:58880\n"
    "buffer_load_dwordx4 v58, s[36:39], s64 offen offset:2048 lds\n"
    "v_mfma_f32_16x16x32_bf16 v[232:235], v[112:115], a[120:123], v[232:235]\n"
    "s_ff0_i32_b64 s30, s[82:83]\n"
    "s_bfm_b64 vcc, s30, 0\n"
    "s_andn2_b64 vcc, vcc, s[80:81]\n"
    "s_bcnt1_i32_b64 s30, vcc\n"
    "s_cmp_lt_u32 s30, 64\n"
    "s_cselect_b32 s22, 0x10000, 0\n"
    "s_waitcnt lgkmcnt(9)\n"
    "v_mfma_f32_16x16x32_bf16 v[236:239], v[120:123], a[120:123], v[236:239]\n"
    "ds_read_b64_tr_b16 v[112:113], v62 offset:49664\n"
    "s_andn2_b64 s[16:17], s[72:73], s[80:81]\n"
    "v_mfma_f32_16x16x32_bf16 v[236:239], v[124:127], a[124:127], v[236:239]\n"
    "s_bcnt1_i32_b64 s30, s[82:83]\n"
    "s_cmp_ge_u32 s30, 56\n"
    "s_cselect_b32 s22, 0, s22\n"
    "v_mfma_f32_16x16x32_bf16 v[204:207], v[124:127], a[116:119], v[204:207]\n"
    "ds_read_b64_tr_b16 v[114:115], v62 offset:50688\n"
    "s_mov_b64 s[14:15], s[16:17]\n"
    "v_mfma_f32_16x16x32_bf16 v[204:207], v[120:123], a[112:115], v[204:207]\n"
    "buffer_load_dword v48, v49, s[20:23], 0 offen sc0 sc1\n"
    "v_mfma_f32_16x16x32_bf16 v[172:175], v[120:123], a[104:107], v[172:175]\n"
    "ds_read_b64_tr_b16 v[116:117], v62 offset:57856\n"
    "s_ff1_i32_b64 s69, s[14:15]\n"
    "v_mfma_f32_16x16x32_bf16 v[172:175], v[124:127], a[108:111], v[172:175]\n"
    "buffer_load_dwordx4 v59, s[36:39], s64 offen offset:3072 lds\n"
    "v_mfma_f32_16x16x32_bf16 v[140:143], v[124:127], a[100:103], v[140:143]\n"
    "ds_read_b64_tr_b16 v[118:119], v62 offset:58880\n"
    "s_bitset0_b64 s[14:15], s69\n"
    "v_mfma_f32_16x16x32_bf16 v[140:143], v[120:123], a[96:99], v[140:143]\n"
    "s_ff1_i32_b64 s69, s[14:15]\n"
    "s_bitset0_b64 s[14:15], s69\n"
    "s_waitcnt lgkmcnt(9)\n"
    "v_mfma_f32_16x16x32_bf16 v[144:147], v[96:99], a[96:99], v[144:147]\n"
    "ds_read_b64_tr_b16 v[120:121], v63 offset:49664\n"
    "buffer_load_dwordx4 a[224:227], v54, s[40:43], s65 offen offset:0\n"
    "v_mfma_f32_16x16x32_bf16 v[144:147], v[100:103], a[100:103], v[144:147]\n"
    "s_ff1_i32_b64 s69, s[14:15]\n"
    "s_bitset0_b64 s[14:15], s69\n"
    "v_mfma_f32_16x16x32_bf16 v[176:179], v[100:103], a[108:111], v[176:179]\n"
    "ds_read_b64_tr_b16 v[122:123], v63 offset:50688\n"
    "s_ff1_i32_b64 s69, s[14:15]\n"
    "v_mfma_f32_16x16x32_bf16 v[176:179], v[96:99], a[104:107], v[176:179]\n"
    "buffer_load_dwordx4 a[228:231], v54, s[40:43], s65 offen offset:1024\n"
    "v_mfma_f32_16x16x32_bf16 v[208:211], v[96:99], a[112:115], v[208:211]\n"
    "ds_read_b64_tr_b16 v[124:125], v63 offset:57856\n"
    "s_bitset0_b64 s[14:15], s69\n"
    "v_mfma_f32_16x16x32_bf16 v[208:211], v[100:103], a[116:119], v[208:211]\n"
    "s_mov_b32 s25, 1573065\n"
    "s_getpc_b64 s[28:29]\n"
    "s_cmp_ge_i32 s25, s24\n"
    "s_cbranch_scc1 90b\n"
    "v_mfma_f32_16x16x32_bf16 v[240:243], v[100:103], a[124:127], v[240:243]\n"
    "ds_read_b64_tr_b16 v[126:127], v63 offset:58880\n"
    "s_xor_b64 s[16:17], s[16:17], s[14:15]\n"
    "v_mfma_f32_16x16x32_bf16 v[240:243], v[96:99], a[120:123], v[240:243]\n"
    "s_mov_b64 s[18:19], s[14:15]\n"
    "s_waitcnt lgkmcnt(8)\n"
    "s_waitcnt vmcnt(32)\n"
    "s_barrier\n"
    "v_mfma_f32_16x16x32_bf16 v[244:247], v[104:107], a[120:123], v[244:247]\n"
    "ds_read_b64_tr_b16 v[96:97], v64 offset:0\n"
    "buffer_load_dwordx4 a[232:235], v54, s[40:43], s65 offen offset:2048\n"
    "v_mfma_f32_16x16x32_bf16 v[244:247], v[108:111], a[124:127], v[244:247]\n"
    "ds_read_b64 v[36:37], v40 offset:8\n"
    "v_mfma_f32_16x16x32_bf16 v[212:215], v[108:111], a[116:119], v[212:215]\n"
    "ds_read_b64_tr_b16 v[98:99], v64 offset:1024\n"
    "s_ff1_i32_b64 s69, s[14:15]\n"
    "v_mfma_f32_16x16x32_bf16 v[212:215], v[104:107], a[112:115], v[212:215]\n"
    "s_bitset0_b64 s[14:15], s69\n"
    "s_ff1_i32_b64 s69, s[14:15]\n"
    "v_mfma_f32_16x16x32_bf16 v[180:183], v[104:107], a[104:107], v[180:183]\n"
    "ds_read_b64_tr_b16 v[100:101], v64 offset:8192\n"
    "buffer_load_dwordx4 a[236:239], v54, s[40:43], s65 offen offset:3072\n"
    "v_mfma_f32_16x16x32_bf16 v[180:183], v[108:111], a[108:111], v[180:183]\n"
    "s_bitset0_b64 s[14:15], s69\n"
    "s_ff1_i32_b64 s69, s[14:15]\n"
    "v_mfma_f32_16x16x32_bf16 v[148:151], v[108:111], a[100:103], v[148:151]\n"
    "ds_read_b64_tr_b16 v[102:103], v64 offset:9216\n"
    "s_bitset0_b64 s[14:15], s69\n"
    "v_mfma_f32_16x16x32_bf16 v[148:151], v[104:107], a[96:99], v[148:151]\n"
    "buffer_load_dwordx4 a[240:243], v55, s[40:43], s65 offen offset:0\n"
    "s_waitcnt lgkmcnt(9)\n"
    "v_mfma_f32_16x16x32_bf16 v[152:155], v[112:115], a[96:99], v[152:155]\n"
    "ds_read_b64_tr_b16 v[104:105], v65 offset:0\n"
    "s_ff1_i32_b64 s69, s[14:15]\n"
    "v_mfma_f32_16x16x32_bf16 v[152:155], v[116:119], a[100:103], v[152:155]\n"
    "s_bitset0_b64 s[14:15], s69\n"
    "s_xor_b64 s[18:19], s[18:19], s[14:15]\n"
    "v_mfma_f32_16x16x32_bf16 v[184:187], v[116:119], a[108:111], v[184:187]\n"
    "ds_read_b64_tr_b16 v[106:107], v65 offset:1024\n"
    "buffer_load_dwordx4 a[244:247], v55, s[40:43], s65 offen offset:1024\n"
    "v_mfma_f32_16x16x32_bf16 v[184:187], v[112:115], a[104:107], v[184:187]\n"
    "s_bcnt1_i32_b64 s69, s[18:19]\n"
    "s_cmp_eq_u32 s69, 4\n"
    "s_cselect_b64 s[16:17], s[16:17], 0\n"
    "s_cselect_b64 s[18:19], s[18:19], 0\n"
    "v_mfma_f32_16x16x32_bf16 v[216:219], v[112:115], a[112:115], v[216:219]\n"
    "ds_read_b64_tr_b16 v[108:109], v65 offset:8192\n"
    "s_cmp_eq_u32 s26, 4\n"
    "s_cselect_b64 s[14:15], s[18:19], s[16:17]\n"
    "s_cselect_b64 s[18:19], s[16:17], s[18:19]\n"
    "s_mov_b64 s[16:17], s[14:15]\n"
    "v_mfma_f32_16x16x32_bf16 v[216:219], v[116:119], a[116:119], v[216:219]\n"
    "buffer_load_dwordx4 a[248:251], v55, s[40:43], s65 offen offset:2048\n"
    "v_mfma_f32_16x16x32_bf16 v[248:251], v[116:119], a[124:127], v[248:251]\n"
    "ds_read_b64_tr_b16 v[110:111], v65 offset:9216\n"
    "buffer_load_dwordx4 a[252:255], v55, s[40:43], s65 offen offset:3072\n"
    "v_mfma_f32_16x16x32_bf16 v[248:251], v[112:115], a[120:123], v[248:251]\n"
    "s_andn2_b64 s[76:77], s[72:73], s[80:81]\n"
    "s_cbranch_scc0 1091f\n"
    "1092:\n"
    "s_andn2_b64 s[14:15], s[16:17], s[80:81]\n"
    "s_cselect_b64 s[76:77], s[14:15], s[76:77]\n"
    "s_ff1_i32_b64 s69, s[76:77]\n"
    "v_readlane_b32 s65, v53, s69\n"
    "s_waitcnt lgkmcnt(9)\n"
    "v_mfma_f32_16x16x32_bf16 v[252:255], v[120:123], a[120:123], v[252:255]\n"
    "ds_read_b64_tr_b16 v[112:113], v66 offset:0\n"
    "s_bitset1_b64 s[80:81], s69\n"
    "v_mfma_f32_16x16x32_bf16 v[252:255], v[124:127], a[124:127], v[252:255]\n"
    "s_nop 0\n"
    "s_nop 0\n"
    "v_mfma_f32_16x16x32_bf16 v[220:223], v[124:127], a[116:119], v[220:223]\n"
    "ds_read_b64_tr_b16 v[114:115], v66 offset:1024\n"
    "s_lshl_b32 s64, s65, 5\n"
    "v_mfma_f32_16x16x32_bf16 v[220:223], v[120:123], a[112:115], v[220:223]\n"
    "s_add_u32 m0, s67, 0\n"
    "s_nop 0\n"
    "v_mfma_f32_16x16x32_bf16 v[188:191], v[120:123], a[104:107], v[188:191]\n"
    "ds_read_b64_tr_b16 v[116:117], v66 offset:8192\n"
    "buffer_load_dwordx4 v56, s[36:39], s64 offen offset:0 lds\n"
    "v_mfma_f32_16x16x32_bf16 v[188:191], v[124:127], a[108:111], v[188:191]\n"
    "v_mfma_f32_16x16x32_bf16 v[156:159], v[124:127], a[100:103], v[156:159]\n"
    "ds_read_b64_tr_b16 v[118:119], v66 offset:9216\n"
    "v_mfma_f32_16x16x32_bf16 v[156:159], v[120:123], a[96:99], v[156:159]\n"
    "buffer_load_dwordx4 v57, s[36:39], s64 offen offset:1024 lds\n"
    "s_waitcnt lgkmcnt(8)\n"
    "v_mfma_f32_16x16x32_bf16 v[128:131], v[96:99], a[128:131], v[128:131]\n"
    "ds_read_b64_tr_b16 v[120:121], v67 offset:0\n"
    "v_mfma_f32_16x16x32_bf16 v[128:131], v[100:103], a[132:135], v[128:131]\n"
    "v_readfirstlane_b32 vcc_lo, v36\n"
    "v_readfirstlane_b32 vcc_hi, v37\n"
    "v_mfma_f32_16x16x32_bf16 v[160:163], v[100:103], a[140:143], v[160:163]\n"
    "ds_read_b64_tr_b16 v[122:123], v67 offset:1024\n"
    "v_mfma_f32_16x16x32_bf16 v[160:163], v[96:99], a[136:139], v[160:163]\n"
    "s_or_b64 s[72:73], s[72:73], vcc\n"
    "v_mfma_f32_16x16x32_bf16 v[192:195], v[96:99], a[144:147], v[192:195]\n"
    "ds_read_b64_tr_b16 v[124:125], v67 offset:8192\n"
    "v_mfma_f32_16x16x32_bf16 v[192:195], v[100:103], a[148:151], v[192:195]\n"
    "buffer_load_dwordx4 v58, s[36:39], s64 offen offset:2048 lds\n"
    "v_mfma_f32_16x16x32_bf16 v[224:227], v[100:103], a[156:159], v[224:227]\n"
    "ds_read_b64_tr_b16 v[126:127], v67 offset:9216\n"
    "v_mfma_f32_16x16x32_bf16 v[224:227], v[96:99], a[152:155], v[224:227]\n"
    "s_waitcnt lgkmcnt(8)\n"
    "v_mfma_f32_16x16x32_bf16 v[228:231], v[104:107], a[152:155], v[228:231]\n"
    "ds_read_b64_tr_b16 v[96:97], v64 offset:512\n"
    "v_mfma_f32_16x16x32_bf16 v[228:231], v[108:111], a[156:159], v[228:231]\n"
    "s_mov_b32 s25, 1638610\n"
    "s_getpc_b64 s[28:29]\n"
    "s_cmp_ge_i32 s25, s24\n"
    "s_cbranch_scc1 90b\n"
    "v_mfma_f32_16x16x32_bf16 v[196:199], v[108:111], a[148:151], v[196:199]\n"
    "ds_read_b64_tr_b16 v[98:99], v64 offset:1536\n"
    "v_mfma_f32_16x16x32_bf16 v[196:199], v[104:107], a[144:147], v[196:199]\n"
    "buffer_load_dwordx4 v59, s[36:39], s64 offen offset:3072 lds\n"
    "v_mfma_f32_16x16x32_bf16 v[164:167], v[104:107], a[136:139], v[164:167]\n"
    "ds_read_b64_tr_b16 v[100:101], v64 offset:8704\n"
    "v_mfma_f32_16x16x32_bf16 v[164:167], v[108:111], a[140:143], v[164:167]\n"
    "v_cmp_le_u32 vcc, s70, v47\n"
    "v_mfma_f32_16x16x32_bf16 v[132:135], v[108:111], a[132:135], v[132:135]\n"
    "ds_read_b64_tr_b16 v[102:103], v64 offset:9728\n"
    "v_mfma_f32_16x16x32_bf16 v[132:135], v[104:107], a[128:131], v[132:135]\n"
    "s_or_b64 s[82:83], s[82:83], vcc\n"
    "v_cmp_le_u32 vcc, s70, v48\n"
    "s_waitcnt lgkmcnt(8)\n"
    "v_mfma_f32_16x16x32_bf16 v[136:139], v[112:115], a[128:131], v[136:139]\n"
    "ds_read_b64_tr_b16 v[104:105], v65 offset:512\n"
    "v_mfma_f32_16x16x32_bf16 v[136:139], v[116:119], a[132:135], v[136:139]\n"
    "s_or_b64 s[82:83], s[82:83], vcc\n"
    "v_mfma_f32_16x16x32_bf16 v[168:171], v[116:119], a[140:143], v[168:171]\n"
    "ds_read_b64_tr_b16 v[106:107], v65 offset:1536\n"
    "buffer_load_dwordx4 a[0:3], v54, s[40:43], s65 offen offset:0\n"
    "v_mfma_f32_16x16x32_bf16 v[168:171], v[112:115], a[136:139], v[168:171]\n"
    "v_mfma_f32_16x16x32_bf16 v[200:203], v[112:115], a[144:147], v[200:203]\n"
    "ds_read_b64_tr_b16 v[108:109], v65 offset:8704\n"
    "v_mfma_f32_16x16x32_bf16 v[200:203], v[116:119], a[148:151], v[200:203]\n"
    "v_mov_b32 v38, s82\n"
    "v_mov_b32 v39, s83\n"
    "ds_write_b64 v41, v[38:39] offset:0\n"
    "v_mfma_f32_16x16x32_bf16 v[232:235], v[116:119], a[156:159], v[232:235]\n"
    "ds_read_b64_tr_b16 v[110:111], v65 offset:9728\n"
    "buffer_load_dwordx4 a[4:7], v54, s[40:43], s65 offen offset:1024\n"
    "v_mfma_f32_16x16x32_bf16 v[232:235], v[112:115], a[152:155], v[232:235]\n"
    "s_ff0_i32_b64 s30, s[82:83]\n"
    "s_bfm_b64 vcc, s30, 0\n"
    "s_andn2_b64 vcc, vcc, s[80:81]\n"
    "s_bcnt1_i32_b64 s30, vcc\n"
    "s_cmp_lt_u32 s30, 64\n"
    "s_cselect_b32 s22, 0x10000, 0\n"
    "s_waitcnt lgkmcnt(9)\n"
    "v_mfma_f32_16x16x32_bf16 v[236:239], v[120:123], a[152:155], v[236:239]\n"
    "ds_read_b64_tr_b16 v[112:113], v66 offset:512\n"
    "v_mfma_f32_16x16x32_bf16 v[236:239], v[124:127], a[156:159], v[236:239]\n"
    "s_bcnt1_i32_b64 s30, s[82:83]\n"
    "s_cmp_ge_u32 s30, 56\n"
    "s_cselect_b32 s22, 0, s22\n"
    "v_mfma_f32_16x16x32_bf16 v[204:207], v[124:127], a[148:151], v[204:207]\n"
    "ds_read_b64_tr_b16 v[114:115], v66 offset:1536\n"
    "v_mfma_f32_16x16x32_bf16 v[204:207], v[120:123], a[144:147], v[204:207]\n"
    "buffer_load_dword v47, v49, s[20:23], 0 offen sc0 sc1\n"
    "v_mfma_f32_16x16x32_bf16 v[172:175], v[120:123], a[136:139], v[172:175]\n"
    "ds_read_b64_tr_b16 v[116:117], v66 offset:8704\n"
    "buffer_load_dwordx4 a[8:11], v54, s[40:43], s65 offen offset:2048\n"
    "v_mfma_f32_16x16x32_bf16 v[172:175], v[124:127], a[140:143], v[172:175]\n"
    "v_mfma_f32_16x16x32_bf16 v[140:143], v[124:127], a[132:135], v[140:143]\n"
    "ds_read_b64_tr_b16 v[118:119], v66 offset:9728\n"
    "v_mfma_f32_16x16x32_bf16 v[140:143], v[120:123], a[128:131], v[140:143]\n"
    "s_waitcnt lgkmcnt(9)\n"
    "v_mfma_f32_16x16x32_bf16 v[144:147], v[96:99], a[128:131], v[144:147]\n"
    "ds_read_b64_tr_b16 v[120:121], v67 offset:512\n"
    "v_mfma_f32_16x16x32_bf16 v[144:147], v[100:103], a[132:135], v[144:147]\n"
    "buffer_load_dwordx4 a[12:15], v54, s[40:43], s65 offen offset:3072\n"
    "v_mfma_f32_16x16x32_bf16 v[176:179], v[100:103], a[140:143], v[176:179]\n"
    "ds_read_b64_tr_b16 v[122:123], v67 offset:1536\n"
    "v_mfma_f32_16x16x32_bf16 v[176:179], v[96:99], a[136:139], v[176:179]\n"
    "v_mfma_f32_16x16x32_bf16 v[208:211], v[96:99], a[144:147], v[208:211]\n"
    "ds_read_b64_tr_b16 v[124:125], v67 offset:8704\n"
    "v_mfma_f32_16x16x32_bf16 v[208:211], v[100:103], a[148:151], v[208:211]\n"
    "s_mov_b32 s25, 1704152\n"
    "s_getpc_b64 s[28:29]\n"
    "s_cmp_ge_i32 s25, s24\n"
    "s_cbranch_scc1 90b\n"
    "v_mfma_f32_16x16x32_bf16 v[240:243], v[100:103], a[156:159], v[240:243]\n"
    "ds_read_b64_tr_b16 v[126:127], v67 offset:9728\n"
    "v_mfma_f32_16x16x32_bf16 v[240:243], v[96:99], a[152:155], v[240:243]\n"
    "buffer_load_dwordx4 a[16:19], v55, s[40:43], s65 offen offset:0\n"
    "s_waitcnt lgkmcnt(8)\n"
    "s_waitcnt vmcnt(35)\n"
    "s_barrier\n"
    "v_mfma_f32_16x16x32_bf16 v[244:247], v[104:107], a[152:155], v[244:247]\n"
    "ds_read_b64_tr_b16 v[96:97], v64 offset:16384\n"
    "v_mfma_f32_16x16x32_bf16 v[244:247], v[108:111], a[156:159], v[244:247]\n"
    "ds_read_b64 v[36:37], v40 offset:0\n"
    "v_mfma_f32_16x16x32_bf16 v[212:215], v[108:111], a[148:151], v[212:215]\n"
    "ds_read_b64_tr_b16 v[98:99], v64 offset:17408\n"
    "v_mfma_f32_16x16x32_bf16 v[212:215], v[104:107], a[144:147], v[212:215]\n"
    "buffer_load_dwordx4 a[20:23], v55, s[40:43], s65 offen offset:1024\n"
    "v_mfma_f32_16x16x32_bf16 v[180:183], v[104:107], a[136:139], v[180:183]\n"
    "ds_read_b64_tr_b16 v[100:101], v64 offset:24576\n"
    "v_mfma_f32_16x16x32_bf16 v[180:183], v[108:111], a[140:143], v[180:183]\n"
    "v_mfma_f32_16x16x32_bf16 v[148:151], v[108:111], a[132:135], v[148:151]\n"
    "ds_read_b64_tr_b16 v[102:103], v64 offset:25600\n"
    "v_mfma_f32_16x16x32_bf16 v[148:151], v[104:107], a[128:131], v[148:151]\n"
    "s_waitcnt lgkmcnt(9)\n"
    "v_mfma_f32_16x16x32_bf16 v[152:155], v[112:115], a[128:131], v[152:155]\n"
    "ds_read_b64_tr_b16 v[104:105], v65 offset:16384\n"
    "v_mfma_f32_16x16x32_bf16 v[152:155], v[116:119], a[132:135], v[152:155]\n"
    "buffer_load_dwordx4 a[24:27], v55, s[40:43], s65 offen offset:2048\n"
    "v_mfma_f32_16x16x32_bf16 v[184:187], v[116:119], a[140:143], v[184:187]\n"
    "ds_read_b64_tr_b16 v[106:107], v65 offset:17408\n"
    "v_mfma_f32_16x16x32_bf16 v[184:187], v[112:115], a[136:139], v[184:187]\n"
    "v_mfma_f32_16x16x32_bf16 v[216:219], v[112:115], a[144:147], v[216:219]\n"
    "ds_read_b64_tr_b16 v[108:109], v65 offset:24576\n"
    "v_mfma_f32_16x16x32_bf16 v[216:219], v[116:119], a[148:151], v[216:219]\n"
    "v_mfma_f32_16x16x32_bf16 v[248:251], v[116:119], a[156:159], v[248:251]\n"
    "ds_read_b64_tr_b16 v[110:111], v65 offset:25600\n"
    "v_mfma_f32_16x16x32_bf16 v[248:251], v[112:115], a[152:155], v[248:251]\n"
    "buffer_load_dwordx4 a[28:31], v55, s[40:43], s65 offen offset:3072\n"
    "s_waitcnt lgkmcnt(9)\n"
    "v_mfma_f32_16x16x32_bf16 v[252:255], v[120:123], a[152:155], v[252:255]\n"
    "ds_read_b64_tr_b16 v[112:113], v66 offset:16384\n"
    "v_mfma_f32_16x16x32_bf16 v[252:255], v[124:127], a[156:159], v[252:255]\n"
    "v_mfma_f32_16x16x32_bf16 v[220:223], v[124:127], a[148:151], v[220:223]\n"
    "ds_read_b64_tr_b16 v[114:115], v66 offset:17408\n"
    "s_andn2_b64 s[76:77], s[72:73], s[80:81]\n"
    "s_cbranch_scc0 1101f\n"
    "1102:\n"
    "s_andn2_b64 s[14:15], s[16:17], s[80:81]\n"
    "s_cselect_b64 s[76:77], s[14:15], s[76:77]\n"
    "s_ff1_i32_b64 s69, s[76:77]\n"
    "v_mfma_f32_16x16x32_bf16 v[220:223], v[120:123], a[144:147], v[220:223]\n"
    "v_readlane_b32 s65, v53, s69\n"
    "s_bitset1_b64 s[80:81], s69\n"
    "v_mfma_f32_16x16x32_bf16 v[188:191], v[120:123], a[136:139], v[188:191]\n"
    "ds_read_b64_tr_b16 v[116:117], v66 offset:24576\n"
    "s_nop 0\n"
    "v_mfma_f32_16x16x32_bf16 v[188:191], v[124:127], a[140:143], v[188:191]\n"
    "s_nop 0\n"
    "s_lshl_b32 s64, s65, 5\n"
    "v_mfma_f32_16x16x32_bf16 v[156:159], v[124:127], a[132:135], v[156:159]\n"
    "ds_read_b64_tr_b16 v[118:119], v66 offset:25600\n"
    "s_add_u32 m0, s67, 16384\n"
    "v_mfma_f32_16x16x32_bf16 v[156:159], v[120:123], a[128:131], v[156:159]\n"
    "s_nop 0\n"
    "s_waitcnt lgkmcnt(8)\n"
    "v_mfma_f32_16x16x32_bf16 v[128:131], v[96:99], a[160:163], v[128:131]\n"
    "ds_read_b64_tr_b16 v[120:121], v67 offset:16384\n"
    "v_mfma_f32_16x16x32_bf16 v[128:131], v[100:103], a[164:167], v[128:131]\n"
    "v_readfirstlane_b32 vcc_lo, v36\n"
    "v_readfirstlane_b32 vcc_hi, v37\n"
    "v_mfma_f32_16x16x32_bf16 v[160:163], v[100:103], a[172:175], v[160:163]\n"
    "ds_read_b64_tr_b16 v[122:123], v67 offset:17408\n"
    "buffer_load_dwordx4 v56, s[36:39], s64 offen offset:0 lds\n"
    "v_mfma_f32_16x16x32_bf16 v[160:163], v[96:99], a[168:171], v[160:163]\n"
    "s_or_b64 s[72:73], s[72:73], vcc\n"
    "v_mfma_f32_16x16x32_bf16 v[192:195], v[96:99], a[176:179], v[192:195]\n"
    "ds_read_b64_tr_b16 v[124:125], v67 offset:24576\n"
    "v_mfma_f32_16x16x32_bf16 v[192:195], v[100:103], a[180:183], v[192:195]\n"
    "buffer_load_dwordx4 v57, s[36:39], s64 offen offset:1024 lds\n"
    "v_mfma_f32_16x16x32_bf16 v[224:227], v[100:103], a[188:191], v[224:227]\n"
    "ds_read_b64_tr_b16 v[126:127], v67 offset:25600\n"
    "v_mfma_f32_16x16x32_bf16 v[224:227], v[96:99], a[184:187], v[224:227]\n"
    "s_waitcnt lgkmcnt(8)\n"
    "v_mfma_f32_16x16x32_bf16 v[228:231], v[104:107], a[184:187], v[228:231]\n"
    "ds_read_b64_tr_b16 v[96:97], v64 offset:16896\n"
    "v_mfma_f32_16x16x32_bf16 v[228:231], v[108:111], a[188:191], v[228:231]\n"
    "s_mov_b32 s25, 1769694\n"
    "s_getpc_b64 s[28:29]\n"
    "s_cmp_ge_i32 s25, s24\n"
    "s_cbranch_scc1 90b\n"
    "v_mfma_f32_16x16x32_bf16 v[196:199], v[108:111], a[180:183], v[196:199]\n"
    "ds_read_b64_tr_b16 v[98:99], v64 offset:17920\n"
    "v_mfma_f32_16x16x32_bf16 v[196:199], v[104:107], a[176:179], v[196:199]\n"
    "buffer_load_dwordx4 v58, s[36:39], s64 offen offset:2048 lds\n"
    "v_mfma_f32_16x16x32_bf16 v[164:167], v[104:107], a[168:171], v[164:167]\n"
    "ds_read_b64_tr_b16 v[100:101], v64 offset:25088\n"
    "v_mfma_f32_16x16x32_bf16 v[164:167], v[108:111], a[172:175], v[164:167]\n"
    "v_cmp_le_u32 vcc, s70, v47\n"
    "v_mfma_f32_16x16x32_bf16 v[132:135], v[108:111], a[164:167], v[132:135]\n"
    "ds_read_b64_tr_b16 v[102:103], v64 offset:26112\n"
    "v_mfma_f32_16x16x32_bf16 v[132:135], v[104:107], a[160:163], v[132:135]\n"
    "s_or_b64 s[82:83], s[82:83], vcc\n"
    "v_cmp_le_u32 vcc, s70, v48\n"
    "s_waitcnt lgkmcnt(8)\n"
    "v_mfma_f32_16x16x32_bf16 v[136:139], v[112:115], a[160:163], v[136:139]\n"
    "ds_read_b64_tr_b16 v[104:105], v65 offset:16896\n"
    "v_mfma_f32_16x16x32_bf16 v[136:139], v[116:119], a[164:167], v[136:139]\n"
    "s_or_b64 s[82:83], s[82:83], vcc\n"
    "v_mfma_f32_16x16x32_bf16 v[168:171], v[116:119], a[172:175], v[168:171]\n"
    "ds_read_b64_tr_b16 v[106:107], v65 offset:17920\n"
    "buffer_load_dwordx4 v59, s[36:39], s64 offen offset:3072 lds\n"
    "v_mfma_f32_16x16x32_bf16 v[168:171], v[112:115], a[168:171], v[168:171]\n"
    "v_mfma_f32_16x16x32_bf16 v[200:203], v[112:115], a[176:179], v[200:203]\n"
    "ds_read_b64_tr_b16 v[108:109], v65 offset:25088\n"
    "v_mfma_f32_16x16x32_bf16 v[200:203], v[116:119], a[180:183], v[200:203]\n"
    "v_mov_b32 v38, s82\n"
    "v_mov_b32 v39, s83\n"
    "ds_write_b64 v41, v[38:39] offset:8\n"
    "v_mfma_f32_16x16x32_bf16 v[232:235], v[116:119], a[188:191], v[232:235]\n"
    "ds_read_b64_tr_b16 v[110:111], v65 offset:26112\n"
    "v_mfma_f32_16x16x32_bf16 v[232:235], v[112:115], a[184:187], v[232:235]\n"
    "s_ff0_i32_b64 s30, s[82:83]\n"
    "s_bfm_b64 vcc, s30, 0\n"
    "s_andn2_b64 vcc, vcc, s[80:81]\n"
    "s_bcnt1_i32_b64 s30, vcc\n"
    "s_cmp_lt_u32 s30, 64\n"
    "s_cselect_b32 s22, 0x10000, 0\n"
    "s_waitcnt lgkmcnt(9)\n"
    "v_mfma_f32_16x16x32_bf16 v[236:239], v[120:123], a[184:187], v[236:239]\n"
    "ds_read_b64_tr_b16 v[112:113], v66 offset:16896\n"
    "buffer_load_dwordx4 a[32:35], v54, s[40:43], s65 offen offset:0\n"
    "v_mfma_f32_16x16x32_bf16 v[236:239], v[124:127], a[188:191], v[236:239]\n"
    "s_bcnt1_i32_b64 s30, s[82:83]\n"
    "s_cmp_ge_u32 s30, 56\n"
    "s_cselect_b32 s22, 0, s22\n"
    "v_mfma_f32_16x16x32_bf16 v[204:207], v[124:127], a[180:183], v[204:207]\n"
    "ds_read_b64_tr_b16 v[114:115], v66 offset:17920\n"
    "v_mfma_f32_16x16x32_bf16 v[204:207], v[120:123], a[176:179], v[204:207]\n"
    "buffer_load_dword v48, v49, s[20:23], 0 offen sc0 sc1\n"
    "v_mfma_f32_16x16x32_bf16 v[172:175], v[120:123], a[168:171], v[172:175]\n"
    "ds_read_b64_tr_b16 v[116:117], v66 offset:25088\n"
    "buffer_load_dwordx4 a[36:39], v54, s[40:43], s65 offen offset:1024\n"
    "v_mfma_f32_16x16x32_bf16 v[172:175], v[124:127], a[172:175], v[172:175]\n"
    "v_mfma_f32_16x16x32_bf16 v[140:143], v[124:127], a[164:167], v[140:143]\n"
    "ds_read_b64_tr_b16 v[118:119], v66 offset:26112\n"
    "v_mfma_f32_16x16x32_bf16 v[140:143], v[120:123], a[160:163], v[140:143]\n"
    "s_waitcnt lgkmcnt(9)\n"
    "v_mfma_f32_16x16x32_bf16 v[144:147], v[96:99], a[160:163], v[144:147]\n"
    "ds_read_b64_tr_b16 v[120:121], v67 offset:16896\n"
    "v_mfma_f32_16x16x32_bf16 v[144:147], v[100:103], a[164:167], v[144:147]\n"
    "buffer_load_dwordx4 a[40:43], v54, s[40:43], s65 offen offset:2048\n"
    "v_mfma_f32_16x16x32_bf16 v[176:179], v[100:103], a[172:175], v[176:179]\n"
    "ds_read_b64_tr_b16 v[122:123], v67 offset:17920\n"
    "v_mfma_f32_16x16x32_bf16 v[176:179], v[96:99], a[168:171], v[176:179]\n"
    "v_mfma_f32_16x16x32_bf16 v[208:211], v[96:99], a[176:179], v[208:211]\n"
    "ds_read_b64_tr_b16 v[124:125], v67 offset:25088\n"
    "v_mfma_f32_16x16x32_bf16 v[208:211], v[100:103], a[180:183], v[208:211]\n"
    "s_mov_b32 s25, 1835236\n"
    "s_getpc_b64 s[28:29]\n"
    "s_cmp_ge_i32 s25, s24\n"
    "s_cbranch_scc1 90b\n"
    "v_mfma_f32_16x16x32_bf16 v[240:243], v[100:103], a[188:191], v[240:243]\n"
    "ds_read_b64_tr_b16 v[126:127], v67 offset:26112\n"
    "v_mfma_f32_16x16x32_bf16 v[240:243], v[96:99], a[184:187], v[240:243]\n"
    "buffer_load_dwordx4 a[44:47], v54, s[40:43], s65 offen offset:3072\n"
    "s_waitcnt lgkmcnt(8)\n"
    "s_waitcnt vmcnt(35)\n"
    "s_barrier\n"
    "v_mfma_f32_16x16x32_bf16 v[244:247], v[104:107], a[184:187], v[244:247]\n"
    "ds_read_b64_tr_b16 v[96:97], v64 offset:32768\n"
    "v_mfma_f32_16x16x32_bf16 v[244:247], v[108:111], a[188:191], v[244:247]\n"
    "ds_read_b64 v[36:37], v40 offset:8\n"
    "v_mfma_f32_16x16x32_bf16 v[212:215], v[108:111], a[180:183], v[212:215]\n"
    "ds_read_b64_tr_b16 v[98:99], v64 offset:33792\n"
    "v_mfma_f32_16x16x32_bf16 v[212:215], v[104:107], a[176:179], v[212:215]\n"
    "buffer_load_dwordx4 a[48:51], v55, s[40:43], s65 offen offset:0\n"
    "v_mfma_f32_16x16x32_bf16 v[180:183], v[104:107], a[168:171], v[180:183]\n"
    "ds_read_b64_tr_b16 v[100:101], v64 offset:40960\n"
    "v_mfma_f32_16x16x32_bf16 v[180:183], v[108:111], a[172:175], v[180:183]\n"
    "v_mfma_f32_16x16x32_bf16 v[148:151], v[108:111], a[164:167], v[148:151]\n"
    "ds_read_b64_tr_b16 v[102:103], v64 offset:41984\n"
    "v_mfma_f32_16x16x32_bf16 v[148:151], v[104:107], a[160:163], v[148:151]\n"
    "s_waitcnt lgkmcnt(9)\n"
    "v_mfma_f32_16x16x32_bf16 v[152:155], v[112:115], a[160:163], v[152:155]\n"
    "ds_read_b64_tr_b16 v[104:105], v65 offset:32768\n"
    "v_mfma_f32_16x16x32_bf16 v[152:155], v[116:119], a[164:167], v[152:155]\n"
    "buffer_load_dwordx4 a[52:55], v55, s[40:43], s65 offen offset:1024\n"
    "v_mfma_f32_16x16x32_bf16 v[184:187], v[116:119], a[172:175], v[184:187]\n"
    "ds_read_b64_tr_b16 v[106:107], v65 offset:33792\n"
    "v_mfma_f32_16x16x32_bf16 v[184:187], v[112:115], a[168:171], v[184:187]\n"
    "v_mfma_f32_16x16x32_bf16 v[216:219], v[112:115], a[176:179], v[216:219]\n"
    "ds_read_b64_tr_b16 v[108:109], v65 offset:40960\n"
    "v_mfma_f32_16x16x32_bf16 v[216:219], v[116:119], a[180:183], v[216:219]\n"
    "v_mfma_f32_16x16x32_bf16 v[248:251], v[116:119], a[188:191], v[248:251]\n"
    "ds_read_b64_tr_b16 v[110:111], v65 offset:41984\n"
    "v_mfma_f32_16x16x32_bf16 v[248:251], v[112:115], a[184:187], v[248:251]\n"
    "buffer_load_dwordx4 a[56:59], v55, s[40:43], s65 offen offset:2048\n"
    "s_waitcnt lgkmcnt(9)\n"
    "v_mfma_f32_16x16x32_bf16 v[252:255], v[120:123], a[184:187], v[252:255]\n"
    "ds_read_b64_tr_b16 v[112:113], v66 offset:32768\n"
    "v_mfma_f32_16x16x32_bf16 v[252:255], v[124:127], a[188:191], v[252:255]\n"
    "v_mfma_f32_16x16x32_bf16 v[220:223], v[124:127], a[180:183], v[220:223]\n"
    "ds_read_b64_tr_b16 v[114:115], v66 offset:33792\n"
    "v_mfma_f32_16x16x32_bf16 v[220:223], v[120:123], a[176:179], v[220:223]\n"
    "v_mfma_f32_16x16x32_bf16 v[188:191], v[120:123], a[168:171], v[188:191]\n"
    "ds_read_b64_tr_b16 v[116:117], v66 offset:40960\n"
    "v_mfma_f32_16x16x32_bf16 v[188:191], v[124:127], a[172:175], v[188:191]\n"
    "buffer_load_dwordx4 a[60:63], v55, s[40:43], s65 offen offset:3072\n"
    "v_mfma_f32_16x16x32_bf16 v[156:159], v[124:127], a[164:167], v[156:159]\n"
    "ds_read_b64_tr_b16 v[118:119], v66 offset:41984\n"
    "v_mfma_f32_16x16x32_bf16 v[156:159], v[120:123], a[160:163], v[156:159]\n"
    "s_waitcnt lgkmcnt(8)\n"
    "v_mfma_f32_16x16x32_bf16 v[128:131], v[96:99], a[192:195], v[128:131]\n"
    "ds_read_b64_tr_b16 v[120:121], v67 offset:32768\n"
    "s_andn2_b64 s[76:77], s[72:73], s[80:81]\n"
    "s_cbranch_scc0 1111f\n"
    "1112:\n"
    "s_andn2_b64 s[14:15], s[16:17], s[80:81]\n"
    "s_cselect_b64 s[76:77], s[14:15], s[76:77]\n"
    "s_ff1_i32_b64 s69, s[76:77]\n"
    "v_mfma_f32_16x16x32_bf16 v[128:131], v[100:103], a[196:199], v[128:131]\n"
    "v_readfirstlane_b32 vcc_lo, v36\n"
    "v_readfirstlane_b32 vcc_hi, v37\n"
    "v_mfma_f32_16x16x32_bf16 v[160:163], v[100:103], a[204:207], v[160:163]\n"
    "ds_read_b64_tr_b16 v[122:123], v67 offset:33792\n"
    "v_readlane_b32 s65, v53, s69\n"
    "v_mfma_f32_16x16x32_bf16 v[160:163], v[96:99], a[200:203], v[160:163]\n"
    "s_or_b64 s[72:73], s[72:73], vcc\n"
    "v_mfma_f32_16x16x32_bf16 v[192:195], v[96:99], a[208:211], v[192:195]\n"
    "ds_read_b64_tr_b16 v[124:125], v67 offset:40960\n"
    "s_bitset1_b64 s[80:81], s69\n"
    "v_mfma_f32_16x16x32_bf16 v[192:195], v[100:103], a[212:215], v[192:195]\n"
    "s_nop 0\n"
    "s_nop 0\n"
    "v_mfma_f32_16x16x32_bf16 v[224:227], v[100:103], a[220:223], v[224:227]\n"
    "ds_read_b64_tr_b16 v[126:127], v67 offset:41984\n"
    "s_lshl_b32 s64, s65, 5\n"
    "v_mfma_f32_16x16x32_bf16 v[224:227], v[96:99], a[216:219], v[224:227]\n"
    "s_add_u32 m0, s67, 32768\n"
    "s_nop 0\n"
    "s_waitcnt lgkmcnt(8)\n"
    "v_mfma_f32_16x16x32_bf16 v[228:231], v[104:107], a[216:219], v[228:231]\n"
    "ds_read_b64_tr_b16 v[96:97], v64 offset:33280\n"
    "buffer_load_dwordx4 v56, s[36:39], s64 offen offset:0 lds\n"
    "v_mfma_f32_16x16x32_bf16 v[228:231], v[108:111], a[220:223], v[228:231]\n"
    "s_mov_b32 s25, 1900778\n"
    "s_getpc_b64 s[28:29]\n"
    "s_cmp_ge_i32 s25, s24\n"
    "s_cbranch_scc1 90b\n"
    "v_mfma_f32_16x16x32_bf16 v[196:199], v[108:111], a[212:215], v[196:199]\n"
    "ds_read_b64_tr_b16 v[98:99], v64 offset:34304\n"
    "v_mfma_f32_16x16x32_bf16 v[196:199], v[104:107], a[208:211], v[196:199]\n"
    "buffer_load_dwordx4 v57, s[36:39], s64 offen offset:1024 lds\n"
    "v_mfma_f32_16x16x32_bf16 v[164:167], v[104:107], a[200:203], v[164:167]\n"
    "ds_read_b64_tr_b16 v[100:101], v64 offset:41472\n"
    "v_mfma_f32_16x16x32_bf16 v[164:167], v[108:111], a[204:207], v[164:167]\n"
    "v_cmp_le_u32 vcc, s70, v47\n"
    "v_mfma_f32_16x16x32_bf16 v[132:135], v[108:111], a[196:199], v[132:135]\n"
    "ds_read_b64_tr_b16 v[102:103], v64 offset:42496\n"
    "v_mfma_f32_16x16x32_bf16 v[132:135], v[104:107], a[192:195], v[132:135]\n"
    "s_or_b64 s[82:83], s[82:83], vcc\n"
    "v_cmp_le_u32 vcc, s70, v48\n"
    "s_waitcnt lgkmcnt(8)\n"
    "v_mfma_f32_16x16x32_bf16 v[136:139], v[112:115], a[192:195], v[136:139]\n"
    "ds_read_b64_tr_b16 v[104:105], v65 offset:33280\n"
    "v_mfma_f32_16x16x32_bf16 v[136:139], v[116:119], a[196:199], v[136:139]\n"
    "s_or_b64 s[82:83], s[82:83], vcc\n"
    "v_mfma_f32_16x16x32_bf16 v[168:171], v[116:119], a[204:207], v[168:171]\n"
    "ds_read_b64_tr_b16 v[106:107], v65 offset:34304\n"
    "buffer_load_dwordx4 v58, s[36:39], s64 offen offset:2048 lds\n"
    "v_mfma_f32_16x16x32_bf16 v[168:171], v[112:115], a[200:203], v[168:171]\n"
    "v_mfma_f32_16x16x32_bf16 v[200:203], v[112:115], a[208:211], v[200:203]\n"
    "ds_read_b64_tr_b16 v[108:109], v65 offset:41472\n"
    "v_mfma_f32_16x16x32_bf16 v[200:203], v[116:119], a[212:215], v[200:203]\n"
    "v_mov_b32 v38, s82\n"
    "v_mov_b32 v39, s83\n"
    "ds_write_b64 v41, v[38:39] offset:0\n"
    "v_mfma_f32_16x16x32_bf16 v[232:235], v[116:119], a[220:223], v[232:235]\n"
    "ds_read_b64_tr_b16 v[110:111], v65 offset:42496\n"
    "v_mfma_f32_16x16x32_bf16 v[232:235], v[112:115], a[216:219], v[232:235]\n"
    "s_ff0_i32_b64 s30, s[82:83]\n"
    "s_bfm_b64 vcc, s30, 0\n"
    "s_andn2_b64 vcc, vcc, s[80:81]\n"
    "s_bcnt1_i32_b64 s30, vcc\n"
    "s_cmp_lt_u32 s30, 64\n"
    "s_cselect_b32 s22, 0x10000, 0\n"
    "s_waitcnt lgkmcnt(9)\n"
    "v_mfma_f32_16x16x32_bf16 v[236:239], v[120:123], a[216:219], v[236:239]\n"
    "ds_read_b64_tr_b16 v[112:113], v66 offset:33280\n"
    "buffer_load_dwordx4 v59, s[36:39], s64 offen offset:3072 lds\n"
    "v_mfma_f32_16x16x32_bf16 v[236:239], v[124:127], a[220:223], v[236:239]\n"
    "s_bcnt1_i32_b64 s30, s[82:83]\n"
    "s_cmp_ge_u32 s30, 56\n"
    "s_cselect_b32 s22, 0, s22\n"
    "v_mfma_f32_16x16x32_bf16 v[204:207], v[124:127], a[212:215], v[204:207]\n"
    "ds_read_b64_tr_b16 v[114:115], v66 offset:34304\n"
    "v_mfma_f32_16x16x32_bf16 v[204:207], v[120:123], a[208:211], v[204:207]\n"
    "buffer_load_dword v48, v49, s[20:23], 0 offen sc0 sc1\n"
    "v_mfma_f32_16x16x32_bf16 v[172:175], v[120:123], a[200:203], v[172:175]\n"
    "ds_read_b64_tr_b16 v[116:117], v66 offset:41472\n"
    "v_mfma_f32_16x16x32_bf16 v[172:175], v[124:127], a[204:207], v[172:175]\n"
    "buffer_load_dwordx4 a[64:67], v54, s[40:43], s65 offen offset:0\n"
    "v_mfma_f32_16x16x32_bf16 v[140:143], v[124:127], a[196:199], v[140:143]\n"
    "ds_read_b64_tr_b16 v[118:119], v66 offset:42496\n"
    "v_mfma_f32_16x16x32_bf16 v[140:143], v[120:123], a[192:195], v[140:143]\n"
    "s_waitcnt lgkmcnt(9)\n"
    "v_mfma_f32_16x16x32_bf16 v[144:147], v[96:99], a[192:195], v[144:147]\n"
    "ds_read_b64_tr_b16 v[120:121], v67 offset:33280\n"
    "v_mfma_f32_16x16x32_bf16 v[144:147], v[100:103], a[196:199], v[144:147]\n"
    "buffer_load_dwordx4 a[68:71], v54, s[40:43], s65 offen offset:1024\n"
    "v_mfma_f32_16x16x32_bf16 v[176:179], v[100:103], a[204:207], v[176:179]\n"
    "ds_read_b64_tr_b16 v[122:123], v67 offset:34304\n"
    "v_mfma_f32_16x16x32_bf16 v[176:179], v[96:99], a[200:203], v[176:179]\n"
    "v_mfma_f32_16x16x32_bf16 v[208:211], v[96:99], a[208:211], v[208:211]\n"
    "ds_read_b64_tr_b16 v[124:125], v67 offset:41472\n"
    "v_mfma_f32_16x16x32_bf16 v[208:211], v[100:103], a[212:215], v[208:211]\n"
    "s_mov_b32 s25, 1966320\n"
    "s_getpc_b64 s[28:29]\n"
    "s_cmp_ge_i32 s25, s24\n"
    "s_cbranch_scc1 90b\n"
    "v_mfma_f32_16x16x32_bf16 v[240:243], v[100:103], a[220:223], v[240:243]\n"
    "ds_read_b64_tr_b16 v[126:127], v67 offset:42496\n"
    "v_mfma_f32_16x16x32_bf16 v[240:243], v[96:99], a[216:219], v[240:243]\n"
    "buffer_load_dwordx4 a[72:75], v54, s[40:43], s65 offen offset:2048\n"
    "s_waitcnt lgkmcnt(8)\n"
    "s_waitcnt vmcnt(34)\n"
    "s_barrier\n"
    "v_mfma_f32_16x16x32_bf16 v[244:247], v[104:107], a[216:219], v[244:247]\n"
    "ds_read_b64_tr_b16 v[96:97], v64 offset:49152\n"
    "v_mfma_f32_16x16x32_bf16 v[244:247], v[108:111], a[220:223], v[244:247]\n"
    "ds_read_b64 v[36:37], v40 offset:0\n"
    "v_mfma_f32_16x16x32_bf16 v[212:215], v[108:111], a[212:215], v[212:215]\n"
    "ds_read_b64_tr_b16 v[98:99], v64 offset:50176\n"
    "v_mfma_f32_16x16x32_bf16 v[212:215], v[104:107], a[208:211], v[212:215]\n"
    "v_mfma_f32_16x16x32_bf16 v[180:183], v[104:107], a[200:203], v[180:183]\n"
    "ds_read_b64_tr_b16 v[100:101], v64 offset:57344\n"
    "v_mfma_f32_16x16x32_bf16 v[180:183], v[108:111], a[204:207], v[180:183]\n"
    "buffer_load_dwordx4 a[76:79], v54, s[40:43], s65 offen offset:3072\n"
    "v_mfma_f32_16x16x32_bf16 v[148:151], v[108:111], a[196:199], v[148:151]\n"
    "ds_read_b64_tr_b16 v[102:103], v64 offset:58368\n"
    "v_mfma_f32_16x16x32_bf16 v[148:151], v[104:107], a[192:195], v[148:151]\n"
    "s_waitcnt lgkmcnt(9)\n"
    "v_mfma_f32_16x16x32_bf16 v[152:155], v[112:115], a[192:195], v[152:155]\n"
    "ds_read_b64_tr_b16 v[104:105], v65 offset:49152\n"
    "v_mfma_f32_16x16x32_bf16 v[152:155], v[116:119], a[196:199], v[152:155]\n"
    "buffer_load_dwordx4 a[80:83], v55, s[40:43], s65 offen offset:0\n"
    "v_mfma_f32_16x16x32_bf16 v[184:187], v[116:119], a[204:207], v[184:187]\n"
    "ds_read_b64_tr_b16 v[106:107], v65 offset:50176\n"
    "v_mfma_f32_16x16x32_bf16 v[184:187], v[112:115], a[200:203], v[184:187]\n"
    "v_mfma_f32_16x16x32_bf16 v[216:219], v[112:115], a[208:211], v[216:219]\n"
    "ds_read_b64_tr_b16 v[108:109], v65 offset:57344\n"
    "v_mfma_f32_16x16x32_bf16 v[216:219], v[116:119], a[212:215], v[216:219]\n"
    "v_mfma_f32_16x16x32_bf16 v[248:251], v[116:119], a[220:223], v[248:251]\n"
    "ds_read_b64_tr_b16 v[110:111], v65 offset:58368\n"
    "v_mfma_f32_16x16x32_bf16 v[248:251], v[112:115], a[216:219], v[248:251]\n"
    "buffer_load_dwordx4 a[84:87], v55, s[40:43], s65 offen offset:1024\n"
    "s_waitcnt lgkmcnt(9)\n"
    "v_mfma_f32_16x16x32_bf16 v[252:255], v[120:123], a[216:219], v[252:255]\n"
    "ds_read_b64_tr_b16 v[112:113], v66 offset:49152\n"
    "v_mfma_f32_16x16x32_bf16 v[252:255], v[124:127], a[220:223], v[252:255]\n"
    "v_mfma_f32_16x16x32_bf16 v[220:223], v[124:127], a[212:215], v[220:223]\n"
    "ds_read_b64_tr_b16 v[114:115], v66 offset:50176\n"
    "v_mfma_f32_16x16x32_bf16 v[220:223], v[120:123], a[208:211], v[220:223]\n"
    "v_mfma_f32_16x16x32_bf16 v[188:191], v[120:123], a[200:203], v[188:191]\n"
    "ds_read_b64_tr_b16 v[116:117], v66 offset:57344\n"
    "v_mfma_f32_16x16x32_bf16 v[188:191], v[124:127], a[204:207], v[188:191]\n"
    "buffer_load_dwordx4 a[88:91], v55, s[40:43], s65 offen offset:2048\n"
    "v_mfma_f32_16x16x32_bf16 v[156:159], v[124:127], a[196:199], v[156:159]\n"
    "ds_read_b64_tr_b16 v[118:119], v66 offset:58368\n"
    "v_mfma_f32_16x16x32_bf16 v[156:159], v[120:123], a[192:195], v[156:159]\n"
    "s_waitcnt lgkmcnt(8)\n"
    "v_mfma_f32_16x16x32_bf16 v[128:131], v[96:99], a[224:227], v[128:131]\n"
    "ds_read_b64_tr_b16 v[120:121], v67 offset:49152\n"
    "v_mfma_f32_16x16x32_bf16 v[128:131], v[100:103], a[228:231], v[128:131]\n"
    "v_readfirstlane_b32 vcc_lo, v36\n"
    "v_readfirstlane_b32 vcc_hi, v37\n"
    "v_mfma_f32_16x16x32_bf16 v[160:163], v[100:103], a[236:239], v[160:163]\n"
    "ds_read_b64_tr_b16 v[122:123], v67 offset:50176\n"
    "v_mfma_f32_16x16x32_bf16 v[160:163], v[96:99], a[232:235], v[160:163]\n"
    "s_or_b64 s[72:73], s[72:73], vcc\n"
    "v_mfma_f32_16x16x32_bf16 v[192:195], v[96:99], a[240:243], v[192:195]\n"
    "ds_read_b64_tr_b16 v[124:125], v67 offset:57344\n"
    "buffer_load_dwordx4 a[92:95], v55, s[40:43], s65 offen offset:3072\n"
    "v_mfma_f32_16x16x32_bf16 v[192:195], v[100:103], a[244:247], v[192:195]\n"
    "v_mfma_f32_16x16x32_bf16 v[224:227], v[100:103], a[252:255], v[224:227]\n"
    "ds_read_b64_tr_b16 v[126:127], v67 offset:58368\n"
    "s_andn2_b64 s[76:77], s[72:73], s[80:81]\n"
    "s_cbranch_scc0 1121f\n"
    "1122:\n"
    "s_andn2_b64 s[14:15], s[16:17], s[80:81]\n"
    "s_cselect_b64 s[76:77], s[14:15], s[76:77]\n"
    "s_ff1_i32_b64 s69, s[76:77]\n"
    "v_mfma_f32_16x16x32_bf16 v[224:227], v[96:99], a[248:251], v[224:227]\n"
    "v_readlane_b32 s65, v53, s69\n"
    "s_bitset1_b64 s[80:81], s69\n"
    "s_waitcnt lgkmcnt(8)\n"
    "v_mfma_f32_16x16x32_bf16 v[228:231], v[104:107], a[248:251], v[228:231]\n"
    "ds_read_b64_tr_b16 v[96:97], v64 offset:49664\n"
    "s_nop 0\n"
    "v_mfma_f32_16x16x32_bf16 v[228:231], v[108:111], a[252:255], v[228:231]\n"
    "s_mov_b32 s25, 2031862\n"
    "s_getpc_b64 s[28:29]\n"
    "s_cmp_ge_i32 s25, s24\n"
    "s_cbranch_scc1 90b\n"
    "v_mfma_f32_16x16x32_bf16 v[196:199], v[108:111], a[244:247], v[196:199]\n"
    "ds_read_b64_tr_b16 v[98:99], v64 offset:50688\n"
    "s_nop 0\n"
    "v_mfma_f32_16x16x32_bf16 v[196:199], v[104:107], a[240:243], v[196:199]\n"
    "s_lshl_b32 s64, s65, 5\n"
    "s_add_u32 m0, s67, 49152\n"
    "v_mfma_f32_16x16x32_bf16 v[164:167], v[104:107], a[232:235], v[164:167]\n"
    "ds_read_b64_tr_b16 v[100:101], v64 offset:57856\n"
    "s_nop 0\n"
    "v_mfma_f32_16x16x32_bf16 v[164:167], v[108:111], a[236:239], v[164:167]\n"
    "buffer_load_dwordx4 v56, s[36:39], s64 offen offset:0 lds\n"
    "v_mfma_f32_16x16x32_bf16 v[132:135], v[108:111], a[228:231], v[132:135]\n"
    "ds_read_b64_tr_b16 v[102:103], v64 offset:58880\n"
    "v_mfma_f32_16x16x32_bf16 v[132:135], v[104:107], a[224:227], v[132:135]\n"
    "s_waitcnt lgkmcnt(8)\n"
    "v_mfma_f32_16x16x32_bf16 v[136:139], v[112:115], a[224:227], v[136:139]\n"
    "ds_read_b64_tr_b16 v[104:105], v65 offset:49664\n"
    "v_mfma_f32_16x16x32_bf16 v[136:139], v[116:119], a[228:231], v[136:139]\n"
    "buffer_load_dwordx4 v57, s[36:39], s64 offen offset:1024 lds\n"
    "v_mfma_f32_16x16x32_bf16 v[168:171], v[116:119], a[236:239], v[168:171]\n"
    "ds_read_b64_tr_b16 v[106:107], v65 offset:50688\n"
    "v_mfma_f32_16x16x32_bf16 v[168:171], v[112:115], a[232:235], v[168:171]\n"
    "v_mfma_f32_16x16x32_bf16 v[200:203], v[112:115], a[240:243], v[200:203]\n"
    "ds_read_b64_tr_b16 v[108:109], v65 offset:57856\n"
    "v_mfma_f32_16x16x32_bf16 v[200:203], v[116:119], a[244:247], v[200:203]\n"
    "v_mfma_f32_16x16x32_bf16 v[232:235], v[116:119], a[252:255], v[232:235]\n"
    "ds_read_b64_tr_b16 v[110:111], v65 offset:58880\n"
    "v_mfma_f32_16x16x32_bf16 v[232:235], v[112:115], a[248:251], v[232:235]\n"
    "buffer_load_dwordx4 v58, s[36:39], s64 offen offset:2048 lds\n"
    "s_waitcnt lgkmcnt(8)\n"
    "v_mfma_f32_16x16x32_bf16 v[236:239], v[120:123], a[248:251], v[236:239]\n"
    "ds_read_b64_tr_b16 v[112:113], v66 offset:49664\n"
    "v_mfma_f32_16x16x32_bf16 v[236:239], v[124:127], a[252:255], v[236:239]\n"
    "v_mfma_f32_16x16x32_bf16 v[204:207], v[124:127], a[244:247], v[204:207]\n"
    "ds_read_b64_tr_b16 v[114:115], v66 offset:50688\n"
    "v_mfma_f32_16x16x32_bf16 v[204:207], v[120:123], a[240:243], v[204:207]\n"
    "buffer_load_dwordx4 v59, s[36:39], s64 offen offset:3072 lds\n"
    "v_mfma_f32_16x16x32_bf16 v[172:175], v[120:123], a[232:235], v[172:175]\n"
    "ds_read_b64_tr_b16 v[116:117], v66 offset:57856\n"
    "v_mfma_f32_16x16x32_bf16 v[172:175], v[124:127], a[236:239], v[172:175]\n"
    "v_mfma_f32_16x16x32_bf16 v[140:143], v[124:127], a[228:231], v[140:143]\n"
    "ds_read_b64_tr_b16 v[118:119], v66 offset:58880\n"
    "v_mfma_f32_16x16x32_bf16 v[140:143], v[120:123], a[224:227], v[140:143]\n"
    "s_waitcnt lgkmcnt(8)\n"
    "v_mfma_f32_16x16x32_bf16 v[144:147], v[96:99], a[224:227], v[144:147]\n"
    "ds_read_b64_tr_b16 v[120:121], v67 offset:49664\n"
    "v_mfma_f32_16x16x32_bf16 v[144:147], v[100:103], a[228:231], v[144:147]\n"
    "buffer_load_dwordx4 a[96:99], v54, s[40:43], s65 offen offset:0\n"
    "v_mfma_f32_16x16x32_bf16 v[176:179], v[100:103], a[236:239], v[176:179]\n"
    "ds_read_b64_tr_b16 v[122:123], v67 offset:50688\n"
    "v_mfma_f32_16x16x32_bf16 v[176:179], v[96:99], a[232:235], v[176:179]\n"
    "v_mfma_f32_16x16x32_bf16 v[208:211], v[96:99], a[240:243], v[208:211]\n"
    "ds_read_b64_tr_b16 v[124:125], v67 offset:57856\n"
    "v_mfma_f32_16x16x32_bf16 v[208:211], v[100:103], a[244:247], v[208:211]\n"
    "s_mov_b32 s25, 2097403\n"
    "s_getpc_b64 s[28:29]\n"
    "s_cmp_ge_i32 s25, s24\n"
    "s_cbranch_scc1 90b\n"
    "v_mfma_f32_16x16x32_bf16 v[240:243], v[100:103], a[252:255], v[240:243]\n"
    "ds_read_b64_tr_b16 v[126:127], v67 offset:58880\n"
    "v_mfma_f32_16x16x32_bf16 v[240:243], v[96:99], a[248:251], v[240:243]\n"
    "buffer_load_dwordx4 a[100:103], v54, s[40:43], s65 offen offset:1024\n"
    "s_waitcnt lgkmcnt(8)\n"
    "s_waitcnt vmcnt(32)\n"
    "s_barrier\n"
    "v_mfma_f32_16x16x32_bf16 v[244:247], v[104:107], a[248:251], v[244:247]\n"
    "ds_read_b64_tr_b16 v[96:97], v60 offset:0\n"
    "v_mfma_f32_16x16x32_bf16 v[244:247], v[108:111], a[252:255], v[244:247]\n"
    "v_mfma_f32_16x16x32_bf16 v[212:215], v[108:111], a[244:247], v[212:215]\n"
    "ds_read_b64_tr_b16 v[98:99], v60 offset:1024\n"
    "v_mfma_f32_16x16x32_bf16 v[212:215], v[104:107], a[240:243], v[212:215]\n"
    "v_mfma_f32_16x16x32_bf16 v[180:183], v[104:107], a[232:235], v[180:183]\n"
    "ds_read_b64_tr_b16 v[100:101], v60 offset:8192\n"
    "v_mfma_f32_16x16x32_bf16 v[180:183], v[108:111], a[236:239], v[180:183]\n"
    "buffer_load_dwordx4 a[104:107], v54, s[40:43], s65 offen offset:2048\n"
    "v_mfma_f32_16x16x32_bf16 v[148:151], v[108:111], a[228:231], v[148:151]\n"
    "ds_read_b64_tr_b16 v[102:103], v60 offset:9216\n"
    "v_mfma_f32_16x16x32_bf16 v[148:151], v[104:107], a[224:227], v[148:151]\n"
    "s_waitcnt lgkmcnt(8)\n"
    "v_mfma_f32_16x16x32_bf16 v[152:155], v[112:115], a[224:227], v[152:155]\n"
    "ds_read_b64_tr_b16 v[104:105], v61 offset:0\n"
    "v_mfma_f32_16x16x32_bf16 v[152:155], v[116:119], a[228:231], v[152:155]\n"
    "v_mfma_f32_16x16x32_bf16 v[184:187], v[116:119], a[236:239], v[184:187]\n"
    "ds_read_b64_tr_b16 v[106:107], v61 offset:1024\n"
    "v_mfma_f32_16x16x32_bf16 v[184:187], v[112:115], a[232:235], v[184:187]\n"
    "buffer_load_dwordx4 a[108:111], v54, s[40:43], s65 offen offset:3072\n"
    "v_mfma_f32_16x16x32_bf16 v[216:219], v[112:115], a[240:243], v[216:219]\n"
    "ds_read_b64_tr_b16 v[108:109], v61 offset:8192\n"
    "v_mfma_f32_16x16x32_bf16 v[216:219], v[116:119], a[244:247], v[216:219]\n"
    "v_mfma_f32_16x16x32_bf16 v[248:251], v[116:119], a[252:255], v[248:251]\n"
    "ds_read_b64_tr_b16 v[110:111], v61 offset:9216\n"
    "v_mfma_f32_16x16x32_bf16 v[248:251], v[112:115], a[248:251], v[248:251]\n"
    "buffer_load_dwordx4 a[112:115], v55, s[40:43], s65 offen offset:0\n"
    "s_waitcnt lgkmcnt(8)\n"
    "v_mfma_f32_16x16x32_bf16 v[252:255], v[120:123], a[248:251], v[252:255]\n"
    "ds_read_b64_tr_b16 v[112:113], v62 offset:0\n"
    "v_mfma_f32_16x16x32_bf16 v[252:255], v[124:127], a[252:255], v[252:255]\n"
    "v_mfma_f32_16x16x32_bf16 v[220:223], v[124:127], a[244:247], v[220:223]\n"
    "ds_read_b64_tr_b16 v[114:115], v62 offset:1024\n"
    "v_mfma_f32_16x16x32_bf16 v[220:223], v[120:123], a[240:243], v[220:223]\n"
    "v_mfma_f32_16x16x32_bf16 v[188:191], v[120:123], a[232:235], v[188:191]\n"
    "ds_read_b64_tr_b16 v[116:117], v62 offset:8192\n"
    "v_mfma_f32_16x16x32_bf16 v[188:191], v[124:127], a[236:239], v[188:191]\n"
    "buffer_load_dwordx4 a[116:119], v55, s[40:43], s65 offen offset:1024\n"
    "v_mfma_f32_16x16x32_bf16 v[156:159], v[124:127], a[228:231], v[156:159]\n"
    "ds_read_b64_tr_b16 v[118:119], v62 offset:9216\n"
    "v_mfma_f32_16x16x32_bf16 v[156:159], v[120:123], a[224:227], v[156:159]\n"
    "s_waitcnt lgkmcnt(8)\n"
    "v_mfma_f32_16x16x32_bf16 v[128:131], v[96:99], a[0:3], v[128:131]\n"
    "ds_read_b64_tr_b16 v[120:121], v63 offset:0\n"
    "v_mfma_f32_16x16x32_bf16 v[128:131], v[100:103], a[4:7], v[128:131]\n"
    "v_mfma_f32_16x16x32_bf16 v[160:163], v[100:103], a[12:15], v[160:163]\n"
    "ds_read_b64_tr_b16 v[122:123], v63 offset:1024\n"
    "v_mfma_f32_16x16x32_bf16 v[160:163], v[96:99], a[8:11], v[160:163]\n"
    "buffer_load_dwordx4 a[120:123], v55, s[40:43], s65 offen offset:2048\n"
    "v_mfma_f32_16x16x32_bf16 v[192:195], v[96:99], a[16:19], v[192:195]\n"
    "ds_read_b64_tr_b16 v[124:125], v63 offset:8192\n"
    "v_mfma_f32_16x16x32_bf16 v[192:195], v[100:103], a[20:23], v[192:195]\n"
    "v_mfma_f32_16x16x32_bf16 v[224:227], v[100:103], a[28:31], v[224:227]\n"
    "ds_read_b64_tr_b16 v[126:127], v63 offset:9216\n"
    "v_mfma_f32_16x16x32_bf16 v[224:227], v[96:99], a[24:27], v[224:227]\n"
    "s_waitcnt lgkmcnt(8)\n"
    "v_mfma_f32_16x16x32_bf16 v[228:231], v[104:107], a[24:27], v[228:231]\n"
    "ds_read_b64_tr_b16 v[96:97], v60 offset:512\n"
    "v_mfma_f32_16x16x32_bf16 v[228:231], v[108:111], a[28:31], v[228:231]\n"
    "s_mov_b32 s25, 2162945\n"
    "s_getpc_b64 s[28:29]\n"
    "s_cmp_ge_i32 s25, s24\n"
    "s_cbranch_scc1 90b\n"
    "v_mfma_f32_16x16x32_bf16 v[196:199], v[108:111], a[20:23], v[196:199]\n"
    "ds_read_b64_tr_b16 v[98:99], v60 offset:1536\n"
    "buffer_load_dwordx4 a[124:127], v55, s[40:43], s65 offen offset:3072\n"
    "v_mfma_f32_16x16x32_bf16 v[196:199], v[104:107], a[16:19], v[196:199]\n"
    "v_mfma_f32_16x16x32_bf16 v[164:167], v[104:107], a[8:11], v[164:167]\n"
    "ds_read_b64_tr_b16 v[100:101], v60 offset:8704\n"
    "s_andn2_b64 s[76:77], s[72:73], s[80:81]\n"
    "s_cbranch_scc0 1131f\n"
    "1132:\n"
    "s_andn2_b64 s[14:15], s[18:19], s[80:81]\n"
    "s_cselect_b64 s[76:77], s[14:15], s[76:77]\n"
    "s_ff1_i32_b64 s69, s[76:77]\n"
    "v_mfma_f32_16x16x32_bf16 v[164:167], v[108:111], a[12:15], v[164:167]\n"
    "v_cmp_le_u32 vcc, s70, v47\n"
    "v_mfma_f32_16x16x32_bf16 v[132:135], v[108:111], a[4:7], v[132:135]\n"
    "ds_read_b64_tr_b16 v[102:103], v60 offset:9728\n"
    "v_readlane_b32 s65, v53, s69\n"
    "v_mfma_f32_16x16x32_bf16 v[132:135], v[104:107], a[0:3], v[132:135]\n"
    "s_or_b64 s[82:83], s[82:83], vcc\n"
    "v_cmp_le_u32 vcc, s70, v48\n"
    "s_waitcnt lgkmcnt(8)\n"
    "v_mfma_f32_16x16x32_bf16 v[136:139], v[112:115], a[0:3], v[136:139]\n"
    "ds_read_b64_tr_b16 v[104:105], v61 offset:512\n"
    "s_bitset1_b64 s[80:81], s69\n"
    "v_mfma_f32_16x16x32_bf16 v[136:139], v[116:119], a[4:7], v[136:139]\n"
    "s_or_b64 s[82:83], s[82:83], vcc\n"
    "v_mfma_f32_16x16x32_bf16 v[168:171], v[116:119], a[12:15], v[168:171]\n"
    "ds_read_b64_tr_b16 v[106:107], v61 offset:1536\n"
    "s_nop 0\n"
    "v_mfma_f32_16x16x32_bf16 v[168:171], v[112:115], a[8:11], v[168:171]\n"
    "s_nop 0\n"
    "s_lshl_b32 s64, s65, 5\n"
    "v_mfma_f32_16x16x32_bf16 v[200:203], v[112:115], a[16:19], v[200:203]\n"
    "ds_read_b64_tr_b16 v[108:109], v61 offset:8704\n"
    "s_add_u32 m0, s67, 65536\n"
    "v_mfma_f32_16x16x32_bf16 v[200:203], v[116:119], a[20:23], v[200:203]\n"
    "v_mov_b32 v38, s82\n"
    "v_mov_b32 v39, s83\n"
    "ds_write_b64 v41, v[38:39] offset:0\n"
    "v_mfma_f32_16x16x32_bf16 v[232:235], v[116:119], a[28:31], v[232:235]\n"
    "ds_read_b64_tr_b16 v[110:111], v61 offset:9728\n"
    "s_nop 0\n"
    "v_mfma_f32_16x16x32_bf16 v[232:235], v[112:115], a[24:27], v[232:235]\n"
    "s_ff0_i32_b64 s30, s[82:83]\n"
    "s_bfm_b64 vcc, s30, 0\n"
    "s_andn2_b64 vcc, vcc, s[80:81]\n"
    "s_bcnt1_i32_b64 s30, vcc\n"
    "s_cmp_lt_u32 s30, 64\n"
    "s_cselect_b32 s22, 0x10000, 0\n"
    "s_waitcnt lgkmcnt(9)\n"
    "v_mfma_f32_16x16x32_bf16 v[236:239], v[120:123], a[24:27], v[236:239]\n"
    "ds_read_b64_tr_b16 v[112:113], v62 offset:512\n"
    "buffer_load_dwordx4 v56, s[36:39], s64 offen offset:0 lds\n"
    "v_mfma_f32_16x16x32_bf16 v[236:239], v[124:127], a[28:31], v[236:239]\n"
    "s_bcnt1_i32_b64 s30, s[82:83]\n"
    "s_cmp_ge_u32 s30, 56\n"
    "s_cselect_b32 s22, 0, s22\n"
    "v_mfma_f32_16x16x32_bf16 v[204:207], v[124:127], a[20:23], v[204:207]\n"
    "ds_read_b64_tr_b16 v[114:115], v62 offset:1536\n"
    "buffer_load_dwordx4 v57, s[36:39], s64 offen offset:1024 lds\n"
    "v_mfma_f32_16x16x32_bf16 v[204:207], v[120:123], a[16:19], v[204:207]\n"
    "buffer_load_dword v47, v49, s[20:23], 0 offen sc0 sc1\n"
    "v_mfma_f32_16x16x32_bf16 v[172:175], v[120:123], a[8:11], v[172:175]\n"
    "ds_read_b64_tr_b16 v[116:117], v62 offset:8704\n"
    "buffer_load_dwordx4 v58, s[36:39], s64 offen offset:2048 lds\n"
    "v_mfma_f32_16x16x32_bf16 v[172:175], v[124:127], a[12:15], v[172:175]\n"
    "buffer_load_dwordx4 v59, s[36:39], s64 offen offset:3072 lds\n"
    "v_mfma_f32_16x16x32_bf16 v[140:143], v[124:127], a[4:7], v[140:143]\n"
    "ds_read_b64_tr_b16 v[118:119], v62 offset:9728\n"
    "buffer_load_dwordx4 a[128:131], v54, s[40:43], s65 offen offset:0\n"
    "v_mfma_f32_16x16x32_bf16 v[140:143], v[120:123], a[0:3], v[140:143]\n"
    "buffer_load_dwordx4 a[132:135], v54, s[40:43], s65 offen offset:1024\n"
    "s_waitcnt lgkmcnt(9)\n"
    "v_mfma_f32_16x16x32_bf16 v[144:147], v[96:99], a[0:3], v[144:147]\n"
    "ds_read_b64_tr_b16 v[120:121], v63 offset:512\n"
    "v_mfma_f32_16x16x32_bf16 v[144:147], v[100:103], a[4:7], v[144:147]\n"
    "buffer_load_dwordx4 a[136:139], v54, s[40:43], s65 offen offset:2048\n"
    "v_mfma_f32_16x16x32_bf16 v[176:179], v[100:103], a[12:15], v[176:179]\n"
    "ds_read_b64_tr_b16 v[122:123], v63 offset:1536\n"
    "v_mfma_f32_16x16x32_bf16 v[176:179], v[96:99], a[8:11], v[176:179]\n"
    "v_mfma_f32_16x16x32_bf16 v[208:211], v[96:99], a[16:19], v[208:211]\n"
    "ds_read_b64_tr_b16 v[124:125], v63 offset:8704\n"
    "v_mfma_f32_16x16x32_bf16 v[208:211], v[100:103], a[20:23], v[208:211]\n"
    "s_mov_b32 s25, 2228490\n"
    "s_getpc_b64 s[28:29]\n"
    "s_cmp_ge_i32 s25, s24\n"
    "s_cbranch_scc1 90b\n"
    "v_mfma_f32_16x16x32_bf16 v[240:243], v[100:103], a[28:31], v[240:243]\n"
    "ds_read_b64_tr_b16 v[126:127], v63 offset:9728\n"
    "buffer_load_dwordx4 a[140:143], v54, s[40:43], s65 offen offset:3072\n"
    "v_mfma_f32_16x16x32_bf16 v[240:243], v[96:99], a[24:27], v[240:243]\n"
    "buffer_load_dwordx4 a[144:147], v55, s[40:43], s65 offen offset:0\n"
    "s_waitcnt lgkmcnt(8)\n"
    "s_waitcnt vmcnt(35)\n"
    "s_barrier\n"
    "v_mfma_f32_16x16x32_bf16 v[244:247], v[104:107], a[24:27], v[244:247]\n"
    "ds_read_b64_tr_b16 v[96:97], v60 offset:16384\n"
    "v_mfma_f32_16x16x32_bf16 v[244:247], v[108:111], a[28:31], v[244:247]\n"
    "ds_read_b64 v[36:37], v40 offset:0\n"
    "v_mfma_f32_16x16x32_bf16 v[212:215], v[108:111], a[20:23], v[212:215]\n"
    "ds_read_b64_tr_b16 v[98:99], v60 offset:17408\n"
    "v_mfma_f32_16x16x32_bf16 v[212:215], v[104:107], a[16:19], v[212:215]\n"
    "buffer_load_dwordx4 a[148:151], v55, s[40:43], s65 offen offset:1024\n"
    "v_mfma_f32_16x16x32_bf16 v[180:183], v[104:107], a[8:11], v[180:183]\n"
    "ds_read_b64_tr_b16 v[100:101], v60 offset:24576\n"
    "v_mfma_f32_16x16x32_bf16 v[180:183], v[108:111], a[12:15], v[180:183]\n"
    "buffer_load_dwordx4 a[152:155], v55, s[40:43], s65 offen offset:2048\n"
    "v_mfma_f32_16x16x32_bf16 v[148:151], v[108:111], a[4:7], v[148:151]\n"
    "ds_read_b64_tr_b16 v[102:103], v60 offset:25600\n"
    "v_mfma_f32_16x16x32_bf16 v[148:151], v[104:107], a[0:3], v[148:151]\n"
    "buffer_load_dwordx4 a[156:159], v55, s[40:43], s65 offen offset:3072\n"
    "s_waitcnt lgkmcnt(9)\n"
    "v_mfma_f32_16x16x32_bf16 v[152:155], v[112:115], a[0:3], v[152:155]\n"
    "ds_read_b64_tr_b16 v[104:105], v61 offset:16384\n"
    "s_andn2_b64 s[76:77], s[72:73], s[80:81]\n"
    "s_cbranch_scc0 1141f\n"
    "1142:\n"
    "s_andn2_b64 s[14:15], s[18:19], s[80:81]\n"
    "s_cselect_b64 s[76:77], s[14:15], s[76:77]\n"
    "s_ff1_i32_b64 s69, s[76:77]\n"
    "v_mfma_f32_16x16x32_bf16 v[152:155], v[116:119], a[4:7], v[152:155]\n"
    "v_readlane_b32 s65, v53, s69\n"
    "s_bitset1_b64 s[80:81], s69\n"
    "v_mfma_f32_16x16x32_bf16 v[184:187], v[116:119], a[12:15], v[184:187]\n"
    "ds_read_b64_tr_b16 v[106:107], v61 offset:17408\n"
    "s_nop 0\n"
    "v_mfma_f32_16x16x32_bf16 v[184:187], v[112:115], a[8:11], v[184:187]\n"
    "s_nop 0\n"
    "s_lshl_b32 s64, s65, 5\n"
    "v_mfma_f32_16x16x32_bf16 v[216:219], v[112:115], a[16:19], v[216:219]\n"
    "ds_read_b64_tr_b16 v[108:109], v61 offset:24576\n"
    "s_add_u32 m0, s67, 81920\n"
    "v_mfma_f32_16x16x32_bf16 v[216:219], v[116:119], a[20:23], v[216:219]\n"
    "s_nop 0\n"
    "v_mfma_f32_16x16x32_bf16 v[248:251], v[116:119], a[28:31], v[248:251]\n"
    "ds_read_b64_tr_b16 v[110:111], v61 offset:25600\n"
    "v_mfma_f32_16x16x32_bf16 v[248:251], v[112:115], a[24:27], v[248:251]\n"
    "buffer_load_dwordx4 v56, s[36:39], s64 offen offset:0 lds\n"
    "s_waitcnt lgkmcnt(9)\n"
    "v_mfma_f32_16x16x32_bf16 v[252:255], v[120:123], a[24:27], v[252:255]\n"
    "ds_read_b64_tr_b16 v[112:113], v62 offset:16384\n"
    "v_mfma_f32_16x16x32_bf16 v[252:255], v[124:127], a[28:31], v[252:255]\n"
    "buffer_load_dwordx4 v57, s[36:39], s64 offen offset:1024 lds\n"
    "v_mfma_f32_16x16x32_bf16 v[220:223], v[124:127], a[20:23], v[220:223]\n"
    "ds_read_b64_tr_b16 v[114:115], v62 offset:17408\n"
    "v_mfma_f32_16x16x32_bf16 v[220:223], v[120:123], a[16:19], v[220:223]\n"
    "buffer_load_dwordx4 v58, s[36:39], s64 offen offset:2048 lds\n"
    "v_mfma_f32_16x16x32_bf16 v[188:191], v[120:123], a[8:11], v[188:191]\n"
    "ds_read_b64_tr_b16 v[116:117], v62 offset:24576\n"
    "v_mfma_f32_16x16x32_bf16 v[188:191], v[124:127], a[12:15], v[188:191]\n"
    "buffer_load_dwordx4 v59, s[36:39], s64 offen offset:3072 lds\n"
    "v_mfma_f32_16x16x32_bf16 v[156:159], v[124:127], a[4:7], v[156:159]\n"
    "ds_read_b64_tr_b16 v[118:119], v62 offset:25600\n"
    "v_mfma_f32_16x16x32_bf16 v[156:159], v[120:123], a[0:3], v[156:159]\n"
    "s_waitcnt lgkmcnt(8)\n"
    "v_mfma_f32_16x16x32_bf16 v[128:131], v[96:99], a[32:35], v[128:131]\n"
    "ds_read_b64_tr_b16 v[120:121], v63 offset:16384\n"
    "v_mfma_f32_16x16x32_bf16 v[128:131], v[100:103], a[36:39], v[128:131]\n"
    "v_readfirstlane_b32 vcc_lo, v36\n"
    "v_readfirstlane_b32 vcc_hi, v37\n"
    "v_mfma_f32_16x16x32_bf16 v[160:163], v[100:103], a[44:47], v[160:163]\n"
    "ds_read_b64_tr_b16 v[122:123], v63 offset:17408\n"
    "buffer_load_dwordx4 a[160:163], v54, s[40:43], s65 offen offset:0\n"
    "v_mfma_f32_16x16x32_bf16 v[160:163], v[96:99], a[40:43], v[160:163]\n"
    "s_or_b64 s[72:73], s[72:73], vcc\n"
    "v_mfma_f32_16x16x32_bf16 v[192:195], v[96:99], a[48:51], v[192:195]\n"
    "ds_read_b64_tr_b16 v[124:125], v63 offset:24576\n"
    "buffer_load_dwordx4 a[164:167], v54, s[40:43], s65 offen offset:1024\n"
    "v_mfma_f32_16x16x32_bf16 v[192:195], v[100:103], a[52:55], v[192:195]\n"
    "buffer_load_dwordx4 a[168:171], v54, s[40:43], s65 offen offset:2048\n"
    "v_mfma_f32_16x16x32_bf16 v[224:227], v[100:103], a[60:63], v[224:227]\n"
    "ds_read_b64_tr_b16 v[126:127], v63 offset:25600\n"
    "v_mfma_f32_16x16x32_bf16 v[224:227], v[96:99], a[56:59], v[224:227]\n"
    "s_waitcnt lgkmcnt(8)\n"
    "v_mfma_f32_16x16x32_bf16 v[228:231], v[104:107], a[56:59], v[228:231]\n"
    "ds_read_b64_tr_b16 v[96:97], v60 offset:16896\n"
    "v_mfma_f32_16x16x32_bf16 v[228:231], v[108:111], a[60:63], v[228:231]\n"
    "s_mov_b32 s25, 2294038\n"
    "s_getpc_b64 s[28:29]\n"
    "s_cmp_ge_i32 s25, s24\n"
    "s_cbranch_scc1 90b\n"
    "v_mfma_f32_16x16x32_bf16 v[196:199], v[108:111], a[52:55], v[196:199]\n"
    "ds_read_b64_tr_b16 v[98:99], v60 offset:17920\n"
    "buffer_load_dwordx4 a[172:175], v54, s[40:43], s65 offen offset:3072\n"
    "v_mfma_f32_16x16x32_bf16 v[196:199], v[104:107], a[48:51], v[196:199]\n"
    "buffer_load_dwordx4 a[176:179], v55, s[40:43], s65 offen offset:0\n"
    "v_mfma_f32_16x16x32_bf16 v[164:167], v[104:107], a[40:43], v[164:167]\n"
    "ds_read_b64_tr_b16 v[100:101], v60 offset:25088\n"
    "v_mfma_f32_16x16x32_bf16 v[164:167], v[108:111], a[44:47], v[164:167]\n"
    "buffer_load_dwordx4 a[180:183], v55, s[40:43], s65 offen offset:1024\n"
    "v_mfma_f32_16x16x32_bf16 v[132:135], v[108:111], a[36:39], v[132:135]\n"
    "ds_read_b64_tr_b16 v[102:103], v60 offset:26112\n"
    "v_mfma_f32_16x16x32_bf16 v[132:135], v[104:107], a[32:35], v[132:135]\n"
    "s_waitcnt lgkmcnt(8)\n"
    "v_mfma_f32_16x16x32_bf16 v[136:139], v[112:115], a[32:35], v[136:139]\n"
    "ds_read_b64_tr_b16 v[104:105], v61 offset:16896\n"
    "v_mfma_f32_16x16x32_bf16 v[136:139], v[116:119], a[36:39], v[136:139]\n"
    "buffer_load_dwordx4 a[184:187], v55, s[40:43], s65 offen offset:2048\n"
    "v_mfma_f32_16x16x32_bf16 v[168:171], v[116:119], a[44:47], v[168:171]\n"
    "ds_read_b64_tr_b16 v[106:107], v61 offset:17920\n"
    "v_mfma_f32_16x16x32_bf16 v[168:171], v[112:115], a[40:43], v[168:171]\n"
    "buffer_load_dwordx4 a[188:191], v55, s[40:43], s65 offen offset:3072\n"
    "v_mfma_f32_16x16x32_bf16 v[200:203], v[112:115], a[48:51], v[200:203]\n"
    "ds_read_b64_tr_b16 v[108:109], v61 offset:25088\n"
    "s_andn2_b64 s[76:77], s[72:73], s[80:81]\n"
    "s_cbranch_scc0 1151f\n"
    "1152:\n"
    "s_andn2_b64 s[14:15], s[18:19], s[80:81]\n"
    "s_cselect_b64 s[76:77], s[14:15], s[76:77]\n"
    "s_ff1_i32_b64 s69, s[76:77]\n"
    "v_mfma_f32_16x16x32_bf16 v[200:203], v[116:119], a[52:55], v[200:203]\n"
    "v_readlane_b32 s65, v53, s69\n"
    "s_bitset1_b64 s[80:81], s69\n"
    "v_mfma_f32_16x16x32_bf16 v[232:235], v[116:119], a[60:63], v[232:235]\n"
    "ds_read_b64_tr_b16 v[110:111], v61 offset:26112\n"
    "s_nop 0\n"
    "v_mfma_f32_16x16x32_bf16 v[232:235], v[112:115], a[56:59], v[232:235]\n"
    "s_nop 0\n"
    "s_lshl_b32 s64, s65, 5\n"
    "s_waitcnt lgkmcnt(8)\n"
    "v_mfma_f32_16x16x32_bf16 v[236:239], v[120:123], a[56:59], v[236:239]\n"
    "ds_read_b64_tr_b16 v[112:113], v62 offset:16896\n"
    "s_add_u32 m0, s67, 98304\n"
    "v_mfma_f32_16x16x32_bf16 v[236:239], v[124:127], a[60:63], v[236:239]\n"
    "s_nop 0\n"
    "v_mfma_f32_16x16x32_bf16 v[204:207], v[124:127], a[52:55], v[204:207]\n"
    "ds_read_b64_tr_b16 v[114:115], v62 offset:17920\n"
    "v_mfma_f32_16x16x32_bf16 v[204:207], v[120:123], a[48:51], v[204:207]\n"
    "buffer_load_dwordx4 v56, s[36:39], s64 offen offset:0 lds\n"
    "v_mfma_f32_16x16x32_bf16 v[172:175], v[120:123], a[40:43], v[172:175]\n"
    "ds_read_b64_tr_b16 v[116:117], v62 offset:25088\n"
    "v_mfma_f32_16x16x32_bf16 v[172:175], v[124:127], a[44:47], v[172:175]\n"
    "buffer_load_dwordx4 v57, s[36:39], s64 offen offset:1024 lds\n"
    "v_mfma_f32_16x16x32_bf16 v[140:143], v[124:127], a[36:39], v[140:143]\n"
    "ds_read_b64_tr_b16 v[118:119], v62 offset:26112\n"
    "v_mfma_f32_16x16x32_bf16 v[140:143], v[120:123], a[32:35], v[140:143]\n"
    "buffer_load_dwordx4 v58, s[36:39], s64 offen offset:2048 lds\n"
    "s_waitcnt lgkmcnt(8)\n"
    "v_mfma_f32_16x16x32_bf16 v[144:147], v[96:99], a[32:35], v[144:147]\n"
    "ds_read_b64_tr_b16 v[120:121], v63 offset:16896\n"
    "v_mfma_f32_16x16x32_bf16 v[144:147], v[100:103], a[36:39], v[144:147]\n"
    "buffer_load_dwordx4 v59, s[36:39], s64 offen offset:3072 lds\n"
    "v_mfma_f32_16x16x32_bf16 v[176:179], v[100:103], a[44:47], v[176:179]\n"
    "ds_read_b64_tr_b16 v[122:123], v63 offset:17920\n"
    "v_mfma_f32_16x16x32_bf16 v[176:179], v[96:99], a[40:43], v[176:179]\n"
    "v_mfma_f32_16x16x32_bf16 v[208:211], v[96:99], a[48:51], v[208:211]\n"
    "ds_read_b64_tr_b16 v[124:125], v63 offset:25088\n"
    "v_mfma_f32_16x16x32_bf16 v[208:211], v[100:103], a[52:55], v[208:211]\n"
    "s_mov_b32 s25, 2359583\n"
    "s_getpc_b64 s[28:29]\n"
    "s_cmp_ge_i32 s25, s24\n"
    "s_cbranch_scc1 90b\n"
    "v_mfma_f32_16x16x32_bf16 v[240:243], v[100:103], a[60:63], v[240:243]\n"
    "ds_read_b64_tr_b16 v[126:127], v63 offset:26112\n"
    "buffer_load_dwordx4 a[192:195], v54, s[40:43], s65 offen offset:0\n"
    "v_mfma_f32_16x16x32_bf16 v[240:243], v[96:99], a[56:59], v[240:243]\n"
    "buffer_load_dwordx4 a[196:199], v54, s[40:43], s65 offen offset:1024\n"
    "s_waitcnt lgkmcnt(8)\n"
    "s_waitcnt vmcnt(43)\n"
    "s_barrier\n"
    "v_mfma_f32_16x16x32_bf16 v[244:247], v[104:107], a[56:59], v[244:247]\n"
    "ds_read_b64_tr_b16 v[96:97], v60 offset:32768\n"
    "v_mfma_f32_16x16x32_bf16 v[244:247], v[108:111], a[60:63], v[244:247]\n"
    "buffer_load_dwordx4 a[200:203], v54, s[40:43], s65 offen offset:2048\n"
    "v_mfma_f32_16x16x32_bf16 v[212:215], v[108:111], a[52:55], v[212:215]\n"
    "ds_read_b64_tr_b16 v[98:99], v60 offset:33792\n"
    "v_mfma_f32_16x16x32_bf16 v[212:215], v[104:107], a[48:51], v[212:215]\n"
    "v_mfma_f32_16x16x32_bf16 v[180:183], v[104:107], a[40:43], v[180:183]\n"
    "ds_read_b64_tr_b16 v[100:101], v60 offset:40960\n"
    "v_mfma_f32_16x16x32_bf16 v[180:183], v[108:111], a[44:47], v[180:183]\n"
    "buffer_load_dwordx4 a[204:207], v54, s[40:43], s65 offen offset:3072\n"
    "v_mfma_f32_16x16x32_bf16 v[148:151], v[108:111], a[36:39], v[148:151]\n"
    "ds_read_b64_tr_b16 v[102:103], v60 offset:41984\n"
    "v_mfma_f32_16x16x32_bf16 v[148:151], v[104:107], a[32:35], v[148:151]\n"
    "buffer_load_dwordx4 a[208:211], v55, s[40:43], s65 offen offset:0\n"
    "s_waitcnt lgkmcnt(8)\n"
    "v_mfma_f32_16x16x32_bf16 v[152:155], v[112:115], a[32:35], v[152:155]\n"
    "ds_read_b64_tr_b16 v[104:105], v61 offset:32768\n"
    "v_mfma_f32_16x16x32_bf16 v[152:155], v[116:119], a[36:39], v[152:155]\n"
    "buffer_load_dwordx4 a[212:215], v55, s[40:43], s65 offen offset:1024\n"
    "v_mfma_f32_16x16x32_bf16 v[184:187], v[116:119], a[44:47], v[184:187]\n"
    "ds_read_b64_tr_b16 v[106:107], v61 offset:33792\n"
    "v_mfma_f32_16x16x32_bf16 v[184:187], v[112:115], a[40:43], v[184:187]\n"
    "v_mfma_f32_16x16x32_bf16 v[216:219], v[112:115], a[48:51], v[216:219]\n"
    "ds_read_b64_tr_b16 v[108:109], v61 offset:40960\n"
    "v_mfma_f32_16x16x32_bf16 v[216:219], v[116:119], a[52:55], v[216:219]\n"
    "buffer_load_dwordx4 a[216:219], v55, s[40:43], s65 offen offset:2048\n"
    "v_mfma_f32_16x16x32_bf16 v[248:251], v[116:119], a[60:63], v[248:251]\n"
    "ds_read_b64_tr_b16 v[110:111], v61 offset:41984\n"
    "v_mfma_f32_16x16x32_bf16 v[248:251], v[112:115], a[56:59], v[248:251]\n"
    "buffer_load_dwordx4 a[220:223], v55, s[40:43], s65 offen offset:3072\n"
    "s_waitcnt lgkmcnt(8)\n"
    "v_mfma_f32_16x16x32_bf16 v[252:255], v[120:123], a[56:59], v[252:255]\n"
    "ds_read_b64_tr_b16 v[112:113], v62 offset:32768\n"
    "s_andn2_b64 s[76:77], s[72:73], s[80:81]\n"
    "s_cbranch_scc0 1161f\n"
    "1162:\n"
    "s_andn2_b64 s[14:15], s[18:19], s[80:81]\n"
    "s_cselect_b64 s[76:77], s[14:15], s[76:77]\n"
    "s_ff1_i32_b64 s69, s[76:77]\n"
    "v_mfma_f32_16x16x32_bf16 v[252:255], v[124:127], a[60:63], v[252:255]\n"
    "v_readlane_b32 s65, v53, s69\n"
    "s_bitset1_b64 s[80:81], s69\n"
    "v_mfma_f32_16x16x32_bf16 v[220:223], v[124:127], a[52:55], v[220:223]\n"
    "ds_read_b64_tr_b16 v[114:115], v62 offset:33792\n"
    "s_nop 0\n"
    "v_mfma_f32_16x16x32_bf16 v[220:223], v[120:123], a[48:51], v[220:223]\n"
    "s_nop 0\n"
    "s_lshl_b32 s64, s65, 5\n"
    "v_mfma_f32_16x16x32_bf16 v[188:191], v[120:123], a[40:43], v[188:191]\n"
    "ds_read_b64_tr_b16 v[116:117], v62 offset:40960\n"
    "s_add_u32 m0, s67, 114688\n"
    "v_mfma_f32_16x16x32_bf16 v[188:191], v[124:127], a[44:47], v[188:191]\n"
    "s_nop 0\n"
    "v_mfma_f32_16x16x32_bf16 v[156:159], v[124:127], a[36:39], v[156:159]\n"
    "ds_read_b64_tr_b16 v[118:119], v62 offset:41984\n"
    "v_mfma_f32_16x16x32_bf16 v[156:159], v[120:123], a[32:35], v[156:159]\n"
    "buffer_load_dwordx4 v56, s[36:39], s64 offen offset:0 lds\n"
    "s_waitcnt lgkmcnt(8)\n"
    "v_mfma_f32_16x16x32_bf16 v[128:131], v[96:99], a[64:67], v[128:131]\n"
    "ds_read_b64_tr_b16 v[120:121], v63 offset:32768\n"
    "v_mfma_f32_16x16x32_bf16 v[128:131], v[100:103], a[68:71], v[128:131]\n"
    "buffer_load_dwordx4 v57, s[36:39], s64 offen offset:1024 lds\n"
    "v_mfma_f32_16x16x32_bf16 v[160:163], v[100:103], a[76:79], v[160:163]\n"
    "ds_read_b64_tr_b16 v[122:123], v63 offset:33792\n"
    "v_mfma_f32_16x16x32_bf16 v[160:163], v[96:99], a[72:75], v[160:163]\n"
    "buffer_load_dwordx4 v58, s[36:39], s64 offen offset:2048 lds\n"
    "v_mfma_f32_16x16x32_bf16 v[192:195], v[96:99], a[80:83], v[192:195]\n"
    "ds_read_b64_tr_b16 v[124:125], v63 offset:40960\n"
    "s_andn2_b64 s[16:17], s[72:73], s[80:81]\n"
    "v_mfma_f32_16x16x32_bf16 v[192:195], v[100:103], a[84:87], v[192:195]\n"
    "s_mov_b64 s[14:15], s[16:17]\n"
    "s_ff1_i32_b64 s69, s[14:15]\n"
    "v_mfma_f32_16x16x32_bf16 v[224:227], v[100:103], a[92:95], v[224:227]\n"
    "ds_read_b64_tr_b16 v[126:127], v63 offset:41984\n"
    "v_mfma_f32_16x16x32_bf16 v[224:227], v[96:99], a[88:91], v[224:227]\n"
    "buffer_load_dwordx4 v59, s[36:39], s64 offen offset:3072 lds\n"
    "s_waitcnt lgkmcnt(8)\n"
    "v_mfma_f32_16x16x32_bf16 v[228:231], v[104:107], a[88:91], v[228:231]\n"
    "ds_read_b64_tr_b16 v[96:97], v60 offset:33280\n"
    "s_bitset0_b64 s[14:15], s69\n"
    "v_mfma_f32_16x16x32_bf16 v[228:231], v[108:111], a[92:95], v[228:231]\n"
    "s_mov_b32 s25, 2425131\n"
    "s_getpc_b64 s[28:29]\n"
    "s_cmp_ge_i32 s25, s24\n"
    "s_cbranch_scc1 90b\n"
    "v_mfma_f32_16x16x32_bf16 v[196:199], v[108:111], a[84:87], v[196:199]\n"
    "ds_read_b64_tr_b16 v[98:99], v60 offset:34304\n"
    "s_ff1_i32_b64 s69, s[14:15]\n"
    "v_mfma_f32_16x16x32_bf16 v[196:199], v[104:107], a[80:83], v[196:199]\n"
    "s_bitset0_b64 s[14:15], s69\n"
    "v_mfma_f32_16x16x32_bf16 v[164:167], v[104:107], a[72:75], v[164:167]\n"
    "ds_read_b64_tr_b16 v[100:101], v60 offset:41472\n"
    "buffer_load_dwordx4 a[224:227], v54, s[40:43], s65 offen offset:0\n"
    "v_mfma_f32_16x16x32_bf16 v[164:167], v[108:111], a[76:79], v[164:167]\n"
    "v_cmp_le_u32 vcc, s70, v47\n"
    "v_mfma_f32_16x16x32_bf16 v[132:135], v[108:111], a[68:71], v[132:135]\n"
    "ds_read_b64_tr_b16 v[102:103], v60 offset:42496\n"
    "s_ff1_i32_b64 s69, s[14:15]\n"
    "v_mfma_f32_16x16x32_bf16 v[132:135], v[104:107], a[64:67], v[132:135]\n"
    "s_or_b64 s[82:83], s[82:83], vcc\n"
    "v_cmp_le_u32 vcc, s70, v48\n"
    "s_waitcnt lgkmcnt(8)\n"
    "v_mfma_f32_16x16x32_bf16 v[136:139], v[112:115], a[64:67], v[136:139]\n"
    "ds_read_b64_tr_b16 v[104:105], v61 offset:33280\n"
    "s_bitset0_b64 s[14:15], s69\n"
    "v_mfma_f32_16x16x32_bf16 v[136:139], v[116:119], a[68:71], v[136:139]\n"
    "s_or_b64 s[82:83], s[82:83], vcc\n"
    "v_mfma_f32_16x16x32_bf16 v[168:171], v[116:119], a[76:79], v[168:171]\n"
    "ds_read_b64_tr_b16 v[106:107], v61 offset:34304\n"
    "s_ff1_i32_b64 s69, s[14:15]\n"
    "v_mfma_f32_16x16x32_bf16 v[168:171], v[112:115], a[72:75], v[168:171]\n"
    "buffer_load_dwordx4 a[228:231], v54, s[40:43], s65 offen offset:1024\n"
    "v_mfma_f32_16x16x32_bf16 v[200:203], v[112:115], a[80:83], v[200:203]\n"
    "ds_read_b64_tr_b16 v[108:109], v61 offset:41472\n"
    "s_bitset0_b64 s[14:15], s69\n"
    "v_mfma_f32_16x16x32_bf16 v[200:203], v[116:119], a[84:87], v[200:203]\n"
    "v_mov_b32 v38, s82\n"
    "v_mov_b32 v39, s83\n"
    "ds_write_b64 v41, v[38:39] offset:0\n"
    "v_mfma_f32_16x16x32_bf16 v[232:235], v[116:119], a[92:95], v[232:235]\n"
    "ds_read_b64_tr_b16 v[110:111], v61 offset:42496\n"
    "s_xor_b64 s[16:17], s[16:17], s[14:15]\n"
    "v_mfma_f32_16x16x32_bf16 v[232:235], v[112:115], a[88:91], v[232:235]\n"
    "s_ff0_i32_b64 s30, s[82:83]\n"
    "s_bfm_b64 vcc, s30, 0\n"
    "s_andn2_b64 vcc, vcc, s[80:81]\n"
    "s_bcnt1_i32_b64 s30, vcc\n"
    "s_cmp_lt_u32 s30, 64\n"
    "s_cselect_b32 s22, 0x10000, 0\n"
    "s_waitcnt lgkmcnt(9)\n"
    "v_mfma_f32_16x16x32_bf16 v[236:239], v[120:123], a[88:91], v[236:239]\n"
    "ds_read_b64_tr_b16 v[112:113], v62 offset:33280\n"
    "s_mov_b64 s[18:19], s[14:15]\n"
    "v_mfma_f32_16x16x32_bf16 v[236:239], v[124:127], a[92:95], v[236:239]\n"
    "s_bcnt1_i32_b64 s30, s[82:83]\n"
    "s_cmp_ge_u32 s30, 56\n"
    "s_cselect_b32 s22, 0, s22\n"
    "v_mfma_f32_16x16x32_bf16 v[204:207], v[124:127], a[84:87], v[204:207]\n"
    "ds_read_b64_tr_b16 v[114:115], v62 offset:34304\n"
    "buffer_load_dwordx4 a[232:235], v54, s[40:43], s65 offen offset:2048\n"
    "v_mfma_f32_16x16x32_bf16 v[204:207], v[120:123], a[80:83], v[204:207]\n"
    "buffer_load_dword v48, v49, s[20:23], 0 offen sc0 sc1\n"
    "v_mfma_f32_16x16x32_bf16 v[172:175], v[120:123], a[72:75], v[172:175]\n"
    "ds_read_b64_tr_b16 v[116:117], v62 offset:41472\n"
    "s_ff1_i32_b64 s69, s[14:15]\n"
    "v_mfma_f32_16x16x32_bf16 v[172:175], v[124:127], a[76:79], v[172:175]\n"
    "s_bitset0_b64 s[14:15], s69\n"
    "s_ff1_i32_b64 s69, s[14:15]\n"
    "v_mfma_f32_16x16x32_bf16 v[140:143], v[124:127], a[68:71], v[140:143]\n"
    "ds_read_b64_tr_b16 v[118:119], v62 offset:42496\n"
    "buffer_load_dwordx4 a[236:239], v54, s[40:43], s65 offen offset:3072\n"
    "v_mfma_f32_16x16x32_bf16 v[140:143], v[120:123], a[64:67], v[140:143]\n"
    "s_bitset0_b64 s[14:15], s69\n"
    "s_ff1_i32_b64 s69, s[14:15]\n"
    "s_waitcnt lgkmcnt(9)\n"
    "v_mfma_f32_16x16x32_bf16 v[144:147], v[96:99], a[64:67], v[144:147]\n"
    "ds_read_b64_tr_b16 v[120:121], v63 offset:33280\n"
    "s_bitset0_b64 s[14:15], s69\n"
    "v_mfma_f32_16x16x32_bf16 v[144:147], v[100:103], a[68:71], v[144:147]\n"
    "buffer_load_dwordx4 a[240:243], v55, s[40:43], s65 offen offset:0\n"
    "v_mfma_f32_16x16x32_bf16 v[176:179], v[100:103], a[76:79], v[176:179]\n"
    "ds_read_b64_tr_b16 v[122:123], v63 offset:34304\n"
    "s_ff1_i32_b64 s69, s[14:15]\n"
    "v_mfma_f32_16x16x32_bf16 v[176:179], v[96:99], a[72:75], v[176:179]\n"
    "s_bitset0_b64 s[14:15], s69\n"
    "s_xor_b64 s[18:19], s[18:19], s[14:15]\n"
    "v_mfma_f32_16x16x32_bf16 v[208:211], v[96:99], a[80:83], v[208:211]\n"
    "ds_read_b64_tr_b16 v[124:125], v63 offset:41472\n"
    "buffer_load_dwordx4 a[244:247], v55, s[40:43], s65 offen offset:1024\n"
    "v_mfma_f32_16x16x32_bf16 v[208:211], v[100:103], a[84:87], v[208:211]\n"
    "s_mov_b32 s25, 2490674\n"
    "s_getpc_b64 s[28:29]\n"
    "s_cmp_ge_i32 s25, s24\n"
    "s_cbranch_scc1 90b\n"
    "v_mfma_f32_16x16x32_bf16 v[240:243], v[100:103], a[92:95], v[240:243]\n"
    "ds_read_b64_tr_b16 v[126:127], v63 offset:42496\n"
    "s_bcnt1_i32_b64 s69, s[18:19]\n"
    "v_mfma_f32_16x16x32_bf16 v[240:243], v[96:99], a[88:91], v[240:243]\n"
    "s_cmp_eq_u32 s69, 4\n"
    "s_cselect_b64 s[16:17], s[16:17], 0\n"
    "s_cselect_b64 s[18:19], s[18:19], 0\n"
    "s_cmp_eq_u32 s26, 4\n"
    "s_cselect_b64 s[14:15], s[18:19], s[16:17]\n"
    "s_cselect_b64 s[18:19], s[16:17], s[18:19]\n"
    "s_mov_b64 s[16:17], s[14:15]\n"
    "s_waitcnt lgkmcnt(8)\n"
    "s_waitcnt vmcnt(48)\n"
    "s_barrier\n"
    "v_mfma_f32_16x16x32_bf16 v[244:247], v[104:107], a[88:91], v[244:247]\n"
    "ds_read_b64_tr_b16 v[96:97], v60 offset:49152\n"
    "buffer_load_dwordx4 a[248:251], v55, s[40:43], s65 offen offset:2048\n"
    "v_mfma_f32_16x16x32_bf16 v[244:247], v[108:111], a[92:95], v[244:247]\n"
    "ds_read_b64 v[36:37], v40 offset:0\n"
    "v_mfma_f32_16x16x32_bf16 v[212:215], v[108:111], a[84:87], v[212:215]\n"
    "ds_read_b64_tr_b16 v[98:99], v60 offset:50176\n"
    "buffer_load_dwordx4 a[252:255], v55, s[40:43], s65 offen offset:3072\n"
    "v_mfma_f32_16x16x32_bf16 v[212:215], v[104:107], a[80:83], v[212:215]\n"
    "s_andn2_b64 s[76:77], s[72:73], s[80:81]\n"
    "s_cbranch_scc0 1171f\n"
    "1172:\n"
    "s_andn2_b64 s[14:15], s[16:17], s[80:81]\n"
    "s_cselect_b64 s[76:77], s[14:15], s[76:77]\n"
    "s_ff1_i32_b64 s69, s[76:77]\n"
    "v_readlane_b32 s65, v53, s69\n"
    "v_mfma_f32_16x16x32_bf16 v[180:183], v[104:107], a[72:75], v[180:183]\n"
    "ds_read_b64_tr_b16 v[100:101], v60 offset:57344\n"
    "s_bitset1_b64 s[80:81], s69\n"
    "v_mfma_f32_16x16x32_bf16 v[180:183], v[108:111], a[76:79], v[180:183]\n"
    "s_nop 0\n"
    "s_nop 0\n"
    "v_mfma_f32_16x16x32_bf16 v[148:151], v[108:111], a[68:71], v[148:151]\n"
    "ds_read_b64_tr_b16 v[102:103], v60 offset:58368\n"
    "s_lshl_b32 s64, s65, 5\n"
    "v_mfma_f32_16x16x32_bf16 v[148:151], v[104:107], a[64:67], v[148:151]\n"
    "s_add_u32 m0, s67, 0\n"
    "s_nop 0\n"
    "s_waitcnt lgkmcnt(9)\n"
    "v_mfma_f32_16x16x32_bf16 v[152:155], v[112:115], a[64:67], v[152:155]\n"
    "ds_read_b64_tr_b16 v[104:105], v61 offset:49152\n"
    "buffer_load_dwordx4 v56, s[36:39], s64 offen offset:0 lds\n"
    "v_mfma_f32_16x16x32_bf16 v[152:155], v[116:119], a[68:71], v[152:155]\n"
    "buffer_load_dwordx4 v57, s[36:39], s64 offen offset:1024 lds\n"
    "v_mfma_f32_16x16x32_bf16 v[184:187], v[116:119], a[76:79], v[184:187]\n"
    "ds_read_b64_tr_b16 v[106:107], v61 offset:50176\n"
    "buffer_load_dwordx4 v58, s[36:39], s64 offen offset:2048 lds\n"
    "v_mfma_f32_16x16x32_bf16 v[184:187], v[112:115], a[72:75], v[184:187]\n"
    "buffer_load_dwordx4 v59, s[36:39], s64 offen offset:3072 lds\n"
    "v_mfma_f32_16x16x32_bf16 v[216:219], v[112:115], a[80:83], v[216:219]\n"
    "ds_read_b64_tr_b16 v[108:109], v61 offset:57344\n"
    "v_mfma_f32_16x16x32_bf16 v[216:219], v[116:119], a[84:87], v[216:219]\n"
    "v_mfma_f32_16x16x32_bf16 v[248:251], v[116:119], a[92:95], v[248:251]\n"
    "ds_read_b64_tr_b16 v[110:111], v61 offset:58368\n"
    "v_mfma_f32_16x16x32_bf16 v[248:251], v[112:115], a[88:91], v[248:251]\n"
    "s_waitcnt lgkmcnt(9)\n"
    "v_mfma_f32_16x16x32_bf16 v[252:255], v[120:123], a[88:91], v[252:255]\n"
    "ds_read_b64_tr_b16 v[112:113], v62 offset:49152\n"
    "v_mfma_f32_16x16x32_bf16 v[252:255], v[124:127], a[92:95], v[252:255]\n"
    "buffer_load_dwordx4 a[0:3], v54, s[40:43], s65 offen offset:0\n"
    "v_mfma_f32_16x16x32_bf16 v[220:223], v[124:127], a[84:87], v[220:223]\n"
    "ds_read_b64_tr_b16 v[114:115], v62 offset:50176\n"
    "v_mfma_f32_16x16x32_bf16 v[220:223], v[120:123], a[80:83], v[220:223]\n"
    "v_mfma_f32_16x16x32_bf16 v[188:191], v[120:123], a[72:75], v[188:191]\n"
    "ds_read_b64_tr_b16 v[116:117], v62 offset:57344\n"
    "v_mfma_f32_16x16x32_bf16 v[188:191], v[124:127], a[76:79], v[188:191]\n"
    "v_mfma_f32_16x16x32_bf16 v[156:159], v[124:127], a[68:71], v[156:159]\n"
    "ds_read_b64_tr_b16 v[118:119], v62 offset:58368\n"
    "v_mfma_f32_16x16x32_bf16 v[156:159], v[120:123], a[64:67], v[156:159]\n"
    "buffer_load_dwordx4 a[4:7], v54, s[40:43], s65 offen offset:1024\n"
    "s_waitcnt lgkmcnt(8)\n"
    "v_mfma_f32_16x16x32_bf16 v[128:131], v[96:99], a[96:99], v[128:131]\n"
    "ds_read_b64_tr_b16 v[120:121], v63 offset:49152\n"
    "v_mfma_f32_16x16x32_bf16 v[128:131], v[100:103], a[100:103], v[128:131]\n"
    "v_readfirstlane_b32 vcc_lo, v36\n"
    "v_readfirstlane_b32 vcc_hi, v37\n"
    "v_mfma_f32_16x16x32_bf16 v[160:163], v[100:103], a[108:111], v[160:163]\n"
    "ds_read_b64_tr_b16 v[122:123], v63 offset:50176\n"
    "v_mfma_f32_16x16x32_bf16 v[160:163], v[96:99], a[104:107], v[160:163]\n"
    "s_or_b64 s[72:73], s[72:73], vcc\n"
    "v_mfma_f32_16x16x32_bf16 v[192:195], v[96:99], a[112:115], v[192:195]\n"
    "ds_read_b64_tr_b16 v[124:125], v63 offset:57344\n"
    "v_mfma_f32_16x16x32_bf16 v[192:195], v[100:103], a[116:119], v[192:195]\n"
    "buffer_load_dwordx4 a[8:11], v54, s[40:43], s65 offen offset:2048\n"
    "v_mfma_f32_16x16x32_bf16 v[224:227], v[100:103], a[124:127], v[224:227]\n"
    "ds_read_b64_tr_b16 v[126:127], v63 offset:58368\n"
    "v_mfma_f32_16x16x32_bf16 v[224:227], v[96:99], a[120:123], v[224:227]\n"
    "s_waitcnt lgkmcnt(8)\n"
    "v_mfma_f32_16x16x32_bf16 v[228:231], v[104:107], a[120:123], v[228:231]\n"
    "ds_read_b64_tr_b16 v[96:97], v60 offset:49664\n"
    "v_mfma_f32_16x16x32_bf16 v[228:231], v[108:111], a[124:127], v[228:231]\n"
    "s_mov_b32 s25, 2556219\n"
    "s_getpc_b64 s[28:29]\n"
    "s_cmp_ge_i32 s25, s24\n"
    "s_cbranch_scc1 90b\n"
    "v_mfma_f32_16x16x32_bf16 v[196:199], v[108:111], a[116:119], v[196:199]\n"
    "ds_read_b64_tr_b16 v[98:99], v60 offset:50688\n"
    "v_mfma_f32_16x16x32_bf16 v[196:199], v[104:107], a[112:115], v[196:199]\n"
    "buffer_load_dwordx4 a[12:15], v54, s[40:43], s65 offen offset:3072\n"
    "v_mfma_f32_16x16x32_bf16 v[164:167], v[104:107], a[104:107], v[164:167]\n"
    "ds_read_b64_tr_b16 v[100:101], v60 offset:57856\n"
    "v_mfma_f32_16x16x32_bf16 v[164:167], v[108:111], a[108:111], v[164:167]\n"
    "v_mfma_f32_16x16x32_bf16 v[132:135], v[108:111], a[100:103], v[132:135]\n"
    "ds_read_b64_tr_b16 v[102:103], v60 offset:58880\n"
    "v_mfma_f32_16x16x32_bf16 v[132:135], v[104:107], a[96:99], v[132:135]\n"
    "s_waitcnt lgkmcnt(8)\n"
    "v_mfma_f32_16x16x32_bf16 v[136:139], v[112:115], a[96:99], v[136:139]\n"
    "ds_read_b64_tr_b16 v[104:105], v61 offset:49664\n"
    "v_mfma_f32_16x16x32_bf16 v[136:139], v[116:119], a[100:103], v[136:139]\n"
    "buffer_load_dwordx4 a[16:19], v55, s[40:43], s65 offen offset:0\n"
    "v_mfma_f32_16x16x32_bf16 v[168:171], v[116:119], a[108:111], v[168:171]\n"
    "ds_read_b64_tr_b16 v[106:107], v61 offset:50688\n"
    "v_mfma_f32_16x16x32_bf16 v[168:171], v[112:115], a[104:107], v[168:171]\n"
    "v_mfma_f32_16x16x32_bf16 v[200:203], v[112:115], a[112:115], v[200:203]\n"
    "ds_read_b64_tr_b16 v[108:109], v61 offset:57856\n"
    "v_mfma_f32_16x16x32_bf16 v[200:203], v[116:119], a[116:119], v[200:203]\n"
    "v_mfma_f32_16x16x32_bf16 v[232:235], v[116:119], a[124:127], v[232:235]\n"
    "ds_read_b64_tr_b16 v[110:111], v61 offset:58880\n"
    "v_mfma_f32_16x16x32_bf16 v[232:235], v[112:115], a[120:123], v[232:235]\n"
    "buffer_load_dwordx4 a[20:23], v55, s[40:43], s65 offen offset:1024\n"
    "s_waitcnt lgkmcnt(8)\n"
    "v_mfma_f32_16x16x32_bf16 v[236:239], v[120:123], a[120:123], v[236:239]\n"
    "ds_read_b64_tr_b16 v[112:113], v62 offset:49664\n"
    "v_mfma_f32_16x16x32_bf16 v[236:239], v[124:127], a[124:127], v[236:239]\n"
    "v_mfma_f32_16x16x32_bf16 v[204:207], v[124:127], a[116:119], v[204:207]\n"
    "ds_read_b64_tr_b16 v[114:115], v62 offset:50688\n"
    "v_mfma_f32_16x16x32_bf16 v[204:207], v[120:123], a[112:115], v[204:207]\n"
    "v_mfma_f32_16x16x32_bf16 v[172:175], v[120:123], a[104:107], v[172:175]\n"
    "ds_read_b64_tr_b16 v[116:117], v62 offset:57856\n"
    "v_mfma_f32_16x16x32_bf16 v[172:175], v[124:127], a[108:111], v[172:175]\n"
    "buffer_load_dwordx4 a[24:27], v55, s[40:43], s65 offen offset:2048\n"
    "v_mfma_f32_16x16x32_bf16 v[140:143], v[124:127], a[100:103], v[140:143]\n"
    "ds_read_b64_tr_b16 v[118:119], v62 offset:58880\n"
    "v_mfma_f32_16x16x32_bf16 v[140:143], v[120:123], a[96:99], v[140:143]\n"
    "s_waitcnt lgkmcnt(8)\n"
    "v_mfma_f32_16x16x32_bf16 v[144:147], v[96:99], a[96:99], v[144:147]\n"
    "ds_read_b64_tr_b16 v[120:121], v63 offset:49664\n"
    "v_mfma_f32_16x16x32_bf16 v[144:147], v[100:103], a[100:103], v[144:147]\n"
    "v_mfma_f32_16x16x32_bf16 v[176:179], v[100:103], a[108:111], v[176:179]\n"
    "ds_read_b64_tr_b16 v[122:123], v63 offset:50688\n"
    "v_mfma_f32_16x16x32_bf16 v[176:179], v[96:99], a[104:107], v[176:179]\n"
    "buffer_load_dwordx4 a[28:31], v55, s[40:43], s65 offen offset:3072\n"
    "v_mfma_f32_16x16x32_bf16 v[208:211], v[96:99], a[112:115], v[208:211]\n"
    "ds_read_b64_tr_b16 v[124:125], v63 offset:57856\n"
    "v_mfma_f32_16x16x32_bf16 v[208:211], v[100:103], a[116:119], v[208:211]\n"
    "s_mov_b32 s25, 2621760\n"
    "s_getpc_b64 s[28:29]\n"
    "s_cmp_ge_i32 s25, s24\n"
    "s_cbranch_scc1 90b\n"
    "v_mfma_f32_16x16x32_bf16 v[240:243], v[100:103], a[124:127], v[240:243]\n"
    "ds_read_b64_tr_b16 v[126:127], v63 offset:58880\n"
    "s_andn2_b64 s[76:77], s[72:73], s[80:81]\n"
    "s_cbranch_scc0 1181f\n"
    "1182:\n"
    "s_andn2_b64 s[14:15], s[16:17], s[80:81]\n"
    "s_cselect_b64 s[76:77], s[14:15], s[76:77]\n"
    "s_ff1_i32_b64 s69, s[76:77]\n"
    "v_mfma_f32_16x16x32_bf16 v[240:243], v[96:99], a[120:123], v[240:243]\n"
    "v_readlane_b32 s65, v53, s69\n"
    "s_bitset1_b64 s[80:81], s69\n"
    "s_waitcnt lgkmcnt(8)\n"
    "s_waitcnt vmcnt(49)\n"
    "s_barrier\n"
    "v_mfma_f32_16x16x32_bf16 v[244:247], v[104:107], a[120:123], v[244:247]\n"
    "ds_read_b64_tr_b16 v[96:97], v64 offset:0\n"
    "s_nop 0\n"
    "v_mfma_f32_16x16x32_bf16 v[244:247], v[108:111], a[124:127], v[244:247]\n"
    "s_nop 0\n"
    "s_lshl_b32 s64, s65, 5\n"
    "v_mfma_f32_16x16x32_bf16 v[212:215], v[108:111], a[116:119], v[212:215]\n"
    "ds_read_b64_tr_b16 v[98:99], v64 offset:1024\n"
    "s_add_u32 m0, s67, 16384\n"
    "v_mfma_f32_16x16x32_bf16 v[212:215], v[104:107], a[112:115], v[212:215]\n"
    "s_nop 0\n"
    "v_mfma_f32_16x16x32_bf16 v[180:183], v[104:107], a[104:107], v[180:183]\n"
    "ds_read_b64_tr_b16 v[100:101], v64 offset:8192\n"
    "v_mfma_f32_16x16x32_bf16 v[180:183], v[108:111], a[108:111], v[180:183]\n"
    "buffer_load_dwordx4 v56, s[36:39], s64 offen offset:0 lds\n"
    "v_mfma_f32_16x16x32_bf16 v[148:151], v[108:111], a[100:103], v[148:151]\n"
    "ds_read_b64_tr_b16 v[102:103], v64 offset:9216\n"
    "v_mfma_f32_16x16x32_bf16 v[148:151], v[104:107], a[96:99], v[148:151]\n"
    "s_waitcnt lgkmcnt(8)\n"
    "v_mfma_f32_16x16x32_bf16 v[152:155], v[112:115], a[96:99], v[152:155]\n"
    "ds_read_b64_tr_b16 v[104:105], v65 offset:0\n"
    "v_mfma_f32_16x16x32_bf16 v[152:155], v[116:119], a[100:103], v[152:155]\n"
    "buffer_load_dwordx4 v57, s[36:39], s64 offen offset:1024 lds\n"
    "v_mfma_f32_16x16x32_bf16 v[184:187], v[116:119], a[108:111], v[184:187]\n"
    "ds_read_b64_tr_b16 v[106:107], v65 offset:1024\n"
    "v_mfma_f32_16x16x32_bf16 v[184:187], v[112:115], a[104:107], v[184:187]\n"
    "v_mfma_f32_16x16x32_bf16 v[216:219], v[112:115], a[112:115], v[216:219]\n"
    "ds_read_b64_tr_b16 v[108:109], v65 offset:8192\n"
    "v_mfma_f32_16x16x32_bf16 v[216:219], v[116:119], a[116:119], v[216:219]\n"
    "v_mfma_f32_16x16x32_bf16 v[248:251], v[116:119], a[124:127], v[248:251]\n"
    "ds_read_b64_tr_b16 v[110:111], v65 offset:9216\n"
    "v_mfma_f32_16x16x32_bf16 v[248:251], v[112:115], a[120:123], v[248:251]\n"
    "buffer_load_dwordx4 v58, s[36:39], s64 offen offset:2048 lds\n"
    "s_waitcnt lgkmcnt(8)\n"
    "v_mfma_f32_16x16x32_bf16 v[252:255], v[120:123], a[120:123], v[252:255]\n"
    "ds_read_b64_tr_b16 v[112:113], v66 offset:0\n"
    "v_mfma_f32_16x16x32_bf16 v[252:255], v[124:127], a[124:127], v[252:255]\n"
    "v_mfma_f32_16x16x32_bf16 v[220:223], v[124:127], a[116:119], v[220:223]\n"
    "ds_read_b64_tr_b16 v[114:115], v66 offset:1024\n"
    "v_mfma_f32_16x16x32_bf16 v[220:223], v[120:123], a[112:115], v[220:223]\n"
    "v_mfma_f32_16x16x32_bf16 v[188:191], v[120:123], a[104:107], v[188:191]\n"
    "ds_read_b64_tr_b16 v[116:117], v66 offset:8192\n"
    "v_mfma_f32_16x16x32_bf16 v[188:191], v[124:127], a[108:111], v[188:191]\n"
    "buffer_load_dwordx4 v59, s[36:39], s64 offen offset:3072 lds\n"
    "v_mfma_f32_16x16x32_bf16 v[156:159], v[124:127], a[100:103], v[156:159]\n"
    "ds_read_b64_tr_b16 v[118:119], v66 offset:9216\n"
    "v_mfma_f32_16x16x32_bf16 v[156:159], v[120:123], a[96:99], v[156:159]\n"
    "s_waitcnt lgkmcnt(8)\n"
    "v_mfma_f32_16x16x32_bf16 v[128:131], v[96:99], a[128:131], v[128:131]\n"
    "ds_read_b64_tr_b16 v[120:121], v67 offset:0\n"
    "v_mfma_f32_16x16x32_bf16 v[128:131], v[100:103], a[132:135], v[128:131]\n"
    "v_mfma_f32_16x16x32_bf16 v[160:163], v[100:103], a[140:143], v[160:163]\n"
    "ds_read_b64_tr_b16 v[122:123], v67 offset:1024\n"
    "v_mfma_f32_16x16x32_bf16 v[160:163], v[96:99], a[136:139], v[160:163]\n"
    "buffer_load_dwordx4 a[32:35], v54, s[40:43], s65 offen offset:0\n"
    "v_mfma_f32_16x16x32_bf16 v[192:195], v[96:99], a[144:147], v[192:195]\n"
    "ds_read_b64_tr_b16 v[124:125], v67 offset:8192\n"
    "v_mfma_f32_16x16x32_bf16 v[192:195], v[100:103], a[148:151], v[192:195]\n"
    "v_mfma_f32_16x16x32_bf16 v[224:227], v[100:103], a[156:159], v[224:227]\n"
    "ds_read_b64_tr_b16 v[126:127], v67 offset:9216\n"
    "v_mfma_f32_16x16x32_bf16 v[224:227], v[96:99], a[152:155], v[224:227]\n"
    "s_waitcnt lgkmcnt(8)\n"
    "v_mfma_f32_16x16x32_bf16 v[228:231], v[104:107], a[152:155], v[228:231]\n"
    "ds_read_b64_tr_b16 v[96:97], v64 offset:512\n"
    "v_mfma_f32_16x16x32_bf16 v[228:231], v[108:111], a[156:159], v[228:231]\n"
    "s_mov_b32 s25, 2687301\n"
    "s_getpc_b64 s[28:29]\n"
    "s_cmp_ge_i32 s25, s24\n"
    "s_cbranch_scc1 90b\n"
    "v_mfma_f32_16x16x32_bf16 v[196:199], v[108:111], a[148:151], v[196:199]\n"
    "ds_read_b64_tr_b16 v[98:99], v64 offset:1536\n"
    "buffer_load_dwordx4 a[36:39], v54, s[40:43], s65 offen offset:1024\n"
    "v_mfma_f32_16x16x32_bf16 v[196:199], v[104:107], a[144:147], v[196:199]\n"
    "v_mfma_f32_16x16x32_bf16 v[164:167], v[104:107], a[136:139], v[164:167]\n"
    "ds_read_b64_tr_b16 v[100:101], v64 offset:8704\n"
    "v_mfma_f32_16x16x32_bf16 v[164:167], v[108:111], a[140:143], v[164:167]\n"
    "v_cmp_le_u32 vcc, s70, v47\n"
    "v_mfma_f32_16x16x32_bf16 v[132:135], v[108:111], a[132:135], v[132:135]\n"
    "ds_read_b64_tr_b16 v[102:103], v64 offset:9728\n"
    "v_mfma_f32_16x16x32_bf16 v[132:135], v[104:107], a[128:131], v[132:135]\n"
    "s_or_b64 s[82:83], s[82:83], vcc\n"
    "v_cmp_le_u32 vcc, s70, v48\n"
    "s_waitcnt lgkmcnt(8)\n"
    "v_mfma_f32_16x16x32_bf16 v[136:139], v[112:115], a[128:131], v[136:139]\n"
    "ds_read_b64_tr_b16 v[104:105], v65 offset:512\n"
    "buffer_load_dwordx4 a[40:43], v54, s[40:43], s65 offen offset:2048\n"
    "v_mfma_f32_16x16x32_bf16 v[136:139], v[116:119], a[132:135], v[136:139]\n"
    "s_or_b64 s[82:83], s[82:83], vcc\n"
    "v_mfma_f32_16x16x32_bf16 v[168:171], v[116:119], a[140:143], v[168:171]\n"
    "ds_read_b64_tr_b16 v[106:107], v65 offset:1536\n"
    "v_mfma_f32_16x16x32_bf16 v[168:171], v[112:115], a[136:139], v[168:171]\n"
    "v_mfma_f32_16x16x32_bf16 v[200:203], v[112:115], a[144:147], v[200:203]\n"
    "ds_read_b64_tr_b16 v[108:109], v65 offset:8704\n"
    "v_mfma_f32_16x16x32_bf16 v[200:203], v[116:119], a[148:151], v[200:203]\n"
    "v_mov_b32 v38, s82\n"
    "v_mov_b32 v39, s83\n"
    "ds_write_b64 v41, v[38:39] offset:0\n"
    "v_mfma_f32_16x16x32_bf16 v[232:235], v[116:119], a[156:159], v[232:235]\n"
    "ds_read_b64_tr_b16 v[110:111], v65 offset:9728\n"
    "buffer_load_dwordx4 a[44:47], v54, s[40:43], s65 offen offset:3072\n"
    "v_mfma_f32_16x16x32_bf16 v[232:235], v[112:115], a[152:155], v[232:235]\n"
    "s_ff0_i32_b64 s30, s[82:83]\n"
    "s_bfm_b64 vcc, s30, 0\n"
    "s_andn2_b64 vcc, vcc, s[80:81]\n"
    "s_bcnt1_i32_b64 s30, vcc\n"
    "s_cmp_lt_u32 s30, 64\n"
    "s_cselect_b32 s22, 0x10000, 0\n"
    "s_waitcnt lgkmcnt(9)\n"
    "v_mfma_f32_16x16x32_bf16 v[236:239], v[120:123], a[152:155], v[236:239]\n"
    "ds_read_b64_tr_b16 v[112:113], v66 offset:512\n"
    "v_mfma_f32_16x16x32_bf16 v[236:239], v[124:127], a[156:159], v[236:239]\n"
    "s_bcnt1_i32_b64 s30, s[82:83]\n"
    "s_cmp_ge_u32 s30, 56\n"
    "s_cselect_b32 s22, 0, s22\n"
    "v_mfma_f32_16x16x32_bf16 v[204:207], v[124:127], a[148:151], v[204:207]\n"
    "ds_read_b64_tr_b16 v[114:115], v66 offset:1536\n"
    "v_mfma_f32_16x16x32_bf16 v[204:207], v[120:123], a[144:147], v[204:207]\n"
    "buffer_load_dword v47, v49, s[20:23], 0 offen sc0 sc1\n"
    "v_mfma_f32_16x16x32_bf16 v[172:175], v[120:123], a[136:139], v[172:175]\n"
    "ds_read_b64_tr_b16 v[116:117], v66 offset:8704\n"
    "buffer_load_dwordx4 a[48:51], v55, s[40:43], s65 offen offset:0\n"
    "v_mfma_f32_16x16x32_bf16 v[172:175], v[124:127], a[140:143], v[172:175]\n"
    "v_mfma_f32_16x16x32_bf16 v[140:143], v[124:127], a[132:135], v[140:143]\n"
    "ds_read_b64_tr_b16 v[118:119], v66 offset:9728\n"
    "v_mfma_f32_16x16x32_bf16 v[140:143], v[120:123], a[128:131], v[140:143]\n"
    "s_waitcnt lgkmcnt(9)\n"
    "v_mfma_f32_16x16x32_bf16 v[144:147], v[96:99], a[128:131], v[144:147]\n"
    "ds_read_b64_tr_b16 v[120:121], v67 offset:512\n"
    "v_mfma_f32_16x16x32_bf16 v[144:147], v[100:103], a[132:135], v[144:147]\n"
    "buffer_load_dwordx4 a[52:55], v55, s[40:43], s65 offen offset:1024\n"
    "v_mfma_f32_16x16x32_bf16 v[176:179], v[100:103], a[140:143], v[176:179]\n"
    "ds_read_b64_tr_b16 v[122:123], v67 offset:1536\n"
    "v_mfma_f32_16x16x32_bf16 v[176:179], v[96:99], a[136:139], v[176:179]\n"
    "v_mfma_f32_16x16x32_bf16 v[208:211], v[96:99], a[144:147], v[208:211]\n"
    "ds_read_b64_tr_b16 v[124:125], v67 offset:8704\n"
    "v_mfma_f32_16x16x32_bf16 v[208:211], v[100:103], a[148:151], v[208:211]\n"
    "s_mov_b32 s25, 2752843\n"
    "s_getpc_b64 s[28:29]\n"
    "s_cmp_ge_i32 s25, s24\n"
    "s_cbranch_scc1 90b\n"
    "v_mfma_f32_16x16x32_bf16 v[240:243], v[100:103], a[156:159], v[240:243]\n"
    "ds_read_b64_tr_b16 v[126:127], v67 offset:9728\n"
    "v_mfma_f32_16x16x32_bf16 v[240:243], v[96:99], a[152:155], v[240:243]\n"
    "buffer_load_dwordx4 a[56:59], v55, s[40:43], s65 offen offset:2048\n"
    "s_waitcnt lgkmcnt(8)\n"
    "s_waitcnt vmcnt(49)\n"
    "s_barrier\n"
    "v_mfma_f32_16x16x32_bf16 v[244:247], v[104:107], a[152:155], v[244:247]\n"
    "ds_read_b64_tr_b16 v[96:97], v64 offset:16384\n"
    "v_mfma_f32_16x16x32_bf16 v[244:247], v[108:111], a[156:159], v[244:247]\n"
    "ds_read_b64 v[36:37], v40 offset:0\n"
    "v_mfma_f32_16x16x32_bf16 v[212:215], v[108:111], a[148:151], v[212:215]\n"
    "ds_read_b64_tr_b16 v[98:99], v64 offset:17408\n"
    "v_mfma_f32_16x16x32_bf16 v[212:215], v[104:107], a[144:147], v[212:215]\n"
    "v_mfma_f32_16x16x32_bf16 v[180:183], v[104:107], a[136:139], v[180:183]\n"
    "ds_read_b64_tr_b16 v[100:101], v64 offset:24576\n"
    "v_mfma_f32_16x16x32_bf16 v[180:183], v[108:111], a[140:143], v[180:183]\n"
    "buffer_load_dwordx4 a[60:63], v55, s[40:43], s65 offen offset:3072\n"
    "v_mfma_f32_16x16x32_bf16 v[148:151], v[108:111], a[132:135], v[148:151]\n"
    "ds_read_b64_tr_b16 v[102:103], v64 offset:25600\n"
    "v_mfma_f32_16x16x32_bf16 v[148:151], v[104:107], a[128:131], v[148:151]\n"
    "s_waitcnt lgkmcnt(9)\n"
    "v_mfma_f32_16x16x32_bf16 v[152:155], v[112:115], a[128:131], v[152:155]\n"
    "ds_read_b64_tr_b16 v[104:105], v65 offset:16384\n"
    "s_andn2_b64 s[76:77], s[72:73], s[80:81]\n"
    "s_cbranch_scc0 1191f\n"
    "1192:\n"
    "s_andn2_b64 s[14:15], s[16:17], s[80:81]\n"
    "s_cselect_b64 s[76:77], s[14:15], s[76:77]\n"
    "s_ff1_i32_b64 s69, s[76:77]\n"
    "v_mfma_f32_16x16x32_bf16 v[152:155], v[116:119], a[132:135], v[152:155]\n"
    "v_readlane_b32 s65, v53, s69\n"
    "s_bitset1_b64 s[80:81], s69\n"
    "v_mfma_f32_16x16x32_bf16 v[184:187], v[116:119], a[140:143], v[184:187]\n"
    "ds_read_b64_tr_b16 v[106:107], v65 offset:17408\n"
    "s_nop 0\n"
    "v_mfma_f32_16x16x32_bf16 v[184:187], v[112:115], a[136:139], v[184:187]\n"
    "s_nop 0\n"
    "s_lshl_b32 s64, s65, 5\n"
    "v_mfma_f32_16x16x32_bf16 v[216:219], v[112:115], a[144:147], v[216:219]\n"
    "ds_read_b64_tr_b16 v[108:109], v65 offset:24576\n"
    "s_add_u32 m0, s67, 32768\n"
    "v_mfma_f32_16x16x32_bf16 v[216:219], v[116:119], a[148:151], v[216:219]\n"
    "s_nop 0\n"
    "v_mfma_f32_16x16x32_bf16 v[248:251], v[116:119], a[156:159], v[248:251]\n"
    "ds_read_b64_tr_b16 v[110:111], v65 offset:25600\n"
    "v_mfma_f32_16x16x32_bf16 v[248:251], v[112:115], a[152:155], v[248:251]\n"
    "buffer_load_dwordx4 v56, s[36:39], s64 offen offset:0 lds\n"
    "s_waitcnt lgkmcnt(9)\n"
    "v_mfma_f32_16x16x32_bf16 v[252:255], v[120:123], a[152:155], v[252:255]\n"
    "ds_read_b64_tr_b16 v[112:113], v66 offset:16384\n"
    "v_mfma_f32_16x16x32_bf16 v[252:255], v[124:127], a[156:159], v[252:255]\n"
    "v_mfma_f32_16x16x32_bf16 v[220:223], v[124:127], a[148:151], v[220:223]\n"
    "ds_read_b64_tr_b16 v[114:115], v66 offset:17408\n"
    "v_mfma_f32_16x16x32_bf16 v[220:223], v[120:123], a[144:147], v[220:223]\n"
    "buffer_load_dwordx4 v57, s[36:39], s64 offen offset:1024 lds\n"
    "v_mfma_f32_16x16x32_bf16 v[188:191], v[120:123], a[136:139], v[188:191]\n"
    "ds_read_b64_tr_b16 v[116:117], v66 offset:24576\n"
    "v_mfma_f32_16x16x32_bf16 v[188:191], v[124:127], a[140:143], v[188:191]\n"
    "v_mfma_f32_16x16x32_bf16 v[156:159], v[124:127], a[132:135], v[156:159]\n"
    "ds_read_b64_tr_b16 v[118:119], v66 offset:25600\n"
    "v_mfma_f32_16x16x32_bf16 v[156:159], v[120:123], a[128:131], v[156:159]\n"
    "s_waitcnt lgkmcnt(8)\n"
    "v_mfma_f32_16x16x32_bf16 v[128:131], v[96:99], a[160:163], v[128:131]\n"
    "ds_read_b64_tr_b16 v[120:121], v67 offset:16384\n"
    "v_mfma_f32_16x16x32_bf16 v[128:131], v[100:103], a[164:167], v[128:131]\n"
    "v_readfirstlane_b32 vcc_lo, v36\n"
    "v_readfirstlane_b32 vcc_hi, v37\n"
    "v_mfma_f32_16x16x32_bf16 v[160:163], v[100:103], a[172:175], v[160:163]\n"
    "ds_read_b64_tr_b16 v[122:123], v67 offset:17408\n"
    "buffer_load_dwordx4 v58, s[36:39], s64 offen offset:2048 lds\n"
    "v_mfma_f32_16x16x32_bf16 v[160:163], v[96:99], a[168:171], v[160:163]\n"
    "s_or_b64 s[72:73], s[72:73], vcc\n"
    "v_mfma_f32_16x16x32_bf16 v[192:195], v[96:99], a[176:179], v[192:195]\n"
    "ds_read_b64_tr_b16 v[124:125], v67 offset:24576\n"
    "v_mfma_f32_16x16x32_bf16 v[192:195], v[100:103], a[180:183], v[192:195]\n"
    "v_mfma_f32_16x16x32_bf16 v[224:227], v[100:103], a[188:191], v[224:227]\n"
    "ds_read_b64_tr_b16 v[126:127], v67 offset:25600\n"
    "v_mfma_f32_16x16x32_bf16 v[224:227], v[96:99], a[184:187], v[224:227]\n"
    "buffer_load_dwordx4 v59, s[36:39], s64 offen offset:3072 lds\n"
    "s_waitcnt lgkmcnt(8)\n"
    "v_mfma_f32_16x16x32_bf16 v[228:231], v[104:107], a[184:187], v[228:231]\n"
    "ds_read_b64_tr_b16 v[96:97], v64 offset:16896\n"
    "v_mfma_f32_16x16x32_bf16 v[228:231], v[108:111], a[188:191], v[228:231]\n"
    "s_mov_b32 s25, 2818385\n"
    "s_getpc_b64 s[28:29]\n"
    "s_cmp_ge_i32 s25, s24\n"
    "s_cbranch_scc1 90b\n"
    "v_mfma_f32_16x16x32_bf16 v[196:199], v[108:111], a[180:183], v[196:199]\n"
    "ds_read_b64_tr_b16 v[98:99], v64 offset:17920\n"
    "v_mfma_f32_16x16x32_bf16 v[196:199], v[104:107], a[176:179], v[196:199]\n"
    "v_mfma_f32_16x16x32_bf16 v[164:167], v[104:107], a[168:171], v[164:167]\n"
    "ds_read_b64_tr_b16 v[100:101], v64 offset:25088\n"
    "v_mfma_f32_16x16x32_bf16 v[164:167], v[108:111], a[172:175], v[164:167]\n"
    "buffer_load_dwordx4 a[64:67], v54, s[40:43], s65 offen offset:0\n"
    "v_mfma_f32_16x16x32_bf16 v[132:135], v[108:111], a[164:167], v[132:135]\n"
    "ds_read_b64_tr_b16 v[102:103], v64 offset:26112\n"
    "v_mfma_f32_16x16x32_bf16 v[132:135], v[104:107], a[160:163], v[132:135]\n"
    "s_waitcnt lgkmcnt(8)\n"
    "v_mfma_f32_16x16x32_bf16 v[136:139], v[112:115], a[160:163], v[136:139]\n"
    "ds_read_b64_tr_b16 v[104:105], v65 offset:16896\n"
    "v_mfma_f32_16x16x32_bf16 v[136:139], v[116:119], a[164:167], v[136:139]\n"
    "v_mfma_f32_16x16x32_bf16 v[168:171], v[116:119], a[172:175], v[168:171]\n"
    "ds_read_b64_tr_b16 v[106:107], v65 offset:17920\n"
    "v_mfma_f32_16x16x32_bf16 v[168:171], v[112:115], a[168:171], v[168:171]\n"
    "buffer_load_dwordx4 a[68:71], v54, s[40:43], s65 offen offset:1024\n"
    "v_mfma_f32_16x16x32_bf16 v[200:203], v[112:115], a[176:179], v[200:203]\n"
    "ds_read_b64_tr_b16 v[108:109], v65 offset:25088\n"
    "v_mfma_f32_16x16x32_bf16 v[200:203], v[116:119], a[180:183], v[200:203]\n"
    "v_mfma_f32_16x16x32_bf16 v[232:235], v[116:119], a[188:191], v[232:235]\n"
    "ds_read_b64_tr_b16 v[110:111], v65 offset:26112\n"
    "v_mfma_f32_16x16x32_bf16 v[232:235], v[112:115], a[184:187], v[232:235]\n"
    "s_waitcnt lgkmcnt(8)\n"
    "v_mfma_f32_16x16x32_bf16 v[236:239], v[120:123], a[184:187], v[236:239]\n"
    "ds_read_b64_tr_b16 v[112:113], v66 offset:16896\n"
    "v_mfma_f32_16x16x32_bf16 v[236:239], v[124:127], a[188:191], v[236:239]\n"
    "buffer_load_dwordx4 a[72:75], v54, s[40:43], s65 offen offset:2048\n"
    "v_mfma_f32_16x16x32_bf16 v[204:207], v[124:127], a[180:183], v[204:207]\n"
    "ds_read_b64_tr_b16 v[114:115], v66 offset:17920\n"
    "v_mfma_f32_16x16x32_bf16 v[204:207], v[120:123], a[176:179], v[204:207]\n"
    "v_mfma_f32_16x16x32_bf16 v[172:175], v[120:123], a[168:171], v[172:175]\n"
    "ds_read_b64_tr_b16 v[116:117], v66 offset:25088\n"
    "v_mfma_f32_16x16x32_bf16 v[172:175], v[124:127], a[172:175], v[172:175]\n"
    "v_mfma_f32_16x16x32_bf16 v[140:143], v[124:127], a[164:167], v[140:143]\n"
    "ds_read_b64_tr_b16 v[118:119], v66 offset:26112\n"
    "v_mfma_f32_16x16x32_bf16 v[140:143], v[120:123], a[160:163], v[140:143]\n"
    "buffer_load_dwordx4 a[76:79], v54, s[40:43], s65 offen offset:3072\n"
    "s_waitcnt lgkmcnt(8)\n"
    "v_mfma_f32_16x16x32_bf16 v[144:147], v[96:99], a[160:163], v[144:147]\n"
    "ds_read_b64_tr_b16 v[120:121], v67 offset:16896\n"
    "v_mfma_f32_16x16x32_bf16 v[144:147], v[100:103], a[164:167], v[144:147]\n"
    "v_mfma_f32_16x16x32_bf16 v[176:179], v[100:103], a[172:175], v[176:179]\n"
    "ds_read_b64_tr_b16 v[122:123], v67 offset:17920\n"
    "v_mfma_f32_16x16x32_bf16 v[176:179], v[96:99], a[168:171], v[176:179]\n"
    "v_mfma_f32_16x16x32_bf16 v[208:211], v[96:99], a[176:179], v[208:211]\n"
    "ds_read_b64_tr_b16 v[124:125], v67 offset:25088\n"
    "v_mfma_f32_16x16x32_bf16 v[208:211], v[100:103], a[180:183], v[208:211]\n"
    "s_mov_b32 s25, 2883925\n"
    "s_getpc_b64 s[28:29]\n"
    "s_cmp_ge_i32 s25, s24\n"
    "s_cbranch_scc1 90b\n"
    "v_mfma_f32_16x16x32_bf16 v[240:243], v[100:103], a[188:191], v[240:243]\n"
    "ds_read_b64_tr_b16 v[126:127], v67 offset:26112\n"
    "buffer_load_dwordx4 a[80:83], v55, s[40:43], s65 offen offset:0\n"
    "v_mfma_f32_16x16x32_bf16 v[240:243], v[96:99], a[184:187], v[240:243]\n"
    "s_waitcnt lgkmcnt(8)\n"
    "s_waitcnt vmcnt(47)\n"
    "s_barrier\n"
    "v_mfma_f32_16x16x32_bf16 v[244:247], v[104:107], a[184:187], v[244:247]\n"
    "ds_read_b64_tr_b16 v[96:97], v64 offset:32768\n"
    "v_mfma_f32_16x16x32_bf16 v[244:247], v[108:111], a[188:191], v[244:247]\n"
    "v_mfma_f32_16x16x32_bf16 v[212:215], v[108:111], a[180:183], v[212:215]\n"
    "ds_read_b64_tr_b16 v[98:99], v64 offset:33792\n"
    "v_mfma_f32_16x16x32_bf16 v[212:215], v[104:107], a[176:179], v[212:215]\n"
    "buffer_load_dwordx4 a[84:87], v55, s[40:43], s65 offen offset:1024\n"
    "v_mfma_f32_16x16x32_bf16 v[180:183], v[104:107], a[168:171], v[180:183]\n"
    "ds_read_b64_tr_b16 v[100:101], v64 offset:40960\n"
    "v_mfma_f32_16x16x32_bf16 v[180:183], v[108:111], a[172:175], v[180:183]\n"
    "v_mfma_f32_16x16x32_bf16 v[148:151], v[108:111], a[164:167], v[148:151]\n"
    "ds_read_b64_tr_b16 v[102:103], v64 offset:41984\n"
    "v_mfma_f32_16x16x32_bf16 v[148:151], v[104:107], a[160:163], v[148:151]\n"
    "s_waitcnt lgkmcnt(8)\n"
    "v_mfma_f32_16x16x32_bf16 v[152:155], v[112:115], a[160:163], v[152:155]\n"
    "ds_read_b64_tr_b16 v[104:105], v65 offset:32768\n"
    "v_mfma_f32_16x16x32_bf16 v[152:155], v[116:119], a[164:167], v[152:155]\n"
    "buffer_load_dwordx4 a[88:91], v55, s[40:43], s65 offen offset:2048\n"
    "v_mfma_f32_16x16x32_bf16 v[184:187], v[116:119], a[172:175], v[184:187]\n"
    "ds_read_b64_tr_b16 v[106:107], v65 offset:33792\n"
    "v_mfma_f32_16x16x32_bf16 v[184:187], v[112:115], a[168:171], v[184:187]\n"
    "v_mfma_f32_16x16x32_bf16 v[216:219], v[112:115], a[176:179], v[216:219]\n"
    "ds_read_b64_tr_b16 v[108:109], v65 offset:40960\n"
    "v_mfma_f32_16x16x32_bf16 v[216:219], v[116:119], a[180:183], v[216:219]\n"
    "v_mfma_f32_16x16x32_bf16 v[248:251], v[116:119], a[188:191], v[248:251]\n"
    "ds_read_b64_tr_b16 v[110:111], v65 offset:41984\n"
    "v_mfma_f32_16x16x32_bf16 v[248:251], v[112:115], a[184:187], v[248:251]\n"
    "buffer_load_dwordx4 a[92:95], v55, s[40:43], s65 offen offset:3072\n"
    "s_waitcnt lgkmcnt(8)\n"
    "v_mfma_f32_16x16x32_bf16 v[252:255], v[120:123], a[184:187], v[252:255]\n"
    "ds_read_b64_tr_b16 v[112:113], v66 offset:32768\n"
    "v_mfma_f32_16x16x32_bf16 v[252:255], v[124:127], a[188:191], v[252:255]\n"
    "v_mfma_f32_16x16x32_bf16 v[220:223], v[124:127], a[180:183], v[220:223]\n"
    "ds_read_b64_tr_b16 v[114:115], v66 offset:33792\n"
    "s_andn2_b64 s[76:77], s[72:73], s[80:81]\n"
    "s_cbranch_scc0 1201f\n"
    "1202:\n"
    "s_andn2_b64 s[14:15], s[16:17], s[80:81]\n"
    "s_cselect_b64 s[76:77], s[14:15], s[76:77]\n"
    "s_ff1_i32_b64 s69, s[76:77]\n"
    "v_mfma_f32_16x16x32_bf16 v[220:223], v[120:123], a[176:179], v[220:223]\n"
    "v_readlane_b32 s65, v53, s69\n"
    "s_bitset1_b64 s[80:81], s69\n"
    "v_mfma_f32_16x16x32_bf16 v[188:191], v[120:123], a[168:171], v[188:191]\n"
    "ds_read_b64_tr_b16 v[116:117], v66 offset:40960\n"
    "s_nop 0\n"
    "v_mfma_f32_16x16x32_bf16 v[188:191], v[124:127], a[172:175], v[188:191]\n"
    "s_nop 0\n"
    "s_lshl_b32 s64, s65, 5\n"
    "v_mfma_f32_16x16x32_bf16 v[156:159], v[124:127], a[164:167], v[156:159]\n"
    "ds_read_b64_tr_b16 v[118:119], v66 offset:41984\n"
    "s_add_u32 m0, s67, 49152\n"
    "v_mfma_f32_16x16x32_bf16 v[156:159], v[120:123], a[160:163], v[156:159]\n"
    "s_nop 0\n"
    "s_waitcnt lgkmcnt(8)\n"
    "v_mfma_f32_16x16x32_bf16 v[128:131], v[96:99], a[192:195], v[128:131]\n"
    "ds_read_b64_tr_b16 v[120:121], v67 offset:32768\n"
    "v_mfma_f32_16x16x32_bf16 v[128:131], v[100:103], a[196:199], v[128:131]\n"
    "buffer_load_dwordx4 v56, s[36:39], s64 offen offset:0 lds\n"
    "v_mfma_f32_16x16x32_bf16 v[160:163], v[100:103], a[204:207], v[160:163]\n"
    "ds_read_b64_tr_b16 v[122:123], v67 offset:33792\n"
    "v_mfma_f32_16x16x32_bf16 v[160:163], v[96:99], a[200:203], v[160:163]\n"
    "v_mfma_f32_16x16x32_bf16 v[192:195], v[96:99], a[208:211], v[192:195]\n"
    "ds_read_b64_tr_b16 v[124:125], v67 offset:40960\n"
    "v_mfma_f32_16x16x32_bf16 v[192:195], v[100:103], a[212:215], v[192:195]\n"
    "buffer_load_dwordx4 v57, s[36:39], s64 offen offset:1024 lds\n"
    "v_mfma_f32_16x16x32_bf16 v[224:227], v[100:103], a[220:223], v[224:227]\n"
    "ds_read_b64_tr_b16 v[126:127], v67 offset:41984\n"
    "v_mfma_f32_16x16x32_bf16 v[224:227], v[96:99], a[216:219], v[224:227]\n"
    "s_waitcnt lgkmcnt(8)\n"
    "v_mfma_f32_16x16x32_bf16 v[228:231], v[104:107], a[216:219], v[228:231]\n"
    "ds_read_b64_tr_b16 v[96:97], v64 offset:33280\n"
    "v_mfma_f32_16x16x32_bf16 v[228:231], v[108:111], a[220:223], v[228:231]\n"
    "s_mov_b32 s25, 2949467\n"
    "s_getpc_b64 s[28:29]\n"
    "s_cmp_ge_i32 s25, s24\n"
    "s_cbranch_scc1 90b\n"
    "v_mfma_f32_16x16x32_bf16 v[196:199], v[108:111], a[212:215], v[196:199]\n"
    "ds_read_b64_tr_b16 v[98:99], v64 offset:34304\n"
    "v_mfma_f32_16x16x32_bf16 v[196:199], v[104:107], a[208:211], v[196:199]\n"
    "buffer_load_dwordx4 v58, s[36:39], s64 offen offset:2048 lds\n"
    "v_mfma_f32_16x16x32_bf16 v[164:167], v[104:107], a[200:203], v[164:167]\n"
    "ds_read_b64_tr_b16 v[100:101], v64 offset:41472\n"
    "v_mfma_f32_16x16x32_bf16 v[164:167], v[108:111], a[204:207], v[164:167]\n"
    "v_cmp_le_u32 vcc, s70, v47\n"
    "v_mfma_f32_16x16x32_bf16 v[132:135], v[108:111], a[196:199], v[132:135]\n"
    "ds_read_b64_tr_b16 v[102:103], v64 offset:42496\n"
    "v_mfma_f32_16x16x32_bf16 v[132:135], v[104:107], a[192:195], v[132:135]\n"
    "s_or_b64 s[82:83], s[82:83], vcc\n"
    "v_cmp_le_u32 vcc, s70, v48\n"
    "s_waitcnt lgkmcnt(8)\n"
    "v_mfma_f32_16x16x32_bf16 v[136:139], v[112:115], a[192:195], v[136:139]\n"
    "ds_read_b64_tr_b16 v[104:105], v65 offset:33280\n"
    "v_mfma_f32_16x16x32_bf16 v[136:139], v[116:119], a[196:199], v[136:139]\n"
    "s_or_b64 s[82:83], s[82:83], vcc\n"
    "v_mfma_f32_16x16x32_bf16 v[168:171], v[116:119], a[204:207], v[168:171]\n"
    "ds_read_b64_tr_b16 v[106:107], v65 offset:34304\n"
    "buffer_load_dwordx4 v59, s[36:39], s64 offen offset:3072 lds\n"
    "v_mfma_f32_16x16x32_bf16 v[168:171], v[112:115], a[200:203], v[168:171]\n"
    "v_mfma_f32_16x16x32_bf16 v[200:203], v[112:115], a[208:211], v[200:203]\n"
    "ds_read_b64_tr_b16 v[108:109], v65 offset:41472\n"
    "v_mfma_f32_16x16x32_bf16 v[200:203], v[116:119], a[212:215], v[200:203]\n"
    "v_mov_b32 v38, s82\n"
    "v_mov_b32 v39, s83\n"
    "ds_write_b64 v41, v[38:39] offset:0\n"
    "v_mfma_f32_16x16x32_bf16 v[232:235], v[116:119], a[220:223], v[232:235]\n"
    "ds_read_b64_tr_b16 v[110:111], v65 offset:42496\n"
    "v_mfma_f32_16x16x32_bf16 v[232:235], v[112:115], a[216:219], v[232:235]\n"
    "s_ff0_i32_b64 s30, s[82:83]\n"
    "s_bfm_b64 vcc, s30, 0\n"
    "s_andn2_b64 vcc, vcc, s[80:81]\n"
    "s_bcnt1_i32_b64 s30, vcc\n"
    "s_cmp_lt_u32 s30, 64\n"
    "s_cselect_b32 s22, 0x10000, 0\n"
    "s_waitcnt lgkmcnt(9)\n"
    "v_mfma_f32_16x16x32_bf16 v[236:239], v[120:123], a[216:219], v[236:239]\n"
    "ds_read_b64_tr_b16 v[112:113], v66 offset:33280\n"
    "buffer_load_dwordx4 a[96:99], v54, s[40:43], s65 offen offset:0\n"
    "v_mfma_f32_16x16x32_bf16 v[236:239], v[124:127], a[220:223], v[236:239]\n"
    "s_bcnt1_i32_b64 s30, s[82:83]\n"
    "s_cmp_ge_u32 s30, 56\n"
    "s_cselect_b32 s22, 0, s22\n"
    "v_mfma_f32_16x16x32_bf16 v[204:207], v[124:127], a[212:215], v[204:207]\n"
    "ds_read_b64_tr_b16 v[114:115], v66 offset:34304\n"
    "v_mfma_f32_16x16x32_bf16 v[204:207], v[120:123], a[208:211], v[204:207]\n"
    "buffer_load_dword v48, v49, s[20:23], 0 offen sc0 sc1\n"
    "v_mfma_f32_16x16x32_bf16 v[172:175], v[120:123], a[200:203], v[172:175]\n"
    "ds_read_b64_tr_b16 v[116:117], v66 offset:41472\n"
    "v_mfma_f32_16x16x32_bf16 v[172:175], v[124:127], a[204:207], v[172:175]\n"
    "buffer_load_dwordx4 a[100:103], v54, s[40:43], s65 offen offset:1024\n"
    "v_mfma_f32_16x16x32_bf16 v[140:143], v[124:127], a[196:199], v[140:143]\n"
    "ds_read_b64_tr_b16 v[118:119], v66 offset:42496\n"
    "v_mfma_f32_16x16x32_bf16 v[140:143], v[120:123], a[192:195], v[140:143]\n"
    "s_waitcnt lgkmcnt(9)\n"
    "v_mfma_f32_16x16x32_bf16 v[144:147], v[96:99], a[192:195], v[144:147]\n"
    "ds_read_b64_tr_b16 v[120:121], v67 offset:33280\n"
    "v_mfma_f32_16x16x32_bf16 v[144:147], v[100:103], a[196:199], v[144:147]\n"
    "v_mfma_f32_16x16x32_bf16 v[176:179], v[100:103], a[204:207], v[176:179]\n"
    "ds_read_b64_tr_b16 v[122:123], v67 offset:34304\n"
    "v_mfma_f32_16x16x32_bf16 v[176:179], v[96:99], a[200:203], v[176:179]\n"
    "buffer_load_dwordx4 a[104:107], v54, s[40:43], s65 offen offset:2048\n"
    "v_mfma_f32_16x16x32_bf16 v[208:211], v[96:99], a[208:211], v[208:211]\n"
    "ds_read_b64_tr_b16 v[124:125], v67 offset:41472\n"
    "v_mfma_f32_16x16x32_bf16 v[208:211], v[100:103], a[212:215], v[208:211]\n"
    "s_mov_b32 s25, 3015009\n"
    "s_getpc_b64 s[28:29]\n"
    "s_cmp_ge_i32 s25, s24\n"
    "s_cbranch_scc1 90b\n"
    "v_mfma_f32_16x16x32_bf16 v[240:243], v[100:103], a[220:223], v[240:243]\n"
    "ds_read_b64_tr_b16 v[126:127], v67 offset:42496\n"
    "v_mfma_f32_16x16x32_bf16 v[240:243], v[96:99], a[216:219], v[240:243]\n"
    "s_waitcnt lgkmcnt(8)\n"
    "s_waitcnt vmcnt(45)\n"
    "s_barrier\n"
    "v_mfma_f32_16x16x32_bf16 v[244:247], v[104:107], a[216:219], v[244:247]\n"
    "ds_read_b64_tr_b16 v[96:97], v64 offset:49152\n"
    "v_mfma_f32_16x16x32_bf16 v[244:247], v[108:111], a[220:223], v[244:247]\n"
    "ds_read_b64 v[36:37], v40 offset:0\n"
    "v_mfma_f32_16x16x32_bf16 v[212:215], v[108:111], a[212:215], v[212:215]\n"
    "ds_read_b64_tr_b16 v[98:99], v64 offset:50176\n"
    "buffer_load_dwordx4 a[108:111], v54, s[40:43], s65 offen offset:3072\n"
    "v_mfma_f32_16x16x32_bf16 v[212:215], v[104:107], a[208:211], v[212:215]\n"
    "v_mfma_f32_16x16x32_bf16 v[180:183], v[104:107], a[200:203], v[180:183]\n"
    "ds_read_b64_tr_b16 v[100:101], v64 offset:57344\n"
    "v_mfma_f32_16x16x32_bf16 v[180:183], v[108:111], a[204:207], v[180:183]\n"
    "v_mfma_f32_16x16x32_bf16 v[148:151], v[108:111], a[196:199], v[148:151]\n"
    "ds_read_b64_tr_b16 v[102:103], v64 offset:58368\n"
    "v_mfma_f32_16x16x32_bf16 v[148:151], v[104:107], a[192:195], v[148:151]\n"
    "buffer_load_dwordx4 a[112:115], v55, s[40:43], s65 offen offset:0\n"
    "s_waitcnt lgkmcnt(9)\n"
    "v_mfma_f32_16x16x32_bf16 v[152:155], v[112:115], a[192:195], v[152:155]\n"
    "ds_read_b64_tr_b16 v[104:105], v65 offset:49152\n"
    "v_mfma_f32_16x16x32_bf16 v[152:155], v[116:119], a[196:199], v[152:155]\n"
    "v_mfma_f32_16x16x32_bf16 v[184:187], v[116:119], a[204:207], v[184:187]\n"
    "ds_read_b64_tr_b16 v[106:107], v65 offset:50176\n"
    "v_mfma_f32_16x16x32_bf16 v[184:187], v[112:115], a[200:203], v[184:187]\n"
    "v_mfma_f32_16x16x32_bf16 v[216:219], v[112:115], a[208:211], v[216:219]\n"
    "ds_read_b64_tr_b16 v[108:109], v65 offset:57344\n"
    "v_mfma_f32_16x16x32_bf16 v[216:219], v[116:119], a[212:215], v[216:219]\n"
    "buffer_load_dwordx4 a[116:119], v55, s[40:43], s65 offen offset:1024\n"
    "v_mfma_f32_16x16x32_bf16 v[248:251], v[116:119], a[220:223], v[248:251]\n"
    "ds_read_b64_tr_b16 v[110:111], v65 offset:58368\n"
    "v_mfma_f32_16x16x32_bf16 v[248:251], v[112:115], a[216:219], v[248:251]\n"
    "s_waitcnt lgkmcnt(9)\n"
    "v_mfma_f32_16x16x32_bf16 v[252:255], v[120:123], a[216:219], v[252:255]\n"
    "ds_read_b64_tr_b16 v[112:113], v66 offset:49152\n"
    "v_mfma_f32_16x16x32_bf16 v[252:255], v[124:127], a[220:223], v[252:255]\n"
    "v_mfma_f32_16x16x32_bf16 v[220:223], v[124:127], a[212:215], v[220:223]\n"
    "ds_read_b64_tr_b16 v[114:115], v66 offset:50176\n"
    "v_mfma_f32_16x16x32_bf16 v[220:223], v[120:123], a[208:211], v[220:223]\n"
    "buffer_load_dwordx4 a[120:123], v55, s[40:43], s65 offen offset:2048\n"
    "v_mfma_f32_16x16x32_bf16 v[188:191], v[120:123], a[200:203], v[188:191]\n"
    "ds_read_b64_tr_b16 v[116:117], v66 offset:57344\n"
    "v_mfma_f32_16x16x32_bf16 v[188:191], v[124:127], a[204:207], v[188:191]\n"
    "v_mfma_f32_16x16x32_bf16 v[156:159], v[124:127], a[196:199], v[156:159]\n"
    "ds_read_b64_tr_b16 v[118:119], v66 offset:58368\n"
    "v_mfma_f32_16x16x32_bf16 v[156:159], v[120:123], a[192:195], v[156:159]\n"
    "s_waitcnt lgkmcnt(8)\n"
    "v_mfma_f32_16x16x32_bf16 v[128:131], v[96:99], a[224:227], v[128:131]\n"
    "ds_read_b64_tr_b16 v[120:121], v67 offset:49152\n"
    "v_mfma_f32_16x16x32_bf16 v[128:131], v[100:103], a[228:231], v[128:131]\n"
    "v_readfirstlane_b32 vcc_lo, v36\n"
    "v_readfirstlane_b32 vcc_hi, v37\n"
    "v_mfma_f32_16x16x32_bf16 v[160:163], v[100:103], a[236:239], v[160:163]\n"
    "ds_read_b64_tr_b16 v[122:123], v67 offset:50176\n"
    "buffer_load_dwordx4 a[124:127], v55, s[40:43], s65 offen offset:3072\n"
    "v_mfma_f32_16x16x32_bf16 v[160:163], v[96:99], a[232:235], v[160:163]\n"
    "s_or_b64 s[72:73], s[72:73], vcc\n"
    "v_mfma_f32_16x16x32_bf16 v[192:195], v[96:99], a[240:243], v[192:195]\n"
    "ds_read_b64_tr_b16 v[124:125], v67 offset:57344\n"
    "s_andn2_b64 s[76:77], s[72:73], s[80:81]\n"
    "s_cbranch_scc0 1211f\n"
    "1212:\n"
    "s_andn2_b64 s[14:15], s[18:19], s[80:81]\n"
    "s_cselect_b64 s[76:77], s[14:15], s[76:77]\n"
    "s_ff1_i32_b64 s69, s[76:77]\n"
    "v_mfma_f32_16x16x32_bf16 v[192:195], v[100:103], a[244:247], v[192:195]\n"
    "v_readlane_b32 s65, v53, s69\n"
    "s_bitset1_b64 s[80:81], s69\n"
    "v_mfma_f32_16x16x32_bf16 v[224:227], v[100:103], a[252:255], v[224:227]\n"
    "ds_read_b64_tr_b16 v[126:127], v67 offset:58368\n"
    "s_nop 0\n"
    "v_mfma_f32_16x16x32_bf16 v[224:227], v[96:99], a[248:251], v[224:227]\n"
    "s_nop 0\n"
    "s_lshl_b32 s64, s65, 5\n"
    "s_waitcnt lgkmcnt(8)\n"
    "v_mfma_f32_16x16x32_bf16 v[228:231], v[104:107], a[248:251], v[228:231]\n"
    "ds_read_b64_tr_b16 v[96:97], v64 offset:49664\n"
    "s_add_u32 m0, s67, 65536\n"
    "v_mfma_f32_16x16x32_bf16 v[228:231], v[108:111], a[252:255], v[228:231]\n"
    "s_mov_b32 s25, 3080550\n"
    "s_getpc_b64 s[28:29]\n"
    "s_cmp_ge_i32 s25, s24\n"
    "s_cbranch_scc1 90b\n"
    "v_mfma_f32_16x16x32_bf16 v[196:199], v[108:111], a[244:247], v[196:199]\n"
    "ds_read_b64_tr_b16 v[98:99], v64 offset:50688\n"
    "s_nop 0\n"
    "v_mfma_f32_16x16x32_bf16 v[196:199], v[104:107], a[240:243], v[196:199]\n"
    "buffer_load_dwordx4 v56, s[36:39], s64 offen offset:0 lds\n"
    "v_mfma_f32_16x16x32_bf16 v[164:167], v[104:107], a[232:235], v[164:167]\n"
    "ds_read_b64_tr_b16 v[100:101], v64 offset:57856\n"
    "v_mfma_f32_16x16x32_bf16 v[164:167], v[108:111], a[236:239], v[164:167]\n"
    "buffer_load_dwordx4 v57, s[36:39], s64 offen offset:1024 lds\n"
    "v_mfma_f32_16x16x32_bf16 v[132:135], v[108:111], a[228:231], v[132:135]\n"
    "ds_read_b64_tr_b16 v[102:103], v64 offset:58880\n"
    "v_mfma_f32_16x16x32_bf16 v[132:135], v[104:107], a[224:227], v[132:135]\n"
    "s_waitcnt lgkmcnt(8)\n"
    "v_mfma_f32_16x16x32_bf16 v[136:139], v[112:115], a[224:227], v[136:139]\n"
    "ds_read_b64_tr_b16 v[104:105], v65 offset:49664\n"
    "v_mfma_f32_16x16x32_bf16 v[136:139], v[116:119], a[228:231], v[136:139]\n"
    "buffer_load_dwordx4 v58, s[36:39], s64 offen offset:2048 lds\n"
    "v_mfma_f32_16x16x32_bf16 v[168:171], v[116:119], a[236:239], v[168:171]\n"
    "ds_read_b64_tr_b16 v[106:107], v65 offset:50688\n"
    "v_mfma_f32_16x16x32_bf16 v[168:171], v[112:115], a[232:235], v[168:171]\n"
    "v_mfma_f32_16x16x32_bf16 v[200:203], v[112:115], a[240:243], v[200:203]\n"
    "ds_read_b64_tr_b16 v[108:109], v65 offset:57856\n"
    "v_mfma_f32_16x16x32_bf16 v[200:203], v[116:119], a[244:247], v[200:203]\n"
    "buffer_load_dwordx4 v59, s[36:39], s64 offen offset:3072 lds\n"
    "v_mfma_f32_16x16x32_bf16 v[232:235], v[116:119], a[252:255], v[232:235]\n"
    "ds_read_b64_tr_b16 v[110:111], v65 offset:58880\n"
    "v_mfma_f32_16x16x32_bf16 v[232:235], v[112:115], a[248:251], v[232:235]\n"
    "s_waitcnt lgkmcnt(8)\n"
    "v_mfma_f32_16x16x32_bf16 v[236:239], v[120:123], a[248:251], v[236:239]\n"
    "ds_read_b64_tr_b16 v[112:113], v66 offset:49664\n"
    "v_mfma_f32_16x16x32_bf16 v[236:239], v[124:127], a[252:255], v[236:239]\n"
    "v_mfma_f32_16x16x32_bf16 v[204:207], v[124:127], a[244:247], v[204:207]\n"
    "ds_read_b64_tr_b16 v[114:115], v66 offset:50688\n"
    "v_mfma_f32_16x16x32_bf16 v[204:207], v[120:123], a[240:243], v[204:207]\n"
    "buffer_load_dwordx4 a[128:131], v54, s[40:43], s65 offen offset:0\n"
    "v_mfma_f32_16x16x32_bf16 v[172:175], v[120:123], a[232:235], v[172:175]\n"
    "ds_read_b64_tr_b16 v[116:117], v66 offset:57856\n"
    "v_mfma_f32_16x16x32_bf16 v[172:175], v[124:127], a[236:239], v[172:175]\n"
    "v_mfma_f32_16x16x32_bf16 v[140:143], v[124:127], a[228:231], v[140:143]\n"
    "ds_read_b64_tr_b16 v[118:119], v66 offset:58880\n"
    "v_mfma_f32_16x16x32_bf16 v[140:143], v[120:123], a[224:227], v[140:143]\n"
    "buffer_load_dwordx4 a[132:135], v54, s[40:43], s65 offen offset:1024\n"
    "s_waitcnt lgkmcnt(8)\n"
    "v_mfma_f32_16x16x32_bf16 v[144:147], v[96:99], a[224:227], v[144:147]\n"
    "ds_read_b64_tr_b16 v[120:121], v67 offset:49664\n"
    "v_mfma_f32_16x16x32_bf16 v[144:147], v[100:103], a[228:231], v[144:147]\n"
    "v_mfma_f32_16x16x32_bf16 v[176:179], v[100:103], a[236:239], v[176:179]\n"
    "ds_read_b64_tr_b16 v[122:123], v67 offset:50688\n"
    "v_mfma_f32_16x16x32_bf16 v[176:179], v[96:99], a[232:235], v[176:179]\n"
    "buffer_load_dwordx4 a[136:139], v54, s[40:43], s65 offen offset:2048\n"
    "v_mfma_f32_16x16x32_bf16 v[208:211], v[96:99], a[240:243], v[208:211]\n"
    "ds_read_b64_tr_b16 v[124:125], v67 offset:57856\n"
    "v_mfma_f32_16x16x32_bf16 v[208:211], v[100:103], a[244:247], v[208:211]\n"
    "s_mov_b32 s25, 3146093\n"
    "s_getpc_b64 s[28:29]\n"
    "s_cmp_ge_i32 s25, s24\n"
    "s_cbranch_scc1 90b\n"
    "v_mfma_f32_16x16x32_bf16 v[240:243], v[100:103], a[252:255], v[240:243]\n"
    "ds_read_b64_tr_b16 v[126:127], v67 offset:58880\n"
    "v_mfma_f32_16x16x32_bf16 v[240:243], v[96:99], a[248:251], v[240:243]\n"
    "buffer_load_dwordx4 a[140:143], v54, s[40:43], s65 offen offset:3072\n"
    "s_waitcnt lgkmcnt(8)\n"
    "s_waitcnt vmcnt(46)\n"
    "s_barrier\n"
    "v_mfma_f32_16x16x32_bf16 v[244:247], v[104:107], a[248:251], v[244:247]\n"
    "ds_read_b64_tr_b16 v[96:97], v60 offset:0\n"
    "v_mfma_f32_16x16x32_bf16 v[244:247], v[108:111], a[252:255], v[244:247]\n"
    "v_mfma_f32_16x16x32_bf16 v[212:215], v[108:111], a[244:247], v[212:215]\n"
    "ds_read_b64_tr_b16 v[98:99], v60 offset:1024\n"
    "v_mfma_f32_16x16x32_bf16 v[212:215], v[104:107], a[240:243], v[212:215]\n"
    "buffer_load_dwordx4 a[144:147], v55, s[40:43], s65 offen offset:0\n"
    "v_mfma_f32_16x16x32_bf16 v[180:183], v[104:107], a[232:235], v[180:183]\n"
    "ds_read_b64_tr_b16 v[100:101], v60 offset:8192\n"
    "v_mfma_f32_16x16x32_bf16 v[180:183], v[108:111], a[236:239], v[180:183]\n"
    "v_mfma_f32_16x16x32_bf16 v[148:151], v[108:111], a[228:231], v[148:151]\n"
    "ds_read_b64_tr_b16 v[102:103], v60 offset:9216\n"
    "v_mfma_f32_16x16x32_bf16 v[148:151], v[104:107], a[224:227], v[148:151]\n"
    "buffer_load_dwordx4 a[148:151], v55, s[40:43], s65 offen offset:1024\n"
    "s_waitcnt lgkmcnt(8)\n"
    "v_mfma_f32_16x16x32_bf16 v[152:155], v[112:115], a[224:227], v[152:155]\n"
    "ds_read_b64_tr_b16 v[104:105], v61 offset:0\n"
    "v_mfma_f32_16x16x32_bf16 v[152:155], v[116:119], a[228:231], v[152:155]\n"
    "v_mfma_f32_16x16x32_bf16 v[184:187], v[116:119], a[236:239], v[184:187]\n"
    "ds_read_b64_tr_b16 v[106:107], v61 offset:1024\n"
    "v_mfma_f32_16x16x32_bf16 v[184:187], v[112:115], a[232:235], v[184:187]\n"
    "buffer_load_dwordx4 a[152:155], v55, s[40:43], s65 offen offset:2048\n"
    "v_mfma_f32_16x16x32_bf16 v[216:219], v[112:115], a[240:243], v[216:219]\n"
    "ds_read_b64_tr_b16 v[108:109], v61 offset:8192\n"
    "v_mfma_f32_16x16x32_bf16 v[216:219], v[116:119], a[244:247], v[216:219]\n"
    "v_mfma_f32_16x16x32_bf16 v[248:251], v[116:119], a[252:255], v[248:251]\n"
    "ds_read_b64_tr_b16 v[110:111], v61 offset:9216\n"
    "v_mfma_f32_16x16x32_bf16 v[248:251], v[112:115], a[248:251], v[248:251]\n"
    "buffer_load_dwordx4 a[156:159], v55, s[40:43], s65 offen offset:3072\n"
    "s_waitcnt lgkmcnt(8)\n"
    "v_mfma_f32_16x16x32_bf16 v[252:255], v[120:123], a[248:251], v[252:255]\n"
    "ds_read_b64_tr_b16 v[112:113], v62 offset:0\n"
    "v_mfma_f32_16x16x32_bf16 v[252:255], v[124:127], a[252:255], v[252:255]\n"
    "v_mfma_f32_16x16x32_bf16 v[220:223], v[124:127], a[244:247], v[220:223]\n"
    "ds_read_b64_tr_b16 v[114:115], v62 offset:1024\n"
    "s_andn2_b64 s[76:77], s[72:73], s[80:81]\n"
    "s_cbranch_scc0 1221f\n"
    "1222:\n"
    "s_andn2_b64 s[14:15], s[18:19], s[80:81]\n"
    "s_cselect_b64 s[76:77], s[14:15], s[76:77]\n"
    "s_ff1_i32_b64 s69, s[76:77]\n"
    "v_mfma_f32_16x16x32_bf16 v[220:223], v[120:123], a[240:243], v[220:223]\n"
    "v_readlane_b32 s65, v53, s69\n"
    "s_bitset1_b64 s[80:81], s69\n"
    "v_mfma_f32_16x16x32_bf16 v[188:191], v[120:123], a[232:235], v[188:191]\n"
    "ds_read_b64_tr_b16 v[116:117], v62 offset:8192\n"
    "s_nop 0\n"
    "v_mfma_f32_16x16x32_bf16 v[188:191], v[124:127], a[236:239], v[188:191]\n"
    "s_nop 0\n"
    "s_lshl_b32 s64, s65, 5\n"
    "v_mfma_f32_16x16x32_bf16 v[156:159], v[124:127], a[228:231], v[156:159]\n"
    "ds_read_b64_tr_b16 v[118:119], v62 offset:9216\n"
    "s_add_u32 m0, s67, 81920\n"
    "v_mfma_f32_16x16x32_bf16 v[156:159], v[120:123], a[224:227], v[156:159]\n"
    "s_nop 0\n"
    "s_mov_b32 s68, 3\n"
    "1:\n"
    "s_waitcnt lgkmcnt(8)\n"
    "v_mfma_f32_16x16x32_bf16 v[128:131], v[96:99], a[0:3], v[128:131]\n"
    "ds_read_b64_tr_b16 v[120:121], v63 offset:0\n"
    "v_mfma_f32_16x16x32_bf16 v[128:131], v[100:103], a[4:7], v[128:131]\n"
    "buffer_load_dwordx4 v56, s[36:39], s64 offen offset:0 lds\n"
    "v_mfma_f32_16x16x32_bf16 v[160:163], v[100:103], a[12:15], v[160:163]\n"
    "ds_read_b64_tr_b16 v[122:123], v63 offset:1024\n"
    "v_mfma_f32_16x16x32_bf16 v[160:163], v[96:99], a[8:11], v[160:163]\n"
    "buffer_load_dwordx4 v57, s[36:39], s64 offen offset:1024 lds\n"
    "v_mfma_f32_16x16x32_bf16 v[192:195], v[96:99], a[16:19], v[192:195]\n"
    "ds_read_b64_tr_b16 v[124:125], v63 offset:8192\n"
    "v_mfma_f32_16x16x32_bf16 v[192:195], v[100:103], a[20:23], v[192:195]\n"
    "v_mfma_f32_16x16x32_bf16 v[224:227], v[100:103], a[28:31], v[224:227]\n"
    "ds_read_b64_tr_b16 v[126:127], v63 offset:9216\n"
    "v_mfma_f32_16x16x32_bf16 v[224:227], v[96:99], a[24:27], v[224:227]\n"
    "buffer_load_dwordx4 v58, s[36:39], s64 offen offset:2048 lds\n"
    "s_waitcnt lgkmcnt(8)\n"
    "v_mfma_f32_16x16x32_bf16 v[228:231], v[104:107], a[24:27], v[228:231]\n"
    "ds_read_b64_tr_b16 v[96:97], v60 offset:512\n"
    "v_mfma_f32_16x16x32_bf16 v[228:231], v[108:111], a[28:31], v[228:231]\n"
    "v_mfma_f32_16x16x32_bf16 v[196:199], v[108:111], a[20:23], v[196:199]\n"
    "ds_read_b64_tr_b16 v[98:99], v60 offset:1536\n"
    "v_mfma_f32_16x16x32_bf16 v[196:199], v[104:107], a[16:19], v[196:199]\n"
    "buffer_load_dwordx4 v59, s[36:39], s64 offen offset:3072 lds\n"
    "v_mfma_f32_16x16x32_bf16 v[164:167], v[104:107], a[8:11], v[164:167]\n"
    "ds_read_b64_tr_b16 v[100:101], v60 offset:8704\n"
    "v_mfma_f32_16x16x32_bf16 v[164:167], v[108:111], a[12:15], v[164:167]\n"
    "v_cmp_le_u32 vcc, s70, v47\n"
    "v_mfma_f32_16x16x32_bf16 v[132:135], v[108:111], a[4:7], v[132:135]\n"
    "ds_read_b64_tr_b16 v[102:103], v60 offset:9728\n"
    "v_mfma_f32_16x16x32_bf16 v[132:135], v[104:107], a[0:3], v[132:135]\n"
    "s_or_b64 s[82:83], s[82:83], vcc\n"
    "v_cmp_le_u32 vcc, s70, v48\n"
    "s_waitcnt lgkmcnt(8)\n"
    "v_mfma_f32_16x16x32_bf16 v[136:139], v[112:115], a[0:3], v[136:139]\n"
    "ds_read_b64_tr_b16 v[104:105], v61 offset:512\n"
    "v_mfma_f32_16x16x32_bf16 v[136:139], v[116:119], a[4:7], v[136:139]\n"
    "s_or_b64 s[82:83], s[82:83], vcc\n"
    "v_mfma_f32_16x16x32_bf16 v[168:171], v[116:119], a[12:15], v[168:171]\n"
    "ds_read_b64_tr_b16 v[106:107], v61 offset:1536\n"
    "buffer_load_dwordx4 a[160:163], v54, s[40:43], s65 offen offset:0\n"
    "v_mfma_f32_16x16x32_bf16 v[168:171], v[112:115], a[8:11], v[168:171]\n"
    "v_mfma_f32_16x16x32_bf16 v[200:203], v[112:115], a[16:19], v[200:203]\n"
    "ds_read_b64_tr_b16 v[108:109], v61 offset:8704\n"
    "v_mfma_f32_16x16x32_bf16 v[200:203], v[116:119], a[20:23], v[200:203]\n"
    "v_mov_b32 v38, s82\n"
    "v_mov_b32 v39, s83\n"
    "ds_write_b64 v41, v[38:39] offset:0\n"
    "v_mfma_f32_16x16x32_bf16 v[232:235], v[116:119], a[28:31], v[232:235]\n"
    "ds_read_b64_tr_b16 v[110:111], v61 offset:9728\n"
    "buffer_load_dwordx4 a[164:167], v54, s[40:43], s65 offen offset:1024\n"
    "v_mfma_f32_16x16x32_bf16 v[232:235], v[112:115], a[24:27], v[232:235]\n"
    "s_ff0_i32_b64 s30, s[82:83]\n"
    "s_bfm_b64 vcc, s30, 0\n"
    "s_andn2_b64 vcc, vcc, s[80:81]\n"
    "s_bcnt1_i32_b64 s30, vcc\n"
    "s_cmp_lt_u32 s30, 64\n"
    "s_cselect_b32 s22, 0x10000, 0\n"
    "s_waitcnt lgkmcnt(9)\n"
    "v_mfma_f32_16x16x32_bf16 v[236:239], v[120:123], a[24:27], v[236:239]\n"
    "ds_read_b64_tr_b16 v[112:113], v62 offset:512\n"
    "v_mfma_f32_16x16x32_bf16 v[236:239], v[124:127], a[28:31], v[236:239]\n"
    "s_bcnt1_i32_b64 s30, s[82:83]\n"
    "s_cmp_ge_u32 s30, 56\n"
    "s_cselect_b32 s22, 0, s22\n"
    "v_mfma_f32_16x16x32_bf16 v[204:207], v[124:127], a[20:23], v[204:207]\n"
    "ds_read_b64_tr_b16 v[114:115], v62 offset:1536\n"
    "buffer_load_dwordx4 a[168:171], v54, s[40:43], s65 offen offset:2048\n"
    "v_mfma_f32_16x16x32_bf16 v[204:207], v[120:123], a[16:19], v[204:207]\n"
    "buffer_load_dword v47, v49, s[20:23], 0 offen sc0 sc1\n"
    "v_mfma_f32_16x16x32_bf16 v[172:175], v[120:123], a[8:11], v[172:175]\n"
    "ds_read_b64_tr_b16 v[116:117], v62 offset:8704\n"
    "v_mfma_f32_16x16x32_bf16 v[172:175], v[124:127], a[12:15], v[172:175]\n"
    "buffer_load_dwordx4 a[172:175], v54, s[40:43], s65 offen offset:3072\n"
    "v_mfma_f32_16x16x32_bf16 v[140:143], v[124:127], a[4:7], v[140:143]\n"
    "ds_read_b64_tr_b16 v[118:119], v62 offset:9728\n"
    "v_mfma_f32_16x16x32_bf16 v[140:143], v[120:123], a[0:3], v[140:143]\n"
    "s_waitcnt lgkmcnt(9)\n"
    "v_mfma_f32_16x16x32_bf16 v[144:147], v[96:99], a[0:3], v[144:147]\n"
    "ds_read_b64_tr_b16 v[120:121], v63 offset:512\n"
    "v_mfma_f32_16x16x32_bf16 v[144:147], v[100:103], a[4:7], v[144:147]\n"
    "buffer_load_dwordx4 a[176:179], v55, s[40:43], s65 offen offset:0\n"
    "v_mfma_f32_16x16x32_bf16 v[176:179], v[100:103], a[12:15], v[176:179]\n"
    "ds_read_b64_tr_b16 v[122:123], v63 offset:1536\n"
    "v_mfma_f32_16x16x32_bf16 v[176:179], v[96:99], a[8:11], v[176:179]\n"
    "v_mfma_f32_16x16x32_bf16 v[208:211], v[96:99], a[16:19], v[208:211]\n"
    "ds_read_b64_tr_b16 v[124:125], v63 offset:8704\n"
    "v_mfma_f32_16x16x32_bf16 v[208:211], v[100:103], a[20:23], v[208:211]\n"
    "buffer_load_dwordx4 a[180:183], v55, s[40:43], s65 offen offset:1024\n"
    "v_mfma_f32_16x16x32_bf16 v[240:243], v[100:103], a[28:31], v[240:243]\n"
    "ds_read_b64_tr_b16 v[126:127], v63 offset:9728\n"
    "v_mfma_f32_16x16x32_bf16 v[240:243], v[96:99], a[24:27], v[240:243]\n"
    "s_waitcnt lgkmcnt(8)\n"
    "s_waitcnt vmcnt(48)\n"
    "s_barrier\n"
    "v_mfma_f32_16x16x32_bf16 v[244:247], v[104:107], a[24:27], v[244:247]\n"
    "ds_read_b64_tr_b16 v[96:97], v60 offset:16384\n"
    "v_mfma_f32_16x16x32_bf16 v[244:247], v[108:111], a[28:31], v[244:247]\n"
    "ds_read_b64 v[36:37], v40 offset:0\n"
    "v_mfma_f32_16x16x32_bf16 v[212:215], v[108:111], a[20:23], v[212:215]\n"
    "ds_read_b64_tr_b16 v[98:99], v60 offset:17408\n"
    "v_mfma_f32_16x16x32_bf16 v[212:215], v[104:107], a[16:19], v[212:215]\n"
    "buffer_load_dwordx4 a[184:187], v55, s[40:43], s65 offen offset:2048\n"
    "v_mfma_f32_16x16x32_bf16 v[180:183], v[104:107], a[8:11], v[180:183]\n"
    "ds_read_b64_tr_b16 v[100:101], v60 offset:24576\n"
    "v_mfma_f32_16x16x32_bf16 v[180:183], v[108:111], a[12:15], v[180:183]\n"
    "buffer_load_dwordx4 a[188:191], v55, s[40:43], s65 offen offset:3072\n"
    "v_mfma_f32_16x16x32_bf16 v[148:151], v[108:111], a[4:7], v[148:151]\n"
    "ds_read_b64_tr_b16 v[102:103], v60 offset:25600\n"
    "v_mfma_f32_16x16x32_bf16 v[148:151], v[104:107], a[0:3], v[148:151]\n"
    "s_waitcnt lgkmcnt(9)\n"
    "v_mfma_f32_16x16x32_bf16 v[152:155], v[112:115], a[0:3], v[152:155]\n"
    "ds_read_b64_tr_b16 v[104:105], v61 offset:16384\n"
    "s_andn2_b64 s[76:77], s[72:73], s[80:81]\n"
    "s_cbranch_scc0 1231f\n"
    "1232:\n"
    "s_andn2_b64 s[14:15], s[18:19], s[80:81]\n"
    "s_cselect_b64 s[76:77], s[14:15], s[76:77]\n"
    "s_ff1_i32_b64 s69, s[76:77]\n"
    "v_mfma_f32_16x16x32_bf16 v[152:155], v[116:119], a[4:7], v[152:155]\n"
    "v_readlane_b32 s65, v53, s69\n"
    "s_bitset1_b64 s[80:81], s69\n"
    "v_mfma_f32_16x16x32_bf16 v[184:187], v[116:119], a[12:15], v[184:187]\n"
    "ds_read_b64_tr_b16 v[106:107], v61 offset:17408\n"
    "s_nop 0\n"
    "v_mfma_f32_16x16x32_bf16 v[184:187], v[112:115], a[8:11], v[184:187]\n"
    "s_nop 0\n"
    "s_lshl_b32 s64, s65, 5\n"
    "v_mfma_f32_16x16x32_bf16 v[216:219], v[112:115], a[16:19], v[216:219]\n"
    "ds_read_b64_tr_b16 v[108:109], v61 offset:24576\n"
    "s_add_u32 m0, s67, 98304\n"
    "v_mfma_f32_16x16x32_bf16 v[216:219], v[116:119], a[20:23], v[216:219]\n"
    "s_nop 0\n"
    "v_mfma_f32_16x16x32_bf16 v[248:251], v[116:119], a[28:31], v[248:251]\n"
    "ds_read_b64_tr_b16 v[110:111], v61 offset:25600\n"
    "v_mfma_f32_16x16x32_bf16 v[248:251], v[112:115], a[24:27], v[248:251]\n"
    "buffer_load_dwordx4 v56, s[36:39], s64 offen offset:0 lds\n"
    "s_waitcnt lgkmcnt(9)\n"
    "v_mfma_f32_16x16x32_bf16 v[252:255], v[120:123], a[24:27], v[252:255]\n"
    "ds_read_b64_tr_b16 v[112:113], v62 offset:16384\n"
    "v_mfma_f32_16x16x32_bf16 v[252:255], v[124:127], a[28:31], v[252:255]\n"
    "buffer_load_dwordx4 v57, s[36:39], s64 offen offset:1024 lds\n"
    "v_mfma_f32_16x16x32_bf16 v[220:223], v[124:127], a[20:23], v[220:223]\n"
    "ds_read_b64_tr_b16 v[114:115], v62 offset:17408\n"
    "v_mfma_f32_16x16x32_bf16 v[220:223], v[120:123], a[16:19], v[220:223]\n"
    "v_mfma_f32_16x16x32_bf16 v[188:191], v[120:123], a[8:11], v[188:191]\n"
    "ds_read_b64_tr_b16 v[116:117], v62 offset:24576\n"
    "v_mfma_f32_16x16x32_bf16 v[188:191], v[124:127], a[12:15], v[188:191]\n"
    "buffer_load_dwordx4 v58, s[36:39], s64 offen offset:2048 lds\n"
    "v_mfma_f32_16x16x32_bf16 v[156:159], v[124:127], a[4:7], v[156:159]\n"
    "ds_read_b64_tr_b16 v[118:119], v62 offset:25600\n"
    "v_mfma_f32_16x16x32_bf16 v[156:159], v[120:123], a[0:3], v[156:159]\n"
    "s_waitcnt lgkmcnt(8)\n"
    "v_mfma_f32_16x16x32_bf16 v[128:131], v[96:99], a[32:35], v[128:131]\n"
    "ds_read_b64_tr_b16 v[120:121], v63 offset:16384\n"
    "v_mfma_f32_16x16x32_bf16 v[128:131], v[100:103], a[36:39], v[128:131]\n"
    "v_readfirstlane_b32 vcc_lo, v36\n"
    "v_readfirstlane_b32 vcc_hi, v37\n"
    "v_mfma_f32_16x16x32_bf16 v[160:163], v[100:103], a[44:47], v[160:163]\n"
    "ds_read_b64_tr_b16 v[122:123], v63 offset:17408\n"
    "v_mfma_f32_16x16x32_bf16 v[160:163], v[96:99], a[40:43], v[160:163]\n"
    "s_or_b64 s[72:73], s[72:73], vcc\n"
    "v_mfma_f32_16x16x32_bf16 v[192:195], v[96:99], a[48:51], v[192:195]\n"
    "ds_read_b64_tr_b16 v[124:125], v63 offset:24576\n"
    "buffer_load_dwordx4 v59, s[36:39], s64 offen offset:3072 lds\n"
    "v_mfma_f32_16x16x32_bf16 v[192:195], v[100:103], a[52:55], v[192:195]\n"
    "v_mfma_f32_16x16x32_bf16 v[224:227], v[100:103], a[60:63], v[224:227]\n"
    "ds_read_b64_tr_b16 v[126:127], v63 offset:25600\n"
    "v_mfma_f32_16x16x32_bf16 v[224:227], v[96:99], a[56:59], v[224:227]\n"
    "buffer_load_dwordx4 a[192:195], v54, s[40:43], s65 offen offset:0\n"
    "s_waitcnt lgkmcnt(8)\n"
    "v_mfma_f32_16x16x32_bf16 v[228:231], v[104:107], a[56:59], v[228:231]\n"
    "ds_read_b64_tr_b16 v[96:97], v60 offset:16896\n"
    "v_mfma_f32_16x16x32_bf16 v[228:231], v[108:111], a[60:63], v[228:231]\n"
    "v_mfma_f32_16x16x32_bf16 v[196:199], v[108:111], a[52:55], v[196:199]\n"
    "ds_read_b64_tr_b16 v[98:99], v60 offset:17920\n"
    "v_mfma_f32_16x16x32_bf16 v[196:199], v[104:107], a[48:51], v[196:199]\n"
    "buffer_load_dwordx4 a[196:199], v54, s[40:43], s65 offen offset:1024\n"
    "v_mfma_f32_16x16x32_bf16 v[164:167], v[104:107], a[40:43], v[164:167]\n"
    "ds_read_b64_tr_b16 v[100:101], v60 offset:25088\n"
    "v_mfma_f32_16x16x32_bf16 v[164:167], v[108:111], a[44:47], v[164:167]\n"
    "v_mfma_f32_16x16x32_bf16 v[132:135], v[108:111], a[36:39], v[132:135]\n"
    "ds_read_b64_tr_b16 v[102:103], v60 offset:26112\n"
    "v_mfma_f32_16x16x32_bf16 v[132:135], v[104:107], a[32:35], v[132:135]\n"
    "buffer_load_dwordx4 a[200:203], v54, s[40:43], s65 offen offset:2048\n"
    "s_waitcnt lgkmcnt(8)\n"
    "v_mfma_f32_16x16x32_bf16 v[136:139], v[112:115], a[32:35], v[136:139]\n"
    "ds_read_b64_tr_b16 v[104:105], v61 offset:16896\n"
    "v_mfma_f32_16x16x32_bf16 v[136:139], v[116:119], a[36:39], v[136:139]\n"
    "v_mfma_f32_16x16x32_bf16 v[168:171], v[116:119], a[44:47], v[168:171]\n"
    "ds_read_b64_tr_b16 v[106:107], v61 offset:17920\n"
    "v_mfma_f32_16x16x32_bf16 v[168:171], v[112:115], a[40:43], v[168:171]\n"
    "buffer_load_dwordx4 a[204:207], v54, s[40:43], s65 offen offset:3072\n"
    "v_mfma_f32_16x16x32_bf16 v[200:203], v[112:115], a[48:51], v[200:203]\n"
    "ds_read_b64_tr_b16 v[108:109], v61 offset:25088\n"
    "v_mfma_f32_16x16x32_bf16 v[200:203], v[116:119], a[52:55], v[200:203]\n"
    "v_mfma_f32_16x16x32_bf16 v[232:235], v[116:119], a[60:63], v[232:235]\n"
    "ds_read_b64_tr_b16 v[110:111], v61 offset:26112\n"
    "v_mfma_f32_16x16x32_bf16 v[232:235], v[112:115], a[56:59], v[232:235]\n"
    "buffer_load_dwordx4 a[208:211], v55, s[40:43], s65 offen offset:0\n"
    "s_waitcnt lgkmcnt(8)\n"
    "v_mfma_f32_16x16x32_bf16 v[236:239], v[120:123], a[56:59], v[236:239]\n"
    "ds_read_b64_tr_b16 v[112:113], v62 offset:16896\n"
    "v_mfma_f32_16x16x32_bf16 v[236:239], v[124:127], a[60:63], v[236:239]\n"
    "v_mfma_f32_16x16x32_bf16 v[204:207], v[124:127], a[52:55], v[204:207]\n"
    "ds_read_b64_tr_b16 v[114:115], v62 offset:17920\n"
    "v_mfma_f32_16x16x32_bf16 v[204:207], v[120:123], a[48:51], v[204:207]\n"
    "buffer_load_dwordx4 a[212:215], v55, s[40:43], s65 offen offset:1024\n"
    "v_mfma_f32_16x16x32_bf16 v[172:175], v[120:123], a[40:43], v[172:175]\n"
    "ds_read_b64_tr_b16 v[116:117], v62 offset:25088\n"
    "v_mfma_f32_16x16x32_bf16 v[172:175], v[124:127], a[44:47], v[172:175]\n"
    "v_mfma_f32_16x16x32_bf16 v[140:143], v[124:127], a[36:39], v[140:143]\n"
    "ds_read_b64_tr_b16 v[118:119], v62 offset:26112\n"
    "v_mfma_f32_16x16x32_bf16 v[140:143], v[120:123], a[32:35], v[140:143]\n"
    "buffer_load_dwordx4 a[216:219], v55, s[40:43], s65 offen offset:2048\n"
    "s_waitcnt lgkmcnt(8)\n"
    "v_mfma_f32_16x16x32_bf16 v[144:147], v[96:99], a[32:35], v[144:147]\n"
    "ds_read_b64_tr_b16 v[120:121], v63 offset:16896\n"
    "v_mfma_f32_16x16x32_bf16 v[144:147], v[100:103], a[36:39], v[144:147]\n"
    "v_mfma_f32_16x16x32_bf16 v[176:179], v[100:103], a[44:47], v[176:179]\n"
    "ds_read_b64_tr_b16 v[122:123], v63 offset:17920\n"
    "v_mfma_f32_16x16x32_bf16 v[176:179], v[96:99], a[40:43], v[176:179]\n"
    "buffer_load_dwordx4 a[220:223], v55, s[40:43], s65 offen offset:3072\n"
    "v_mfma_f32_16x16x32_bf16 v[208:211], v[96:99], a[48:51], v[208:211]\n"
    "ds_read_b64_tr_b16 v[124:125], v63 offset:25088\n"
    "v_mfma_f32_16x16x32_bf16 v[208:211], v[100:103], a[52:55], v[208:211]\n"
    "v_mfma_f32_16x16x32_bf16 v[240:243], v[100:103], a[60:63], v[240:243]\n"
    "ds_read_b64_tr_b16 v[126:127], v63 offset:26112\n"
    "s_andn2_b64 s[76:77], s[72:73], s[80:81]\n"
    "s_cbranch_scc0 1241f\n"
    "1242:\n"
    "s_andn2_b64 s[14:15], s[18:19], s[80:81]\n"
    "s_cselect_b64 s[76:77], s[14:15], s[76:77]\n"
    "s_ff1_i32_b64 s69, s[76:77]\n"
    "v_mfma_f32_16x16x32_bf16 v[240:243], v[96:99], a[56:59], v[240:243]\n"
    "v_readlane_b32 s65, v53, s69\n"
    "s_bitset1_b64 s[80:81], s69\n"
    "s_waitcnt lgkmcnt(8)\n"
    "s_waitcnt vmcnt(50)\n"
    "s_barrier\n"
    "v_mfma_f32_16x16x32_bf16 v[244:247], v[104:107], a[56:59], v[244:247]\n"
    "ds_read_b64_tr_b16 v[96:97], v60 offset:32768\n"
    "s_nop 0\n"
    "v_mfma_f32_16x16x32_bf16 v[244:247], v[108:111], a[60:63], v[244:247]\n"
    "s_nop 0\n"
    "s_lshl_b32 s64, s65, 5\n"
    "v_mfma_f32_16x16x32_bf16 v[212:215], v[108:111], a[52:55], v[212:215]\n"
    "ds_read_b64_tr_b16 v[98:99], v60 offset:33792\n"
    "s_add_u32 m0, s67, 114688\n"
    "v_mfma_f32_16x16x32_bf16 v[212:215], v[104:107], a[48:51], v[212:215]\n"
    "s_nop 0\n"
    "v_mfma_f32_16x16x32_bf16 v[180:183], v[104:107], a[40:43], v[180:183]\n"
    "ds_read_b64_tr_b16 v[100:101], v60 offset:40960\n"
    "v_mfma_f32_16x16x32_bf16 v[180:183], v[108:111], a[44:47], v[180:183]\n"
    "buffer_load_dwordx4 v56, s[36:39], s64 offen offset:0 lds\n"
    "v_mfma_f32_16x16x32_bf16 v[148:151], v[108:111], a[36:39], v[148:151]\n"
    "ds_read_b64_tr_b16 v[102:103], v60 offset:41984\n"
    "v_mfma_f32_16x16x32_bf16 v[148:151], v[104:107], a[32:35], v[148:151]\n"
    "buffer_load_dwordx4 v57, s[36:39], s64 offen offset:1024 lds\n"
    "s_waitcnt lgkmcnt(8)\n"
    "v_mfma_f32_16x16x32_bf16 v[152:155], v[112:115], a[32:35], v[152:155]\n"
    "ds_read_b64_tr_b16 v[104:105], v61 offset:32768\n"
    "v_mfma_f32_16x16x32_bf16 v[152:155], v[116:119], a[36:39], v[152:155]\n"
    "v_mfma_f32_16x16x32_bf16 v[184:187], v[116:119], a[44:47], v[184:187]\n"
    "ds_read_b64_tr_b16 v[106:107], v61 offset:33792\n"
    "v_mfma_f32_16x16x32_bf16 v[184:187], v[112:115], a[40:43], v[184:187]\n"
    "buffer_load_dwordx4 v58, s[36:39], s64 offen offset:2048 lds\n"
    "v_mfma_f32_16x16x32_bf16 v[216:219], v[112:115], a[48:51], v[216:219]\n"
    "ds_read_b64_tr_b16 v[108:109], v61 offset:40960\n"
    "s_andn2_b64 s[16:17], s[72:73], s[80:81]\n"
    "v_mfma_f32_16x16x32_bf16 v[216:219], v[116:119], a[52:55], v[216:219]\n"
    "s_mov_b64 s[14:15], s[16:17]\n"
    "s_ff1_i32_b64 s69, s[14:15]\n"
    "v_mfma_f32_16x16x32_bf16 v[248:251], v[116:119], a[60:63], v[248:251]\n"
    "ds_read_b64_tr_b16 v[110:111], v61 offset:41984\n"
    "v_mfma_f32_16x16x32_bf16 v[248:251], v[112:115], a[56:59], v[248:251]\n"
    "buffer_load_dwordx4 v59, s[36:39], s64 offen offset:3072 lds\n"
    "s_waitcnt lgkmcnt(8)\n"
    "v_mfma_f32_16x16x32_bf16 v[252:255], v[120:123], a[56:59], v[252:255]\n"
    "ds_read_b64_tr_b16 v[112:113], v62 offset:32768\n"
    "s_bitset0_b64 s[14:15], s69\n"
    "v_mfma_f32_16x16x32_bf16 v[252:255], v[124:127], a[60:63], v[252:255]\n"
    "s_ff1_i32_b64 s69, s[14:15]\n"
    "s_bitset0_b64 s[14:15], s69\n"
    "v_mfma_f32_16x16x32_bf16 v[220:223], v[124:127], a[52:55], v[220:223]\n"
    "ds_read_b64_tr_b16 v[114:115], v62 offset:33792\n"
    "v_mfma_f32_16x16x32_bf16 v[220:223], v[120:123], a[48:51], v[220:223]\n"
    "v_mfma_f32_16x16x32_bf16 v[188:191], v[120:123], a[40:43], v[188:191]\n"
    "ds_read_b64_tr_b16 v[116:117], v62 offset:40960\n"
    "v_mfma_f32_16x16x32_bf16 v[188:191], v[124:127], a[44:47], v[188:191]\n"
    "buffer_load_dwordx4 a[224:227], v54, s[40:43], s65 offen offset:0\n"
    "v_mfma_f32_16x16x32_bf16 v[156:159], v[124:127], a[36:39], v[156:159]\n"
    "ds_read_b64_tr_b16 v[118:119], v62 offset:41984\n"
    "s_ff1_i32_b64 s69, s[14:15]\n"
    "v_mfma_f32_16x16x32_bf16 v[156:159], v[120:123], a[32:35], v[156:159]\n"
    "s_bitset0_b64 s[14:15], s69\n"
    "s_ff1_i32_b64 s69, s[14:15]\n"
    "s_waitcnt lgkmcnt(8)\n"
    "v_mfma_f32_16x16x32_bf16 v[128:131], v[96:99], a[64:67], v[128:131]\n"
    "ds_read_b64_tr_b16 v[120:121], v63 offset:32768\n"
    "v_mfma_f32_16x16x32_bf16 v[128:131], v[100:103], a[68:71], v[128:131]\n"
    "buffer_load_dwordx4 a[228:231], v54, s[40:43], s65 offen offset:1024\n"
    "v_mfma_f32_16x16x32_bf16 v[160:163], v[100:103], a[76:79], v[160:163]\n"
    "ds_read_b64_tr_b16 v[122:123], v63 offset:33792\n"
    "s_bitset0_b64 s[14:15], s69\n"
    "v_mfma_f32_16x16x32_bf16 v[160:163], v[96:99], a[72:75], v[160:163]\n"
    "s_xor_b64 s[16:17], s[16:17], s[14:15]\n"
    "s_mov_b64 s[18:19], s[14:15]\n"
    "v_mfma_f32_16x16x32_bf16 v[192:195], v[96:99], a[80:83], v[192:195]\n"
    "ds_read_b64_tr_b16 v[124:125], v63 offset:40960\n"
    "v_mfma_f32_16x16x32_bf16 v[192:195], v[100:103], a[84:87], v[192:195]\n"
    "buffer_load_dwordx4 a[232:235], v54, s[40:43], s65 offen offset:2048\n"
    "v_mfma_f32_16x16x32_bf16 v[224:227], v[100:103], a[92:95], v[224:227]\n"
    "ds_read_b64_tr_b16 v[126:127], v63 offset:41984\n"
    "s_ff1_i32_b64 s69, s[14:15]\n"
    "v_mfma_f32_16x16x32_bf16 v[224:227], v[96:99], a[88:91], v[224:227]\n"
    "s_bitset0_b64 s[14:15], s69\n"
    "s_ff1_i32_b64 s69, s[14:15]\n"
    "s_waitcnt lgkmcnt(8)\n"
    "v_mfma_f32_16x16x32_bf16 v[228:231], v[104:107], a[88:91], v[228:231]\n"
    "ds_read_b64_tr_b16 v[96:97], v60 offset:33280\n"
    "v_mfma_f32_16x16x32_bf16 v[228:231], v[108:111], a[92:95], v[228:231]\n"
    "buffer_load_dwordx4 a[236:239], v54, s[40:43], s65 offen offset:3072\n"
    "v_mfma_f32_16x16x32_bf16 v[196:199], v[108:111], a[84:87], v[196:199]\n"
    "ds_read_b64_tr_b16 v[98:99], v60 offset:34304\n"
    "s_bitset0_b64 s[14:15], s69\n"
    "v_mfma_f32_16x16x32_bf16 v[196:199], v[104:107], a[80:83], v[196:199]\n"
    "s_ff1_i32_b64 s69, s[14:15]\n"
    "s_bitset0_b64 s[14:15], s69\n"
    "v_mfma_f32_16x16x32_bf16 v[164:167], v[104:107], a[72:75], v[164:167]\n"
    "ds_read_b64_tr_b16 v[100:101], v60 offset:41472\n"
    "v_mfma_f32_16x16x32_bf16 v[164:167], v[108:111], a[76:79], v[164:167]\n"
    "v_cmp_le_u32 vcc, s70, v47\n"
    "v_mfma_f32_16x16x32_bf16 v[132:135], v[108:111], a[68:71], v[132:135]\n"
    "ds_read_b64_tr_b16 v[102:103], v60 offset:42496\n"
    "v_mfma_f32_16x16x32_bf16 v[132:135], v[104:107], a[64:67], v[132:135]\n"
    "s_or_b64 s[82:83], s[82:83], vcc\n"
    "v_cmp_le_u32 vcc, s70, v48\n"
    "s_waitcnt lgkmcnt(8)\n"
    "v_mfma_f32_16x16x32_bf16 v[136:139], v[112:115], a[64:67], v[136:139]\n"
    "ds_read_b64_tr_b16 v[104:105], v61 offset:33280\n"
    "buffer_load_dwordx4 a[240:243], v55, s[40:43], s65 offen offset:0\n"
    "v_mfma_f32_16x16x32_bf16 v[136:139], v[116:119], a[68:71], v[136:139]\n"
    "s_or_b64 s[82:83], s[82:83], vcc\n"
    "v_mfma_f32_16x16x32_bf16 v[168:171], v[116:119], a[76:79], v[168:171]\n"
    "ds_read_b64_tr_b16 v[106:107], v61 offset:34304\n"
    "s_ff1_i32_b64 s69, s[14:15]\n"
    "v_mfma_f32_16x16x32_bf16 v[168:171], v[112:115], a[72:75], v[168:171]\n"
    "s_bitset0_b64 s[14:15], s69\n"
    "s_xor_b64 s[18:19], s[18:19], s[14:15]\n"
    "v_mfma_f32_16x16x32_bf16 v[200:203], v[112:115], a[80:83], v[200:203]\n"
    "ds_read_b64_tr_b16 v[108:109], v61 offset:41472\n"
    "buffer_load_dwordx4 a[244:247], v55, s[40:43], s65 offen offset:1024\n"
    "v_mfma_f32_16x16x32_bf16 v[200:203], v[116:119], a[84:87], v[200:203]\n"
    "v_mov_b32 v38, s82\n"
    "v_mov_b32 v39, s83\n"
    "ds_write_b64 v41, v[38:39] offset:0\n"
    "v_mfma_f32_16x16x32_bf16 v[232:235], v[116:119], a[92:95], v[232:235]\n"
    "ds_read_b64_tr_b16 v[110:111], v61 offset:42496\n"
    "s_bcnt1_i32_b64 s69, s[18:19]\n"
    "v_mfma_f32_16x16x32_bf16 v[232:235], v[112:115], a[88:91], v[232:235]\n"
    "s_ff0_i32_b64 s30, s[82:83]\n"
    "s_bfm_b64 vcc, s30, 0\n"
    "s_andn2_b64 vcc, vcc, s[80:81]\n"
    "s_bcnt1_i32_b64 s30, vcc\n"
    "s_cmp_lt_u32 s30, 64\n"
    "s_cselect_b32 s22, 0x10000, 0\n"
    "s_waitcnt lgkmcnt(9)\n"
    "v_mfma_f32_16x16x32_bf16 v[236:239], v[120:123], a[88:91], v[236:239]\n"
    "ds_read_b64_tr_b16 v[112:113], v62 offset:33280\n"
    "s_cmp_eq_u32 s69, 4\n"
    "s_cselect_b64 s[16:17], s[16:17], 0\n"
    "s_cselect_b64 s[18:19], s[18:19], 0\n"
    "v_mfma_f32_16x16x32_bf16 v[236:239], v[124:127], a[92:95], v[236:239]\n"
    "s_bcnt1_i32_b64 s30, s[82:83]\n"
    "s_cmp_ge_u32 s30, 56\n"
    "s_cselect_b32 s22, 0, s22\n"
    "v_mfma_f32_16x16x32_bf16 v[204:207], v[124:127], a[84:87], v[204:207]\n"
    "ds_read_b64_tr_b16 v[114:115], v62 offset:34304\n"
    "s_cmp_eq_u32 s26, 4\n"
    "s_cselect_b64 s[14:15], s[18:19], s[16:17]\n"
    "s_cselect_b64 s[18:19], s[16:17], s[18:19]\n"
    "s_mov_b64 s[16:17], s[14:15]\n"
    "v_mfma_f32_16x16x32_bf16 v[204:207], v[120:123], a[80:83], v[204:207]\n"
    "buffer_load_dword v48, v49, s[20:23], 0 offen sc0 sc1\n"
    "v_mfma_f32_16x16x32_bf16 v[172:175], v[120:123], a[72:75], v[172:175]\n"
    "ds_read_b64_tr_b16 v[116:117], v62 offset:41472\n"
    "buffer_load_dwordx4 a[248:251], v55, s[40:43], s65 offen offset:2048\n"
    "v_mfma_f32_16x16x32_bf16 v[172:175], v[124:127], a[76:79], v[172:175]\n"
    "buffer_load_dwordx4 a[252:255], v55, s[40:43], s65 offen offset:3072\n"
    "v_mfma_f32_16x16x32_bf16 v[140:143], v[124:127], a[68:71], v[140:143]\n"
    "ds_read_b64_tr_b16 v[118:119], v62 offset:42496\n"
    "s_andn2_b64 s[76:77], s[72:73], s[80:81]\n"
    "s_cbranch_scc0 1251f\n"
    "1252:\n"
    "s_andn2_b64 s[14:15], s[16:17], s[80:81]\n"
    "s_cselect_b64 s[76:77], s[14:15], s[76:77]\n"
    "s_ff1_i32_b64 s69, s[76:77]\n"
    "v_mfma_f32_16x16x32_bf16 v[140:143], v[120:123], a[64:67], v[140:143]\n"
    "v_readlane_b32 s65, v53, s69\n"
    "s_bitset1_b64 s[80:81], s69\n"
    "s_waitcnt lgkmcnt(9)\n"
    "v_mfma_f32_16x16x32_bf16 v[144:147], v[96:99], a[64:67], v[144:147]\n"
    "ds_read_b64_tr_b16 v[120:121], v63 offset:33280\n"
    "s_nop 0\n"
    "v_mfma_f32_16x16x32_bf16 v[144:147], v[100:103], a[68:71], v[144:147]\n"
    "s_nop 0\n"
    "s_lshl_b32 s64, s65, 5\n"
    "v_mfma_f32_16x16x32_bf16 v[176:179], v[100:103], a[76:79], v[176:179]\n"
    "ds_read_b64_tr_b16 v[122:123], v63 offset:34304\n"
    "s_add_u32 m0, s67, 0\n"
    "v_mfma_f32_16x16x32_bf16 v[176:179], v[96:99], a[72:75], v[176:179]\n"
    "s_nop 0\n"
    "v_mfma_f32_16x16x32_bf16 v[208:211], v[96:99], a[80:83], v[208:211]\n"
    "ds_read_b64_tr_b16 v[124:125], v63 offset:41472\n"
    "buffer_load_dwordx4 v56, s[36:39], s64 offen offset:0 lds\n"
    "v_mfma_f32_16x16x32_bf16 v[208:211], v[100:103], a[84:87], v[208:211]\n"
    "v_mfma_f32_16x16x32_bf16 v[240:243], v[100:103], a[92:95], v[240:243]\n"
    "ds_read_b64_tr_b16 v[126:127], v63 offset:42496\n"
    "v_mfma_f32_16x16x32_bf16 v[240:243], v[96:99], a[88:91], v[240:243]\n"
    "buffer_load_dwordx4 v57, s[36:39], s64 offen offset:1024 lds\n"
    "s_waitcnt lgkmcnt(8)\n"
    "s_waitcnt vmcnt(52)\n"
    "s_barrier\n"
    "v_mfma_f32_16x16x32_bf16 v[244:247], v[104:107], a[88:91], v[244:247]\n"
    "ds_read_b64_tr_b16 v[96:97], v60 offset:49152\n"
    "v_mfma_f32_16x16x32_bf16 v[244:247], v[108:111], a[92:95], v[244:247]\n"
    "ds_read_b64 v[36:37], v40 offset:0\n"
    "v_mfma_f32_16x16x32_bf16 v[212:215], v[108:111], a[84:87], v[212:215]\n"
    "ds_read_b64_tr_b16 v[98:99], v60 offset:50176\n"
    "v_mfma_f32_16x16x32_bf16 v[212:215], v[104:107], a[80:83], v[212:215]\n"
    "v_mfma_f32_16x16x32_bf16 v[180:183], v[104:107], a[72:75], v[180:183]\n"
    "ds_read_b64_tr_b16 v[100:101], v60 offset:57344\n"
    "v_mfma_f32_16x16x32_bf16 v[180:183], v[108:111], a[76:79], v[180:183]\n"
    "buffer_load_dwordx4 v58, s[36:39], s64 offen offset:2048 lds\n"
    "v_mfma_f32_16x16x32_bf16 v[148:151], v[108:111], a[68:71], v[148:151]\n"
    "ds_read_b64_tr_b16 v[102:103], v60 offset:58368\n"
    "v_mfma_f32_16x16x32_bf16 v[148:151], v[104:107], a[64:67], v[148:151]\n"
    "s_waitcnt lgkmcnt(9)\n"
    "v_mfma_f32_16x16x32_bf16 v[152:155], v[112:115], a[64:67], v[152:155]\n"
    "ds_read_b64_tr_b16 v[104:105], v61 offset:49152\n"
    "v_mfma_f32_16x16x32_bf16 v[152:155], v[116:119], a[68:71], v[152:155]\n"
    "v_mfma_f32_16x16x32_bf16 v[184:187], v[116:119], a[76:79], v[184:187]\n"
    "ds_read_b64_tr_b16 v[106:107], v61 offset:50176\n"
    "v_mfma_f32_16x16x32_bf16 v[184:187], v[112:115], a[72:75], v[184:187]\n"
    "buffer_load_dwordx4 v59, s[36:39], s64 offen offset:3072 lds\n"
    "v_mfma_f32_16x16x32_bf16 v[216:219], v[112:115], a[80:83], v[216:219]\n"
    "ds_read_b64_tr_b16 v[108:109], v61 offset:57344\n"
    "v_mfma_f32_16x16x32_bf16 v[216:219], v[116:119], a[84:87], v[216:219]\n"
    "v_mfma_f32_16x16x32_bf16 v[248:251], v[116:119], a[92:95], v[248:251]\n"
    "ds_read_b64_tr_b16 v[110:111], v61 offset:58368\n"
    "v_mfma_f32_16x16x32_bf16 v[248:251], v[112:115], a[88:91], v[248:251]\n"
    "s_waitcnt lgkmcnt(9)\n"
    "v_mfma_f32_16x16x32_bf16 v[252:255], v[120:123], a[88:91], v[252:255]\n"
    "ds_read_b64_tr_b16 v[112:113], v62 offset:49152\n"
    "v_mfma_f32_16x16x32_bf16 v[252:255], v[124:127], a[92:95], v[252:255]\n"
    "buffer_load_dwordx4 a[0:3], v54, s[40:43], s65 offen offset:0\n"
    "v_mfma_f32_16x16x32_bf16 v[220:223], v[124:127], a[84:87], v[220:223]\n"
    "ds_read_b64_tr_b16 v[114:115], v62 offset:50176\n"
    "v_mfma_f32_16x16x32_bf16 v[220:223], v[120:123], a[80:83], v[220:223]\n"
    "v_mfma_f32_16x16x32_bf16 v[188:191], v[120:123], a[72:75], v[188:191]\n"
    "ds_read_b64_tr_b16 v[116:117], v62 offset:57344\n"
    "v_mfma_f32_16x16x32_bf16 v[188:191], v[124:127], a[76:79], v[188:191]\n"
    "v_mfma_f32_16x16x32_bf16 v[156:159], v[124:127], a[68:71], v[156:159]\n"
    "ds_read_b64_tr_b16 v[118:119], v62 offset:58368\n"
    "v_mfma_f32_16x16x32_bf16 v[156:159], v[120:123], a[64:67], v[156:159]\n"
    "buffer_load_dwordx4 a[4:7], v54, s[40:43], s65 offen offset:1024\n"
    "s_waitcnt lgkmcnt(8)\n"
    "v_mfma_f32_16x16x32_bf16 v[128:131], v[96:99], a[96:99], v[128:131]\n"
    "ds_read_b64_tr_b16 v[120:121], v63 offset:49152\n"
    "v_mfma_f32_16x16x32_bf16 v[128:131], v[100:103], a[100:103], v[128:131]\n"
    "v_readfirstlane_b32 vcc_lo, v36\n"
    "v_readfirstlane_b32 vcc_hi, v37\n"
    "v_mfma_f32_16x16x32_bf16 v[160:163], v[100:103], a[108:111], v[160:163]\n"
    "ds_read_b64_tr_b16 v[122:123], v63 offset:50176\n"
    "v_mfma_f32_16x16x32_bf16 v[160:163], v[96:99], a[104:107], v[160:163]\n"
    "s_or_b64 s[72:73], s[72:73], vcc\n"
    "v_mfma_f32_16x16x32_bf16 v[192:195], v[96:99], a[112:115], v[192:195]\n"
    "ds_read_b64_tr_b16 v[124:125], v63 offset:57344\n"
    "v_mfma_f32_16x16x32_bf16 v[192:195], v[100:103], a[116:119], v[192:195]\n"
    "buffer_load_dwordx4 a[8:11], v54, s[40:43], s65 offen offset:2048\n"
    "v_mfma_f32_16x16x32_bf16 v[224:227], v[100:103], a[124:127], v[224:227]\n"
    "ds_read_b64_tr_b16 v[126:127], v63 offset:58368\n"
    "v_mfma_f32_16x16x32_bf16 v[224:227], v[96:99], a[120:123], v[224:227]\n"
    "s_waitcnt lgkmcnt(8)\n"
    "v_mfma_f32_16x16x32_bf16 v[228:231], v[104:107], a[120:123], v[228:231]\n"
    "ds_read_b64_tr_b16 v[96:97], v60 offset:49664\n"
    "v_mfma_f32_16x16x32_bf16 v[228:231], v[108:111], a[124:127], v[228:231]\n"
    "v_mfma_f32_16x16x32_bf16 v[196:199], v[108:111], a[116:119], v[196:199]\n"
    "ds_read_b64_tr_b16 v[98:99], v60 offset:50688\n"
    "v_mfma_f32_16x16x32_bf16 v[196:199], v[104:107], a[112:115], v[196:199]\n"
    "buffer_load_dwordx4 a[12:15], v54, s[40:43], s65 offen offset:3072\n"
    "v_mfma_f32_16x16x32_bf16 v[164:167], v[104:107], a[104:107], v[164:167]\n"
    "ds_read_b64_tr_b16 v[100:101], v60 offset:57856\n"
    "v_mfma_f32_16x16x32_bf16 v[164:167], v[108:111], a[108:111], v[164:167]\n"
    "v_mfma_f32_16x16x32_bf16 v[132:135], v[108:111], a[100:103], v[132:135]\n"
    "ds_read_b64_tr_b16 v[102:103], v60 offset:58880\n"
    "v_mfma_f32_16x16x32_bf16 v[132:135], v[104:107], a[96:99], v[132:135]\n"
    "s_waitcnt lgkmcnt(8)\n"
    "v_mfma_f32_16x16x32_bf16 v[136:139], v[112:115], a[96:99], v[136:139]\n"
    "ds_read_b64_tr_b16 v[104:105], v61 offset:49664\n"
    "v_mfma_f32_16x16x32_bf16 v[136:139], v[116:119], a[100:103], v[136:139]\n"
    "buffer_load_dwordx4 a[16:19], v55, s[40:43], s65 offen offset:0\n"
    "v_mfma_f32_16x16x32_bf16 v[168:171], v[116:119], a[108:111], v[168:171]\n"
    "ds_read_b64_tr_b16 v[106:107], v61 offset:50688\n"
    "v_mfma_f32_16x16x32_bf16 v[168:171], v[112:115], a[104:107], v[168:171]\n"
    "v_mfma_f32_16x16x32_bf16 v[200:203], v[112:115], a[112:115], v[200:203]\n"
    "ds_read_b64_tr_b16 v[108:109], v61 offset:57856\n"
    "v_mfma_f32_16x16x32_bf16 v[200:203], v[116:119], a[116:119], v[200:203]\n"
    "v_mfma_f32_16x16x32_bf16 v[232:235], v[116:119], a[124:127], v[232:235]\n"
    "ds_read_b64_tr_b16 v[110:111], v61 offset:58880\n"
    "v_mfma_f32_16x16x32_bf16 v[232:235], v[112:115], a[120:123], v[232:235]\n"
    "buffer_load_dwordx4 a[20:23], v55, s[40:43], s65 offen offset:1024\n"
    "s_waitcnt lgkmcnt(8)\n"
    "v_mfma_f32_16x16x32_bf16 v[236:239], v[120:123], a[120:123], v[236:239]\n"
    "ds_read_b64_tr_b16 v[112:113], v62 offset:49664\n"
    "v_mfma_f32_16x16x32_bf16 v[236:239], v[124:127], a[124:127], v[236:239]\n"
    "v_mfma_f32_16x16x32_bf16 v[204:207], v[124:127], a[116:119], v[204:207]\n"
    "ds_read_b64_tr_b16 v[114:115], v62 offset:50688\n"
    "v_mfma_f32_16x16x32_bf16 v[204:207], v[120:123], a[112:115], v[204:207]\n"
    "v_mfma_f32_16x16x32_bf16 v[172:175], v[120:123], a[104:107], v[172:175]\n"
    "ds_read_b64_tr_b16 v[116:117], v62 offset:57856\n"
    "v_mfma_f32_16x16x32_bf16 v[172:175], v[124:127], a[108:111], v[172:175]\n"
    "buffer_load_dwordx4 a[24:27], v55, s[40:43], s65 offen offset:2048\n"
    "v_mfma_f32_16x16x32_bf16 v[140:143], v[124:127], a[100:103], v[140:143]\n"
    "ds_read_b64_tr_b16 v[118:119], v62 offset:58880\n"
    "v_mfma_f32_16x16x32_bf16 v[140:143], v[120:123], a[96:99], v[140:143]\n"
    "s_waitcnt lgkmcnt(8)\n"
    "v_mfma_f32_16x16x32_bf16 v[144:147], v[96:99], a[96:99], v[144:147]\n"
    "ds_read_b64_tr_b16 v[120:121], v63 offset:49664\n"
    "v_mfma_f32_16x16x32_bf16 v[144:147], v[100:103], a[100:103], v[144:147]\n"
    "v_mfma_f32_16x16x32_bf16 v[176:179], v[100:103], a[108:111], v[176:179]\n"
    "ds_read_b64_tr_b16 v[122:123], v63 offset:50688\n"
    "v_mfma_f32_16x16x32_bf16 v[176:179], v[96:99], a[104:107], v[176:179]\n"
    "buffer_load_dwordx4 a[28:31], v55, s[40:43], s65 offen offset:3072\n"
    "v_mfma_f32_16x16x32_bf16 v[208:211], v[96:99], a[112:115], v[208:211]\n"
    "ds_read_b64_tr_b16 v[124:125], v63 offset:57856\n"
    "v_mfma_f32_16x16x32_bf16 v[208:211], v[100:103], a[116:119], v[208:211]\n"
    "v_mfma_f32_16x16x32_bf16 v[240:243], v[100:103], a[124:127], v[240:243]\n"
    "ds_read_b64_tr_b16 v[126:127], v63 offset:58880\n"
    "s_andn2_b64 s[76:77], s[72:73], s[80:81]\n"
    "s_cbranch_scc0 1261f\n"
    "1262:\n"
    "s_andn2_b64 s[14:15], s[16:17], s[80:81]\n"
    "s_cselect_b64 s[76:77], s[14:15], s[76:77]\n"
    "s_ff1_i32_b64 s69, s[76:77]\n"
    "v_mfma_f32_16x16x32_bf16 v[240:243], v[96:99], a[120:123], v[240:243]\n"
    "v_readlane_b32 s65, v53, s69\n"
    "s_bitset1_b64 s[80:81], s69\n"
    "s_waitcnt lgkmcnt(8)\n"
    "s_waitcnt vmcnt(50)\n"
    "s_barrier\n"
    "v_mfma_f32_16x16x32_bf16 v[244:247], v[104:107], a[120:123], v[244:247]\n"
    "ds_read_b64_tr_b16 v[96:97], v64 offset:0\n"
    "s_nop 0\n"
    "v_mfma_f32_16x16x32_bf16 v[244:247], v[108:111], a[124:127], v[244:247]\n"
    "s_nop 0\n"
    "s_lshl_b32 s64, s65, 5\n"
    "v_mfma_f32_16x16x32_bf16 v[212:215], v[108:111], a[116:119], v[212:215]\n"
    "ds_read_b64_tr_b16 v[98:99], v64 offset:1024\n"
    "s_add_u32 m0, s67, 16384\n"
    "v_mfma_f32_16x16x32_bf16 v[212:215], v[104:107], a[112:115], v[212:215]\n"
    "s_nop 0\n"
    "v_mfma_f32_16x16x32_bf16 v[180:183], v[104:107], a[104:107], v[180:183]\n"
    "ds_read_b64_tr_b16 v[100:101], v64 offset:8192\n"
    "v_mfma_f32_16x16x32_bf16 v[180:183], v[108:111], a[108:111], v[180:183]\n"
    "buffer_load_dwordx4 v56, s[36:39], s64 offen offset:0 lds\n"
    "v_mfma_f32_16x16x32_bf16 v[148:151], v[108:111], a[100:103], v[148:151]\n"
    "ds_read_b64_tr_b16 v[102:103], v64 offset:9216\n"
    "v_mfma_f32_16x16x32_bf16 v[148:151], v[104:107], a[96:99], v[148:151]\n"
    "s_waitcnt lgkmcnt(8)\n"
    "v_mfma_f32_16x16x32_bf16 v[152:155], v[112:115], a[96:99], v[152:155]\n"
    "ds_read_b64_tr_b16 v[104:105], v65 offset:0\n"
    "v_mfma_f32_16x16x32_bf16 v[152:155], v[116:119], a[100:103], v[152:155]\n"
    "buffer_load_dwordx4 v57, s[36:39], s64 offen offset:1024 lds\n"
    "v_mfma_f32_16x16x32_bf16 v[184:187], v[116:119], a[108:111], v[184:187]\n"
    "ds_read_b64_tr_b16 v[106:107], v65 offset:1024\n"
    "v_mfma_f32_16x16x32_bf16 v[184:187], v[112:115], a[104:107], v[184:187]\n"
    "v_mfma_f32_16x16x32_bf16 v[216:219], v[112:115], a[112:115], v[216:219]\n"
    "ds_read_b64_tr_b16 v[108:109], v65 offset:8192\n"
    "v_mfma_f32_16x16x32_bf16 v[216:219], v[116:119], a[116:119], v[216:219]\n"
    "v_mfma_f32_16x16x32_bf16 v[248:251], v[116:119], a[124:127], v[248:251]\n"
    "ds_read_b64_tr_b16 v[110:111], v65 offset:9216\n"
    "v_mfma_f32_16x16x32_bf16 v[248:251], v[112:115], a[120:123], v[248:251]\n"
    "buffer_load_dwordx4 v58, s[36:39], s64 offen offset:2048 lds\n"
    "s_waitcnt lgkmcnt(8)\n"
    "v_mfma_f32_16x16x32_bf16 v[252:255], v[120:123], a[120:123], v[252:255]\n"
    "ds_read_b64_tr_b16 v[112:113], v66 offset:0\n"
    "v_mfma_f32_16x16x32_bf16 v[252:255], v[124:127], a[124:127], v[252:255]\n"
    "v_mfma_f32_16x16x32_bf16 v[220:223], v[124:127], a[116:119], v[220:223]\n"
    "ds_read_b64_tr_b16 v[114:115], v66 offset:1024\n"
    "v_mfma_f32_16x16x32_bf16 v[220:223], v[120:123], a[112:115], v[220:223]\n"
    "v_mfma_f32_16x16x32_bf16 v[188:191], v[120:123], a[104:107], v[188:191]\n"
    "ds_read_b64_tr_b16 v[116:117], v66 offset:8192\n"
    "v_mfma_f32_16x16x32_bf16 v[188:191], v[124:127], a[108:111], v[188:191]\n"
    "buffer_load_dwordx4 v59, s[36:39], s64 offen offset:3072 lds\n"
    "v_mfma_f32_16x16x32_bf16 v[156:159], v[124:127], a[100:103], v[156:159]\n"
    "ds_read_b64_tr_b16 v[118:119], v66 offset:9216\n"
    "v_mfma_f32_16x16x32_bf16 v[156:159], v[120:123], a[96:99], v[156:159]\n"
    "s_waitcnt lgkmcnt(8)\n"
    "v_mfma_f32_16x16x32_bf16 v[128:131], v[96:99], a[128:131], v[128:131]\n"
    "ds_read_b64_tr_b16 v[120:121], v67 offset:0\n"
    "v_mfma_f32_16x16x32_bf16 v[128:131], v[100:103], a[132:135], v[128:131]\n"
    "v_mfma_f32_16x16x32_bf16 v[160:163], v[100:103], a[140:143], v[160:163]\n"
    "ds_read_b64_tr_b16 v[122:123], v67 offset:1024\n"
    "v_mfma_f32_16x16x32_bf16 v[160:163], v[96:99], a[136:139], v[160:163]\n"
    "buffer_load_dwordx4 a[32:35], v54, s[40:43], s65 offen offset:0\n"
    "v_mfma_f32_16x16x32_bf16 v[192:195], v[96:99], a[144:147], v[192:195]\n"
    "ds_read_b64_tr_b16 v[124:125], v67 offset:8192\n"
    "v_mfma_f32_16x16x32_bf16 v[192:195], v[100:103], a[148:151], v[192:195]\n"
    "v_mfma_f32_16x16x32_bf16 v[224:227], v[100:103], a[156:159], v[224:227]\n"
    "ds_read_b64_tr_b16 v[126:127], v67 offset:9216\n"
    "v_mfma_f32_16x16x32_bf16 v[224:227], v[96:99], a[152:155], v[224:227]\n"
    "s_waitcnt lgkmcnt(8)\n"
    "v_mfma_f32_16x16x32_bf16 v[228:231], v[104:107], a[152:155], v[228:231]\n"
    "ds_read_b64_tr_b16 v[96:97], v64 offset:512\n"
    "v_mfma_f32_16x16x32_bf16 v[228:231], v[108:111], a[156:159], v[228:231]\n"
    "buffer_load_dwordx4 a[36:39], v54, s[40:43], s65 offen offset:1024\n"
    "v_mfma_f32_16x16x32_bf16 v[196:199], v[108:111], a[148:151], v[196:199]\n"
    "ds_read_b64_tr_b16 v[98:99], v64 offset:1536\n"
    "v_mfma_f32_16x16x32_bf16 v[196:199], v[104:107], a[144:147], v[196:199]\n"
    "v_mfma_f32_16x16x32_bf16 v[164:167], v[104:107], a[136:139], v[164:167]\n"
    "ds_read_b64_tr_b16 v[100:101], v64 offset:8704\n"
    "v_mfma_f32_16x16x32_bf16 v[164:167], v[108:111], a[140:143], v[164:167]\n"
    "v_cmp_le_u32 vcc, s70, v47\n"
    "v_mfma_f32_16x16x32_bf16 v[132:135], v[108:111], a[132:135], v[132:135]\n"
    "ds_read_b64_tr_b16 v[102:103], v64 offset:9728\n"
    "v_mfma_f32_16x16x32_bf16 v[132:135], v[104:107], a[128:131], v[132:135]\n"
    "s_or_b64 s[82:83], s[82:83], vcc\n"
    "v_cmp_le_u32 vcc, s70, v48\n"
    "s_waitcnt lgkmcnt(8)\n"
    "v_mfma_f32_16x16x32_bf16 v[136:139], v[112:115], a[128:131], v[136:139]\n"
    "ds_read_b64_tr_b16 v[104:105], v65 offset:512\n"
    "buffer_load_dwordx4 a[40:43], v54, s[40:43], s65 offen offset:2048\n"
    "v_mfma_f32_16x16x32_bf16 v[136:139], v[116:119], a[132:135], v[136:139]\n"
    "s_or_b64 s[82:83], s[82:83], vcc\n"
    "v_mfma_f32_16x16x32_bf16 v[168:171], v[116:119], a[140:143], v[168:171]\n"
    "ds_read_b64_tr_b16 v[106:107], v65 offset:1536\n"
    "v_mfma_f32_16x16x32_bf16 v[168:171], v[112:115], a[136:139], v[168:171]\n"
    "v_mfma_f32_16x16x32_bf16 v[200:203], v[112:115], a[144:147], v[200:203]\n"
    "ds_read_b64_tr_b16 v[108:109], v65 offset:8704\n"
    "v_mfma_f32_16x16x32_bf16 v[200:203], v[116:119], a[148:151], v[200:203]\n"
    "v_mov_b32 v38, s82\n"
    "v_mov_b32 v39, s83\n"
    "ds_write_b64 v41, v[38:39] offset:0\n"
    "v_mfma_f32_16x16x32_bf16 v[232:235], v[116:119], a[156:159], v[232:235]\n"
    "ds_read_b64_tr_b16 v[110:111], v65 offset:9728\n"
    "buffer_load_dwordx4 a[44:47], v54, s[40:43], s65 offen offset:3072\n"
    "v_mfma_f32_16x16x32_bf16 v[232:235], v[112:115], a[152:155], v[232:235]\n"
    "s_ff0_i32_b64 s30, s[82:83]\n"
    "s_bfm_b64 vcc, s30, 0\n"
    "s_andn2_b64 vcc, vcc, s[80:81]\n"
    "s_bcnt1_i32_b64 s30, vcc\n"
    "s_cmp_lt_u32 s30, 64\n"
    "s_cselect_b32 s22, 0x10000, 0\n"
    "s_waitcnt lgkmcnt(9)\n"
    "v_mfma_f32_16x16x32_bf16 v[236:239], v[120:123], a[152:155], v[236:239]\n"
    "ds_read_b64_tr_b16 v[112:113], v66 offset:512\n"
    "v_mfma_f32_16x16x32_bf16 v[236:239], v[124:127], a[156:159], v[236:239]\n"
    "s_bcnt1_i32_b64 s30, s[82:83]\n"
    "s_cmp_ge_u32 s30, 56\n"
    "s_cselect_b32 s22, 0, s22\n"
    "v_mfma_f32_16x16x32_bf16 v[204:207], v[124:127], a[148:151], v[204:207]\n"
    "ds_read_b64_tr_b16 v[114:115], v66 offset:1536\n"
    "v_mfma_f32_16x16x32_bf16 v[204:207], v[120:123], a[144:147], v[204:207]\n"
    "buffer_load_dword v47, v49, s[20:23], 0 offen sc0 sc1\n"
    "v_mfma_f32_16x16x32_bf16 v[172:175], v[120:123], a[136:139], v[172:175]\n"
    "ds_read_b64_tr_b16 v[116:117], v66 offset:8704\n"
    "buffer_load_dwordx4 a[48:51], v55, s[40:43], s65 offen offset:0\n"
    "v_mfma_f32_16x16x32_bf16 v[172:175], v[124:127], a[140:143], v[172:175]\n"
    "v_mfma_f32_16x16x32_bf16 v[140:143], v[124:127], a[132:135], v[140:143]\n"
    "ds_read_b64_tr_b16 v[118:119], v66 offset:9728\n"
    "v_mfma_f32_16x16x32_bf16 v[140:143], v[120:123], a[128:131], v[140:143]\n"
    "s_waitcnt lgkmcnt(9)\n"
    "v_mfma_f32_16x16x32_bf16 v[144:147], v[96:99], a[128:131], v[144:147]\n"
    "ds_read_b64_tr_b16 v[120:121], v67 offset:512\n"
    "v_mfma_f32_16x16x32_bf16 v[144:147], v[100:103], a[132:135], v[144:147]\n"
    "buffer_load_dwordx4 a[52:55], v55, s[40:43], s65 offen offset:1024\n"
    "v_mfma_f32_16x16x32_bf16 v[176:179], v[100:103], a[140:143], v[176:179]\n"
    "ds_read_b64_tr_b16 v[122:123], v67 offset:1536\n"
    "v_mfma_f32_16x16x32_bf16 v[176:179], v[96:99], a[136:139], v[176:179]\n"
    "v_mfma_f32_16x16x32_bf16 v[208:211], v[96:99], a[144:147], v[208:211]\n"
    "ds_read_b64_tr_b16 v[124:125], v67 offset:8704\n"
    "v_mfma_f32_16x16x32_bf16 v[208:211], v[100:103], a[148:151], v[208:211]\n"
    "v_mfma_f32_16x16x32_bf16 v[240:243], v[100:103], a[156:159], v[240:243]\n"
    "ds_read_b64_tr_b16 v[126:127], v67 offset:9728\n"
    "v_mfma_f32_16x16x32_bf16 v[240:243], v[96:99], a[152:155], v[240:243]\n"
    "buffer_load_dwordx4 a[56:59], v55, s[40:43], s65 offen offset:2048\n"
    "s_waitcnt lgkmcnt(8)\n"
    "s_waitcnt vmcnt(49)\n"
    "s_barrier\n"
    "v_mfma_f32_16x16x32_bf16 v[244:247], v[104:107], a[152:155], v[244:247]\n"
    "ds_read_b64_tr_b16 v[96:97], v64 offset:16384\n"
    "v_mfma_f32_16x16x32_bf16 v[244:247], v[108:111], a[156:159], v[244:247]\n"
    "ds_read_b64 v[36:37], v40 offset:0\n"
    "v_mfma_f32_16x16x32_bf16 v[212:215], v[108:111], a[148:151], v[212:215]\n"
    "ds_read_b64_tr_b16 v[98:99], v64 offset:17408\n"
    "v_mfma_f32_16x16x32_bf16 v[212:215], v[104:107], a[144:147], v[212:215]\n"
    "v_mfma_f32_16x16x32_bf16 v[180:183], v[104:107], a[136:139], v[180:183]\n"
    "ds_read_b64_tr_b16 v[100:101], v64 offset:24576\n"
    "v_mfma_f32_16x16x32_bf16 v[180:183], v[108:111], a[140:143], v[180:183]\n"
    "buffer_load_dwordx4 a[60:63], v55, s[40:43], s65 offen offset:3072\n"
    "v_mfma_f32_16x16x32_bf16 v[148:151], v[108:111], a[132:135], v[148:151]\n"
    "ds_read_b64_tr_b16 v[102:103], v64 offset:25600\n"
    "v_mfma_f32_16x16x32_bf16 v[148:151], v[104:107], a[128:131], v[148:151]\n"
    "s_waitcnt lgkmcnt(9)\n"
    "v_mfma_f32_16x16x32_bf16 v[152:155], v[112:115], a[128:131], v[152:155]\n"
    "ds_read_b64_tr_b16 v[104:105], v65 offset:16384\n"
    "s_andn2_b64 s[76:77], s[72:73], s[80:81]\n"
    "s_cbranch_scc0 1271f\n"
    "1272:\n"
    "s_andn2_b64 s[14:15], s[16:17], s[80:81]\n"
    "s_cselect_b64 s[76:77], s[14:15], s[76:77]\n"
    "s_ff1_i32_b64 s69, s[76:77]\n"
    "v_mfma_f32_16x16x32_bf16 v[152:155], v[116:119], a[132:135], v[152:155]\n"
    "v_readlane_b32 s65, v53, s69\n"
    "s_bitset1_b64 s[80:81], s69\n"
    "v_mfma_f32_16x16x32_bf16 v[184:187], v[116:119], a[140:143], v[184:187]\n"
    "ds_read_b64_tr_b16 v[106:107], v65 offset:17408\n"
    "s_nop 0\n"
    "v_mfma_f32_16x16x32_bf16 v[184:187], v[112:115], a[136:139], v[184:187]\n"
    "s_nop 0\n"
    "s_lshl_b32 s64, s65, 5\n"
    "v_mfma_f32_16x16x32_bf16 v[216:219], v[112:115], a[144:147], v[216:219]\n"
    "ds_read_b64_tr_b16 v[108:109], v65 offset:24576\n"
    "s_add_u32 m0, s67, 32768\n"
    "v_mfma_f32_16x16x32_bf16 v[216:219], v[116:119], a[148:151], v[216:219]\n"
    "s_nop 0\n"
    "v_mfma_f32_16x16x32_bf16 v[248:251], v[116:119], a[156:159], v[248:251]\n"
    "ds_read_b64_tr_b16 v[110:111], v65 offset:25600\n"
    "v_mfma_f32_16x16x32_bf16 v[248:251], v[112:115], a[152:155], v[248:251]\n"
    "buffer_load_dwordx4 v56, s[36:39], s64 offen offset:0 lds\n"
    "s_waitcnt lgkmcnt(9)\n"
    "v_mfma_f32_16x16x32_bf16 v[252:255], v[120:123], a[152:155], v[252:255]\n"
    "ds_read_b64_tr_b16 v[112:113], v66 offset:16384\n"
    "v_mfma_f32_16x16x32_bf16 v[252:255], v[124:127], a[156:159], v[252:255]\n"
    "v_mfma_f32_16x16x32_bf16 v[220:223], v[124:127], a[148:151], v[220:223]\n"
    "ds_read_b64_tr_b16 v[114:115], v66 offset:17408\n"
    "v_mfma_f32_16x16x32_bf16 v[220:223], v[120:123], a[144:147], v[220:223]\n"
    "buffer_load_dwordx4 v57, s[36:39], s64 offen offset:1024 lds\n"
    "v_mfma_f32_16x16x32_bf16 v[188:191], v[120:123], a[136:139], v[188:191]\n"
    "ds_read_b64_tr_b16 v[116:117], v66 offset:24576\n"
    "v_mfma_f32_16x16x32_bf16 v[188:191], v[124:127], a[140:143], v[188:191]\n"
    "v_mfma_f32_16x16x32_bf16 v[156:159], v[124:127], a[132:135], v[156:159]\n"
    "ds_read_b64_tr_b16 v[118:119], v66 offset:25600\n"
    "v_mfma_f32_16x16x32_bf16 v[156:159], v[120:123], a[128:131], v[156:159]\n"
    "s_waitcnt lgkmcnt(8)\n"
    "v_mfma_f32_16x16x32_bf16 v[128:131], v[96:99], a[160:163], v[128:131]\n"
    "ds_read_b64_tr_b16 v[120:121], v67 offset:16384\n"
    "v_mfma_f32_16x16x32_bf16 v[128:131], v[100:103], a[164:167], v[128:131]\n"
    "v_readfirstlane_b32 vcc_lo, v36\n"
    "v_readfirstlane_b32 vcc_hi, v37\n"
    "v_mfma_f32_16x16x32_bf16 v[160:163], v[100:103], a[172:175], v[160:163]\n"
    "ds_read_b64_tr_b16 v[122:123], v67 offset:17408\n"
    "buffer_load_dwordx4 v58, s[36:39], s64 offen offset:2048 lds\n"
    "v_mfma_f32_16x16x32_bf16 v[160:163], v[96:99], a[168:171], v[160:163]\n"
    "s_or_b64 s[72:73], s[72:73], vcc\n"
    "v_mfma_f32_16x16x32_bf16 v[192:195], v[96:99], a[176:179], v[192:195]\n"
    "ds_read_b64_tr_b16 v[124:125], v67 offset:24576\n"
    "v_mfma_f32_16x16x32_bf16 v[192:195], v[100:103], a[180:183], v[192:195]\n"
    "v_mfma_f32_16x16x32_bf16 v[224:227], v[100:103], a[188:191], v[224:227]\n"
    "ds_read_b64_tr_b16 v[126:127], v67 offset:25600\n"
    "v_mfma_f32_16x16x32_bf16 v[224:227], v[96:99], a[184:187], v[224:227]\n"
    "buffer_load_dwordx4 v59, s[36:39], s64 offen offset:3072 lds\n"
    "s_waitcnt lgkmcnt(8)\n"
    "v_mfma_f32_16x16x32_bf16 v[228:231], v[104:107], a[184:187], v[228:231]\n"
    "ds_read_b64_tr_b16 v[96:97], v64 offset:16896\n"
    "v_mfma_f32_16x16x32_bf16 v[228:231], v[108:111], a[188:191], v[228:231]\n"
    "v_mfma_f32_16x16x32_bf16 v[196:199], v[108:111], a[180:183], v[196:199]\n"
    "ds_read_b64_tr_b16 v[98:99], v64 offset:17920\n"
    "v_mfma_f32_16x16x32_bf16 v[196:199], v[104:107], a[176:179], v[196:199]\n"
    "v_mfma_f32_16x16x32_bf16 v[164:167], v[104:107], a[168:171], v[164:167]\n"
    "ds_read_b64_tr_b16 v[100:101], v64 offset:25088\n"
    "v_mfma_f32_16x16x32_bf16 v[164:167], v[108:111], a[172:175], v[164:167]\n"
    "buffer_load_dwordx4 a[64:67], v54, s[40:43], s65 offen offset:0\n"
    "v_mfma_f32_16x16x32_bf16 v[132:135], v[108:111], a[164:167], v[132:135]\n"
    "ds_read_b64_tr_b16 v[102:103], v64 offset:26112\n"
    "v_mfma_f32_16x16x32_bf16 v[132:135], v[104:107], a[160:163], v[132:135]\n"
    "s_waitcnt lgkmcnt(8)\n"
    "v_mfma_f32_16x16x32_bf16 v[136:139], v[112:115], a[160:163], v[136:139]\n"
    "ds_read_b64_tr_b16 v[104:105], v65 offset:16896\n"
    "v_mfma_f32_16x16x32_bf16 v[136:139], v[116:119], a[164:167], v[136:139]\n"
    "v_mfma_f32_16x16x32_bf16 v[168:171], v[116:119], a[172:175], v[168:171]\n"
    "ds_read_b64_tr_b16 v[106:107], v65 offset:17920\n"
    "v_mfma_f32_16x16x32_bf16 v[168:171], v[112:115], a[168:171], v[168:171]\n"
    "buffer_load_dwordx4 a[68:71], v54, s[40:43], s65 offen offset:1024\n"
    "v_mfma_f32_16x16x32_bf16 v[200:203], v[112:115], a[176:179], v[200:203]\n"
    "ds_read_b64_tr_b16 v[108:109], v65 offset:25088\n"
    "v_mfma_f32_16x16x32_bf16 v[200:203], v[116:119], a[180:183], v[200:203]\n"
    "v_mfma_f32_16x16x32_bf16 v[232:235], v[116:119], a[188:191], v[232:235]\n"
    "ds_read_b64_tr_b16 v[110:111], v65 offset:26112\n"
    "v_mfma_f32_16x16x32_bf16 v[232:235], v[112:115], a[184:187], v[232:235]\n"
    "s_waitcnt lgkmcnt(8)\n"
    "v_mfma_f32_16x16x32_bf16 v[236:239], v[120:123], a[184:187], v[236:239]\n"
    "ds_read_b64_tr_b16 v[112:113], v66 offset:16896\n"
    "v_mfma_f32_16x16x32_bf16 v[236:239], v[124:127], a[188:191], v[236:239]\n"
    "buffer_load_dwordx4 a[72:75], v54, s[40:43], s65 offen offset:2048\n"
    "v_mfma_f32_16x16x32_bf16 v[204:207], v[124:127], a[180:183], v[204:207]\n"
    "ds_read_b64_tr_b16 v[114:115], v66 offset:17920\n"
    "v_mfma_f32_16x16x32_bf16 v[204:207], v[120:123], a[176:179], v[204:207]\n"
    "v_mfma_f32_16x16x32_bf16 v[172:175], v[120:123], a[168:171], v[172:175]\n"
    "ds_read_b64_tr_b16 v[116:117], v66 offset:25088\n"
    "v_mfma_f32_16x16x32_bf16 v[172:175], v[124:127], a[172:175], v[172:175]\n"
    "v_mfma_f32_16x16x32_bf16 v[140:143], v[124:127], a[164:167], v[140:143]\n"
    "ds_read_b64_tr_b16 v[118:119], v66 offset:26112\n"
    "v_mfma_f32_16x16x32_bf16 v[140:143], v[120:123], a[160:163], v[140:143]\n"
    "buffer_load_dwordx4 a[76:79], v54, s[40:43], s65 offen offset:3072\n"
    "s_waitcnt lgkmcnt(8)\n"
    "v_mfma_f32_16x16x32_bf16 v[144:147], v[96:99], a[160:163], v[144:147]\n"
    "ds_read_b64_tr_b16 v[120:121], v67 offset:16896\n"
    "v_mfma_f32_16x16x32_bf16 v[144:147], v[100:103], a[164:167], v[144:147]\n"
    "v_mfma_f32_16x16x32_bf16 v[176:179], v[100:103], a[172:175], v[176:179]\n"
    "ds_read_b64_tr_b16 v[122:123], v67 offset:17920\n"
    "v_mfma_f32_16x16x32_bf16 v[176:179], v[96:99], a[168:171], v[176:179]\n"
    "v_mfma_f32_16x16x32_bf16 v[208:211], v[96:99], a[176:179], v[208:211]\n"
    "ds_read_b64_tr_b16 v[124:125], v67 offset:25088\n"
    "v_mfma_f32_16x16x32_bf16 v[208:211], v[100:103], a[180:183], v[208:211]\n"
    "buffer_load_dwordx4 a[80:83], v55, s[40:43], s65 offen offset:0\n"
    "v_mfma_f32_16x16x32_bf16 v[240:243], v[100:103], a[188:191], v[240:243]\n"
    "ds_read_b64_tr_b16 v[126:127], v67 offset:26112\n"
    "v_mfma_f32_16x16x32_bf16 v[240:243], v[96:99], a[184:187], v[240:243]\n"
    "s_waitcnt lgkmcnt(8)\n"
    "s_waitcnt vmcnt(47)\n"
    "s_barrier\n"
    "v_mfma_f32_16x16x32_bf16 v[244:247], v[104:107], a[184:187], v[244:247]\n"
    "ds_read_b64_tr_b16 v[96:97], v64 offset:32768\n"
    "v_mfma_f32_16x16x32_bf16 v[244:247], v[108:111], a[188:191], v[244:247]\n"
    "v_mfma_f32_16x16x32_bf16 v[212:215], v[108:111], a[180:183], v[212:215]\n"
    "ds_read_b64_tr_b16 v[98:99], v64 offset:33792\n"
    "v_mfma_f32_16x16x32_bf16 v[212:215], v[104:107], a[176:179], v[212:215]\n"
    "buffer_load_dwordx4 a[84:87], v55, s[40:43], s65 offen offset:1024\n"
    "v_mfma_f32_16x16x32_bf16 v[180:183], v[104:107], a[168:171], v[180:183]\n"
    "ds_read_b64_tr_b16 v[100:101], v64 offset:40960\n"
    "v_mfma_f32_16x16x32_bf16 v[180:183], v[108:111], a[172:175], v[180:183]\n"
    "v_mfma_f32_16x16x32_bf16 v[148:151], v[108:111], a[164:167], v[148:151]\n"
    "ds_read_b64_tr_b16 v[102:103], v64 offset:41984\n"
    "v_mfma_f32_16x16x32_bf16 v[148:151], v[104:107], a[160:163], v[148:151]\n"
    "s_waitcnt lgkmcnt(8)\n"
    "v_mfma_f32_16x16x32_bf16 v[152:155], v[112:115], a[160:163], v[152:155]\n"
    "ds_read_b64_tr_b16 v[104:105], v65 offset:32768\n"
    "v_mfma_f32_16x16x32_bf16 v[152:155], v[116:119], a[164:167], v[152:155]\n"
    "buffer_load_dwordx4 a[88:91], v55, s[40:43], s65 offen offset:2048\n"
    "v_mfma_f32_16x16x32_bf16 v[184:187], v[116:119], a[172:175], v[184:187]\n"
    "ds_read_b64_tr_b16 v[106:107], v65 offset:33792\n"
    "v_mfma_f32_16x16x32_bf16 v[184:187], v[112:115], a[168:171], v[184:187]\n"
    "v_mfma_f32_16x16x32_bf16 v[216:219], v[112:115], a[176:179], v[216:219]\n"
    "ds_read_b64_tr_b16 v[108:109], v65 offset:40960\n"
    "v_mfma_f32_16x16x32_bf16 v[216:219], v[116:119], a[180:183], v[216:219]\n"
    "v_mfma_f32_16x16x32_bf16 v[248:251], v[116:119], a[188:191], v[248:251]\n"
    "ds_read_b64_tr_b16 v[110:111], v65 offset:41984\n"
    "v_mfma_f32_16x16x32_bf16 v[248:251], v[112:115], a[184:187], v[248:251]\n"
    "buffer_load_dwordx4 a[92:95], v55, s[40:43], s65 offen offset:3072\n"
    "s_waitcnt lgkmcnt(8)\n"
    "v_mfma_f32_16x16x32_bf16 v[252:255], v[120:123], a[184:187], v[252:255]\n"
    "ds_read_b64_tr_b16 v[112:113], v66 offset:32768\n"
    "v_mfma_f32_16x16x32_bf16 v[252:255], v[124:127], a[188:191], v[252:255]\n"
    "v_mfma_f32_16x16x32_bf16 v[220:223], v[124:127], a[180:183], v[220:223]\n"
    "ds_read_b64_tr_b16 v[114:115], v66 offset:33792\n"
    "s_andn2_b64 s[76:77], s[72:73], s[80:81]\n"
    "s_cbranch_scc0 1281f\n"
    "1282:\n"
    "s_andn2_b64 s[14:15], s[16:17], s[80:81]\n"
    "s_cselect_b64 s[76:77], s[14:15], s[76:77]\n"
    "s_ff1_i32_b64 s69, s[76:77]\n"
    "v_mfma_f32_16x16x32_bf16 v[220:223], v[120:123], a[176:179], v[220:223]\n"
    "v_readlane_b32 s65, v53, s69\n"
    "s_bitset1_b64 s[80:81], s69\n"
    "v_mfma_f32_16x16x32_bf16 v[188:191], v[120:123], a[168:171], v[188:191]\n"
    "ds_read_b64_tr_b16 v[116:117], v66 offset:40960\n"
    "s_nop 0\n"
    "v_mfma_f32_16x16x32_bf16 v[188:191], v[124:127], a[172:175], v[188:191]\n"
    "s_nop 0\n"
    "s_lshl_b32 s64, s65, 5\n"
    "v_mfma_f32_16x16x32_bf16 v[156:159], v[124:127], a[164:167], v[156:159]\n"
    "ds_read_b64_tr_b16 v[118:119], v66 offset:41984\n"
    "s_add_u32 m0, s67, 49152\n"
    "v_mfma_f32_16x16x32_bf16 v[156:159], v[120:123], a[160:163], v[156:159]\n"
    "s_nop 0\n"
    "s_waitcnt lgkmcnt(8)\n"
    "v_mfma_f32_16x16x32_bf16 v[128:131], v[96:99], a[192:195], v[128:131]\n"
    "ds_read_b64_tr_b16 v[120:121], v67 offset:32768\n"
    "v_mfma_f32_16x16x32_bf16 v[128:131], v[100:103], a[196:199], v[128:131]\n"
    "buffer_load_dwordx4 v56, s[36:39], s64 offen offset:0 lds\n"
    "v_mfma_f32_16x16x32_bf16 v[160:163], v[100:103], a[204:207], v[160:163]\n"
    "ds_read_b64_tr_b16 v[122:123], v67 offset:33792\n"
    "v_mfma_f32_16x16x32_bf16 v[160:163], v[96:99], a[200:203], v[160:163]\n"
    "v_mfma_f32_16x16x32_bf16 v[192:195], v[96:99], a[208:211], v[192:195]\n"
    "ds_read_b64_tr_b16 v[124:125], v67 offset:40960\n"
    "v_mfma_f32_16x16x32_bf16 v[192:195], v[100:103], a[212:215], v[192:195]\n"
    "buffer_load_dwordx4 v57, s[36:39], s64 offen offset:1024 lds\n"
    "v_mfma_f32_16x16x32_bf16 v[224:227], v[100:103], a[220:223], v[224:227]\n"
    "ds_read_b64_tr_b16 v[126:127], v67 offset:41984\n"
    "v_mfma_f32_16x16x32_bf16 v[224:227], v[96:99], a[216:219], v[224:227]\n"
    "s_waitcnt lgkmcnt(8)\n"
    "v_mfma_f32_16x16x32_bf16 v[228:231], v[104:107], a[216:219], v[228:231]\n"
    "ds_read_b64_tr_b16 v[96:97], v64 offset:33280\n"
    "v_mfma_f32_16x16x32_bf16 v[228:231], v[108:111], a[220:223], v[228:231]\n"
    "v_mfma_f32_16x16x32_bf16 v[196:199], v[108:111], a[212:215], v[196:199]\n"
    "ds_read_b64_tr_b16 v[98:99], v64 offset:34304\n"
    "v_mfma_f32_16x16x32_bf16 v[196:199], v[104:107], a[208:211], v[196:199]\n"
    "buffer_load_dwordx4 v58, s[36:39], s64 offen offset:2048 lds\n"
    "v_mfma_f32_16x16x32_bf16 v[164:167], v[104:107], a[200:203], v[164:167]\n"
    "ds_read_b64_tr_b16 v[100:101], v64 offset:41472\n"
    "v_mfma_f32_16x16x32_bf16 v[164:167], v[108:111], a[204:207], v[164:167]\n"
    "v_cmp_le_u32 vcc, s70, v47\n"
    "v_mfma_f32_16x16x32_bf16 v[132:135], v[108:111], a[196:199], v[132:135]\n"
    "ds_read_b64_tr_b16 v[102:103], v64 offset:42496\n"
    "v_mfma_f32_16x16x32_bf16 v[132:135], v[104:107], a[192:195], v[132:135]\n"
    "s_or_b64 s[82:83], s[82:83], vcc\n"
    "v_cmp_le_u32 vcc, s70, v48\n"
    "s_waitcnt lgkmcnt(8)\n"
    "v_mfma_f32_16x16x32_bf16 v[136:139], v[112:115], a[192:195], v[136:139]\n"
    "ds_read_b64_tr_b16 v[104:105], v65 offset:33280\n"
    "v_mfma_f32_16x16x32_bf16 v[136:139], v[116:119], a[196:199], v[136:139]\n"
    "s_or_b64 s[82:83], s[82:83], vcc\n"
    "v_mfma_f32_16x16x32_bf16 v[168:171], v[116:119], a[204:207], v[168:171]\n"
    "ds_read_b64_tr_b16 v[106:107], v65 offset:34304\n"
    "buffer_load_dwordx4 v59, s[36:39], s64 offen offset:3072 lds\n"
    "v_mfma_f32_16x16x32_bf16 v[168:171], v[112:115], a[200:203], v[168:171]\n"
    "v_mfma_f32_16x16x32_bf16 v[200:203], v[112:115], a[208:211], v[200:203]\n"
    "ds_read_b64_tr_b16 v[108:109], v65 offset:41472\n"
    "v_mfma_f32_16x16x32_bf16 v[200:203], v[116:119], a[212:215], v[200:203]\n"
    "v_mov_b32 v38, s82\n"
    "v_mov_b32 v39, s83\n"
    "ds_write_b64 v41, v[38:39] offset:0\n"
    "v_mfma_f32_16x16x32_bf16 v[232:235], v[116:119], a[220:223], v[232:235]\n"
    "ds_read_b64_tr_b16 v[110:111], v65 offset:42496\n"
    "v_mfma_f32_16x16x32_bf16 v[232:235], v[112:115], a[216:219], v[232:235]\n"
    "s_ff0_i32_b64 s30, s[82:83]\n"
    "s_bfm_b64 vcc, s30, 0\n"
    "s_andn2_b64 vcc, vcc, s[80:81]\n"
    "s_bcnt1_i32_b64 s30, vcc\n"
    "s_cmp_lt_u32 s30, 64\n"
    "s_cselect_b32 s22, 0x10000, 0\n"
    "s_waitcnt lgkmcnt(9)\n"
    "v_mfma_f32_16x16x32_bf16 v[236:239], v[120:123], a[216:219], v[236:239]\n"
    "ds_read_b64_tr_b16 v[112:113], v66 offset:33280\n"
    "buffer_load_dwordx4 a[96:99], v54, s[40:43], s65 offen offset:0\n"
    "v_mfma_f32_16x16x32_bf16 v[236:239], v[124:127], a[220:223], v[236:239]\n"
    "s_bcnt1_i32_b64 s30, s[82:83]\n"
    "s_cmp_ge_u32 s30, 56\n"
    "s_cselect_b32 s22, 0, s22\n"
    "v_mfma_f32_16x16x32_bf16 v[204:207], v[124:127], a[212:215], v[204:207]\n"
    "ds_read_b64_tr_b16 v[114:115], v66 offset:34304\n"
    "v_mfma_f32_16x16x32_bf16 v[204:207], v[120:123], a[208:211], v[204:207]\n"
    "buffer_load_dword v48, v49, s[20:23], 0 offen sc0 sc1\n"
    "v_mfma_f32_16x16x32_bf16 v[172:175], v[120:123], a[200:203], v[172:175]\n"
    "ds_read_b64_tr_b16 v[116:117], v66 offset:41472\n"
    "v_mfma_f32_16x16x32_bf16 v[172:175], v[124:127], a[204:207], v[172:175]\n"
    "buffer_load_dwordx4 a[100:103], v54, s[40:43], s65 offen offset:1024\n"
    "v_mfma_f32_16x16x32_bf16 v[140:143], v[124:127], a[196:199], v[140:143]\n"
    "ds_read_b64_tr_b16 v[118:119], v66 offset:42496\n"
    "v_mfma_f32_16x16x32_bf16 v[140:143], v[120:123], a[192:195], v[140:143]\n"
    "s_waitcnt lgkmcnt(9)\n"
    "v_mfma_f32_16x16x32_bf16 v[144:147], v[96:99], a[192:195], v[144:147]\n"
    "ds_read_b64_tr_b16 v[120:121], v67 offset:33280\n"
    "v_mfma_f32_16x16x32_bf16 v[144:147], v[100:103], a[196:199], v[144:147]\n"
    "v_mfma_f32_16x16x32_bf16 v[176:179], v[100:103], a[204:207], v[176:179]\n"
    "ds_read_b64_tr_b16 v[122:123], v67 offset:34304\n"
    "v_mfma_f32_16x16x32_bf16 v[176:179], v[96:99], a[200:203], v[176:179]\n"
    "buffer_load_dwordx4 a[104:107], v54, s[40:43], s65 offen offset:2048\n"
    "v_mfma_f32_16x16x32_bf16 v[208:211], v[96:99], a[208:211], v[208:211]\n"
    "ds_read_b64_tr_b16 v[124:125], v67 offset:41472\n"
    "v_mfma_f32_16x16x32_bf16 v[208:211], v[100:103], a[212:215], v[208:211]\n"
    "v_mfma_f32_16x16x32_bf16 v[240:243], v[100:103], a[220:223], v[240:243]\n"
    "ds_read_b64_tr_b16 v[126:127], v67 offset:42496\n"
    "v_mfma_f32_16x16x32_bf16 v[240:243], v[96:99], a[216:219], v[240:243]\n"
    "s_waitcnt lgkmcnt(8)\n"
    "s_waitcnt vmcnt(45)\n"
    "s_barrier\n"
    "v_mfma_f32_16x16x32_bf16 v[244:247], v[104:107], a[216:219], v[244:247]\n"
    "ds_read_b64_tr_b16 v[96:97], v64 offset:49152\n"
    "v_mfma_f32_16x16x32_bf16 v[244:247], v[108:111], a[220:223], v[244:247]\n"
    "ds_read_b64 v[36:37], v40 offset:0\n"
    "v_mfma_f32_16x16x32_bf16 v[212:215], v[108:111], a[212:215], v[212:215]\n"
    "ds_read_b64_tr_b16 v[98:99], v64 offset:50176\n"
    "buffer_load_dwordx4 a[108:111], v54, s[40:43], s65 offen offset:3072\n"
    "v_mfma_f32_16x16x32_bf16 v[212:215], v[104:107], a[208:211], v[212:215]\n"
    "v_mfma_f32_16x16x32_bf16 v[180:183], v[104:107], a[200:203], v[180:183]\n"
    "ds_read_b64_tr_b16 v[100:101], v64 offset:57344\n"
    "v_mfma_f32_16x16x32_bf16 v[180:183], v[108:111], a[204:207], v[180:183]\n"
    "v_mfma_f32_16x16x32_bf16 v[148:151], v[108:111], a[196:199], v[148:151]\n"
    "ds_read_b64_tr_b16 v[102:103], v64 offset:58368\n"
    "v_mfma_f32_16x16x32_bf16 v[148:151], v[104:107], a[192:195], v[148:151]\n"
    "buffer_load_dwordx4 a[112:115], v55, s[40:43], s65 offen offset:0\n"
    "s_waitcnt lgkmcnt(9)\n"
    "v_mfma_f32_16x16x32_bf16 v[152:155], v[112:115], a[192:195], v[152:155]\n"
    "ds_read_b64_tr_b16 v[104:105], v65 offset:49152\n"
    "v_mfma_f32_16x16x32_bf16 v[152:155], v[116:119], a[196:199], v[152:155]\n"
    "v_mfma_f32_16x16x32_bf16 v[184:187], v[116:119], a[204:207], v[184:187]\n"
    "ds_read_b64_tr_b16 v[106:107], v65 offset:50176\n"
    "v_mfma_f32_16x16x32_bf16 v[184:187], v[112:115], a[200:203], v[184:187]\n"
    "v_mfma_f32_16x16x32_bf16 v[216:219], v[112:115], a[208:211], v[216:219]\n"
    "ds_read_b64_tr_b16 v[108:109], v65 offset:57344\n"
    "v_mfma_f32_16x16x32_bf16 v[216:219], v[116:119], a[212:215], v[216:219]\n"
    "buffer_load_dwordx4 a[116:119], v55, s[40:43], s65 offen offset:1024\n"
    "v_mfma_f32_16x16x32_bf16 v[248:251], v[116:119], a[220:223], v[248:251]\n"
    "ds_read_b64_tr_b16 v[110:111], v65 offset:58368\n"
    "v_mfma_f32_16x16x32_bf16 v[248:251], v[112:115], a[216:219], v[248:251]\n"
    "s_waitcnt lgkmcnt(9)\n"
    "v_mfma_f32_16x16x32_bf16 v[252:255], v[120:123], a[216:219], v[252:255]\n"
    "ds_read_b64_tr_b16 v[112:113], v66 offset:49152\n"
    "v_mfma_f32_16x16x32_bf16 v[252:255], v[124:127], a[220:223], v[252:255]\n"
    "v_mfma_f32_16x16x32_bf16 v[220:223], v[124:127], a[212:215], v[220:223]\n"
    "ds_read_b64_tr_b16 v[114:115], v66 offset:50176\n"
    "v_mfma_f32_16x16x32_bf16 v[220:223], v[120:123], a[208:211], v[220:223]\n"
    "buffer_load_dwordx4 a[120:123], v55, s[40:43], s65 offen offset:2048\n"
    "v_mfma_f32_16x16x32_bf16 v[188:191], v[120:123], a[200:203], v[188:191]\n"
    "ds_read_b64_tr_b16 v[116:117], v66 offset:57344\n"
    "v_mfma_f32_16x16x32_bf16 v[188:191], v[124:127], a[204:207], v[188:191]\n"
    "v_mfma_f32_16x16x32_bf16 v[156:159], v[124:127], a[196:199], v[156:159]\n"
    "ds_read_b64_tr_b16 v[118:119], v66 offset:58368\n"
    "v_mfma_f32_16x16x32_bf16 v[156:159], v[120:123], a[192:195], v[156:159]\n"
    "s_waitcnt lgkmcnt(8)\n"
    "v_mfma_f32_16x16x32_bf16 v[128:131], v[96:99], a[224:227], v[128:131]\n"
    "ds_read_b64_tr_b16 v[120:121], v67 offset:49152\n"
    "v_mfma_f32_16x16x32_bf16 v[128:131], v[100:103], a[228:231], v[128:131]\n"
    "v_readfirstlane_b32 vcc_lo, v36\n"
    "v_readfirstlane_b32 vcc_hi, v37\n"
    "v_mfma_f32_16x16x32_bf16 v[160:163], v[100:103], a[236:239], v[160:163]\n"
    "ds_read_b64_tr_b16 v[122:123], v67 offset:50176\n"
    "buffer_load_dwordx4 a[124:127], v55, s[40:43], s65 offen offset:3072\n"
    "v_mfma_f32_16x16x32_bf16 v[160:163], v[96:99], a[232:235], v[160:163]\n"
    "s_or_b64 s[72:73], s[72:73], vcc\n"
    "v_mfma_f32_16x16x32_bf16 v[192:195], v[96:99], a[240:243], v[192:195]\n"
    "ds_read_b64_tr_b16 v[124:125], v67 offset:57344\n"
    "s_andn2_b64 s[76:77], s[72:73], s[80:81]\n"
    "s_cbranch_scc0 1291f\n"
    "1292:\n"
    "s_andn2_b64 s[14:15], s[18:19], s[80:81]\n"
    "s_cselect_b64 s[76:77], s[14:15], s[76:77]\n"
    "s_ff1_i32_b64 s69, s[76:77]\n"
    "v_mfma_f32_16x16x32_bf16 v[192:195], v[100:103], a[244:247], v[192:195]\n"
    "v_readlane_b32 s65, v53, s69\n"
    "s_bitset1_b64 s[80:81], s69\n"
    "v_mfma_f32_16x16x32_bf16 v[224:227], v[100:103], a[252:255], v[224:227]\n"
    "ds_read_b64_tr_b16 v[126:127], v67 offset:58368\n"
    "s_nop 0\n"
    "v_mfma_f32_16x16x32_bf16 v[224:227], v[96:99], a[248:251], v[224:227]\n"
    "s_nop 0\n"
    "s_lshl_b32 s64, s65, 5\n"
    "s_waitcnt lgkmcnt(8)\n"
    "v_mfma_f32_16x16x32_bf16 v[228:231], v[104:107], a[248:251], v[228:231]\n"
    "ds_read_b64_tr_b16 v[96:97], v64 offset:49664\n"
    "s_add_u32 m0, s67, 65536\n"
    "v_mfma_f32_16x16x32_bf16 v[228:231], v[108:111], a[252:255], v[228:231]\n"
    "s_nop 0\n"
    "v_mfma_f32_16x16x32_bf16 v[196:199], v[108:111], a[244:247], v[196:199]\n"
    "ds_read_b64_tr_b16 v[98:99], v64 offset:50688\n"
    "v_mfma_f32_16x16x32_bf16 v[196:199], v[104:107], a[240:243], v[196:199]\n"
    "buffer_load_dwordx4 v56, s[36:39], s64 offen offset:0 lds\n"
    "v_mfma_f32_16x16x32_bf16 v[164:167], v[104:107], a[232:235], v[164:167]\n"
    "ds_read_b64_tr_b16 v[100:101], v64 offset:57856\n"
    "v_mfma_f32_16x16x32_bf16 v[164:167], v[108:111], a[236:239], v[164:167]\n"
    "buffer_load_dwordx4 v57, s[36:39], s64 offen offset:1024 lds\n"
    "v_mfma_f32_16x16x32_bf16 v[132:135], v[108:111], a[228:231], v[132:135]\n"
    "ds_read_b64_tr_b16 v[102:103], v64 offset:58880\n"
    "v_mfma_f32_16x16x32_bf16 v[132:135], v[104:107], a[224:227], v[132:135]\n"
    "s_waitcnt lgkmcnt(8)\n"
    "v_mfma_f32_16x16x32_bf16 v[136:139], v[112:115], a[224:227], v[136:139]\n"
    "ds_read_b64_tr_b16 v[104:105], v65 offset:49664\n"
    "v_mfma_f32_16x16x32_bf16 v[136:139], v[116:119], a[228:231], v[136:139]\n"
    "buffer_load_dwordx4 v58, s[36:39], s64 offen offset:2048 lds\n"
    "v_mfma_f32_16x16x32_bf16 v[168:171], v[116:119], a[236:239], v[168:171]\n"
    "ds_read_b64_tr_b16 v[106:107], v65 offset:50688\n"
    "v_mfma_f32_16x16x32_bf16 v[168:171], v[112:115], a[232:235], v[168:171]\n"
    "v_mfma_f32_16x16x32_bf16 v[200:203], v[112:115], a[240:243], v[200:203]\n"
    "ds_read_b64_tr_b16 v[108:109], v65 offset:57856\n"
    "v_mfma_f32_16x16x32_bf16 v[200:203], v[116:119], a[244:247], v[200:203]\n"
    "buffer_load_dwordx4 v59, s[36:39], s64 offen offset:3072 lds\n"
    "v_mfma_f32_16x16x32_bf16 v[232:235], v[116:119], a[252:255], v[232:235]\n"
    "ds_read_b64_tr_b16 v[110:111], v65 offset:58880\n"
    "v_mfma_f32_16x16x32_bf16 v[232:235], v[112:115], a[248:251], v[232:235]\n"
    "s_waitcnt lgkmcnt(8)\n"
    "v_mfma_f32_16x16x32_bf16 v[236:239], v[120:123], a[248:251], v[236:239]\n"
    "ds_read_b64_tr_b16 v[112:113], v66 offset:49664\n"
    "v_mfma_f32_16x16x32_bf16 v[236:239], v[124:127], a[252:255], v[236:239]\n"
    "v_mfma_f32_16x16x32_bf16 v[204:207], v[124:127], a[244:247], v[204:207]\n"
    "ds_read_b64_tr_b16 v[114:115], v66 offset:50688\n"
    "v_mfma_f32_16x16x32_bf16 v[204:207], v[120:123], a[240:243], v[204:207]\n"
    "buffer_load_dwordx4 a[128:131], v54, s[40:43], s65 offen offset:0\n"
    "v_mfma_f32_16x16x32_bf16 v[172:175], v[120:123], a[232:235], v[172:175]\n"
    "ds_read_b64_tr_b16 v[116:117], v66 offset:57856\n"
    "v_mfma_f32_16x16x32_bf16 v[172:175], v[124:127], a[236:239], v[172:175]\n"
    "v_mfma_f32_16x16x32_bf16 v[140:143], v[124:127], a[228:231], v[140:143]\n"
    "ds_read_b64_tr_b16 v[118:119], v66 offset:58880\n"
    "v_mfma_f32_16x16x32_bf16 v[140:143], v[120:123], a[224:227], v[140:143]\n"
    "buffer_load_dwordx4 a[132:135], v54, s[40:43], s65 offen offset:1024\n"
    "s_waitcnt lgkmcnt(8)\n"
    "v_mfma_f32_16x16x32_bf16 v[144:147], v[96:99], a[224:227], v[144:147]\n"
    "ds_read_b64_tr_b16 v[120:121], v67 offset:49664\n"
    "v_mfma_f32_16x16x32_bf16 v[144:147], v[100:103], a[228:231], v[144:147]\n"
    "v_mfma_f32_16x16x32_bf16 v[176:179], v[100:103], a[236:239], v[176:179]\n"
    "ds_read_b64_tr_b16 v[122:123], v67 offset:50688\n"
    "v_mfma_f32_16x16x32_bf16 v[176:179], v[96:99], a[232:235], v[176:179]\n"
    "buffer_load_dwordx4 a[136:139], v54, s[40:43], s65 offen offset:2048\n"
    "v_mfma_f32_16x16x32_bf16 v[208:211], v[96:99], a[240:243], v[208:211]\n"
    "ds_read_b64_tr_b16 v[124:125], v67 offset:57856\n"
    "v_mfma_f32_16x16x32_bf16 v[208:211], v[100:103], a[244:247], v[208:211]\n"
    "v_mfma_f32_16x16x32_bf16 v[240:243], v[100:103], a[252:255], v[240:243]\n"
    "ds_read_b64_tr_b16 v[126:127], v67 offset:58880\n"
    "v_mfma_f32_16x16x32_bf16 v[240:243], v[96:99], a[248:251], v[240:243]\n"
    "buffer_load_dwordx4 a[140:143], v54, s[40:43], s65 offen offset:3072\n"
    "s_waitcnt lgkmcnt(8)\n"
    "s_waitcnt vmcnt(46)\n"
    "s_barrier\n"
    "v_mfma_f32_16x16x32_bf16 v[244:247], v[104:107], a[248:251], v[244:247]\n"
    "ds_read_b64_tr_b16 v[96:97], v60 offset:0\n"
    "v_mfma_f32_16x16x32_bf16 v[244:247], v[108:111], a[252:255], v[244:247]\n"
    "v_mfma_f32_16x16x32_bf16 v[212:215], v[108:111], a[244:247], v[212:215]\n"
    "ds_read_b64_tr_b16 v[98:99], v60 offset:1024\n"
    "v_mfma_f32_16x16x32_bf16 v[212:215], v[104:107], a[240:243], v[212:215]\n"
    "buffer_load_dwordx4 a[144:147], v55, s[40:43], s65 offen offset:0\n"
    "v_mfma_f32_16x16x32_bf16 v[180:183], v[104:107], a[232:235], v[180:183]\n"
    "ds_read_b64_tr_b16 v[100:101], v60 offset:8192\n"
    "v_mfma_f32_16x16x32_bf16 v[180:183], v[108:111], a[236:239], v[180:183]\n"
    "v_mfma_f32_16x16x32_bf16 v[148:151], v[108:111], a[228:231], v[148:151]\n"
    "ds_read_b64_tr_b16 v[102:103], v60 offset:9216\n"
    "v_mfma_f32_16x16x32_bf16 v[148:151], v[104:107], a[224:227], v[148:151]\n"
    "buffer_load_dwordx4 a[148:151], v55, s[40:43], s65 offen offset:1024\n"
    "s_waitcnt lgkmcnt(8)\n"
    "v_mfma_f32_16x16x32_bf16 v[152:155], v[112:115], a[224:227], v[152:155]\n"
    "ds_read_b64_tr_b16 v[104:105], v61 offset:0\n"
    "v_mfma_f32_16x16x32_bf16 v[152:155], v[116:119], a[228:231], v[152:155]\n"
    "v_mfma_f32_16x16x32_bf16 v[184:187], v[116:119], a[236:239], v[184:187]\n"
    "ds_read_b64_tr_b16 v[106:107], v61 offset:1024\n"
    "v_mfma_f32_16x16x32_bf16 v[184:187], v[112:115], a[232:235], v[184:187]\n"
    "buffer_load_dwordx4 a[152:155], v55, s[40:43], s65 offen offset:2048\n"
    "v_mfma_f32_16x16x32_bf16 v[216:219], v[112:115], a[240:243], v[216:219]\n"
    "ds_read_b64_tr_b16 v[108:109], v61 offset:8192\n"
    "v_mfma_f32_16x16x32_bf16 v[216:219], v[116:119], a[244:247], v[216:219]\n"
    "v_mfma_f32_16x16x32_bf16 v[248:251], v[116:119], a[252:255], v[248:251]\n"
    "ds_read_b64_tr_b16 v[110:111], v61 offset:9216\n"
    "v_mfma_f32_16x16x32_bf16 v[248:251], v[112:115], a[248:251], v[248:251]\n"
    "buffer_load_dwordx4 a[156:159], v55, s[40:43], s65 offen offset:3072\n"
    "s_waitcnt lgkmcnt(8)\n"
    "v_mfma_f32_16x16x32_bf16 v[252:255], v[120:123], a[248:251], v[252:255]\n"
    "ds_read_b64_tr_b16 v[112:113], v62 offset:0\n"
    "v_mfma_f32_16x16x32_bf16 v[252:255], v[124:127], a[252:255], v[252:255]\n"
    "v_mfma_f32_16x16x32_bf16 v[220:223], v[124:127], a[244:247], v[220:223]\n"
    "ds_read_b64_tr_b16 v[114:115], v62 offset:1024\n"
    "s_andn2_b64 s[76:77], s[72:73], s[80:81]\n"
    "s_cbranch_scc0 1301f\n"
    "1302:\n"
    "s_andn2_b64 s[14:15], s[18:19], s[80:81]\n"
    "s_cselect_b64 s[76:77], s[14:15], s[76:77]\n"
    "s_ff1_i32_b64 s69, s[76:77]\n"
    "v_mfma_f32_16x16x32_bf16 v[220:223], v[120:123], a[240:243], v[220:223]\n"
    "v_readlane_b32 s65, v53, s69\n"
    "s_bitset1_b64 s[80:81], s69\n"
    "v_mfma_f32_16x16x32_bf16 v[188:191], v[120:123], a[232:235], v[188:191]\n"
    "ds_read_b64_tr_b16 v[116:117], v62 offset:8192\n"
    "s_nop 0\n"
    "v_mfma_f32_16x16x32_bf16 v[188:191], v[124:127], a[236:239], v[188:191]\n"
    "s_nop 0\n"
    "s_lshl_b32 s64, s65, 5\n"
    "v_mfma_f32_16x16x32_bf16 v[156:159], v[124:127], a[228:231], v[156:159]\n"
    "ds_read_b64_tr_b16 v[118:119], v62 offset:9216\n"
    "s_add_u32 m0, s67, 81920\n"
    "v_mfma_f32_16x16x32_bf16 v[156:159], v[120:123], a[224:227], v[156:159]\n"
    "s_nop 0\n"
    "s_sub_u32 s68, s68, 1\n"
    "s_cmp_lg_u32 s68, 0\n"
    "s_cbranch_scc1 1b\n"
    "s_waitcnt lgkmcnt(8)\n"
    "v_mfma_f32_16x16x32_bf16 v[128:131], v[96:99], a[0:3], v[128:131]\n"
    "ds_read_b64_tr_b16 v[120:121], v63 offset:0\n"
    "v_mfma_f32_16x16x32_bf16 v[128:131], v[100:103], a[4:7], v[128:131]\n"
    "buffer_load_dwordx4 v56, s[36:39], s64 offen offset:0 lds\n"
    "v_mfma_f32_16x16x32_bf16 v[160:163], v[100:103], a[12:15], v[160:163]\n"
    "ds_read_b64_tr_b16 v[122:123], v63 offset:1024\n"
    "v_mfma_f32_16x16x32_bf16 v[160:163], v[96:99], a[8:11], v[160:163]\n"
    "buffer_load_dwordx4 v57, s[36:39], s64 offen offset:1024 lds\n"
    "v_mfma_f32_16x16x32_bf16 v[192:195], v[96:99], a[16:19], v[192:195]\n"
    "ds_read_b64_tr_b16 v[124:125], v63 offset:8192\n"
    "v_mfma_f32_16x16x32_bf16 v[192:195], v[100:103], a[20:23], v[192:195]\n"
    "v_mfma_f32_16x16x32_bf16 v[224:227], v[100:103], a[28:31], v[224:227]\n"
    "ds_read_b64_tr_b16 v[126:127], v63 offset:9216\n"
    "v_mfma_f32_16x16x32_bf16 v[224:227], v[96:99], a[24:27], v[224:227]\n"
    "buffer_load_dwordx4 v58, s[36:39], s64 offen offset:2048 lds\n"
    "s_waitcnt lgkmcnt(8)\n"
    "v_mfma_f32_16x16x32_bf16 v[228:231], v[104:107], a[24:27], v[228:231]\n"
    "ds_read_b64_tr_b16 v[96:97], v60 offset:512\n"
    "v_mfma_f32_16x16x32_bf16 v[228:231], v[108:111], a[28:31], v[228:231]\n"
    "v_mfma_f32_16x16x32_bf16 v[196:199], v[108:111], a[20:23], v[196:199]\n"
    "ds_read_b64_tr_b16 v[98:99], v60 offset:1536\n"
    "v_mfma_f32_16x16x32_bf16 v[196:199], v[104:107], a[16:19], v[196:199]\n"
    "buffer_load_dwordx4 v59, s[36:39], s64 offen offset:3072 lds\n"
    "v_mfma_f32_16x16x32_bf16 v[164:167], v[104:107], a[8:11], v[164:167]\n"
    "ds_read_b64_tr_b16 v[100:101], v60 offset:8704\n"
    "v_mfma_f32_16x16x32_bf16 v[164:167], v[108:111], a[12:15], v[164:167]\n"
    "v_cmp_le_u32 vcc, s70, v47\n"
    "v_mfma_f32_16x16x32_bf16 v[132:135], v[108:111], a[4:7], v[132:135]\n"
    "ds_read_b64_tr_b16 v[102:103], v60 offset:9728\n"
    "v_mfma_f32_16x16x32_bf16 v[132:135], v[104:107], a[0:3], v[132:135]\n"
    "s_or_b64 s[82:83], s[82:83], vcc\n"
    "v_cmp_le_u32 vcc, s70, v48\n"
    "s_waitcnt lgkmcnt(8)\n"
    "v_mfma_f32_16x16x32_bf16 v[136:139], v[112:115], a[0:3], v[136:139]\n"
    "ds_read_b64_tr_b16 v[104:105], v61 offset:512\n"
    "v_mfma_f32_16x16x32_bf16 v[136:139], v[116:119], a[4:7], v[136:139]\n"
    "s_or_b64 s[82:83], s[82:83], vcc\n"
    "v_mfma_f32_16x16x32_bf16 v[168:171], v[116:119], a[12:15], v[168:171]\n"
    "ds_read_b64_tr_b16 v[106:107], v61 offset:1536\n"
    "buffer_load_dwordx4 a[160:163], v54, s[40:43], s65 offen offset:0\n"
    "v_mfma_f32_16x16x32_bf16 v[168:171], v[112:115], a[8:11], v[168:171]\n"
    "v_mfma_f32_16x16x32_bf16 v[200:203], v[112:115], a[16:19], v[200:203]\n"
    "ds_read_b64_tr_b16 v[108:109], v61 offset:8704\n"
    "v_mfma_f32_16x16x32_bf16 v[200:203], v[116:119], a[20:23], v[200:203]\n"
    "v_mov_b32 v38, s82\n"
    "v_mov_b32 v39, s83\n"
    "ds_write_b64 v41, v[38:39] offset:0\n"
    "v_mfma_f32_16x16x32_bf16 v[232:235], v[116:119], a[28:31], v[232:235]\n"
    "ds_read_b64_tr_b16 v[110:111], v61 offset:9728\n"
    "buffer_load_dwordx4 a[164:167], v54, s[40:43], s65 offen offset:1024\n"
    "v_mfma_f32_16x16x32_bf16 v[232:235], v[112:115], a[24:27], v[232:235]\n"
    "s_ff0_i32_b64 s30, s[82:83]\n"
    "s_bfm_b64 vcc, s30, 0\n"
    "s_andn2_b64 vcc, vcc, s[80:81]\n"
    "s_bcnt1_i32_b64 s30, vcc\n"
    "s_cmp_lt_u32 s30, 64\n"
    "s_cselect_b32 s22, 0x10000, 0\n"
    "s_waitcnt lgkmcnt(9)\n"
    "v_mfma_f32_16x16x32_bf16 v[236:239], v[120:123], a[24:27], v[236:239]\n"
    "ds_read_b64_tr_b16 v[112:113], v62 offset:512\n"
    "v_mfma_f32_16x16x32_bf16 v[236:239], v[124:127], a[28:31], v[236:239]\n"
    "s_bcnt1_i32_b64 s30, s[82:83]\n"
    "s_cmp_ge_u32 s30, 56\n"
    "s_cselect_b32 s22, 0, s22\n"
    "v_mfma_f32_16x16x32_bf16 v[204:207], v[124:127], a[20:23], v[204:207]\n"
    "ds_read_b64_tr_b16 v[114:115], v62 offset:1536\n"
    "buffer_load_dwordx4 a[168:171], v54, s[40:43], s65 offen offset:2048\n"
    "v_mfma_f32_16x16x32_bf16 v[204:207], v[120:123], a[16:19], v[204:207]\n"
    "buffer_load_dword v47, v49, s[20:23], 0 offen sc0 sc1\n"
    "v_mfma_f32_16x16x32_bf16 v[172:175], v[120:123], a[8:11], v[172:175]\n"
    "ds_read_b64_tr_b16 v[116:117], v62 offset:8704\n"
    "v_mfma_f32_16x16x32_bf16 v[172:175], v[124:127], a[12:15], v[172:175]\n"
    "buffer_load_dwordx4 a[172:175], v54, s[40:43], s65 offen offset:3072\n"
    "v_mfma_f32_16x16x32_bf16 v[140:143], v[124:127], a[4:7], v[140:143]\n"
    "ds_read_b64_tr_b16 v[118:119], v62 offset:9728\n"
    "v_mfma_f32_16x16x32_bf16 v[140:143], v[120:123], a[0:3], v[140:143]\n"
    "s_waitcnt lgkmcnt(9)\n"
    "v_mfma_f32_16x16x32_bf16 v[144:147], v[96:99], a[0:3], v[144:147]\n"
    "ds_read_b64_tr_b16 v[120:121], v63 offset:512\n"
    "v_mfma_f32_16x16x32_bf16 v[144:147], v[100:103], a[4:7], v[144:147]\n"
    "buffer_load_dwordx4 a[176:179], v55, s[40:43], s65 offen offset:0\n"
    "v_mfma_f32_16x16x32_bf16 v[176:179], v[100:103], a[12:15], v[176:179]\n"
    "ds_read_b64_tr_b16 v[122:123], v63 offset:1536\n"
    "v_mfma_f32_16x16x32_bf16 v[176:179], v[96:99], a[8:11], v[176:179]\n"
    "v_mfma_f32_16x16x32_bf16 v[208:211], v[96:99], a[16:19], v[208:211]\n"
    "ds_read_b64_tr_b16 v[124:125], v63 offset:8704\n"
    "v_mfma_f32_16x16x32_bf16 v[208:211], v[100:103], a[20:23], v[208:211]\n"
    "buffer_load_dwordx4 a[180:183], v55, s[40:43], s65 offen offset:1024\n"
    "v_mfma_f32_16x16x32_bf16 v[240:243], v[100:103], a[28:31], v[240:243]\n"
    "ds_read_b64_tr_b16 v[126:127], v63 offset:9728\n"
    "v_mfma_f32_16x16x32_bf16 v[240:243], v[96:99], a[24:27], v[240:243]\n"
    "s_waitcnt lgkmcnt(8)\n"
    "s_waitcnt vmcnt(48)\n"
    "s_barrier\n"
    "v_mfma_f32_16x16x32_bf16 v[244:247], v[104:107], a[24:27], v[244:247]\n"
    "ds_read_b64_tr_b16 v[96:97], v60 offset:16384\n"
    "v_mfma_f32_16x16x32_bf16 v[244:247], v[108:111], a[28:31], v[244:247]\n"
    "ds_read_b64 v[36:37], v40 offset:0\n"
    "v_mfma_f32_16x16x32_bf16 v[212:215], v[108:111], a[20:23], v[212:215]\n"
    "ds_read_b64_tr_b16 v[98:99], v60 offset:17408\n"
    "v_mfma_f32_16x16x32_bf16 v[212:215], v[104:107], a[16:19], v[212:215]\n"
    "buffer_load_dwordx4 a[184:187], v55, s[40:43], s65 offen offset:2048\n"
    "v_mfma_f32_16x16x32_bf16 v[180:183], v[104:107], a[8:11], v[180:183]\n"
    "ds_read_b64_tr_b16 v[100:101], v60 offset:24576\n"
    "v_mfma_f32_16x16x32_bf16 v[180:183], v[108:111], a[12:15], v[180:183]\n"
    "buffer_load_dwordx4 a[188:191], v55, s[40:43], s65 offen offset:3072\n"
    "v_mfma_f32_16x16x32_bf16 v[148:151], v[108:111], a[4:7], v[148:151]\n"
    "ds_read_b64_tr_b16 v[102:103], v60 offset:25600\n"
    "v_mfma_f32_16x16x32_bf16 v[148:151], v[104:107], a[0:3], v[148:151]\n"
    "s_waitcnt lgkmcnt(9)\n"
    "v_mfma_f32_16x16x32_bf16 v[152:155], v[112:115], a[0:3], v[152:155]\n"
    "ds_read_b64_tr_b16 v[104:105], v61 offset:16384\n"
    "s_andn2_b64 s[76:77], s[72:73], s[80:81]\n"
    "s_cbranch_scc0 1311f\n"
    "1312:\n"
    "s_andn2_b64 s[14:15], s[18:19], s[80:81]\n"
    "s_cselect_b64 s[76:77], s[14:15], s[76:77]\n"
    "s_ff1_i32_b64 s69, s[76:77]\n"
    "v_mfma_f32_16x16x32_bf16 v[152:155], v[116:119], a[4:7], v[152:155]\n"
    "v_readlane_b32 s65, v53, s69\n"
    "s_bitset1_b64 s[80:81], s69\n"
    "v_mfma_f32_16x16x32_bf16 v[184:187], v[116:119], a[12:15], v[184:187]\n"
    "ds_read_b64_tr_b16 v[106:107], v61 offset:17408\n"
    "s_nop 0\n"
    "v_mfma_f32_16x16x32_bf16 v[184:187], v[112:115], a[8:11], v[184:187]\n"
    "s_nop 0\n"
    "s_lshl_b32 s64, s65, 5\n"
    "v_mfma_f32_16x16x32_bf16 v[216:219], v[112:115], a[16:19], v[216:219]\n"
    "ds_read_b64_tr_b16 v[108:109], v61 offset:24576\n"
    "s_add_u32 m0, s67, 98304\n"
    "v_mfma_f32_16x16x32_bf16 v[216:219], v[116:119], a[20:23], v[216:219]\n"
    "s_nop 0\n"
    "v_mfma_f32_16x16x32_bf16 v[248:251], v[116:119], a[28:31], v[248:251]\n"
    "ds_read_b64_tr_b16 v[110:111], v61 offset:25600\n"
    "v_mfma_f32_16x16x32_bf16 v[248:251], v[112:115], a[24:27], v[248:251]\n"
    "buffer_load_dwordx4 v56, s[36:39], s64 offen offset:0 lds\n"
    "s_waitcnt lgkmcnt(9)\n"
    "v_mfma_f32_16x16x32_bf16 v[252:255], v[120:123], a[24:27], v[252:255]\n"
    "ds_read_b64_tr_b16 v[112:113], v62 offset:16384\n"
    "v_mfma_f32_16x16x32_bf16 v[252:255], v[124:127], a[28:31], v[252:255]\n"
    "buffer_load_dwordx4 v57, s[36:39], s64 offen offset:1024 lds\n"
    "v_mfma_f32_16x16x32_bf16 v[220:223], v[124:127], a[20:23], v[220:223]\n"
    "ds_read_b64_tr_b16 v[114:115], v62 offset:17408\n"
    "v_mfma_f32_16x16x32_bf16 v[220:223], v[120:123], a[16:19], v[220:223]\n"
    "v_mfma_f32_16x16x32_bf16 v[188:191], v[120:123], a[8:11], v[188:191]\n"
    "ds_read_b64_tr_b16 v[116:117], v62 offset:24576\n"
    "v_mfma_f32_16x16x32_bf16 v[188:191], v[124:127], a[12:15], v[188:191]\n"
    "buffer_load_dwordx4 v58, s[36:39], s64 offen offset:2048 lds\n"
    "v_mfma_f32_16x16x32_bf16 v[156:159], v[124:127], a[4:7], v[156:159]\n"
    "ds_read_b64_tr_b16 v[118:119], v62 offset:25600\n"
    "v_mfma_f32_16x16x32_bf16 v[156:159], v[120:123], a[0:3], v[156:159]\n"
    "s_waitcnt lgkmcnt(8)\n"
    "v_mfma_f32_16x16x32_bf16 v[128:131], v[96:99], a[32:35], v[128:131]\n"
    "ds_read_b64_tr_b16 v[120:121], v63 offset:16384\n"
    "v_mfma_f32_16x16x32_bf16 v[128:131], v[100:103], a[36:39], v[128:131]\n"
    "v_readfirstlane_b32 vcc_lo, v36\n"
    "v_readfirstlane_b32 vcc_hi, v37\n"
    "v_mfma_f32_16x16x32_bf16 v[160:163], v[100:103], a[44:47], v[160:163]\n"
    "ds_read_b64_tr_b16 v[122:123], v63 offset:17408\n"
    "v_mfma_f32_16x16x32_bf16 v[160:163], v[96:99], a[40:43], v[160:163]\n"
    "s_or_b64 s[72:73], s[72:73], vcc\n"
    "v_mfma_f32_16x16x32_bf16 v[192:195], v[96:99], a[48:51], v[192:195]\n"
    "ds_read_b64_tr_b16 v[124:125], v63 offset:24576\n"
    "buffer_load_dwordx4 v59, s[36:39], s64 offen offset:3072 lds\n"
    "v_mfma_f32_16x16x32_bf16 v[192:195], v[100:103], a[52:55], v[192:195]\n"
    "v_mfma_f32_16x16x32_bf16 v[224:227], v[100:103], a[60:63], v[224:227]\n"
    "ds_read_b64_tr_b16 v[126:127], v63 offset:25600\n"
    "v_mfma_f32_16x16x32_bf16 v[224:227], v[96:99], a[56:59], v[224:227]\n"
    "buffer_load_dwordx4 a[192:195], v54, s[40:43], s65 offen offset:0\n"
    "s_waitcnt lgkmcnt(8)\n"
    "v_mfma_f32_16x16x32_bf16 v[228:231], v[104:107], a[56:59], v[228:231]\n"
    "ds_read_b64_tr_b16 v[96:97], v60 offset:16896\n"
    "v_mfma_f32_16x16x32_bf16 v[228:231], v[108:111], a[60:63], v[228:231]\n"
    "v_mfma_f32_16x16x32_bf16 v[196:199], v[108:111], a[52:55], v[196:199]\n"
    "ds_read_b64_tr_b16 v[98:99], v60 offset:17920\n"
    "v_mfma_f32_16x16x32_bf16 v[196:199], v[104:107], a[48:51], v[196:199]\n"
    "buffer_load_dwordx4 a[196:199], v54, s[40:43], s65 offen offset:1024\n"
    "v_mfma_f32_16x16x32_bf16 v[164:167], v[104:107], a[40:43], v[164:167]\n"
    "ds_read_b64_tr_b16 v[100:101], v60 offset:25088\n"
    "v_mfma_f32_16x16x32_bf16 v[164:167], v[108:111], a[44:47], v[164:167]\n"
    "v_mfma_f32_16x16x32_bf16 v[132:135], v[108:111], a[36:39], v[132:135]\n"
    "ds_read_b64_tr_b16 v[102:103], v60 offset:26112\n"
    "v_mfma_f32_16x16x32_bf16 v[132:135], v[104:107], a[32:35], v[132:135]\n"
    "buffer_load_dwordx4 a[200:203], v54, s[40:43], s65 offen offset:2048\n"
    "s_waitcnt lgkmcnt(8)\n"
    "v_mfma_f32_16x16x32_bf16 v[136:139], v[112:115], a[32:35], v[136:139]\n"
    "ds_read_b64_tr_b16 v[104:105], v61 offset:16896\n"
    "v_mfma_f32_16x16x32_bf16 v[136:139], v[116:119], a[36:39], v[136:139]\n"
    "v_mfma_f32_16x16x32_bf16 v[168:171], v[116:119], a[44:47], v[168:171]\n"
    "ds_read_b64_tr_b16 v[106:107], v61 offset:17920\n"
    "v_mfma_f32_16x16x32_bf16 v[168:171], v[112:115], a[40:43], v[168:171]\n"
    "buffer_load_dwordx4 a[204:207], v54, s[40:43], s65 offen offset:3072\n"
    "v_mfma_f32_16x16x32_bf16 v[200:203], v[112:115], a[48:51], v[200:203]\n"
    "ds_read_b64_tr_b16 v[108:109], v61 offset:25088\n"
    "v_mfma_f32_16x16x32_bf16 v[200:203], v[116:119], a[52:55], v[200:203]\n"
    "v_mfma_f32_16x16x32_bf16 v[232:235], v[116:119], a[60:63], v[232:235]\n"
    "ds_read_b64_tr_b16 v[110:111], v61 offset:26112\n"
    "v_mfma_f32_16x16x32_bf16 v[232:235], v[112:115], a[56:59], v[232:235]\n"
    "buffer_load_dwordx4 a[208:211], v55, s[40:43], s65 offen offset:0\n"
    "s_waitcnt lgkmcnt(8)\n"
    "v_mfma_f32_16x16x32_bf16 v[236:239], v[120:123], a[56:59], v[236:239]\n"
    "ds_read_b64_tr_b16 v[112:113], v62 offset:16896\n"
    "v_mfma_f32_16x16x32_bf16 v[236:239], v[124:127], a[60:63], v[236:239]\n"
    "v_mfma_f32_16x16x32_bf16 v[204:207], v[124:127], a[52:55], v[204:207]\n"
    "ds_read_b64_tr_b16 v[114:115], v62 offset:17920\n"
    "v_mfma_f32_16x16x32_bf16 v[204:207], v[120:123], a[48:51], v[204:207]\n"
    "buffer_load_dwordx4 a[212:215], v55, s[40:43], s65 offen offset:1024\n"
    "v_mfma_f32_16x16x32_bf16 v[172:175], v[120:123], a[40:43], v[172:175]\n"
    "ds_read_b64_tr_b16 v[116:117], v62 offset:25088\n"
    "v_mfma_f32_16x16x32_bf16 v[172:175], v[124:127], a[44:47], v[172:175]\n"
    "v_mfma_f32_16x16x32_bf16 v[140:143], v[124:127], a[36:39], v[140:143]\n"
    "ds_read_b64_tr_b16 v[118:119], v62 offset:26112\n"
    "v_mfma_f32_16x16x32_bf16 v[140:143], v[120:123], a[32:35], v[140:143]\n"
    "buffer_load_dwordx4 a[216:219], v55, s[40:43], s65 offen offset:2048\n"
    "s_waitcnt lgkmcnt(8)\n"
    "v_mfma_f32_16x16x32_bf16 v[144:147], v[96:99], a[32:35], v[144:147]\n"
    "ds_read_b64_tr_b16 v[120:121], v63 offset:16896\n"
    "v_mfma_f32_16x16x32_bf16 v[144:147], v[100:103], a[36:39], v[144:147]\n"
    "v_mfma_f32_16x16x32_bf16 v[176:179], v[100:103], a[44:47], v[176:179]\n"
    "ds_read_b64_tr_b16 v[122:123], v63 offset:17920\n"
    "v_mfma_f32_16x16x32_bf16 v[176:179], v[96:99], a[40:43], v[176:179]\n"
    "buffer_load_dwordx4 a[220:223], v55, s[40:43], s65 offen offset:3072\n"
    "v_mfma_f32_16x16x32_bf16 v[208:211], v[96:99], a[48:51], v[208:211]\n"
    "ds_read_b64_tr_b16 v[124:125], v63 offset:25088\n"
    "v_mfma_f32_16x16x32_bf16 v[208:211], v[100:103], a[52:55], v[208:211]\n"
    "v_mfma_f32_16x16x32_bf16 v[240:243], v[100:103], a[60:63], v[240:243]\n"
    "ds_read_b64_tr_b16 v[126:127], v63 offset:26112\n"
    "s_andn2_b64 s[76:77], s[72:73], s[80:81]\n"
    "s_cbranch_scc0 1321f\n"
    "1322:\n"
    "s_andn2_b64 s[14:15], s[18:19], s[80:81]\n"
    "s_cselect_b64 s[76:77], s[14:15], s[76:77]\n"
    "s_ff1_i32_b64 s69, s[76:77]\n"
    "v_mfma_f32_16x16x32_bf16 v[240:243], v[96:99], a[56:59], v[240:243]\n"
    "v_readlane_b32 s65, v53, s69\n"
    "s_bitset1_b64 s[80:81], s69\n"
    "s_waitcnt lgkmcnt(8)\n"
    "s_waitcnt vmcnt(50)\n"
    "s_barrier\n"
    "v_mfma_f32_16x16x32_bf16 v[244:247], v[104:107], a[56:59], v[244:247]\n"
    "ds_read_b64_tr_b16 v[96:97], v60 offset:32768\n"
    "s_nop 0\n"
    "v_mfma_f32_16x16x32_bf16 v[244:247], v[108:111], a[60:63], v[244:247]\n"
    "s_nop 0\n"
    "s_lshl_b32 s64, s65, 5\n"
    "v_mfma_f32_16x16x32_bf16 v[212:215], v[108:111], a[52:55], v[212:215]\n"
    "ds_read_b64_tr_b16 v[98:99], v60 offset:33792\n"
    "s_add_u32 m0, s67, 114688\n"
    "v_mfma_f32_16x16x32_bf16 v[212:215], v[104:107], a[48:51], v[212:215]\n"
    "s_nop 0\n"
    "v_mfma_f32_16x16x32_bf16 v[180:183], v[104:107], a[40:43], v[180:183]\n"
    "ds_read_b64_tr_b16 v[100:101], v60 offset:40960\n"
    "v_mfma_f32_16x16x32_bf16 v[180:183], v[108:111], a[44:47], v[180:183]\n"
    "buffer_load_dwordx4 v56, s[36:39], s64 offen offset:0 lds\n"
    "v_mfma_f32_16x16x32_bf16 v[148:151], v[108:111], a[36:39], v[148:151]\n"
    "ds_read_b64_tr_b16 v[102:103], v60 offset:41984\n"
    "v_mfma_f32_16x16x32_bf16 v[148:151], v[104:107], a[32:35], v[148:151]\n"
    "buffer_load_dwordx4 v57, s[36:39], s64 offen offset:1024 lds\n"
    "s_waitcnt lgkmcnt(8)\n"
    "v_mfma_f32_16x16x32_bf16 v[152:155], v[112:115], a[32:35], v[152:155]\n"
    "ds_read_b64_tr_b16 v[104:105], v61 offset:32768\n"
    "v_mfma_f32_16x16x32_bf16 v[152:155], v[116:119], a[36:39], v[152:155]\n"
    "v_mfma_f32_16x16x32_bf16 v[184:187], v[116:119], a[44:47], v[184:187]\n"
    "ds_read_b64_tr_b16 v[106:107], v61 offset:33792\n"
    "v_mfma_f32_16x16x32_bf16 v[184:187], v[112:115], a[40:43], v[184:187]\n"
    "buffer_load_dwordx4 v58, s[36:39], s64 offen offset:2048 lds\n"
    "v_mfma_f32_16x16x32_bf16 v[216:219], v[112:115], a[48:51], v[216:219]\n"
    "ds_read_b64_tr_b16 v[108:109], v61 offset:40960\n"
    "s_andn2_b64 s[16:17], s[72:73], s[80:81]\n"
    "v_mfma_f32_16x16x32_bf16 v[216:219], v[116:119], a[52:55], v[216:219]\n"
    "s_mov_b64 s[14:15], s[16:17]\n"
    "s_ff1_i32_b64 s69, s[14:15]\n"
    "v_mfma_f32_16x16x32_bf16 v[248:251], v[116:119], a[60:63], v[248:251]\n"
    "ds_read_b64_tr_b16 v[110:111], v61 offset:41984\n"
    "v_mfma_f32_16x16x32_bf16 v[248:251], v[112:115], a[56:59], v[248:251]\n"
    "buffer_load_dwordx4 v59, s[36:39], s64 offen offset:3072 lds\n"
    "s_waitcnt lgkmcnt(8)\n"
    "v_mfma_f32_16x16x32_bf16 v[252:255], v[120:123], a[56:59], v[252:255]\n"
    "ds_read_b64_tr_b16 v[112:113], v62 offset:32768\n"
    "s_bitset0_b64 s[14:15], s69\n"
    "v_mfma_f32_16x16x32_bf16 v[252:255], v[124:127], a[60:63], v[252:255]\n"
    "s_ff1_i32_b64 s69, s[14:15]\n"
    "s_bitset0_b64 s[14:15], s69\n"
    "v_mfma_f32_16x16x32_bf16 v[220:223], v[124:127], a[52:55], v[220:223]\n"
    "ds_read_b64_tr_b16 v[114:115], v62 offset:33792\n"
    "v_mfma_f32_16x16x32_bf16 v[220:223], v[120:123], a[48:51], v[220:223]\n"
    "v_mfma_f32_16x16x32_bf16 v[188:191], v[120:123], a[40:43], v[188:191]\n"
    "ds_read_b64_tr_b16 v[116:117], v62 offset:40960\n"
    "v_mfma_f32_16x16x32_bf16 v[188:191], v[124:127], a[44:47], v[188:191]\n"
    "buffer_load_dwordx4 a[224:227], v54, s[40:43], s65 offen offset:0\n"
    "v_mfma_f32_16x16x32_bf16 v[156:159], v[124:127], a[36:39], v[156:159]\n"
    "ds_read_b64_tr_b16 v[118:119], v62 offset:41984\n"
    "s_ff1_i32_b64 s69, s[14:15]\n"
    "v_mfma_f32_16x16x32_bf16 v[156:159], v[120:123], a[32:35], v[156:159]\n"
    "s_bitset0_b64 s[14:15], s69\n"
    "s_ff1_i32_b64 s69, s[14:15]\n"
    "s_waitcnt lgkmcnt(8)\n"
    "v_mfma_f32_16x16x32_bf16 v[128:131], v[96:99], a[64:67], v[128:131]\n"
    "ds_read_b64_tr_b16 v[120:121], v63 offset:32768\n"
    "v_mfma_f32_16x16x32_bf16 v[128:131], v[100:103], a[68:71], v[128:131]\n"
    "buffer_load_dwordx4 a[228:231], v54, s[40:43], s65 offen offset:1024\n"
    "v_mfma_f32_16x16x32_bf16 v[160:163], v[100:103], a[76:79], v[160:163]\n"
    "ds_read_b64_tr_b16 v[122:123], v63 offset:33792\n"
    "s_bitset0_b64 s[14:15], s69\n"
    "v_mfma_f32_16x16x32_bf16 v[160:163], v[96:99], a[72:75], v[160:163]\n"
    "s_xor_b64 s[16:17], s[16:17], s[14:15]\n"
    "s_mov_b64 s[18:19], s[14:15]\n"
    "v_mfma_f32_16x16x32_bf16 v[192:195], v[96:99], a[80:83], v[192:195]\n"
    "ds_read_b64_tr_b16 v[124:125], v63 offset:40960\n"
    "v_mfma_f32_16x16x32_bf16 v[192:195], v[100:103], a[84:87], v[192:195]\n"
    "buffer_load_dwordx4 a[232:235], v54, s[40:43], s65 offen offset:2048\n"
    "v_mfma_f32_16x16x32_bf16 v[224:227], v[100:103], a[92:95], v[224:227]\n"
    "ds_read_b64_tr_b16 v[126:127], v63 offset:41984\n"
    "s_ff1_i32_b64 s69, s[14:15]\n"
    "v_mfma_f32_16x16x32_bf16 v[224:227], v[96:99], a[88:91], v[224:227]\n"
    "s_bitset0_b64 s[14:15], s69\n"
    "s_ff1_i32_b64 s69, s[14:15]\n"
    "s_waitcnt lgkmcnt(8)\n"
    "v_mfma_f32_16x16x32_bf16 v[228:231], v[104:107], a[88:91], v[228:231]\n"
    "ds_read_b64_tr_b16 v[96:97], v60 offset:33280\n"
    "v_mfma_f32_16x16x32_bf16 v[228:231], v[108:111], a[92:95], v[228:231]\n"
    "buffer_load_dwordx4 a[236:239], v54, s[40:43], s65 offen offset:3072\n"
    "v_mfma_f32_16x16x32_bf16 v[196:199], v[108:111], a[84:87], v[196:199]\n"
    "ds_read_b64_tr_b16 v[98:99], v60 offset:34304\n"
    "s_bitset0_b64 s[14:15], s69\n"
    "v_mfma_f32_16x16x32_bf16 v[196:199], v[104:107], a[80:83], v[196:199]\n"
    "s_ff1_i32_b64 s69, s[14:15]\n"
    "s_bitset0_b64 s[14:15], s69\n"
    "v_mfma_f32_16x16x32_bf16 v[164:167], v[104:107], a[72:75], v[164:167]\n"
    "ds_read_b64_tr_b16 v[100:101], v60 offset:41472\n"
    "v_mfma_f32_16x16x32_bf16 v[164:167], v[108:111], a[76:79], v[164:167]\n"
    "v_cmp_le_u32 vcc, s70, v47\n"
    "v_mfma_f32_16x16x32_bf16 v[132:135], v[108:111], a[68:71], v[132:135]\n"
    "ds_read_b64_tr_b16 v[102:103], v60 offset:42496\n"
    "v_mfma_f32_16x16x32_bf16 v[132:135], v[104:107], a[64:67], v[132:135]\n"
    "s_or_b64 s[82:83], s[82:83], vcc\n"
    "v_cmp_le_u32 vcc, s70, v48\n"
    "s_waitcnt lgkmcnt(8)\n"
    "v_mfma_f32_16x16x32_bf16 v[136:139], v[112:115], a[64:67], v[136:139]\n"
    "ds_read_b64_tr_b16 v[104:105], v61 offset:33280\n"
    "buffer_load_dwordx4 a[240:243], v55, s[40:43], s65 offen offset:0\n"
    "v_mfma_f32_16x16x32_bf16 v[136:139], v[116:119], a[68:71], v[136:139]\n"
    "s_or_b64 s[82:83], s[82:83], vcc\n"
    "v_mfma_f32_16x16x32_bf16 v[168:171], v[116:119], a[76:79], v[168:171]\n"
    "ds_read_b64_tr_b16 v[106:107], v61 offset:34304\n"
    "s_ff1_i32_b64 s69, s[14:15]\n"
    "v_mfma_f32_16x16x32_bf16 v[168:171], v[112:115], a[72:75], v[168:171]\n"
    "s_bitset0_b64 s[14:15], s69\n"
    "s_xor_b64 s[18:19], s[18:19], s[14:15]\n"
    "v_mfma_f32_16x16x32_bf16 v[200:203], v[112:115], a[80:83], v[200:203]\n"
    "ds_read_b64_tr_b16 v[108:109], v61 offset:41472\n"
    "buffer_load_dwordx4 a[244:247], v55, s[40:43], s65 offen offset:1024\n"
    "v_mfma_f32_16x16x32_bf16 v[200:203], v[116:119], a[84:87], v[200:203]\n"
    "v_mov_b32 v38, s82\n"
    "v_mov_b32 v39, s83\n"
    "ds_write_b64 v41, v[38:39] offset:0\n"
    "v_mfma_f32_16x16x32_bf16 v[232:235], v[116:119], a[92:95], v[232:235]\n"
    "ds_read_b64_tr_b16 v[110:111], v61 offset:42496\n"
    "s_bcnt1_i32_b64 s69, s[18:19]\n"
    "v_mfma_f32_16x16x32_bf16 v[232:235], v[112:115], a[88:91], v[232:235]\n"
    "s_ff0_i32_b64 s30, s[82:83]\n"
    "s_bfm_b64 vcc, s30, 0\n"
    "s_andn2_b64 vcc, vcc, s[80:81]\n"
    "s_bcnt1_i32_b64 s30, vcc\n"
    "s_cmp_lt_u32 s30, 64\n"
    "s_cselect_b32 s22, 0x10000, 0\n"
    "s_waitcnt lgkmcnt(9)\n"
    "v_mfma_f32_16x16x32_bf16 v[236:239], v[120:123], a[88:91], v[236:239]\n"
    "ds_read_b64_tr_b16 v[112:113], v62 offset:33280\n"
    "s_cmp_eq_u32 s69, 4\n"
    "s_cselect_b64 s[16:17], s[16:17], 0\n"
    "s_cselect_b64 s[18:19], s[18:19], 0\n"
    "v_mfma_f32_16x16x32_bf16 v[236:239], v[124:127], a[92:95], v[236:239]\n"
    "s_bcnt1_i32_b64 s30, s[82:83]\n"
    "s_cmp_ge_u32 s30, 56\n"
    "s_cselect_b32 s22, 0, s22\n"
    "v_mfma_f32_16x16x32_bf16 v[204:207], v[124:127], a[84:87], v[204:207]\n"
    "ds_read_b64_tr_b16 v[114:115], v62 offset:34304\n"
    "s_cmp_eq_u32 s26, 4\n"
    "s_cselect_b64 s[14:15], s[18:19], s[16:17]\n"
    "s_cselect_b64 s[18:19], s[16:17], s[18:19]\n"
    "s_mov_b64 s[16:17], s[14:15]\n"
    "v_mfma_f32_16x16x32_bf16 v[204:207], v[120:123], a[80:83], v[204:207]\n"
    "buffer_load_dword v48, v49, s[20:23], 0 offen sc0 sc1\n"
    "v_mfma_f32_16x16x32_bf16 v[172:175], v[120:123], a[72:75], v[172:175]\n"
    "ds_read_b64_tr_b16 v[116:117], v62 offset:41472\n"
    "buffer_load_dwordx4 a[248:251], v55, s[40:43], s65 offen offset:2048\n"
    "v_mfma_f32_16x16x32_bf16 v[172:175], v[124:127], a[76:79], v[172:175]\n"
    "buffer_load_dwordx4 a[252:255], v55, s[40:43], s65 offen offset:3072\n"
    "v_mfma_f32_16x16x32_bf16 v[140:143], v[124:127], a[68:71], v[140:143]\n"
    "ds_read_b64_tr_b16 v[118:119], v62 offset:42496\n"
    "s_andn2_b64 s[76:77], s[72:73], s[80:81]\n"
    "s_cbranch_scc0 1331f\n"
    "1332:\n"
    "s_andn2_b64 s[14:15], s[16:17], s[80:81]\n"
    "s_cselect_b64 s[76:77], s[14:15], s[76:77]\n"
    "s_ff1_i32_b64 s69, s[76:77]\n"
    "v_mfma_f32_16x16x32_bf16 v[140:143], v[120:123], a[64:67], v[140:143]\n"
    "v_readlane_b32 s65, v53, s69\n"
    "s_bitset1_b64 s[80:81], s69\n"
    "s_waitcnt lgkmcnt(9)\n"
    "v_mfma_f32_16x16x32_bf16 v[144:147], v[96:99], a[64:67], v[144:147]\n"
    "ds_read_b64_tr_b16 v[120:121], v63 offset:33280\n"
    "s_nop 0\n"
    "v_mfma_f32_16x16x32_bf16 v[144:147], v[100:103], a[68:71], v[144:147]\n"
    "s_nop 0\n"
    "s_lshl_b32 s64, s65, 5\n"
    "v_mfma_f32_16x16x32_bf16 v[176:179], v[100:103], a[76:79], v[176:179]\n"
    "ds_read_b64_tr_b16 v[122:123], v63 offset:34304\n"
    "s_add_u32 m0, s67, 0\n"
    "v_mfma_f32_16x16x32_bf16 v[176:179], v[96:99], a[72:75], v[176:179]\n"
    "s_nop 0\n"
    "v_mfma_f32_16x16x32_bf16 v[208:211], v[96:99], a[80:83], v[208:211]\n"
    "ds_read_b64_tr_b16 v[124:125], v63 offset:41472\n"
    "buffer_load_dwordx4 v56, s[36:39], s64 offen offset:0 lds\n"
    "v_mfma_f32_16x16x32_bf16 v[208:211], v[100:103], a[84:87], v[208:211]\n"
    "v_mfma_f32_16x16x32_bf16 v[240:243], v[100:103], a[92:95], v[240:243]\n"
    "ds_read_b64_tr_b16 v[126:127], v63 offset:42496\n"
    "v_mfma_f32_16x16x32_bf16 v[240:243], v[96:99], a[88:91], v[240:243]\n"
    "buffer_load_dwordx4 v57, s[36:39], s64 offen offset:1024 lds\n"
    "s_waitcnt lgkmcnt(8)\n"
    "s_waitcnt vmcnt(52)\n"
    "s_barrier\n"
    "v_mfma_f32_16x16x32_bf16 v[244:247], v[104:107], a[88:91], v[244:247]\n"
    "ds_read_b64_tr_b16 v[96:97], v60 offset:49152\n"
    "v_mfma_f32_16x16x32_bf16 v[244:247], v[108:111], a[92:95], v[244:247]\n"
    "ds_read_b64 v[36:37], v40 offset:0\n"
    "v_mfma_f32_16x16x32_bf16 v[212:215], v[108:111], a[84:87], v[212:215]\n"
    "ds_read_b64_tr_b16 v[98:99], v60 offset:50176\n"
    "v_mfma_f32_16x16x32_bf16 v[212:215], v[104:107], a[80:83], v[212:215]\n"
    "v_mfma_f32_16x16x32_bf16 v[180:183], v[104:107], a[72:75], v[180:183]\n"
    "ds_read_b64_tr_b16 v[100:101], v60 offset:57344\n"
    "v_mfma_f32_16x16x32_bf16 v[180:183], v[108:111], a[76:79], v[180:183]\n"
    "buffer_load_dwordx4 v58, s[36:39], s64 offen offset:2048 lds\n"
    "v_mfma_f32_16x16x32_bf16 v[148:151], v[108:111], a[68:71], v[148:151]\n"
    "ds_read_b64_tr_b16 v[102:103], v60 offset:58368\n"
    "v_mfma_f32_16x16x32_bf16 v[148:151], v[104:107], a[64:67], v[148:151]\n"
    "s_waitcnt lgkmcnt(9)\n"
    "v_mfma_f32_16x16x32_bf16 v[152:155], v[112:115], a[64:67], v[152:155]\n"
    "ds_read_b64_tr_b16 v[104:105], v61 offset:49152\n"
    "v_mfma_f32_16x16x32_bf16 v[152:155], v[116:119], a[68:71], v[152:155]\n"
    "v_mfma_f32_16x16x32_bf16 v[184:187], v[116:119], a[76:79], v[184:187]\n"
    "ds_read_b64_tr_b16 v[106:107], v61 offset:50176\n"
    "v_mfma_f32_16x16x32_bf16 v[184:187], v[112:115], a[72:75], v[184:187]\n"
    "buffer_load_dwordx4 v59, s[36:39], s64 offen offset:3072 lds\n"
    "v_mfma_f32_16x16x32_bf16 v[216:219], v[112:115], a[80:83], v[216:219]\n"
    "ds_read_b64_tr_b16 v[108:109], v61 offset:57344\n"
    "v_mfma_f32_16x16x32_bf16 v[216:219], v[116:119], a[84:87], v[216:219]\n"
    "v_mfma_f32_16x16x32_bf16 v[248:251], v[116:119], a[92:95], v[248:251]\n"
    "ds_read_b64_tr_b16 v[110:111], v61 offset:58368\n"
    "v_mfma_f32_16x16x32_bf16 v[248:251], v[112:115], a[88:91], v[248:251]\n"
    "s_waitcnt lgkmcnt(9)\n"
    "v_mfma_f32_16x16x32_bf16 v[252:255], v[120:123], a[88:91], v[252:255]\n"
    "ds_read_b64_tr_b16 v[112:113], v62 offset:49152\n"
    "v_mfma_f32_16x16x32_bf16 v[252:255], v[124:127], a[92:95], v[252:255]\n"
    "buffer_load_dwordx4 a[0:3], v54, s[40:43], s65 offen offset:0\n"
    "v_mfma_f32_16x16x32_bf16 v[220:223], v[124:127], a[84:87], v[220:223]\n"
    "ds_read_b64_tr_b16 v[114:115], v62 offset:50176\n"
    "v_mfma_f32_16x16x32_bf16 v[220:223], v[120:123], a[80:83], v[220:223]\n"
    "v_mfma_f32_16x16x32_bf16 v[188:191], v[120:123], a[72:75], v[188:191]\n"
    "ds_read_b64_tr_b16 v[116:117], v62 offset:57344\n"
    "v_mfma_f32_16x16x32_bf16 v[188:191], v[124:127], a[76:79], v[188:191]\n"
    "v_mfma_f32_16x16x32_bf16 v[156:159], v[124:127], a[68:71], v[156:159]\n"
    "ds_read_b64_tr_b16 v[118:119], v62 offset:58368\n"
    "v_mfma_f32_16x16x32_bf16 v[156:159], v[120:123], a[64:67], v[156:159]\n"
    "buffer_load_dwordx4 a[4:7], v54, s[40:43], s65 offen offset:1024\n"
    "s_waitcnt lgkmcnt(8)\n"
    "v_mfma_f32_16x16x32_bf16 v[128:131], v[96:99], a[96:99], v[128:131]\n"
    "ds_read_b64_tr_b16 v[120:121], v63 offset:49152\n"
    "v_mfma_f32_16x16x32_bf16 v[128:131], v[100:103], a[100:103], v[128:131]\n"
    "v_readfirstlane_b32 vcc_lo, v36\n"
    "v_readfirstlane_b32 vcc_hi, v37\n"
    "v_mfma_f32_16x16x32_bf16 v[160:163], v[100:103], a[108:111], v[160:163]\n"
    "ds_read_b64_tr_b16 v[122:123], v63 offset:50176\n"
    "v_mfma_f32_16x16x32_bf16 v[160:163], v[96:99], a[104:107], v[160:163]\n"
    "s_or_b64 s[72:73], s[72:73], vcc\n"
    "v_mfma_f32_16x16x32_bf16 v[192:195], v[96:99], a[112:115], v[192:195]\n"
    "ds_read_b64_tr_b16 v[124:125], v63 offset:57344\n"
    "v_mfma_f32_16x16x32_bf16 v[192:195], v[100:103], a[116:119], v[192:195]\n"
    "buffer_load_dwordx4 a[8:11], v54, s[40:43], s65 offen offset:2048\n"
    "v_mfma_f32_16x16x32_bf16 v[224:227], v[100:103], a[124:127], v[224:227]\n"
    "ds_read_b64_tr_b16 v[126:127], v63 offset:58368\n"
    "v_mfma_f32_16x16x32_bf16 v[224:227], v[96:99], a[120:123], v[224:227]\n"
    "s_waitcnt lgkmcnt(8)\n"
    "v_mfma_f32_16x16x32_bf16 v[228:231], v[104:107], a[120:123], v[228:231]\n"
    "ds_read_b64_tr_b16 v[96:97], v60 offset:49664\n"
    "v_mfma_f32_16x16x32_bf16 v[228:231], v[108:111], a[124:127], v[228:231]\n"
    "v_mfma_f32_16x16x32_bf16 v[196:199], v[108:111], a[116:119], v[196:199]\n"
    "ds_read_b64_tr_b16 v[98:99], v60 offset:50688\n"
    "v_mfma_f32_16x16x32_bf16 v[196:199], v[104:107], a[112:115], v[196:199]\n"
    "buffer_load_dwordx4 a[12:15], v54, s[40:43], s65 offen offset:3072\n"
    "v_mfma_f32_16x16x32_bf16 v[164:167], v[104:107], a[104:107], v[164:167]\n"
    "ds_read_b64_tr_b16 v[100:101], v60 offset:57856\n"
    "v_mfma_f32_16x16x32_bf16 v[164:167], v[108:111], a[108:111], v[164:167]\n"
    "v_mfma_f32_16x16x32_bf16 v[132:135], v[108:111], a[100:103], v[132:135]\n"
    "ds_read_b64_tr_b16 v[102:103], v60 offset:58880\n"
    "v_mfma_f32_16x16x32_bf16 v[132:135], v[104:107], a[96:99], v[132:135]\n"
    "s_waitcnt lgkmcnt(8)\n"
    "v_mfma_f32_16x16x32_bf16 v[136:139], v[112:115], a[96:99], v[136:139]\n"
    "ds_read_b64_tr_b16 v[104:105], v61 offset:49664\n"
    "v_mfma_f32_16x16x32_bf16 v[136:139], v[116:119], a[100:103], v[136:139]\n"
    "buffer_load_dwordx4 a[16:19], v55, s[40:43], s65 offen offset:0\n"
    "v_mfma_f32_16x16x32_bf16 v[168:171], v[116:119], a[108:111], v[168:171]\n"
    "ds_read_b64_tr_b16 v[106:107], v61 offset:50688\n"
    "v_mfma_f32_16x16x32_bf16 v[168:171], v[112:115], a[104:107], v[168:171]\n"
    "v_mfma_f32_16x16x32_bf16 v[200:203], v[112:115], a[112:115], v[200:203]\n"
    "ds_read_b64_tr_b16 v[108:109], v61 offset:57856\n"
    "v_mfma_f32_16x16x32_bf16 v[200:203], v[116:119], a[116:119], v[200:203]\n"
    "v_mfma_f32_16x16x32_bf16 v[232:235], v[116:119], a[124:127], v[232:235]\n"
    "ds_read_b64_tr_b16 v[110:111], v61 offset:58880\n"
    "v_mfma_f32_16x16x32_bf16 v[232:235], v[112:115], a[120:123], v[232:235]\n"
    "buffer_load_dwordx4 a[20:23], v55, s[40:43], s65 offen offset:1024\n"
    "s_waitcnt lgkmcnt(8)\n"
    "v_mfma_f32_16x16x32_bf16 v[236:239], v[120:123], a[120:123], v[236:239]\n"
    "ds_read_b64_tr_b16 v[112:113], v62 offset:49664\n"
    "v_mfma_f32_16x16x32_bf16 v[236:239], v[124:127], a[124:127], v[236:239]\n"
    "v_mfma_f32_16x16x32_bf16 v[204:207], v[124:127], a[116:119], v[204:207]\n"
    "ds_read_b64_tr_b16 v[114:115], v62 offset:50688\n"
    "v_mfma_f32_16x16x32_bf16 v[204:207], v[120:123], a[112:115], v[204:207]\n"
    "v_mfma_f32_16x16x32_bf16 v[172:175], v[120:123], a[104:107], v[172:175]\n"
    "ds_read_b64_tr_b16 v[116:117], v62 offset:57856\n"
    "v_mfma_f32_16x16x32_bf16 v[172:175], v[124:127], a[108:111], v[172:175]\n"
    "buffer_load_dwordx4 a[24:27], v55, s[40:43], s65 offen offset:2048\n"
    "v_mfma_f32_16x16x32_bf16 v[140:143], v[124:127], a[100:103], v[140:143]\n"
    "ds_read_b64_tr_b16 v[118:119], v62 offset:58880\n"
    "v_mfma_f32_16x16x32_bf16 v[140:143], v[120:123], a[96:99], v[140:143]\n"
    "s_waitcnt lgkmcnt(8)\n"
    "v_mfma_f32_16x16x32_bf16 v[144:147], v[96:99], a[96:99], v[144:147]\n"
    "ds_read_b64_tr_b16 v[120:121], v63 offset:49664\n"
    "v_mfma_f32_16x16x32_bf16 v[144:147], v[100:103], a[100:103], v[144:147]\n"
    "v_mfma_f32_16x16x32_bf16 v[176:179], v[100:103], a[108:111], v[176:179]\n"
    "ds_read_b64_tr_b16 v[122:123], v63 offset:50688\n"
    "v_mfma_f32_16x16x32_bf16 v[176:179], v[96:99], a[104:107], v[176:179]\n"
    "buffer_load_dwordx4 a[28:31], v55, s[40:43], s65 offen offset:3072\n"
    "v_mfma_f32_16x16x32_bf16 v[208:211], v[96:99], a[112:115], v[208:211]\n"
    "ds_read_b64_tr_b16 v[124:125], v63 offset:57856\n"
    "v_mfma_f32_16x16x32_bf16 v[208:211], v[100:103], a[116:119], v[208:211]\n"
    "v_mfma_f32_16x16x32_bf16 v[240:243], v[100:103], a[124:127], v[240:243]\n"
    "ds_read_b64_tr_b16 v[126:127], v63 offset:58880\n"
    "s_andn2_b64 s[76:77], s[72:73], s[80:81]\n"
    "s_cbranch_scc0 1341f\n"
    "1342:\n"
    "s_andn2_b64 s[14:15], s[16:17], s[80:81]\n"
    "s_cselect_b64 s[76:77], s[14:15], s[76:77]\n"
    "s_ff1_i32_b64 s69, s[76:77]\n"
    "v_mfma_f32_16x16x32_bf16 v[240:243], v[96:99], a[120:123], v[240:243]\n"
    "v_readlane_b32 s65, v53, s69\n"
    "s_bitset1_b64 s[80:81], s69\n"
    "s_waitcnt lgkmcnt(8)\n"
    "s_waitcnt vmcnt(50)\n"
    "s_barrier\n"
    "v_mfma_f32_16x16x32_bf16 v[244:247], v[104:107], a[120:123], v[244:247]\n"
    "ds_read_b64_tr_b16 v[96:97], v64 offset:0\n"
    "s_nop 0\n"
    "v_mfma_f32_16x16x32_bf16 v[244:247], v[108:111], a[124:127], v[244:247]\n"
    "s_nop 0\n"
    "s_lshl_b32 s64, s65, 5\n"
    "v_mfma_f32_16x16x32_bf16 v[212:215], v[108:111], a[116:119], v[212:215]\n"
    "ds_read_b64_tr_b16 v[98:99], v64 offset:1024\n"
    "s_add_u32 m0, s67, 16384\n"
    "v_mfma_f32_16x16x32_bf16 v[212:215], v[104:107], a[112:115], v[212:215]\n"
    "s_nop 0\n"
    "v_mfma_f32_16x16x32_bf16 v[180:183], v[104:107], a[104:107], v[180:183]\n"
    "ds_read_b64_tr_b16 v[100:101], v64 offset:8192\n"
    "v_mfma_f32_16x16x32_bf16 v[180:183], v[108:111], a[108:111], v[180:183]\n"
    "buffer_load_dwordx4 v56, s[36:39], s64 offen offset:0 lds\n"
    "v_mfma_f32_16x16x32_bf16 v[148:151], v[108:111], a[100:103], v[148:151]\n"
    "ds_read_b64_tr_b16 v[102:103], v64 offset:9216\n"
    "v_mfma_f32_16x16x32_bf16 v[148:151], v[104:107], a[96:99], v[148:151]\n"
    "s_waitcnt lgkmcnt(8)\n"
    "v_mfma_f32_16x16x32_bf16 v[152:155], v[112:115], a[96:99], v[152:155]\n"
    "ds_read_b64_tr_b16 v[104:105], v65 offset:0\n"
    "v_mfma_f32_16x16x32_bf16 v[152:155], v[116:119], a[100:103], v[152:155]\n"
    "buffer_load_dwordx4 v57, s[36:39], s64 offen offset:1024 lds\n"
    "v_mfma_f32_16x16x32_bf16 v[184:187], v[116:119], a[108:111], v[184:187]\n"
    "ds_read_b64_tr_b16 v[106:107], v65 offset:1024\n"
    "v_mfma_f32_16x16x32_bf16 v[184:187], v[112:115], a[104:107], v[184:187]\n"
    "v_mfma_f32_16x16x32_bf16 v[216:219], v[112:115], a[112:115], v[216:219]\n"
    "ds_read_b64_tr_b16 v[108:109], v65 offset:8192\n"
    "v_mfma_f32_16x16x32_bf16 v[216:219], v[116:119], a[116:119], v[216:219]\n"
    "v_mfma_f32_16x16x32_bf16 v[248:251], v[116:119], a[124:127], v[248:251]\n"
    "ds_read_b64_tr_b16 v[110:111], v65 offset:9216\n"
    "v_mfma_f32_16x16x32_bf16 v[248:251], v[112:115], a[120:123], v[248:251]\n"
    "buffer_load_dwordx4 v58, s[36:39], s64 offen offset:2048 lds\n"
    "s_waitcnt lgkmcnt(8)\n"
    "v_mfma_f32_16x16x32_bf16 v[252:255], v[120:123], a[120:123], v[252:255]\n"
    "ds_read_b64_tr_b16 v[112:113], v66 offset:0\n"
    "v_mfma_f32_16x16x32_bf16 v[252:255], v[124:127], a[124:127], v[252:255]\n"
    "v_mfma_f32_16x16x32_bf16 v[220:223], v[124:127], a[116:119], v[220:223]\n"
    "ds_read_b64_tr_b16 v[114:115], v66 offset:1024\n"
    "v_mfma_f32_16x16x32_bf16 v[220:223], v[120:123], a[112:115], v[220:223]\n"
    "v_mfma_f32_16x16x32_bf16 v[188:191], v[120:123], a[104:107], v[188:191]\n"
    "ds_read_b64_tr_b16 v[116:117], v66 offset:8192\n"
    "v_mfma_f32_16x16x32_bf16 v[188:191], v[124:127], a[108:111], v[188:191]\n"
    "buffer_load_dwordx4 v59, s[36:39], s64 offen offset:3072 lds\n"
    "v_mfma_f32_16x16x32_bf16 v[156:159], v[124:127], a[100:103], v[156:159]\n"
    "ds_read_b64_tr_b16 v[118:119], v66 offset:9216\n"
    "v_mfma_f32_16x16x32_bf16 v[156:159], v[120:123], a[96:99], v[156:159]\n"
    "s_waitcnt lgkmcnt(8)\n"
    "v_mfma_f32_16x16x32_bf16 v[128:131], v[96:99], a[128:131], v[128:131]\n"
    "ds_read_b64_tr_b16 v[120:121], v67 offset:0\n"
    "v_mfma_f32_16x16x32_bf16 v[128:131], v[100:103], a[132:135], v[128:131]\n"
    "v_mfma_f32_16x16x32_bf16 v[160:163], v[100:103], a[140:143], v[160:163]\n"
    "ds_read_b64_tr_b16 v[122:123], v67 offset:1024\n"
    "v_mfma_f32_16x16x32_bf16 v[160:163], v[96:99], a[136:139], v[160:163]\n"
    "buffer_load_dwordx4 a[32:35], v54, s[40:43], s65 offen offset:0\n"
    "v_mfma_f32_16x16x32_bf16 v[192:195], v[96:99], a[144:147], v[192:195]\n"
    "ds_read_b64_tr_b16 v[124:125], v67 offset:8192\n"
    "v_mfma_f32_16x16x32_bf16 v[192:195], v[100:103], a[148:151], v[192:195]\n"
    "v_mfma_f32_16x16x32_bf16 v[224:227], v[100:103], a[156:159], v[224:227]\n"
    "ds_read_b64_tr_b16 v[126:127], v67 offset:9216\n"
    "v_mfma_f32_16x16x32_bf16 v[224:227], v[96:99], a[152:155], v[224:227]\n"
    "s_waitcnt lgkmcnt(8)\n"
    "v_mfma_f32_16x16x32_bf16 v[228:231], v[104:107], a[152:155], v[228:231]\n"
    "ds_read_b64_tr_b16 v[96:97], v64 offset:512\n"
    "v_mfma_f32_16x16x32_bf16 v[228:231], v[108:111], a[156:159], v[228:231]\n"
    "buffer_load_dwordx4 a[36:39], v54, s[40:43], s65 offen offset:1024\n"
    "v_mfma_f32_16x16x32_bf16 v[196:199], v[108:111], a[148:151], v[196:199]\n"
    "ds_read_b64_tr_b16 v[98:99], v64 offset:1536\n"
    "v_mfma_f32_16x16x32_bf16 v[196:199], v[104:107], a[144:147], v[196:199]\n"
    "v_mfma_f32_16x16x32_bf16 v[164:167], v[104:107], a[136:139], v[164:167]\n"
    "ds_read_b64_tr_b16 v[100:101], v64 offset:8704\n"
    "v_mfma_f32_16x16x32_bf16 v[164:167], v[108:111], a[140:143], v[164:167]\n"
    "v_cmp_le_u32 vcc, s70, v47\n"
    "v_mfma_f32_16x16x32_bf16 v[132:135], v[108:111], a[132:135], v[132:135]\n"
    "ds_read_b64_tr_b16 v[102:103], v64 offset:9728\n"
    "v_mfma_f32_16x16x32_bf16 v[132:135], v[104:107], a[128:131], v[132:135]\n"
    "s_or_b64 s[82:83], s[82:83], vcc\n"
    "v_cmp_le_u32 vcc, s70, v48\n"
    "s_waitcnt lgkmcnt(8)\n"
    "v_mfma_f32_16x16x32_bf16 v[136:139], v[112:115], a[128:131], v[136:139]\n"
    "ds_read_b64_tr_b16 v[104:105], v65 offset:512\n"
    "buffer_load_dwordx4 a[40:43], v54, s[40:43], s65 offen offset:2048\n"
    "v_mfma_f32_16x16x32_bf16 v[136:139], v[116:119], a[132:135], v[136:139]\n"
    "s_or_b64 s[82:83], s[82:83], vcc\n"
    "v_mfma_f32_16x16x32_bf16 v[168:171], v[116:119], a[140:143], v[168:171]\n"
    "ds_read_b64_tr_b16 v[106:107], v65 offset:1536\n"
    "v_mfma_f32_16x16x32_bf16 v[168:171], v[112:115], a[136:139], v[168:171]\n"
    "v_mfma_f32_16x16x32_bf16 v[200:203], v[112:115], a[144:147], v[200:203]\n"
    "ds_read_b64_tr_b16 v[108:109], v65 offset:8704\n"
    "v_mfma_f32_16x16x32_bf16 v[200:203], v[116:119], a[148:151], v[200:203]\n"
    "v_mov_b32 v38, s82\n"
    "v_mov_b32 v39, s83\n"
    "ds_write_b64 v41, v[38:39] offset:0\n"
    "v_mfma_f32_16x16x32_bf16 v[232:235], v[116:119], a[156:159], v[232:235]\n"
    "ds_read_b64_tr_b16 v[110:111], v65 offset:9728\n"
    "buffer_load_dwordx4 a[44:47], v54, s[40:43], s65 offen offset:3072\n"
    "v_mfma_f32_16x16x32_bf16 v[232:235], v[112:115], a[152:155], v[232:235]\n"
    "s_ff0_i32_b64 s30, s[82:83]\n"
    "s_bfm_b64 vcc, s30, 0\n"
    "s_andn2_b64 vcc, vcc, s[80:81]\n"
    "s_bcnt1_i32_b64 s30, vcc\n"
    "s_cmp_lt_u32 s30, 64\n"
    "s_cselect_b32 s22, 0x10000, 0\n"
    "s_waitcnt lgkmcnt(9)\n"
    "v_mfma_f32_16x16x32_bf16 v[236:239], v[120:123], a[152:155], v[236:239]\n"
    "ds_read_b64_tr_b16 v[112:113], v66 offset:512\n"
    "v_mfma_f32_16x16x32_bf16 v[236:239], v[124:127], a[156:159], v[236:239]\n"
    "s_bcnt1_i32_b64 s30, s[82:83]\n"
    "s_cmp_ge_u32 s30, 56\n"
    "s_cselect_b32 s22, 0, s22\n"
    "v_mfma_f32_16x16x32_bf16 v[204:207], v[124:127], a[148:151], v[204:207]\n"
    "ds_read_b64_tr_b16 v[114:115], v66 offset:1536\n"
    "v_mfma_f32_16x16x32_bf16 v[204:207], v[120:123], a[144:147], v[204:207]\n"
    "buffer_load_dword v47, v49, s[20:23], 0 offen sc0 sc1\n"
    "v_mfma_f32_16x16x32_bf16 v[172:175], v[120:123], a[136:139], v[172:175]\n"
    "ds_read_b64_tr_b16 v[116:117], v66 offset:8704\n"
    "buffer_load_dwordx4 a[48:51], v55, s[40:43], s65 offen offset:0\n"
    "v_mfma_f32_16x16x32_bf16 v[172:175], v[124:127], a[140:143], v[172:175]\n"
    "v_mfma_f32_16x16x32_bf16 v[140:143], v[124:127], a[132:135], v[140:143]\n"
    "ds_read_b64_tr_b16 v[118:119], v66 offset:9728\n"
    "v_mfma_f32_16x16x32_bf16 v[140:143], v[120:123], a[128:131], v[140:143]\n"
    "s_waitcnt lgkmcnt(9)\n"
    "v_mfma_f32_16x16x32_bf16 v[144:147], v[96:99], a[128:131], v[144:147]\n"
    "ds_read_b64_tr_b16 v[120:121], v67 offset:512\n"
    "v_mfma_f32_16x16x32_bf16 v[144:147], v[100:103], a[132:135], v[144:147]\n"
    "buffer_load_dwordx4 a[52:55], v55, s[40:43], s65 offen offset:1024\n"
    "v_mfma_f32_16x16x32_bf16 v[176:179], v[100:103], a[140:143], v[176:179]\n"
    "ds_read_b64_tr_b16 v[122:123], v67 offset:1536\n"
    "v_mfma_f32_16x16x32_bf16 v[176:179], v[96:99], a[136:139], v[176:179]\n"
    "v_mfma_f32_16x16x32_bf16 v[208:211], v[96:99], a[144:147], v[208:211]\n"
    "ds_read_b64_tr_b16 v[124:125], v67 offset:8704\n"
    "v_mfma_f32_16x16x32_bf16 v[208:211], v[100:103], a[148:151], v[208:211]\n"
    "v_mfma_f32_16x16x32_bf16 v[240:243], v[100:103], a[156:159], v[240:243]\n"
    "ds_read_b64_tr_b16 v[126:127], v67 offset:9728\n"
    "v_mfma_f32_16x16x32_bf16 v[240:243], v[96:99], a[152:155], v[240:243]\n"
    "buffer_load_dwordx4 a[56:59], v55, s[40:43], s65 offen offset:2048\n"
    "s_waitcnt lgkmcnt(8)\n"
    "s_waitcnt vmcnt(49)\n"
    "s_barrier\n"
    "v_mfma_f32_16x16x32_bf16 v[244:247], v[104:107], a[152:155], v[244:247]\n"
    "ds_read_b64_tr_b16 v[96:97], v64 offset:16384\n"
    "v_mfma_f32_16x16x32_bf16 v[244:247], v[108:111], a[156:159], v[244:247]\n"
    "ds_read_b64 v[36:37], v40 offset:0\n"
    "v_mfma_f32_16x16x32_bf16 v[212:215], v[108:111], a[148:151], v[212:215]\n"
    "ds_read_b64_tr_b16 v[98:99], v64 offset:17408\n"
    "v_mfma_f32_16x16x32_bf16 v[212:215], v[104:107], a[144:147], v[212:215]\n"
    "v_mfma_f32_16x16x32_bf16 v[180:183], v[104:107], a[136:139], v[180:183]\n"
    "ds_read_b64_tr_b16 v[100:101], v64 offset:24576\n"
    "v_mfma_f32_16x16x32_bf16 v[180:183], v[108:111], a[140:143], v[180:183]\n"
    "buffer_load_dwordx4 a[60:63], v55, s[40:43], s65 offen offset:3072\n"
    "v_mfma_f32_16x16x32_bf16 v[148:151], v[108:111], a[132:135], v[148:151]\n"
    "ds_read_b64_tr_b16 v[102:103], v64 offset:25600\n"
    "v_mfma_f32_16x16x32_bf16 v[148:151], v[104:107], a[128:131], v[148:151]\n"
    "s_waitcnt lgkmcnt(9)\n"
    "v_mfma_f32_16x16x32_bf16 v[152:155], v[112:115], a[128:131], v[152:155]\n"
    "ds_read_b64_tr_b16 v[104:105], v65 offset:16384\n"
    "s_andn2_b64 s[76:77], s[72:73], s[80:81]\n"
    "s_cbranch_scc0 1351f\n"
    "1352:\n"
    "s_andn2_b64 s[14:15], s[16:17], s[80:81]\n"
    "s_cselect_b64 s[76:77], s[14:15], s[76:77]\n"
    "s_ff1_i32_b64 s69, s[76:77]\n"
    "v_mfma_f32_16x16x32_bf16 v[152:155], v[116:119], a[132:135], v[152:155]\n"
    "v_readlane_b32 s65, v53, s69\n"
    "s_bitset1_b64 s[80:81], s69\n"
    "v_mfma_f32_16x16x32_bf16 v[184:187], v[116:119], a[140:143], v[184:187]\n"
    "ds_read_b64_tr_b16 v[106:107], v65 offset:17408\n"
    "s_nop 0\n"
    "v_mfma_f32_16x16x32_bf16 v[184:187], v[112:115], a[136:139], v[184:187]\n"
    "s_nop 0\n"
    "s_lshl_b32 s64, s65, 5\n"
    "v_mfma_f32_16x16x32_bf16 v[216:219], v[112:115], a[144:147], v[216:219]\n"
    "ds_read_b64_tr_b16 v[108:109], v65 offset:24576\n"
    "s_add_u32 m0, s67, 32768\n"
    "v_mfma_f32_16x16x32_bf16 v[216:219], v[116:119], a[148:151], v[216:219]\n"
    "s_nop 0\n"
    "v_mfma_f32_16x16x32_bf16 v[248:251], v[116:119], a[156:159], v[248:251]\n"
    "ds_read_b64_tr_b16 v[110:111], v65 offset:25600\n"
    "v_mfma_f32_16x16x32_bf16 v[248:251], v[112:115], a[152:155], v[248:251]\n"
    "buffer_load_dwordx4 v56, s[36:39], s64 offen offset:0 lds\n"
    "s_waitcnt lgkmcnt(9)\n"
    "v_mfma_f32_16x16x32_bf16 v[252:255], v[120:123], a[152:155], v[252:255]\n"
    "ds_read_b64_tr_b16 v[112:113], v66 offset:16384\n"
    "v_mfma_f32_16x16x32_bf16 v[252:255], v[124:127], a[156:159], v[252:255]\n"
    "v_mfma_f32_16x16x32_bf16 v[220:223], v[124:127], a[148:151], v[220:223]\n"
    "ds_read_b64_tr_b16 v[114:115], v66 offset:17408\n"
    "v_mfma_f32_16x16x32_bf16 v[220:223], v[120:123], a[144:147], v[220:223]\n"
    "buffer_load_dwordx4 v57, s[36:39], s64 offen offset:1024 lds\n"
    "v_mfma_f32_16x16x32_bf16 v[188:191], v[120:123], a[136:139], v[188:191]\n"
    "ds_read_b64_tr_b16 v[116:117], v66 offset:24576\n"
    "v_mfma_f32_16x16x32_bf16 v[188:191], v[124:127], a[140:143], v[188:191]\n"
    "v_mfma_f32_16x16x32_bf16 v[156:159], v[124:127], a[132:135], v[156:159]\n"
    "ds_read_b64_tr_b16 v[118:119], v66 offset:25600\n"
    "v_mfma_f32_16x16x32_bf16 v[156:159], v[120:123], a[128:131], v[156:159]\n"
    "s_waitcnt lgkmcnt(8)\n"
    "v_mfma_f32_16x16x32_bf16 v[128:131], v[96:99], a[160:163], v[128:131]\n"
    "ds_read_b64_tr_b16 v[120:121], v67 offset:16384\n"
    "v_mfma_f32_16x16x32_bf16 v[128:131], v[100:103], a[164:167], v[128:131]\n"
    "v_readfirstlane_b32 vcc_lo, v36\n"
    "v_readfirstlane_b32 vcc_hi, v37\n"
    "v_mfma_f32_16x16x32_bf16 v[160:163], v[100:103], a[172:175], v[160:163]\n"
    "ds_read_b64_tr_b16 v[122:123], v67 offset:17408\n"
    "buffer_load_dwordx4 v58, s[36:39], s64 offen offset:2048 lds\n"
    "v_mfma_f32_16x16x32_bf16 v[160:163], v[96:99], a[168:171], v[160:163]\n"
    "s_or_b64 s[72:73], s[72:73], vcc\n"
    "v_mfma_f32_16x16x32_bf16 v[192:195], v[96:99], a[176:179], v[192:195]\n"
    "ds_read_b64_tr_b16 v[124:125], v67 offset:24576\n"
    "v_mfma_f32_16x16x32_bf16 v[192:195], v[100:103], a[180:183], v[192:195]\n"
    "v_mfma_f32_16x16x32_bf16 v[224:227], v[100:103], a[188:191], v[224:227]\n"
    "ds_read_b64_tr_b16 v[126:127], v67 offset:25600\n"
    "v_mfma_f32_16x16x32_bf16 v[224:227], v[96:99], a[184:187], v[224:227]\n"
    "buffer_load_dwordx4 v59, s[36:39], s64 offen offset:3072 lds\n"
    "s_waitcnt lgkmcnt(8)\n"
    "v_mfma_f32_16x16x32_bf16 v[228:231], v[104:107], a[184:187], v[228:231]\n"
    "ds_read_b64_tr_b16 v[96:97], v64 offset:16896\n"
    "v_mfma_f32_16x16x32_bf16 v[228:231], v[108:111], a[188:191], v[228:231]\n"
    "v_mfma_f32_16x16x32_bf16 v[196:199], v[108:111], a[180:183], v[196:199]\n"
    "ds_read_b64_tr_b16 v[98:99], v64 offset:17920\n"
    "v_mfma_f32_16x16x32_bf16 v[196:199], v[104:107], a[176:179], v[196:199]\n"
    "v_mfma_f32_16x16x32_bf16 v[164:167], v[104:107], a[168:171], v[164:167]\n"
    "ds_read_b64_tr_b16 v[100:101], v64 offset:25088\n"
    "v_mfma_f32_16x16x32_bf16 v[164:167], v[108:111], a[172:175], v[164:167]\n"
    "buffer_load_dwordx4 a[64:67], v54, s[40:43], s65 offen offset:0\n"
    "v_mfma_f32_16x16x32_bf16 v[132:135], v[108:111], a[164:167], v[132:135]\n"
    "ds_read_b64_tr_b16 v[102:103], v64 offset:26112\n"
    "v_mfma_f32_16x16x32_bf16 v[132:135], v[104:107], a[160:163], v[132:135]\n"
    "s_waitcnt lgkmcnt(8)\n"
    "v_mfma_f32_16x16x32_bf16 v[136:139], v[112:115], a[160:163], v[136:139]\n"
    "ds_read_b64_tr_b16 v[104:105], v65 offset:16896\n"
    "v_mfma_f32_16x16x32_bf16 v[136:139], v[116:119], a[164:167], v[136:139]\n"
    "v_mfma_f32_16x16x32_bf16 v[168:171], v[116:119], a[172:175], v[168:171]\n"
    "ds_read_b64_tr_b16 v[106:107], v65 offset:17920\n"
    "v_mfma_f32_16x16x32_bf16 v[168:171], v[112:115], a[168:171], v[168:171]\n"
    "buffer_load_dwordx4 a[68:71], v54, s[40:43], s65 offen offset:1024\n"
    "v_mfma_f32_16x16x32_bf16 v[200:203], v[112:115], a[176:179], v[200:203]\n"
    "ds_read_b64_tr_b16 v[108:109], v65 offset:25088\n"
    "v_mfma_f32_16x16x32_bf16 v[200:203], v[116:119], a[180:183], v[200:203]\n"
    "v_mfma_f32_16x16x32_bf16 v[232:235], v[116:119], a[188:191], v[232:235]\n"
    "ds_read_b64_tr_b16 v[110:111], v65 offset:26112\n"
    "v_mfma_f32_16x16x32_bf16 v[232:235], v[112:115], a[184:187], v[232:235]\n"
    "s_waitcnt lgkmcnt(8)\n"
    "v_mfma_f32_16x16x32_bf16 v[236:239], v[120:123], a[184:187], v[236:239]\n"
    "ds_read_b64_tr_b16 v[112:113], v66 offset:16896\n"
    "v_mfma_f32_16x16x32_bf16 v[236:239], v[124:127], a[188:191], v[236:239]\n"
    "buffer_load_dwordx4 a[72:75], v54, s[40:43], s65 offen offset:2048\n"
    "v_mfma_f32_16x16x32_bf16 v[204:207], v[124:127], a[180:183], v[204:207]\n"
    "ds_read_b64_tr_b16 v[114:115], v66 offset:17920\n"
    "v_mfma_f32_16x16x32_bf16 v[204:207], v[120:123], a[176:179], v[204:207]\n"
    "v_mfma_f32_16x16x32_bf16 v[172:175], v[120:123], a[168:171], v[172:175]\n"
    "ds_read_b64_tr_b16 v[116:117], v66 offset:25088\n"
    "v_mfma_f32_16x16x32_bf16 v[172:175], v[124:127], a[172:175], v[172:175]\n"
    "v_mfma_f32_16x16x32_bf16 v[140:143], v[124:127], a[164:167], v[140:143]\n"
    "ds_read_b64_tr_b16 v[118:119], v66 offset:26112\n"
    "v_mfma_f32_16x16x32_bf16 v[140:143], v[120:123], a[160:163], v[140:143]\n"
    "buffer_load_dwordx4 a[76:79], v54, s[40:43], s65 offen offset:3072\n"
    "s_waitcnt lgkmcnt(8)\n"
    "v_mfma_f32_16x16x32_bf16 v[144:147], v[96:99], a[160:163], v[144:147]\n"
    "ds_read_b64_tr_b16 v[120:121], v67 offset:16896\n"
    "v_mfma_f32_16x16x32_bf16 v[144:147], v[100:103], a[164:167], v[144:147]\n"
    "v_mfma_f32_16x16x32_bf16 v[176:179], v[100:103], a[172:175], v[176:179]\n"
    "ds_read_b64_tr_b16 v[122:123], v67 offset:17920\n"
    "v_mfma_f32_16x16x32_bf16 v[176:179], v[96:99], a[168:171], v[176:179]\n"
    "v_mfma_f32_16x16x32_bf16 v[208:211], v[96:99], a[176:179], v[208:211]\n"
    "ds_read_b64_tr_b16 v[124:125], v67 offset:25088\n"
    "v_mfma_f32_16x16x32_bf16 v[208:211], v[100:103], a[180:183], v[208:211]\n"
    "buffer_load_dwordx4 a[80:83], v55, s[40:43], s65 offen offset:0\n"
    "v_mfma_f32_16x16x32_bf16 v[240:243], v[100:103], a[188:191], v[240:243]\n"
    "ds_read_b64_tr_b16 v[126:127], v67 offset:26112\n"
    "v_mfma_f32_16x16x32_bf16 v[240:243], v[96:99], a[184:187], v[240:243]\n"
    "s_waitcnt lgkmcnt(8)\n"
    "s_waitcnt vmcnt(47)\n"
    "s_barrier\n"
    "v_mfma_f32_16x16x32_bf16 v[244:247], v[104:107], a[184:187], v[244:247]\n"
    "ds_read_b64_tr_b16 v[96:97], v64 offset:32768\n"
    "v_mfma_f32_16x16x32_bf16 v[244:247], v[108:111], a[188:191], v[244:247]\n"
    "v_mfma_f32_16x16x32_bf16 v[212:215], v[108:111], a[180:183], v[212:215]\n"
    "ds_read_b64_tr_b16 v[98:99], v64 offset:33792\n"
    "v_mfma_f32_16x16x32_bf16 v[212:215], v[104:107], a[176:179], v[212:215]\n"
    "buffer_load_dwordx4 a[84:87], v55, s[40:43], s65 offen offset:1024\n"
    "v_mfma_f32_16x16x32_bf16 v[180:183], v[104:107], a[168:171], v[180:183]\n"
    "ds_read_b64_tr_b16 v[100:101], v64 offset:40960\n"
    "v_mfma_f32_16x16x32_bf16 v[180:183], v[108:111], a[172:175], v[180:183]\n"
    "v_mfma_f32_16x16x32_bf16 v[148:151], v[108:111], a[164:167], v[148:151]\n"
    "ds_read_b64_tr_b16 v[102:103], v64 offset:41984\n"
    "v_mfma_f32_16x16x32_bf16 v[148:151], v[104:107], a[160:163], v[148:151]\n"
    "s_waitcnt lgkmcnt(8)\n"
    "v_mfma_f32_16x16x32_bf16 v[152:155], v[112:115], a[160:163], v[152:155]\n"
    "ds_read_b64_tr_b16 v[104:105], v65 offset:32768\n"
    "v_mfma_f32_16x16x32_bf16 v[152:155], v[116:119], a[164:167], v[152:155]\n"
    "buffer_load_dwordx4 a[88:91], v55, s[40:43], s65 offen offset:2048\n"
    "v_mfma_f32_16x16x32_bf16 v[184:187], v[116:119], a[172:175], v[184:187]\n"
    "ds_read_b64_tr_b16 v[106:107], v65 offset:33792\n"
    "v_mfma_f32_16x16x32_bf16 v[184:187], v[112:115], a[168:171], v[184:187]\n"
    "v_mfma_f32_16x16x32_bf16 v[216:219], v[112:115], a[176:179], v[216:219]\n"
    "ds_read_b64_tr_b16 v[108:109], v65 offset:40960\n"
    "v_mfma_f32_16x16x32_bf16 v[216:219], v[116:119], a[180:183], v[216:219]\n"
    "v_mfma_f32_16x16x32_bf16 v[248:251], v[116:119], a[188:191], v[248:251]\n"
    "ds_read_b64_tr_b16 v[110:111], v65 offset:41984\n"
    "v_mfma_f32_16x16x32_bf16 v[248:251], v[112:115], a[184:187], v[248:251]\n"
    "buffer_load_dwordx4 a[92:95], v55, s[40:43], s65 offen offset:3072\n"
    "s_waitcnt lgkmcnt(8)\n"
    "v_mfma_f32_16x16x32_bf16 v[252:255], v[120:123], a[184:187], v[252:255]\n"
    "ds_read_b64_tr_b16 v[112:113], v66 offset:32768\n"
    "v_mfma_f32_16x16x32_bf16 v[252:255], v[124:127], a[188:191], v[252:255]\n"
    "v_mfma_f32_16x16x32_bf16 v[220:223], v[124:127], a[180:183], v[220:223]\n"
    "ds_read_b64_tr_b16 v[114:115], v66 offset:33792\n"
    "s_andn2_b64 s[76:77], s[72:73], s[80:81]\n"
    "s_cbranch_scc0 1361f\n"
    "1362:\n"
    "s_andn2_b64 s[14:15], s[16:17], s[80:81]\n"
    "s_cselect_b64 s[76:77], s[14:15], s[76:77]\n"
    "s_ff1_i32_b64 s69, s[76:77]\n"
    "v_mfma_f32_16x16x32_bf16 v[220:223], v[120:123], a[176:179], v[220:223]\n"
    "v_readlane_b32 s65, v53, s69\n"
    "s_bitset1_b64 s[80:81], s69\n"
    "v_mfma_f32_16x16x32_bf16 v[188:191], v[120:123], a[168:171], v[188:191]\n"
    "ds_read_b64_tr_b16 v[116:117], v66 offset:40960\n"
    "s_nop 0\n"
    "v_mfma_f32_16x16x32_bf16 v[188:191], v[124:127], a[172:175], v[188:191]\n"
    "s_nop 0\n"
    "s_lshl_b32 s64, s65, 5\n"
    "v_mfma_f32_16x16x32_bf16 v[156:159], v[124:127], a[164:167], v[156:159]\n"
    "ds_read_b64_tr_b16 v[118:119], v66 offset:41984\n"
    "s_add_u32 m0, s67, 49152\n"
    "v_mfma_f32_16x16x32_bf16 v[156:159], v[120:123], a[160:163], v[156:159]\n"
    "s_nop 0\n"
    "s_waitcnt lgkmcnt(8)\n"
    "v_mfma_f32_16x16x32_bf16 v[128:131], v[96:99], a[192:195], v[128:131]\n"
    "ds_read_b64_tr_b16 v[120:121], v67 offset:32768\n"
    "v_mfma_f32_16x16x32_bf16 v[128:131], v[100:103], a[196:199], v[128:131]\n"
    "buffer_load_dwordx4 v56, s[36:39], s64 offen offset:0 lds\n"
    "v_mfma_f32_16x16x32_bf16 v[160:163], v[100:103], a[204:207], v[160:163]\n"
    "ds_read_b64_tr_b16 v[122:123], v67 offset:33792\n"
    "v_mfma_f32_16x16x32_bf16 v[160:163], v[96:99], a[200:203], v[160:163]\n"
    "v_mfma_f32_16x16x32_bf16 v[192:195], v[96:99], a[208:211], v[192:195]\n"
    "ds_read_b64_tr_b16 v[124:125], v67 offset:40960\n"
    "v_mfma_f32_16x16x32_bf16 v[192:195], v[100:103], a[212:215], v[192:195]\n"
    "buffer_load_dwordx4 v57, s[36:39], s64 offen offset:1024 lds\n"
    "v_mfma_f32_16x16x32_bf16 v[224:227], v[100:103], a[220:223], v[224:227]\n"
    "ds_read_b64_tr_b16 v[126:127], v67 offset:41984\n"
    "v_mfma_f32_16x16x32_bf16 v[224:227], v[96:99], a[216:219], v[224:227]\n"
    "s_waitcnt lgkmcnt(8)\n"
    "v_mfma_f32_16x16x32_bf16 v[228:231], v[104:107], a[216:219], v[228:231]\n"
    "ds_read_b64_tr_b16 v[96:97], v64 offset:33280\n"
    "v_mfma_f32_16x16x32_bf16 v[228:231], v[108:111], a[220:223], v[228:231]\n"
    "v_mfma_f32_16x16x32_bf16 v[196:199], v[108:111], a[212:215], v[196:199]\n"
    "ds_read_b64_tr_b16 v[98:99], v64 offset:34304\n"
    "v_mfma_f32_16x16x32_bf16 v[196:199], v[104:107], a[208:211], v[196:199]\n"
    "buffer_load_dwordx4 v58, s[36:39], s64 offen offset:2048 lds\n"
    "v_mfma_f32_16x16x32_bf16 v[164:167], v[104:107], a[200:203], v[164:167]\n"
    "ds_read_b64_tr_b16 v[100:101], v64 offset:41472\n"
    "v_mfma_f32_16x16x32_bf16 v[164:167], v[108:111], a[204:207], v[164:167]\n"
    "v_cmp_le_u32 vcc, s70, v47\n"
    "v_mfma_f32_16x16x32_bf16 v[132:135], v[108:111], a[196:199], v[132:135]\n"
    "ds_read_b64_tr_b16 v[102:103], v64 offset:42496\n"
    "v_mfma_f32_16x16x32_bf16 v[132:135], v[104:107], a[192:195], v[132:135]\n"
    "s_or_b64 s[82:83], s[82:83], vcc\n"
    "v_cmp_le_u32 vcc, s70, v48\n"
    "s_waitcnt lgkmcnt(8)\n"
    "v_mfma_f32_16x16x32_bf16 v[136:139], v[112:115], a[192:195], v[136:139]\n"
    "ds_read_b64_tr_b16 v[104:105], v65 offset:33280\n"
    "v_mfma_f32_16x16x32_bf16 v[136:139], v[116:119], a[196:199], v[136:139]\n"
    "s_or_b64 s[82:83], s[82:83], vcc\n"
    "v_mfma_f32_16x16x32_bf16 v[168:171], v[116:119], a[204:207], v[168:171]\n"
    "ds_read_b64_tr_b16 v[106:107], v65 offset:34304\n"
    "buffer_load_dwordx4 v59, s[36:39], s64 offen offset:3072 lds\n"
    "v_mfma_f32_16x16x32_bf16 v[168:171], v[112:115], a[200:203], v[168:171]\n"
    "v_mfma_f32_16x16x32_bf16 v[200:203], v[112:115], a[208:211], v[200:203]\n"
    "ds_read_b64_tr_b16 v[108:109], v65 offset:41472\n"
    "v_mfma_f32_16x16x32_bf16 v[200:203], v[116:119], a[212:215], v[200:203]\n"
    "v_mov_b32 v38, s82\n"
    "v_mov_b32 v39, s83\n"
    "ds_write_b64 v41, v[38:39] offset:0\n"
    "v_mfma_f32_16x16x32_bf16 v[232:235], v[116:119], a[220:223], v[232:235]\n"
    "ds_read_b64_tr_b16 v[110:111], v65 offset:42496\n"
    "v_mfma_f32_16x16x32_bf16 v[232:235], v[112:115], a[216:219], v[232:235]\n"
    "buffer_load_dwordx4 a[96:99], v54, s[40:43], s65 offen offset:0\n"
    "s_waitcnt lgkmcnt(9)\n"
    "v_mfma_f32_16x16x32_bf16 v[236:239], v[120:123], a[216:219], v[236:239]\n"
    "ds_read_b64_tr_b16 v[112:113], v66 offset:33280\n"
    "v_mfma_f32_16x16x32_bf16 v[236:239], v[124:127], a[220:223], v[236:239]\n"
    "v_mfma_f32_16x16x32_bf16 v[204:207], v[124:127], a[212:215], v[204:207]\n"
    "ds_read_b64_tr_b16 v[114:115], v66 offset:34304\n"
    "v_mfma_f32_16x16x32_bf16 v[204:207], v[120:123], a[208:211], v[204:207]\n"
    "v_mfma_f32_16x16x32_bf16 v[172:175], v[120:123], a[200:203], v[172:175]\n"
    "ds_read_b64_tr_b16 v[116:117], v66 offset:41472\n"
    "v_mfma_f32_16x16x32_bf16 v[172:175], v[124:127], a[204:207], v[172:175]\n"
    "buffer_load_dwordx4 a[100:103], v54, s[40:43], s65 offen offset:1024\n"
    "v_mfma_f32_16x16x32_bf16 v[140:143], v[124:127], a[196:199], v[140:143]\n"
    "ds_read_b64_tr_b16 v[118:119], v66 offset:42496\n"
    "v_mfma_f32_16x16x32_bf16 v[140:143], v[120:123], a[192:195], v[140:143]\n"
    "s_waitcnt lgkmcnt(9)\n"
    "v_mfma_f32_16x16x32_bf16 v[144:147], v[96:99], a[192:195], v[144:147]\n"
    "ds_read_b64_tr_b16 v[120:121], v67 offset:33280\n"
    "v_mfma_f32_16x16x32_bf16 v[144:147], v[100:103], a[196:199], v[144:147]\n"
    "v_mfma_f32_16x16x32_bf16 v[176:179], v[100:103], a[204:207], v[176:179]\n"
    "ds_read_b64_tr_b16 v[122:123], v67 offset:34304\n"
    "v_mfma_f32_16x16x32_bf16 v[176:179], v[96:99], a[200:203], v[176:179]\n"
    "buffer_load_dwordx4 a[104:107], v54, s[40:43], s65 offen offset:2048\n"
    "v_mfma_f32_16x16x32_bf16 v[208:211], v[96:99], a[208:211], v[208:211]\n"
    "ds_read_b64_tr_b16 v[124:125], v67 offset:41472\n"
    "v_mfma_f32_16x16x32_bf16 v[208:211], v[100:103], a[212:215], v[208:211]\n"
    "v_mfma_f32_16x16x32_bf16 v[240:243], v[100:103], a[220:223], v[240:243]\n"
    "ds_read_b64_tr_b16 v[126:127], v67 offset:42496\n"
    "v_mfma_f32_16x16x32_bf16 v[240:243], v[96:99], a[216:219], v[240:243]\n"
    "s_waitcnt lgkmcnt(8)\n"
    "s_waitcnt vmcnt(44)\n"
    "s_barrier\n"
    "v_mfma_f32_16x16x32_bf16 v[244:247], v[104:107], a[216:219], v[244:247]\n"
    "ds_read_b64_tr_b16 v[96:97], v64 offset:49152\n"
    "v_mfma_f32_16x16x32_bf16 v[244:247], v[108:111], a[220:223], v[244:247]\n"
    "ds_read_b64 v[36:37], v40 offset:0\n"
    "v_mfma_f32_16x16x32_bf16 v[212:215], v[108:111], a[212:215], v[212:215]\n"
    "ds_read_b64_tr_b16 v[98:99], v64 offset:50176\n"
    "buffer_load_dwordx4 a[108:111], v54, s[40:43], s65 offen offset:3072\n"
    "v_mfma_f32_16x16x32_bf16 v[212:215], v[104:107], a[208:211], v[212:215]\n"
    "v_mfma_f32_16x16x32_bf16 v[180:183], v[104:107], a[200:203], v[180:183]\n"
    "ds_read_b64_tr_b16 v[100:101], v64 offset:57344\n"
    "v_mfma_f32_16x16x32_bf16 v[180:183], v[108:111], a[204:207], v[180:183]\n"
    "v_mfma_f32_16x16x32_bf16 v[148:151], v[108:111], a[196:199], v[148:151]\n"
    "ds_read_b64_tr_b16 v[102:103], v64 offset:58368\n"
    "v_mfma_f32_16x16x32_bf16 v[148:151], v[104:107], a[192:195], v[148:151]\n"
    "buffer_load_dwordx4 a[112:115], v55, s[40:43], s65 offen offset:0\n"
    "s_waitcnt lgkmcnt(9)\n"
    "v_mfma_f32_16x16x32_bf16 v[152:155], v[112:115], a[192:195], v[152:155]\n"
    "ds_read_b64_tr_b16 v[104:105], v65 offset:49152\n"
    "v_mfma_f32_16x16x32_bf16 v[152:155], v[116:119], a[196:199], v[152:155]\n"
    "v_mfma_f32_16x16x32_bf16 v[184:187], v[116:119], a[204:207], v[184:187]\n"
    "ds_read_b64_tr_b16 v[106:107], v65 offset:50176\n"
    "v_mfma_f32_16x16x32_bf16 v[184:187], v[112:115], a[200:203], v[184:187]\n"
    "v_mfma_f32_16x16x32_bf16 v[216:219], v[112:115], a[208:211], v[216:219]\n"
    "ds_read_b64_tr_b16 v[108:109], v65 offset:57344\n"
    "v_mfma_f32_16x16x32_bf16 v[216:219], v[116:119], a[212:215], v[216:219]\n"
    "buffer_load_dwordx4 a[116:119], v55, s[40:43], s65 offen offset:1024\n"
    "v_mfma_f32_16x16x32_bf16 v[248:251], v[116:119], a[220:223], v[248:251]\n"
    "ds_read_b64_tr_b16 v[110:111], v65 offset:58368\n"
    "v_mfma_f32_16x16x32_bf16 v[248:251], v[112:115], a[216:219], v[248:251]\n"
    "s_waitcnt lgkmcnt(9)\n"
    "v_mfma_f32_16x16x32_bf16 v[252:255], v[120:123], a[216:219], v[252:255]\n"
    "ds_read_b64_tr_b16 v[112:113], v66 offset:49152\n"
    "v_mfma_f32_16x16x32_bf16 v[252:255], v[124:127], a[220:223], v[252:255]\n"
    "v_mfma_f32_16x16x32_bf16 v[220:223], v[124:127], a[212:215], v[220:223]\n"
    "ds_read_b64_tr_b16 v[114:115], v66 offset:50176\n"
    "v_mfma_f32_16x16x32_bf16 v[220:223], v[120:123], a[208:211], v[220:223]\n"
    "buffer_load_dwordx4 a[120:123], v55, s[40:43], s65 offen offset:2048\n"
    "v_mfma_f32_16x16x32_bf16 v[188:191], v[120:123], a[200:203], v[188:191]\n"
    "ds_read_b64_tr_b16 v[116:117], v66 offset:57344\n"
    "v_mfma_f32_16x16x32_bf16 v[188:191], v[124:127], a[204:207], v[188:191]\n"
    "v_mfma_f32_16x16x32_bf16 v[156:159], v[124:127], a[196:199], v[156:159]\n"
    "ds_read_b64_tr_b16 v[118:119], v66 offset:58368\n"
    "v_mfma_f32_16x16x32_bf16 v[156:159], v[120:123], a[192:195], v[156:159]\n"
    "s_waitcnt lgkmcnt(8)\n"
    "v_mfma_f32_16x16x32_bf16 v[128:131], v[96:99], a[224:227], v[128:131]\n"
    "ds_read_b64_tr_b16 v[120:121], v67 offset:49152\n"
    "v_mfma_f32_16x16x32_bf16 v[128:131], v[100:103], a[228:231], v[128:131]\n"
    "v_readfirstlane_b32 vcc_lo, v36\n"
    "v_readfirstlane_b32 vcc_hi, v37\n"
    "v_mfma_f32_16x16x32_bf16 v[160:163], v[100:103], a[236:239], v[160:163]\n"
    "ds_read_b64_tr_b16 v[122:123], v67 offset:50176\n"
    "buffer_load_dwordx4 a[124:127], v55, s[40:43], s65 offen offset:3072\n"
    "v_mfma_f32_16x16x32_bf16 v[160:163], v[96:99], a[232:235], v[160:163]\n"
    "s_or_b64 s[72:73], s[72:73], vcc\n"
    "v_mfma_f32_16x16x32_bf16 v[192:195], v[96:99], a[240:243], v[192:195]\n"
    "ds_read_b64_tr_b16 v[124:125], v67 offset:57344\n"
    "s_andn2_b64 s[76:77], s[72:73], s[80:81]\n"
    "s_cbranch_scc0 1371f\n"
    "1372:\n"
    "s_andn2_b64 s[14:15], s[18:19], s[80:81]\n"
    "s_cselect_b64 s[76:77], s[14:15], s[76:77]\n"
    "s_ff1_i32_b64 s69, s[76:77]\n"
    "v_mfma_f32_16x16x32_bf16 v[192:195], v[100:103], a[244:247], v[192:195]\n"
    "v_readlane_b32 s65, v53, s69\n"
    "s_bitset1_b64 s[80:81], s69\n"
    "v_mfma_f32_16x16x32_bf16 v[224:227], v[100:103], a[252:255], v[224:227]\n"
    "ds_read_b64_tr_b16 v[126:127], v67 offset:58368\n"
    "s_nop 0\n"
    "v_mfma_f32_16x16x32_bf16 v[224:227], v[96:99], a[248:251], v[224:227]\n"
    "s_nop 0\n"
    "s_lshl_b32 s64, s65, 5\n"
    "s_waitcnt lgkmcnt(8)\n"
    "v_mfma_f32_16x16x32_bf16 v[228:231], v[104:107], a[248:251], v[228:231]\n"
    "ds_read_b64_tr_b16 v[96:97], v64 offset:49664\n"
    "s_add_u32 m0, s67, 65536\n"
    "v_mfma_f32_16x16x32_bf16 v[228:231], v[108:111], a[252:255], v[228:231]\n"
    "s_nop 0\n"
    "v_mfma_f32_16x16x32_bf16 v[196:199], v[108:111], a[244:247], v[196:199]\n"
    "ds_read_b64_tr_b16 v[98:99], v64 offset:50688\n"
    "v_mfma_f32_16x16x32_bf16 v[196:199], v[104:107], a[240:243], v[196:199]\n"
    "buffer_load_dwordx4 v56, s[36:39], s64 offen offset:0 lds\n"
    "v_mfma_f32_16x16x32_bf16 v[164:167], v[104:107], a[232:235], v[164:167]\n"
    "ds_read_b64_tr_b16 v[100:101], v64 offset:57856\n"
    "v_mfma_f32_16x16x32_bf16 v[164:167], v[108:111], a[236:239], v[164:167]\n"
    "buffer_load_dwordx4 v57, s[36:39], s64 offen offset:1024 lds\n"
    "v_mfma_f32_16x16x32_bf16 v[132:135], v[108:111], a[228:231], v[132:135]\n"
    "ds_read_b64_tr_b16 v[102:103], v64 offset:58880\n"
    "v_mfma_f32_16x16x32_bf16 v[132:135], v[104:107], a[224:227], v[132:135]\n"
    "buffer_load_dwordx4 v58, s[36:39], s64 offen offset:2048 lds\n"
    "s_waitcnt lgkmcnt(8)\n"
    "v_mfma_f32_16x16x32_bf16 v[136:139], v[112:115], a[224:227], v[136:139]\n"
    "ds_read_b64_tr_b16 v[104:105], v65 offset:49664\n"
    "v_mfma_f32_16x16x32_bf16 v[136:139], v[116:119], a[228:231], v[136:139]\n"
    "v_mfma_f32_16x16x32_bf16 v[168:171], v[116:119], a[236:239], v[168:171]\n"
    "ds_read_b64_tr_b16 v[106:107], v65 offset:50688\n"
    "v_mfma_f32_16x16x32_bf16 v[168:171], v[112:115], a[232:235], v[168:171]\n"
    "buffer_load_dwordx4 v59, s[36:39], s64 offen offset:3072 lds\n"
    "v_mfma_f32_16x16x32_bf16 v[200:203], v[112:115], a[240:243], v[200:203]\n"
    "ds_read_b64_tr_b16 v[108:109], v65 offset:57856\n"
    "v_mfma_f32_16x16x32_bf16 v[200:203], v[116:119], a[244:247], v[200:203]\n"
    "v_mfma_f32_16x16x32_bf16 v[232:235], v[116:119], a[252:255], v[232:235]\n"
    "ds_read_b64_tr_b16 v[110:111], v65 offset:58880\n"
    "v_mfma_f32_16x16x32_bf16 v[232:235], v[112:115], a[248:251], v[232:235]\n"
    "buffer_load_dwordx4 a[128:131], v54, s[40:43], s65 offen offset:0\n"
    "s_waitcnt lgkmcnt(8)\n"
    "v_mfma_f32_16x16x32_bf16 v[236:239], v[120:123], a[248:251], v[236:239]\n"
    "ds_read_b64_tr_b16 v[112:113], v66 offset:49664\n"
    "v_mfma_f32_16x16x32_bf16 v[236:239], v[124:127], a[252:255], v[236:239]\n"
    "v_mfma_f32_16x16x32_bf16 v[204:207], v[124:127], a[244:247], v[204:207]\n"
    "ds_read_b64_tr_b16 v[114:115], v66 offset:50688\n"
    "v_mfma_f32_16x16x32_bf16 v[204:207], v[120:123], a[240:243], v[204:207]\n"
    "buffer_load_dwordx4 a[132:135], v54, s[40:43], s65 offen offset:1024\n"
    "v_mfma_f32_16x16x32_bf16 v[172:175], v[120:123], a[232:235], v[172:175]\n"
    "ds_read_b64_tr_b16 v[116:117], v66 offset:57856\n"
    "v_mfma_f32_16x16x32_bf16 v[172:175], v[124:127], a[236:239], v[172:175]\n"
    "buffer_load_dwordx4 a[136:139], v54, s[40:43], s65 offen offset:2048\n"
    "v_mfma_f32_16x16x32_bf16 v[140:143], v[124:127], a[228:231], v[140:143]\n"
    "ds_read_b64_tr_b16 v[118:119], v66 offset:58880\n"
    "v_mfma_f32_16x16x32_bf16 v[140:143], v[120:123], a[224:227], v[140:143]\n"
    "s_waitcnt lgkmcnt(8)\n"
    "v_mfma_f32_16x16x32_bf16 v[144:147], v[96:99], a[224:227], v[144:147]\n"
    "ds_read_b64_tr_b16 v[120:121], v67 offset:49664\n"
    "v_mfma_f32_16x16x32_bf16 v[144:147], v[100:103], a[228:231], v[144:147]\n"
    "buffer_load_dwordx4 a[140:143], v54, s[40:43], s65 offen offset:3072\n"
    "v_mfma_f32_16x16x32_bf16 v[176:179], v[100:103], a[236:239], v[176:179]\n"
    "ds_read_b64_tr_b16 v[122:123], v67 offset:50688\n"
    "v_mfma_f32_16x16x32_bf16 v[176:179], v[96:99], a[232:235], v[176:179]\n"
    "v_mfma_f32_16x16x32_bf16 v[208:211], v[96:99], a[240:243], v[208:211]\n"
    "ds_read_b64_tr_b16 v[124:125], v67 offset:57856\n"
    "v_mfma_f32_16x16x32_bf16 v[208:211], v[100:103], a[244:247], v[208:211]\n"
    "buffer_load_dwordx4 a[144:147], v55, s[40:43], s65 offen offset:0\n"
    "v_mfma_f32_16x16x32_bf16 v[240:243], v[100:103], a[252:255], v[240:243]\n"
    "ds_read_b64_tr_b16 v[126:127], v67 offset:58880\n"
    "v_mfma_f32_16x16x32_bf16 v[240:243], v[96:99], a[248:251], v[240:243]\n"
    "buffer_load_dwordx4 a[148:151], v55, s[40:43], s65 offen offset:1024\n"
    "s_waitcnt lgkmcnt(8)\n"
    "s_waitcnt vmcnt(47)\n"
    "s_barrier\n"
    "v_mfma_f32_16x16x32_bf16 v[244:247], v[104:107], a[248:251], v[244:247]\n"
    "ds_read_b64_tr_b16 v[96:97], v60 offset:0\n"
    "v_mfma_f32_16x16x32_bf16 v[244:247], v[108:111], a[252:255], v[244:247]\n"
    "v_mfma_f32_16x16x32_bf16 v[212:215], v[108:111], a[244:247], v[212:215]\n"
    "ds_read_b64_tr_b16 v[98:99], v60 offset:1024\n"
    "v_mfma_f32_16x16x32_bf16 v[212:215], v[104:107], a[240:243], v[212:215]\n"
    "buffer_load_dwordx4 a[152:155], v55, s[40:43], s65 offen offset:2048\n"
    "v_mfma_f32_16x16x32_bf16 v[180:183], v[104:107], a[232:235], v[180:183]\n"
    "ds_read_b64_tr_b16 v[100:101], v60 offset:8192\n"
    "v_mfma_f32_16x16x32_bf16 v[180:183], v[108:111], a[236:239], v[180:183]\n"
    "v_mfma_f32_16x16x32_bf16 v[148:151], v[108:111], a[228:231], v[148:151]\n"
    "ds_read_b64_tr_b16 v[102:103], v60 offset:9216\n"
    "v_mfma_f32_16x16x32_bf16 v[148:151], v[104:107], a[224:227], v[148:151]\n"
    "buffer_load_dwordx4 a[156:159], v55, s[40:43], s65 offen offset:3072\n"
    "s_waitcnt lgkmcnt(8)\n"
    "v_mfma_f32_16x16x32_bf16 v[152:155], v[112:115], a[224:227], v[152:155]\n"
    "ds_read_b64_tr_b16 v[104:105], v61 offset:0\n"
    "s_andn2_b64 s[76:77], s[72:73], s[80:81]\n"
    "s_cbranch_scc0 1381f\n"
    "1382:\n"
    "s_andn2_b64 s[14:15], s[18:19], s[80:81]\n"
    "s_cselect_b64 s[76:77], s[14:15], s[76:77]\n"
    "s_ff1_i32_b64 s69, s[76:77]\n"
    "v_mfma_f32_16x16x32_bf16 v[152:155], v[116:119], a[228:231], v[152:155]\n"
    "v_readlane_b32 s65, v53, s69\n"
    "s_bitset1_b64 s[80:81], s69\n"
    "v_mfma_f32_16x16x32_bf16 v[184:187], v[116:119], a[236:239], v[184:187]\n"
    "ds_read_b64_tr_b16 v[106:107], v61 offset:1024\n"
    "s_nop 0\n"
    "v_mfma_f32_16x16x32_bf16 v[184:187], v[112:115], a[232:235], v[184:187]\n"
    "s_nop 0\n"
    "s_lshl_b32 s64, s65, 5\n"
    "v_mfma_f32_16x16x32_bf16 v[216:219], v[112:115], a[240:243], v[216:219]\n"
    "ds_read_b64_tr_b16 v[108:109], v61 offset:8192\n"
    "s_add_u32 m0, s67, 81920\n"
    "v_mfma_f32_16x16x32_bf16 v[216:219], v[116:119], a[244:247], v[216:219]\n"
    "s_nop 0\n"
    "v_mfma_f32_16x16x32_bf16 v[248:251], v[116:119], a[252:255], v[248:251]\n"
    "ds_read_b64_tr_b16 v[110:111], v61 offset:9216\n"
    "v_mfma_f32_16x16x32_bf16 v[248:251], v[112:115], a[248:251], v[248:251]\n"
    "buffer_load_dwordx4 v56, s[36:39], s64 offen offset:0 lds\n"
    "s_waitcnt lgkmcnt(8)\n"
    "v_mfma_f32_16x16x32_bf16 v[252:255], v[120:123], a[248:251], v[252:255]\n"
    "ds_read_b64_tr_b16 v[112:113], v62 offset:0\n"
    "v_mfma_f32_16x16x32_bf16 v[252:255], v[124:127], a[252:255], v[252:255]\n"
    "buffer_load_dwordx4 v57, s[36:39], s64 offen offset:1024 lds\n"
    "v_mfma_f32_16x16x32_bf16 v[220:223], v[124:127], a[244:247], v[220:223]\n"
    "ds_read_b64_tr_b16 v[114:115], v62 offset:1024\n"
    "v_mfma_f32_16x16x32_bf16 v[220:223], v[120:123], a[240:243], v[220:223]\n"
    "buffer_load_dwordx4 v58, s[36:39], s64 offen offset:2048 lds\n"
    "v_mfma_f32_16x16x32_bf16 v[188:191], v[120:123], a[232:235], v[188:191]\n"
    "ds_read_b64_tr_b16 v[116:117], v62 offset:8192\n"
    "v_mfma_f32_16x16x32_bf16 v[188:191], v[124:127], a[236:239], v[188:191]\n"
    "v_mfma_f32_16x16x32_bf16 v[156:159], v[124:127], a[228:231], v[156:159]\n"
    "ds_read_b64_tr_b16 v[118:119], v62 offset:9216\n"
    "v_mfma_f32_16x16x32_bf16 v[156:159], v[120:123], a[224:227], v[156:159]\n"
    "buffer_load_dwordx4 v59, s[36:39], s64 offen offset:3072 lds\n"
    "s_waitcnt lgkmcnt(8)\n"
    "v_mfma_f32_16x16x32_bf16 v[128:131], v[96:99], a[0:3], v[128:131]\n"
    "ds_read_b64_tr_b16 v[120:121], v63 offset:0\n"
    "v_mfma_f32_16x16x32_bf16 v[128:131], v[100:103], a[4:7], v[128:131]\n"
    "v_mfma_f32_16x16x32_bf16 v[160:163], v[100:103], a[12:15], v[160:163]\n"
    "ds_read_b64_tr_b16 v[122:123], v63 offset:1024\n"
    "v_mfma_f32_16x16x32_bf16 v[160:163], v[96:99], a[8:11], v[160:163]\n"
    "buffer_load_dwordx4 a[160:163], v54, s[40:43], s65 offen offset:0\n"
    "v_mfma_f32_16x16x32_bf16 v[192:195], v[96:99], a[16:19], v[192:195]\n"
    "ds_read_b64_tr_b16 v[124:125], v63 offset:8192\n"
    "v_mfma_f32_16x16x32_bf16 v[192:195], v[100:103], a[20:23], v[192:195]\n"
    "v_mfma_f32_16x16x32_bf16 v[224:227], v[100:103], a[28:31], v[224:227]\n"
    "ds_read_b64_tr_b16 v[126:127], v63 offset:9216\n"
    "v_mfma_f32_16x16x32_bf16 v[224:227], v[96:99], a[24:27], v[224:227]\n"
    "buffer_load_dwordx4 a[164:167], v54, s[40:43], s65 offen offset:1024\n"
    "s_waitcnt lgkmcnt(8)\n"
    "v_mfma_f32_16x16x32_bf16 v[228:231], v[104:107], a[24:27], v[228:231]\n"
    "ds_read_b64_tr_b16 v[96:97], v60 offset:512\n"
    "v_mfma_f32_16x16x32_bf16 v[228:231], v[108:111], a[28:31], v[228:231]\n"
    "buffer_load_dwordx4 a[168:171], v54, s[40:43], s65 offen offset:2048\n"
    "v_mfma_f32_16x16x32_bf16 v[196:199], v[108:111], a[20:23], v[196:199]\n"
    "ds_read_b64_tr_b16 v[98:99], v60 offset:1536\n"
    "v_mfma_f32_16x16x32_bf16 v[196:199], v[104:107], a[16:19], v[196:199]\n"
    "v_mfma_f32_16x16x32_bf16 v[164:167], v[104:107], a[8:11], v[164:167]\n"
    "ds_read_b64_tr_b16 v[100:101], v60 offset:8704\n"
    "v_mfma_f32_16x16x32_bf16 v[164:167], v[108:111], a[12:15], v[164:167]\n"
    "buffer_load_dwordx4 a[172:175], v54, s[40:43], s65 offen offset:3072\n"
    "v_mfma_f32_16x16x32_bf16 v[132:135], v[108:111], a[4:7], v[132:135]\n"
    "ds_read_b64_tr_b16 v[102:103], v60 offset:9728\n"
    "v_mfma_f32_16x16x32_bf16 v[132:135], v[104:107], a[0:3], v[132:135]\n"
    "s_waitcnt lgkmcnt(8)\n"
    "v_mfma_f32_16x16x32_bf16 v[136:139], v[112:115], a[0:3], v[136:139]\n"
    "ds_read_b64_tr_b16 v[104:105], v61 offset:512\n"
    "v_mfma_f32_16x16x32_bf16 v[136:139], v[116:119], a[4:7], v[136:139]\n"
    "buffer_load_dwordx4 a[176:179], v55, s[40:43], s65 offen offset:0\n"
    "v_mfma_f32_16x16x32_bf16 v[168:171], v[116:119], a[12:15], v[168:171]\n"
    "ds_read_b64_tr_b16 v[106:107], v61 offset:1536\n"
    "v_mfma_f32_16x16x32_bf16 v[168:171], v[112:115], a[8:11], v[168:171]\n"
    "buffer_load_dwordx4 a[180:183], v55, s[40:43], s65 offen offset:1024\n"
    "v_mfma_f32_16x16x32_bf16 v[200:203], v[112:115], a[16:19], v[200:203]\n"
    "ds_read_b64_tr_b16 v[108:109], v61 offset:8704\n"
    "v_mfma_f32_16x16x32_bf16 v[200:203], v[116:119], a[20:23], v[200:203]\n"
    "v_mfma_f32_16x16x32_bf16 v[232:235], v[116:119], a[28:31], v[232:235]\n"
    "ds_read_b64_tr_b16 v[110:111], v61 offset:9728\n"
    "v_mfma_f32_16x16x32_bf16 v[232:235], v[112:115], a[24:27], v[232:235]\n"
    "buffer_load_dwordx4 a[184:187], v55, s[40:43], s65 offen offset:2048\n"
    "s_waitcnt lgkmcnt(8)\n"
    "v_mfma_f32_16x16x32_bf16 v[236:239], v[120:123], a[24:27], v[236:239]\n"
    "ds_read_b64_tr_b16 v[112:113], v62 offset:512\n"
    "v_mfma_f32_16x16x32_bf16 v[236:239], v[124:127], a[28:31], v[236:239]\n"
    "v_mfma_f32_16x16x32_bf16 v[204:207], v[124:127], a[20:23], v[204:207]\n"
    "ds_read_b64_tr_b16 v[114:115], v62 offset:1536\n"
    "v_mfma_f32_16x16x32_bf16 v[204:207], v[120:123], a[16:19], v[204:207]\n"
    "buffer_load_dwordx4 a[188:191], v55, s[40:43], s65 offen offset:3072\n"
    "v_mfma_f32_16x16x32_bf16 v[172:175], v[120:123], a[8:11], v[172:175]\n"
    "ds_read_b64_tr_b16 v[116:117], v62 offset:8704\n"
    "s_andn2_b64 s[76:77], s[72:73], s[80:81]\n"
    "s_cbranch_scc0 1391f\n"
    "1392:\n"
    "s_andn2_b64 s[14:15], s[18:19], s[80:81]\n"
    "s_cselect_b64 s[76:77], s[14:15], s[76:77]\n"
    "s_ff1_i32_b64 s69, s[76:77]\n"
    "v_mfma_f32_16x16x32_bf16 v[172:175], v[124:127], a[12:15], v[172:175]\n"
    "v_readlane_b32 s65, v53, s69\n"
    "s_bitset1_b64 s[80:81], s69\n"
    "v_mfma_f32_16x16x32_bf16 v[140:143], v[124:127], a[4:7], v[140:143]\n"
    "ds_read_b64_tr_b16 v[118:119], v62 offset:9728\n"
    "s_nop 0\n"
    "v_mfma_f32_16x16x32_bf16 v[140:143], v[120:123], a[0:3], v[140:143]\n"
    "s_nop 0\n"
    "s_lshl_b32 s64, s65, 5\n"
    "s_waitcnt lgkmcnt(8)\n"
    "v_mfma_f32_16x16x32_bf16 v[144:147], v[96:99], a[0:3], v[144:147]\n"
    "ds_read_b64_tr_b16 v[120:121], v63 offset:512\n"
    "s_add_u32 m0, s67, 98304\n"
    "v_mfma_f32_16x16x32_bf16 v[144:147], v[100:103], a[4:7], v[144:147]\n"
    "s_nop 0\n"
    "v_mfma_f32_16x16x32_bf16 v[176:179], v[100:103], a[12:15], v[176:179]\n"
    "ds_read_b64_tr_b16 v[122:123], v63 offset:1536\n"
    "v_mfma_f32_16x16x32_bf16 v[176:179], v[96:99], a[8:11], v[176:179]\n"
    "buffer_load_dwordx4 v56, s[36:39], s64 offen offset:0 lds\n"
    "v_mfma_f32_16x16x32_bf16 v[208:211], v[96:99], a[16:19], v[208:211]\n"
    "ds_read_b64_tr_b16 v[124:125], v63 offset:8704\n"
    "v_mfma_f32_16x16x32_bf16 v[208:211], v[100:103], a[20:23], v[208:211]\n"
    "buffer_load_dwordx4 v57, s[36:39], s64 offen offset:1024 lds\n"
    "v_mfma_f32_16x16x32_bf16 v[240:243], v[100:103], a[28:31], v[240:243]\n"
    "ds_read_b64_tr_b16 v[126:127], v63 offset:9728\n"
    "v_mfma_f32_16x16x32_bf16 v[240:243], v[96:99], a[24:27], v[240:243]\n"
    "buffer_load_dwordx4 v58, s[36:39], s64 offen offset:2048 lds\n"
    "s_waitcnt lgkmcnt(8)\n"
    "s_waitcnt vmcnt(51)\n"
    "s_barrier\n"
    "v_mfma_f32_16x16x32_bf16 v[244:247], v[104:107], a[24:27], v[244:247]\n"
    "ds_read_b64_tr_b16 v[96:97], v60 offset:16384\n"
    "v_mfma_f32_16x16x32_bf16 v[244:247], v[108:111], a[28:31], v[244:247]\n"
    "v_mfma_f32_16x16x32_bf16 v[212:215], v[108:111], a[20:23], v[212:215]\n"
    "ds_read_b64_tr_b16 v[98:99], v60 offset:17408\n"
    "v_mfma_f32_16x16x32_bf16 v[212:215], v[104:107], a[16:19], v[212:215]\n"
    "buffer_load_dwordx4 v59, s[36:39], s64 offen offset:3072 lds\n"
    "v_mfma_f32_16x16x32_bf16 v[180:183], v[104:107], a[8:11], v[180:183]\n"
    "ds_read_b64_tr_b16 v[100:101], v60 offset:24576\n"
    "v_mfma_f32_16x16x32_bf16 v[180:183], v[108:111], a[12:15], v[180:183]\n"
    "v_mfma_f32_16x16x32_bf16 v[148:151], v[108:111], a[4:7], v[148:151]\n"
    "ds_read_b64_tr_b16 v[102:103], v60 offset:25600\n"
    "v_mfma_f32_16x16x32_bf16 v[148:151], v[104:107], a[0:3], v[148:151]\n"
    "buffer_load_dwordx4 a[192:195], v54, s[40:43], s65 offen offset:0\n"
    "s_waitcnt lgkmcnt(8)\n"
    "v_mfma_f32_16x16x32_bf16 v[152:155], v[112:115], a[0:3], v[152:155]\n"
    "ds_read_b64_tr_b16 v[104:105], v61 offset:16384\n"
    "v_mfma_f32_16x16x32_bf16 v[152:155], v[116:119], a[4:7], v[152:155]\n"
    "v_mfma_f32_16x16x32_bf16 v[184:187], v[116:119], a[12:15], v[184:187]\n"
    "ds_read_b64_tr_b16 v[106:107], v61 offset:17408\n"
    "v_mfma_f32_16x16x32_bf16 v[184:187], v[112:115], a[8:11], v[184:187]\n"
    "buffer_load_dwordx4 a[196:199], v54, s[40:43], s65 offen offset:1024\n"
    "v_mfma_f32_16x16x32_bf16 v[216:219], v[112:115], a[16:19], v[216:219]\n"
    "ds_read_b64_tr_b16 v[108:109], v61 offset:24576\n"
    "v_mfma_f32_16x16x32_bf16 v[216:219], v[116:119], a[20:23], v[216:219]\n"
    "buffer_load_dwordx4 a[200:203], v54, s[40:43], s65 offen offset:2048\n"
    "v_mfma_f32_16x16x32_bf16 v[248:251], v[116:119], a[28:31], v[248:251]\n"
    "ds_read_b64_tr_b16 v[110:111], v61 offset:25600\n"
    "v_mfma_f32_16x16x32_bf16 v[248:251], v[112:115], a[24:27], v[248:251]\n"
    "s_waitcnt lgkmcnt(8)\n"
    "v_mfma_f32_16x16x32_bf16 v[252:255], v[120:123], a[24:27], v[252:255]\n"
    "ds_read_b64_tr_b16 v[112:113], v62 offset:16384\n"
    "v_mfma_f32_16x16x32_bf16 v[252:255], v[124:127], a[28:31], v[252:255]\n"
    "buffer_load_dwordx4 a[204:207], v54, s[40:43], s65 offen offset:3072\n"
    "v_mfma_f32_16x16x32_bf16 v[220:223], v[124:127], a[20:23], v[220:223]\n"
    "ds_read_b64_tr_b16 v[114:115], v62 offset:17408\n"
    "v_mfma_f32_16x16x32_bf16 v[220:223], v[120:123], a[16:19], v[220:223]\n"
    "v_mfma_f32_16x16x32_bf16 v[188:191], v[120:123], a[8:11], v[188:191]\n"
    "ds_read_b64_tr_b16 v[116:117], v62 offset:24576\n"
    "v_mfma_f32_16x16x32_bf16 v[188:191], v[124:127], a[12:15], v[188:191]\n"
    "buffer_load_dwordx4 a[208:211], v55, s[40:43], s65 offen offset:0\n"
    "v_mfma_f32_16x16x32_bf16 v[156:159], v[124:127], a[4:7], v[156:159]\n"
    "ds_read_b64_tr_b16 v[118:119], v62 offset:25600\n"
    "v_mfma_f32_16x16x32_bf16 v[156:159], v[120:123], a[0:3], v[156:159]\n"
    "buffer_load_dwordx4 a[212:215], v55, s[40:43], s65 offen offset:1024\n"
    "s_waitcnt lgkmcnt(8)\n"
    "v_mfma_f32_16x16x32_bf16 v[128:131], v[96:99], a[32:35], v[128:131]\n"
    "ds_read_b64_tr_b16 v[120:121], v63 offset:16384\n"
    "v_mfma_f32_16x16x32_bf16 v[128:131], v[100:103], a[36:39], v[128:131]\n"
    "v_mfma_f32_16x16x32_bf16 v[160:163], v[100:103], a[44:47], v[160:163]\n"
    "ds_read_b64_tr_b16 v[122:123], v63 offset:17408\n"
    "v_mfma_f32_16x16x32_bf16 v[160:163], v[96:99], a[40:43], v[160:163]\n"
    "buffer_load_dwordx4 a[216:219], v55, s[40:43], s65 offen offset:2048\n"
    "v_mfma_f32_16x16x32_bf16 v[192:195], v[96:99], a[48:51], v[192:195]\n"
    "ds_read_b64_tr_b16 v[124:125], v63 offset:24576\n"
    "v_mfma_f32_16x16x32_bf16 v[192:195], v[100:103], a[52:55], v[192:195]\n"
    "v_mfma_f32_16x16x32_bf16 v[224:227], v[100:103], a[60:63], v[224:227]\n"
    "ds_read_b64_tr_b16 v[126:127], v63 offset:25600\n"
    "v_mfma_f32_16x16x32_bf16 v[224:227], v[96:99], a[56:59], v[224:227]\n"
    "buffer_load_dwordx4 a[220:223], v55, s[40:43], s65 offen offset:3072\n"
    "s_waitcnt lgkmcnt(8)\n"
    "v_mfma_f32_16x16x32_bf16 v[228:231], v[104:107], a[56:59], v[228:231]\n"
    "ds_read_b64_tr_b16 v[96:97], v60 offset:16896\n"
    "s_andn2_b64 s[76:77], s[72:73], s[80:81]\n"
    "s_cbranch_scc0 1401f\n"
    "1402:\n"
    "s_andn2_b64 s[14:15], s[18:19], s[80:81]\n"
    "s_cselect_b64 s[76:77], s[14:15], s[76:77]\n"
    "s_ff1_i32_b64 s69, s[76:77]\n"
    "v_mfma_f32_16x16x32_bf16 v[228:231], v[108:111], a[60:63], v[228:231]\n"
    "v_readlane_b32 s65, v53, s69\n"
    "s_bitset1_b64 s[80:81], s69\n"
    "v_mfma_f32_16x16x32_bf16 v[196:199], v[108:111], a[52:55], v[196:199]\n"
    "ds_read_b64_tr_b16 v[98:99], v60 offset:17920\n"
    "s_nop 0\n"
    "v_mfma_f32_16x16x32_bf16 v[196:199], v[104:107], a[48:51], v[196:199]\n"
    "s_nop 0\n"
    "s_lshl_b32 s64, s65, 5\n"
    "v_mfma_f32_16x16x32_bf16 v[164:167], v[104:107], a[40:43], v[164:167]\n"
    "ds_read_b64_tr_b16 v[100:101], v60 offset:25088\n"
    "s_add_u32 m0, s67, 131072\n"
    "v_mfma_f32_16x16x32_bf16 v[164:167], v[108:111], a[44:47], v[164:167]\n"
    "s_nop 0\n"
    "v_mfma_f32_16x16x32_bf16 v[132:135], v[108:111], a[36:39], v[132:135]\n"
    "ds_read_b64_tr_b16 v[102:103], v60 offset:26112\n"
    "v_mfma_f32_16x16x32_bf16 v[132:135], v[104:107], a[32:35], v[132:135]\n"
    "buffer_load_dwordx4 v56, s[36:39], s64 offen offset:0 lds\n"
    "s_waitcnt lgkmcnt(8)\n"
    "v_mfma_f32_16x16x32_bf16 v[136:139], v[112:115], a[32:35], v[136:139]\n"
    "ds_read_b64_tr_b16 v[104:105], v61 offset:16896\n"
    "v_mfma_f32_16x16x32_bf16 v[136:139], v[116:119], a[36:39], v[136:139]\n"
    "buffer_load_dwordx4 v57, s[36:39], s64 offen offset:1024 lds\n"
    "v_mfma_f32_16x16x32_bf16 v[168:171], v[116:119], a[44:47], v[168:171]\n"
    "ds_read_b64_tr_b16 v[106:107], v61 offset:17920\n"
    "v_mfma_f32_16x16x32_bf16 v[168:171], v[112:115], a[40:43], v[168:171]\n"
    "buffer_load_dwordx4 v58, s[36:39], s64 offen offset:2048 lds\n"
    "v_mfma_f32_16x16x32_bf16 v[200:203], v[112:115], a[48:51], v[200:203]\n"
    "ds_read_b64_tr_b16 v[108:109], v61 offset:25088\n"
    "v_mfma_f32_16x16x32_bf16 v[200:203], v[116:119], a[52:55], v[200:203]\n"
    "v_mfma_f32_16x16x32_bf16 v[232:235], v[116:119], a[60:63], v[232:235]\n"
    "ds_read_b64_tr_b16 v[110:111], v61 offset:26112\n"
    "v_mfma_f32_16x16x32_bf16 v[232:235], v[112:115], a[56:59], v[232:235]\n"
    "buffer_load_dwordx4 v59, s[36:39], s64 offen offset:3072 lds\n"
    "s_waitcnt lgkmcnt(8)\n"
    "v_mfma_f32_16x16x32_bf16 v[236:239], v[120:123], a[56:59], v[236:239]\n"
    "ds_read_b64_tr_b16 v[112:113], v62 offset:16896\n"
    "v_mfma_f32_16x16x32_bf16 v[236:239], v[124:127], a[60:63], v[236:239]\n"
    "v_mfma_f32_16x16x32_bf16 v[204:207], v[124:127], a[52:55], v[204:207]\n"
    "ds_read_b64_tr_b16 v[114:115], v62 offset:17920\n"
    "v_mfma_f32_16x16x32_bf16 v[204:207], v[120:123], a[48:51], v[204:207]\n"
    "buffer_load_dwordx4 a[224:227], v54, s[40:43], s65 offen offset:0\n"
    "v_mfma_f32_16x16x32_bf16 v[172:175], v[120:123], a[40:43], v[172:175]\n"
    "ds_read_b64_tr_b16 v[116:117], v62 offset:25088\n"
    "v_mfma_f32_16x16x32_bf16 v[172:175], v[124:127], a[44:47], v[172:175]\n"
    "v_mfma_f32_16x16x32_bf16 v[140:143], v[124:127], a[36:39], v[140:143]\n"
    "ds_read_b64_tr_b16 v[118:119], v62 offset:26112\n"
    "v_mfma_f32_16x16x32_bf16 v[140:143], v[120:123], a[32:35], v[140:143]\n"
    "buffer_load_dwordx4 a[228:231], v54, s[40:43], s65 offen offset:1024\n"
    "s_waitcnt lgkmcnt(8)\n"
    "v_mfma_f32_16x16x32_bf16 v[144:147], v[96:99], a[32:35], v[144:147]\n"
    "ds_read_b64_tr_b16 v[120:121], v63 offset:16896\n"
    "v_mfma_f32_16x16x32_bf16 v[144:147], v[100:103], a[36:39], v[144:147]\n"
    "buffer_load_dwordx4 a[232:235], v54, s[40:43], s65 offen offset:2048\n"
    "v_mfma_f32_16x16x32_bf16 v[176:179], v[100:103], a[44:47], v[176:179]\n"
    "ds_read_b64_tr_b16 v[122:123], v63 offset:17920\n"
    "v_mfma_f32_16x16x32_bf16 v[176:179], v[96:99], a[40:43], v[176:179]\n"
    "v_mfma_f32_16x16x32_bf16 v[208:211], v[96:99], a[48:51], v[208:211]\n"
    "ds_read_b64_tr_b16 v[124:125], v63 offset:25088\n"
    "v_mfma_f32_16x16x32_bf16 v[208:211], v[100:103], a[52:55], v[208:211]\n"
    "buffer_load_dwordx4 a[236:239], v54, s[40:43], s65 offen offset:3072\n"
    "v_mfma_f32_16x16x32_bf16 v[240:243], v[100:103], a[60:63], v[240:243]\n"
    "ds_read_b64_tr_b16 v[126:127], v63 offset:26112\n"
    "v_mfma_f32_16x16x32_bf16 v[240:243], v[96:99], a[56:59], v[240:243]\n"
    "s_waitcnt lgkmcnt(8)\n"
    "s_waitcnt vmcnt(56)\n"
    "s_barrier\n"
    "v_mfma_f32_16x16x32_bf16 v[244:247], v[104:107], a[56:59], v[244:247]\n"
    "ds_read_b64_tr_b16 v[96:97], v60 offset:32768\n"
    "v_mfma_f32_16x16x32_bf16 v[244:247], v[108:111], a[60:63], v[244:247]\n"
    "buffer_load_dwordx4 a[240:243], v55, s[40:43], s65 offen offset:0\n"
    "v_mfma_f32_16x16x32_bf16 v[212:215], v[108:111], a[52:55], v[212:215]\n"
    "ds_read_b64_tr_b16 v[98:99], v60 offset:33792\n"
    "v_mfma_f32_16x16x32_bf16 v[212:215], v[104:107], a[48:51], v[212:215]\n"
    "buffer_load_dwordx4 a[244:247], v55, s[40:43], s65 offen offset:1024\n"
    "v_mfma_f32_16x16x32_bf16 v[180:183], v[104:107], a[40:43], v[180:183]\n"
    "ds_read_b64_tr_b16 v[100:101], v60 offset:40960\n"
    "v_mfma_f32_16x16x32_bf16 v[180:183], v[108:111], a[44:47], v[180:183]\n"
    "v_mfma_f32_16x16x32_bf16 v[148:151], v[108:111], a[36:39], v[148:151]\n"
    "ds_read_b64_tr_b16 v[102:103], v60 offset:41984\n"
    "v_mfma_f32_16x16x32_bf16 v[148:151], v[104:107], a[32:35], v[148:151]\n"
    "buffer_load_dwordx4 a[248:251], v55, s[40:43], s65 offen offset:2048\n"
    "s_waitcnt lgkmcnt(8)\n"
    "v_mfma_f32_16x16x32_bf16 v[152:155], v[112:115], a[32:35], v[152:155]\n"
    "ds_read_b64_tr_b16 v[104:105], v61 offset:32768\n"
    "v_mfma_f32_16x16x32_bf16 v[152:155], v[116:119], a[36:39], v[152:155]\n"
    "v_mfma_f32_16x16x32_bf16 v[184:187], v[116:119], a[44:47], v[184:187]\n"
    "ds_read_b64_tr_b16 v[106:107], v61 offset:33792\n"
    "v_mfma_f32_16x16x32_bf16 v[184:187], v[112:115], a[40:43], v[184:187]\n"
    "buffer_load_dwordx4 a[252:255], v55, s[40:43], s65 offen offset:3072\n"
    "v_mfma_f32_16x16x32_bf16 v[216:219], v[112:115], a[48:51], v[216:219]\n"
    "ds_read_b64_tr_b16 v[108:109], v61 offset:40960\n"
    "v_mfma_f32_16x16x32_bf16 v[216:219], v[116:119], a[52:55], v[216:219]\n"
    "v_mfma_f32_16x16x32_bf16 v[248:251], v[116:119], a[60:63], v[248:251]\n"
    "ds_read_b64_tr_b16 v[110:111], v61 offset:41984\n"
    "v_mfma_f32_16x16x32_bf16 v[248:251], v[112:115], a[56:59], v[248:251]\n"
    "s_waitcnt lgkmcnt(8)\n"
    "s_waitcnt vmcnt(48)\n"
    "s_barrier\n"
    "v_mfma_f32_16x16x32_bf16 v[252:255], v[120:123], a[56:59], v[252:255]\n"
    "ds_read_b64_tr_b16 v[112:113], v60 offset:49152\n"
    "v_mfma_f32_16x16x32_bf16 v[252:255], v[124:127], a[60:63], v[252:255]\n"
    "v_mfma_f32_16x16x32_bf16 v[220:223], v[124:127], a[52:55], v[220:223]\n"
    "ds_read_b64_tr_b16 v[114:115], v60 offset:50176\n"
    "v_mfma_f32_16x16x32_bf16 v[220:223], v[120:123], a[48:51], v[220:223]\n"
    "v_mfma_f32_16x16x32_bf16 v[188:191], v[120:123], a[40:43], v[188:191]\n"
    "ds_read_b64_tr_b16 v[116:117], v60 offset:57344\n"
    "v_mfma_f32_16x16x32_bf16 v[188:191], v[124:127], a[44:47], v[188:191]\n"
    "v_mfma_f32_16x16x32_bf16 v[156:159], v[124:127], a[36:39], v[156:159]\n"
    "ds_read_b64_tr_b16 v[118:119], v60 offset:58368\n"
    "v_mfma_f32_16x16x32_bf16 v[156:159], v[120:123], a[32:35], v[156:159]\n"
    "v_mov_b32 v88, v45\n"
    "v_mov_b32 v89, v45\n"
    "v_mov_b32 v90, v44\n"
    "v_mov_b32 v91, v44\n"
    "v_mov_b32 v92, 1.0\n"
    "v_mov_b32 v93, 1.0\n"
    "s_waitcnt lgkmcnt(8)\n"
    "v_mfma_f32_16x16x32_bf16 v[128:131], v[96:99], a[64:67], v[128:131]\n"
    "ds_read_b64_tr_b16 v[120:121], v61 offset:49152\n"
    "v_mfma_f32_16x16x32_bf16 v[128:131], v[100:103], a[68:71], v[128:131]\n"
    "v_mfma_f32_16x16x32_bf16 v[160:163], v[100:103], a[76:79], v[160:163]\n"
    "ds_read_b64_tr_b16 v[122:123], v61 offset:50176\n"
    "v_mfma_f32_16x16x32_bf16 v[160:163], v[96:99], a[72:75], v[160:163]\n"
    "v_mfma_f32_16x16x32_bf16 v[192:195], v[96:99], a[80:83], v[192:195]\n"
    "ds_read_b64_tr_b16 v[124:125], v61 offset:57344\n"
    "v_mfma_f32_16x16x32_bf16 v[192:195], v[100:103], a[84:87], v[192:195]\n"
    "v_mfma_f32_16x16x32_bf16 v[224:227], v[100:103], a[92:95], v[224:227]\n"
    "ds_read_b64_tr_b16 v[126:127], v61 offset:58368\n"
    "v_mfma_f32_16x16x32_bf16 v[224:227], v[96:99], a[88:91], v[224:227]\n"
    "s_waitcnt lgkmcnt(8)\n"
    "s_waitcnt vmcnt(36)\n"
    "s_barrier\n"
    "v_mfma_f32_16x16x32_bf16 v[228:231], v[104:107], a[88:91], v[228:231]\n"
    "ds_read_b64_tr_b16 v[96:97], v64 offset:0\n"
    "v_mfma_f32_16x16x32_bf16 v[228:231], v[108:111], a[92:95], v[228:231]\n"
    "v_mfma_f32_16x16x32_bf16 v[196:199], v[108:111], a[84:87], v[196:199]\n"
    "ds_read_b64_tr_b16 v[98:99], v64 offset:1024\n"
    "v_mfma_f32_16x16x32_bf16 v[196:199], v[104:107], a[80:83], v[196:199]\n"
    "v_mfma_f32_16x16x32_bf16 v[164:167], v[104:107], a[72:75], v[164:167]\n"
    "ds_read_b64_tr_b16 v[100:101], v64 offset:8192\n"
    "v_mfma_f32_16x16x32_bf16 v[164:167], v[108:111], a[76:79], v[164:167]\n"
    "v_mfma_f32_16x16x32_bf16 v[132:135], v[108:111], a[68:71], v[132:135]\n"
    "ds_read_b64_tr_b16 v[102:103], v64 offset:9216\n"
    "v_mfma_f32_16x16x32_bf16 v[132:135], v[104:107], a[64:67], v[132:135]\n"
    "s_waitcnt lgkmcnt(8)\n"
    "v_mfma_f32_16x16x32_bf16 v[128:131], v[112:115], a[96:99], v[128:131]\n"
    "ds_read_b64_tr_b16 v[104:105], v65 offset:0\n"
    "v_mfma_f32_16x16x32_bf16 v[128:131], v[116:119], a[100:103], v[128:131]\n"
    "v_mfma_f32_16x16x32_bf16 v[160:163], v[116:119], a[108:111], v[160:163]\n"
    "ds_read_b64_tr_b16 v[106:107], v65 offset:1024\n"
    "v_mfma_f32_16x16x32_bf16 v[160:163], v[112:115], a[104:107], v[160:163]\n"
    "v_mfma_f32_16x16x32_bf16 v[192:195], v[112:115], a[112:115], v[192:195]\n"
    "ds_read_b64_tr_b16 v[108:109], v65 offset:8192\n"
    "v_mfma_f32_16x16x32_bf16 v[192:195], v[116:119], a[116:119], v[192:195]\n"
    "v_mfma_f32_16x16x32_bf16 v[224:227], v[116:119], a[124:127], v[224:227]\n"
    "ds_read_b64_tr_b16 v[110:111], v65 offset:9216\n"
    "v_mfma_f32_16x16x32_bf16 v[224:227], v[112:115], a[120:123], v[224:227]\n"
    "s_waitcnt lgkmcnt(8)\n"
    "s_waitcnt vmcnt(24)\n"
    "s_barrier\n"
    "v_mfma_f32_16x16x32_bf16 v[228:231], v[120:123], a[120:123], v[228:231]\n"
    "ds_read_b64_tr_b16 v[112:113], v64 offset:16384\n"
    "v_mfma_f32_16x16x32_bf16 v[228:231], v[124:127], a[124:127], v[228:231]\n"
    "v_mfma_f32_16x16x32_bf16 v[196:199], v[124:127], a[116:119], v[196:199]\n"
    "ds_read_b64_tr_b16 v[114:115], v64 offset:17408\n"
    "v_mfma_f32_16x16x32_bf16 v[196:199], v[120:123], a[112:115], v[196:199]\n"
    "v_mfma_f32_16x16x32_bf16 v[164:167], v[120:123], a[104:107], v[164:167]\n"
    "ds_read_b64_tr_b16 v[116:117], v64 offset:24576\n"
    "v_mfma_f32_16x16x32_bf16 v[164:167], v[124:127], a[108:111], v[164:167]\n"
    "v_mfma_f32_16x16x32_bf16 v[132:135], v[124:127], a[100:103], v[132:135]\n"
    "ds_read_b64_tr_b16 v[118:119], v64 offset:25600\n"
    "v_mfma_f32_16x16x32_bf16 v[132:135], v[120:123], a[96:99], v[132:135]\n"
    "s_waitcnt lgkmcnt(8)\n"
    "v_mfma_f32_16x16x32_bf16 v[128:131], v[96:99], a[128:131], v[128:131]\n"
    "ds_read_b64_tr_b16 v[120:121], v65 offset:16384\n"
    "v_mfma_f32_16x16x32_bf16 v[128:131], v[100:103], a[132:135], v[128:131]\n"
    "v_mfma_f32_16x16x32_bf16 v[160:163], v[100:103], a[140:143], v[160:163]\n"
    "ds_read_b64_tr_b16 v[122:123], v65 offset:17408\n"
    "v_mfma_f32_16x16x32_bf16 v[160:163], v[96:99], a[136:139], v[160:163]\n"
    "v_mfma_f32_16x16x32_bf16 v[192:195], v[96:99], a[144:147], v[192:195]\n"
    "ds_read_b64_tr_b16 v[124:125], v65 offset:24576\n"
    "v_mfma_f32_16x16x32_bf16 v[192:195], v[100:103], a[148:151], v[192:195]\n"
    "v_mfma_f32_16x16x32_bf16 v[224:227], v[100:103], a[156:159], v[224:227]\n"
    "ds_read_b64_tr_b16 v[126:127], v65 offset:25600\n"
    "v_mfma_f32_16x16x32_bf16 v[224:227], v[96:99], a[152:155], v[224:227]\n"
    "s_waitcnt lgkmcnt(8)\n"
    "s_waitcnt vmcnt(12)\n"
    "s_barrier\n"
    "v_mfma_f32_16x16x32_bf16 v[228:231], v[104:107], a[152:155], v[228:231]\n"
    "ds_read_b64_tr_b16 v[96:97], v64 offset:32768\n"
    "v_mfma_f32_16x16x32_bf16 v[228:231], v[108:111], a[156:159], v[228:231]\n"
    "v_mfma_f32_16x16x32_bf16 v[196:199], v[108:111], a[148:151], v[196:199]\n"
    "ds_read_b64_tr_b16 v[98:99], v64 offset:33792\n"
    "v_mfma_f32_16x16x32_bf16 v[196:199], v[104:107], a[144:147], v[196:199]\n"
    "v_mfma_f32_16x16x32_bf16 v[164:167], v[104:107], a[136:139], v[164:167]\n"
    "ds_read_b64_tr_b16 v[100:101], v64 offset:40960\n"
    "v_mfma_f32_16x16x32_bf16 v[164:167], v[108:111], a[140:143], v[164:167]\n"
    "v_mfma_f32_16x16x32_bf16 v[132:135], v[108:111], a[132:135], v[132:135]\n"
    "ds_read_b64_tr_b16 v[102:103], v64 offset:41984\n"
    "v_mfma_f32_16x16x32_bf16 v[132:135], v[104:107], a[128:131], v[132:135]\n"
    "s_waitcnt lgkmcnt(8)\n"
    "v_mfma_f32_16x16x32_bf16 v[128:131], v[112:115], a[160:163], v[128:131]\n"
    "ds_read_b64_tr_b16 v[104:105], v65 offset:32768\n"
    "v_mfma_f32_16x16x32_bf16 v[128:131], v[116:119], a[164:167], v[128:131]\n"
    "v_mfma_f32_16x16x32_bf16 v[160:163], v[116:119], a[172:175], v[160:163]\n"
    "ds_read_b64_tr_b16 v[106:107], v65 offset:33792\n"
    "v_mfma_f32_16x16x32_bf16 v[160:163], v[112:115], a[168:171], v[160:163]\n"
    "v_mfma_f32_16x16x32_bf16 v[192:195], v[112:115], a[176:179], v[192:195]\n"
    "ds_read_b64_tr_b16 v[108:109], v65 offset:40960\n"
    "v_mfma_f32_16x16x32_bf16 v[192:195], v[116:119], a[180:183], v[192:195]\n"
    "v_mfma_f32_16x16x32_bf16 v[224:227], v[116:119], a[188:191], v[224:227]\n"
    "ds_read_b64_tr_b16 v[110:111], v65 offset:41984\n"
    "v_mfma_f32_16x16x32_bf16 v[224:227], v[112:115], a[184:187], v[224:227]\n"
    "s_waitcnt lgkmcnt(8)\n"
    "s_waitcnt vmcnt(0)\n"
    "s_barrier\n"
    "v_mfma_f32_16x16x32_bf16 v[228:231], v[120:123], a[184:187], v[228:231]\n"
    "ds_read_b64_tr_b16 v[112:113], v68 offset:0\n"
    "v_mfma_f32_16x16x32_bf16 v[228:231], v[124:127], a[188:191], v[228:231]\n"
    "v_mfma_f32_16x16x32_bf16 v[196:199], v[124:127], a[180:183], v[196:199]\n"
    "ds_read_b64_tr_b16 v[114:115], v68 offset:1024\n"
    "v_mfma_f32_16x16x32_bf16 v[196:199], v[120:123], a[176:179], v[196:199]\n"
    "v_mfma_f32_16x16x32_bf16 v[164:167], v[120:123], a[168:171], v[164:167]\n"
    "ds_read_b64_tr_b16 v[116:117], v68 offset:8192\n"
    "v_mfma_f32_16x16x32_bf16 v[164:167], v[124:127], a[172:175], v[164:167]\n"
    "v_mfma_f32_16x16x32_bf16 v[132:135], v[124:127], a[164:167], v[132:135]\n"
    "ds_read_b64_tr_b16 v[118:119], v68 offset:9216\n"
    "v_mfma_f32_16x16x32_bf16 v[132:135], v[120:123], a[160:163], v[132:135]\n"
    "s_waitcnt lgkmcnt(8)\n"
    "v_mfma_f32_16x16x32_bf16 v[128:131], v[96:99], a[192:195], v[128:131]\n"
    "ds_read_b64_tr_b16 v[120:121], v69 offset:0\n"
    "v_mfma_f32_16x16x32_bf16 v[128:131], v[100:103], a[196:199], v[128:131]\n"
    "v_mfma_f32_16x16x32_bf16 v[160:163], v[100:103], a[204:207], v[160:163]\n"
    "ds_read_b64_tr_b16 v[122:123], v69 offset:1024\n"
    "v_mfma_f32_16x16x32_bf16 v[160:163], v[96:99], a[200:203], v[160:163]\n"
    "v_mfma_f32_16x16x32_bf16 v[192:195], v[96:99], a[208:211], v[192:195]\n"
    "ds_read_b64_tr_b16 v[124:125], v69 offset:8192\n"
    "v_mfma_f32_16x16x32_bf16 v[192:195], v[100:103], a[212:215], v[192:195]\n"
    "v_mfma_f32_16x16x32_bf16 v[224:227], v[100:103], a[220:223], v[224:227]\n"
    "ds_read_b64_tr_b16 v[126:127], v69 offset:9216\n"
    "v_mfma_f32_16x16x32_bf16 v[224:227], v[96:99], a[216:219], v[224:227]\n"
    "s_waitcnt lgkmcnt(8)\n"
    "v_mfma_f32_16x16x32_bf16 v[228:231], v[104:107], a[216:219], v[228:231]\n"
    "ds_read_b64_tr_b16 v[96:97], v62 offset:32768\n"
    "v_mfma_f32_16x16x32_bf16 v[228:231], v[108:111], a[220:223], v[228:231]\n"
    "v_mfma_f32_16x16x32_bf16 v[196:199], v[108:111], a[212:215], v[196:199]\n"
    "ds_read_b64_tr_b16 v[98:99], v62 offset:33792\n"
    "v_mfma_f32_16x16x32_bf16 v[196:199], v[104:107], a[208:211], v[196:199]\n"
    "v_mfma_f32_16x16x32_bf16 v[164:167], v[104:107], a[200:203], v[164:167]\n"
    "ds_read_b64_tr_b16 v[100:101], v62 offset:40960\n"
    "v_mfma_f32_16x16x32_bf16 v[164:167], v[108:111], a[204:207], v[164:167]\n"
    "v_mfma_f32_16x16x32_bf16 v[132:135], v[108:111], a[196:199], v[132:135]\n"
    "ds_read_b64_tr_b16 v[102:103], v62 offset:41984\n"
    "v_mfma_f32_16x16x32_bf16 v[132:135], v[104:107], a[192:195], v[132:135]\n"
    "s_waitcnt lgkmcnt(8)\n"
    "v_mfma_f32_16x16x32_bf16 v[128:131], v[112:115], a[224:227], v[128:131]\n"
    "ds_read_b64_tr_b16 v[104:105], v63 offset:32768\n"
    "v_mfma_f32_16x16x32_bf16 v[128:131], v[116:119], a[228:231], v[128:131]\n"
    "v_mfma_f32_16x16x32_bf16 v[160:163], v[116:119], a[236:239], v[160:163]\n"
    "ds_read_b64_tr_b16 v[106:107], v63 offset:33792\n"
    "v_mfma_f32_16x16x32_bf16 v[160:163], v[112:115], a[232:235], v[160:163]\n"
    "v_mfma_f32_16x16x32_bf16 v[192:195], v[112:115], a[240:243], v[192:195]\n"
    "ds_read_b64_tr_b16 v[108:109], v63 offset:40960\n"
    "v_mfma_f32_16x16x32_bf16 v[192:195], v[116:119], a[244:247], v[192:195]\n"
    "v_mfma_f32_16x16x32_bf16 v[224:227], v[116:119], a[252:255], v[224:227]\n"
    "ds_read_b64_tr_b16 v[110:111], v63 offset:41984\n"
    "v_mfma_f32_16x16x32_bf16 v[224:227], v[112:115], a[248:251], v[224:227]\n"
    "s_waitcnt lgkmcnt(8)\n"
    "v_mfma_f32_16x16x32_bf16 v[228:231], v[120:123], a[248:251], v[228:231]\n"
    "ds_read_b64_tr_b16 v[112:113], v62 offset:49152\n"
    "v_mfma_f32_16x16x32_bf16 v[228:231], v[124:127], a[252:255], v[228:231]\n"
    "v_mfma_f32_16x16x32_bf16 v[196:199], v[124:127], a[244:247], v[196:199]\n"
    "ds_read_b64_tr_b16 v[114:115], v62 offset:50176\n"
    "v_mfma_f32_16x16x32_bf16 v[196:199], v[120:123], a[240:243], v[196:199]\n"
    "v_mfma_f32_16x16x32_bf16 v[164:167], v[120:123], a[232:235], v[164:167]\n"
    "ds_read_b64_tr_b16 v[116:117], v62 offset:57344\n"
    "v_mfma_f32_16x16x32_bf16 v[164:167], v[124:127], a[236:239], v[164:167]\n"
    "v_mfma_f32_16x16x32_bf16 v[132:135], v[124:127], a[228:231], v[132:135]\n"
    "ds_read_b64_tr_b16 v[118:119], v62 offset:58368\n"
    "v_mfma_f32_16x16x32_bf16 v[132:135], v[120:123], a[224:227], v[132:135]\n"
    "s_waitcnt lgkmcnt(8)\n"
    "v_mfma_f32_16x16x32_bf16 v[136:139], v[96:99], a[64:67], v[136:139]\n"
    "ds_read_b64_tr_b16 v[120:121], v63 offset:49152\n"
    "s_nop 0\n"
    "s_nop 0\n"
    "s_nop 0\n"
    "v_mfma_f32_16x16x32_bf16 v[136:139], v[100:103], a[68:71], v[136:139]\n"
    "s_nop 0\n"
    "s_nop 0\n"
    "s_nop 0\n"
    "v_mfma_f32_16x16x32_bf16 v[168:171], v[100:103], a[76:79], v[168:171]\n"
    "ds_read_b64_tr_b16 v[122:123], v63 offset:50176\n"
    "v_mul_f32 v72, v128, v128\n"
    "v_mul_f32 v73, v129, v129\n"
    "v_mul_f32 v74, v130, v130\n"
    "v_mfma_f32_16x16x32_bf16 v[168:171], v[96:99], a[72:75], v[168:171]\n"
    "v_mul_f32 v75, v131, v131\n"
    "v_mul_f32 v76, v132, v132\n"
    "v_mul_f32 v77, v133, v133\n"
    "v_mfma_f32_16x16x32_bf16 v[200:203], v[96:99], a[80:83], v[200:203]\n"
    "ds_read_b64_tr_b16 v[124:125], v63 offset:57344\n"
    "v_mul_f32 v78, v134, v134\n"
    "v_mul_f32 v79, v135, v135\n"
    "v_fma_f32 v72, v72, v44, v45\n"
    "v_mfma_f32_16x16x32_bf16 v[200:203], v[100:103], a[84:87], v[200:203]\n"
    "v_fma_f32 v73, v73, v44, v45\n"
    "v_fma_f32 v74, v74, v44, v45\n"
    "v_fma_f32 v75, v75, v44, v45\n"
    "v_mfma_f32_16x16x32_bf16 v[232:235], v[100:103], a[92:95], v[232:235]\n"
    "ds_read_b64_tr_b16 v[126:127], v63 offset:58368\n"
    "v_fma_f32 v76, v76, v44, v45\n"
    "v_fma_f32 v77, v77, v44, v45\n"
    "v_fma_f32 v78, v78, v44, v45\n"
    "v_mfma_f32_16x16x32_bf16 v[232:235], v[96:99], a[88:91], v[232:235]\n"
    "v_fma_f32 v79, v79, v44, v45\n"
    "v_mul_f32 v72, v72, v128\n"
    "v_mul_f32 v73, v73, v129\n"
    "s_waitcnt lgkmcnt(8)\n"
    "v_mfma_f32_16x16x32_bf16 v[236:239], v[104:107], a[88:91], v[236:239]\n"
    "ds_read_b64_tr_b16 v[96:97], v66 offset:0\n"
    "v_mul_f32 v74, v74, v130\n"
    "v_mul_f32 v75, v75, v131\n"
    "v_mul_f32 v76, v76, v132\n"
    "v_mfma_f32_16x16x32_bf16 v[236:239], v[108:111], a[92:95], v[236:239]\n"
    "v_mul_f32 v77, v77, v133\n"
    "v_mul_f32 v78, v78, v134\n"
    "v_mul_f32 v79, v79, v135\n"
    "v_mfma_f32_16x16x32_bf16 v[204:207], v[108:111], a[84:87], v[204:207]\n"
    "ds_read_b64_tr_b16 v[98:99], v66 offset:1024\n"
    "v_exp_f32 v72, v72\n"
    "v_exp_f32 v73, v73\n"
    "v_exp_f32 v74, v74\n"
    "v_mfma_f32_16x16x32_bf16 v[204:207], v[104:107], a[80:83], v[204:207]\n"
    "v_exp_f32 v75, v75\n"
    "v_exp_f32 v76, v76\n"
    "v_exp_f32 v77, v77\n"
    "v_mfma_f32_16x16x32_bf16 v[172:175], v[104:107], a[72:75], v[172:175]\n"
    "ds_read_b64_tr_b16 v[100:101], v66 offset:8192\n"
    "v_exp_f32 v78, v78\n"
    "v_exp_f32 v79, v79\n"
    "v_add_f32 v72, 1.0, v72\n"
    "v_mfma_f32_16x16x32_bf16 v[172:175], v[108:111], a[76:79], v[172:175]\n"
    "v_add_f32 v73, 1.0, v73\n"
    "v_add_f32 v74, 1.0, v74\n"
    "v_add_f32 v75, 1.0, v75\n"
    "v_mfma_f32_16x16x32_bf16 v[140:143], v[108:111], a[68:71], v[140:143]\n"
    "ds_read_b64_tr_b16 v[102:103], v66 offset:9216\n"
    "v_add_f32 v76, 1.0, v76\n"
    "v_add_f32 v77, 1.0, v77\n"
    "v_add_f32 v78, 1.0, v78\n"
    "v_mfma_f32_16x16x32_bf16 v[140:143], v[104:107], a[64:67], v[140:143]\n"
    "v_add_f32 v79, 1.0, v79\n"
    "v_rcp_f32 v72, v72\n"
    "v_rcp_f32 v73, v73\n"
    "s_waitcnt lgkmcnt(8)\n"
    "v_mfma_f32_16x16x32_bf16 v[136:139], v[112:115], a[96:99], v[136:139]\n"
    "ds_read_b64_tr_b16 v[104:105], v67 offset:0\n"
    "v_rcp_f32 v74, v74\n"
    "v_rcp_f32 v75, v75\n"
    "v_rcp_f32 v76, v76\n"
    "v_mfma_f32_16x16x32_bf16 v[136:139], v[116:119], a[100:103], v[136:139]\n"
    "v_rcp_f32 v77, v77\n"
    "v_rcp_f32 v78, v78\n"
    "v_rcp_f32 v79, v79\n"
    "v_mfma_f32_16x16x32_bf16 v[168:171], v[116:119], a[108:111], v[168:171]\n"
    "ds_read_b64_tr_b16 v[106:107], v67 offset:1024\n"
    "v_mul_f32 v72, v72, v128\n"
    "v_mul_f32 v73, v73, v129\n"
    "v_mul_f32 v74, v74, v130\n"
    "v_mfma_f32_16x16x32_bf16 v[168:171], v[112:115], a[104:107], v[168:171]\n"
    "v_mul_f32 v75, v75, v131\n"
    "v_mul_f32 v76, v76, v132\n"
    "v_mul_f32 v77, v77, v133\n"
    "v_mfma_f32_16x16x32_bf16 v[200:203], v[112:115], a[112:115], v[200:203]\n"
    "ds_read_b64_tr_b16 v[108:109], v67 offset:8192\n"
    "v_mul_f32 v78, v78, v134\n"
    "v_mul_f32 v79, v79, v135\n"
    "v_cvt_pk_bf16_f32 v128, v72, v73\n"
    "v_mfma_f32_16x16x32_bf16 v[200:203], v[116:119], a[116:119], v[200:203]\n"
    "v_cvt_pk_bf16_f32 v129, v74, v75\n"
    "v_cvt_pk_bf16_f32 v130, v76, v77\n"
    "v_cvt_pk_bf16_f32 v131, v78, v79\n"
    "v_mfma_f32_16x16x32_bf16 v[232:235], v[116:119], a[124:127], v[232:235]\n"
    "ds_read_b64_tr_b16 v[110:111], v67 offset:9216\n"
    "v_cndmask_b32 v132, v128, v130, s[92:93]\n"
    "v_cndmask_b32 v133, v129, v131, s[92:93]\n"
    "v_cndmask_b32 v134, v130, v128, s[92:93]\n"
    "v_mfma_f32_16x16x32_bf16 v[232:235], v[112:115], a[120:123], v[232:235]\n"
    "v_cndmask_b32 v135, v131, v129, s[92:93]\n"
    "s_waitcnt lgkmcnt(8)\n"
    "v_mfma_f32_16x16x32_bf16 v[236:239], v[120:123], a[120:123], v[236:239]\n"
    "ds_read_b64_tr_b16 v[112:113], v66 offset:16384\n"
    "v_mfma_f32_16x16x32_bf16 v[236:239], v[124:127], a[124:127], v[236:239]\n"
    "buffer_store_dwordx4 v[132:135], v46, s[60:63], s88 offen offset:0 sc1\n"
    "v_mfma_f32_16x16x32_bf16 v[204:207], v[124:127], a[116:119], v[204:207]\n"
    "ds_read_b64_tr_b16 v[114:115], v66 offset:17408\n"
    "v_mul_f32 v80, v160, v160\n"
    "v_mul_f32 v81, v161, v161\n"
    "v_mul_f32 v82, v162, v162\n"
    "v_mfma_f32_16x16x32_bf16 v[204:207], v[120:123], a[112:115], v[204:207]\n"
    "v_mul_f32 v83, v163, v163\n"
    "v_mul_f32 v84, v164, v164\n"
    "v_mul_f32 v85, v165, v165\n"
    "v_mfma_f32_16x16x32_bf16 v[172:175], v[120:123], a[104:107], v[172:175]\n"
    "ds_read_b64_tr_b16 v[116:117], v66 offset:24576\n"
    "v_mul_f32 v86, v166, v166\n"
    "v_mul_f32 v87, v167, v167\n"
    "v_fma_f32 v80, v80, v44, v45\n"
    "v_mfma_f32_16x16x32_bf16 v[172:175], v[124:127], a[108:111], v[172:175]\n"
    "v_fma_f32 v81, v81, v44, v45\n"
    "v_fma_f32 v82, v82, v44, v45\n"
    "v_fma_f32 v83, v83, v44, v45\n"
    "v_mfma_f32_16x16x32_bf16 v[140:143], v[124:127], a[100:103], v[140:143]\n"
    "ds_read_b64_tr_b16 v[118:119], v66 offset:25600\n"
    "v_fma_f32 v84, v84, v44, v45\n"
    "v_fma_f32 v85, v85, v44, v45\n"
    "v_fma_f32 v86, v86, v44, v45\n"
    "v_mfma_f32_16x16x32_bf16 v[140:143], v[120:123], a[96:99], v[140:143]\n"
    "v_fma_f32 v87, v87, v44, v45\n"
    "v_mul_f32 v80, v80, v160\n"
    "v_mul_f32 v81, v81, v161\n"
    "s_waitcnt lgkmcnt(8)\n"
    "v_mfma_f32_16x16x32_bf16 v[136:139], v[96:99], a[128:131], v[136:139]\n"
    "ds_read_b64_tr_b16 v[120:121], v67 offset:16384\n"
    "v_mul_f32 v82, v82, v162\n"
    "v_mul_f32 v83, v83, v163\n"
    "v_mul_f32 v84, v84, v164\n"
    "v_mfma_f32_16x16x32_bf16 v[136:139], v[100:103], a[132:135], v[136:139]\n"
    "v_mul_f32 v85, v85, v165\n"
    "v_mul_f32 v86, v86, v166\n"
    "v_mul_f32 v87, v87, v167\n"
    "v_mfma_f32_16x16x32_bf16 v[168:171], v[100:103], a[140:143], v[168:171]\n"
    "ds_read_b64_tr_b16 v[122:123], v67 offset:17408\n"
    "v_exp_f32 v80, v80\n"
    "v_exp_f32 v81, v81\n"
    "v_exp_f32 v82, v82\n"
    "v_mfma_f32_16x16x32_bf16 v[168:171], v[96:99], a[136:139], v[168:171]\n"
    "v_exp_f32 v83, v83\n"
    "v_exp_f32 v84, v84\n"
    "v_exp_f32 v85, v85\n"
    "v_mfma_f32_16x16x32_bf16 v[200:203], v[96:99], a[144:147], v[200:203]\n"
    "ds_read_b64_tr_b16 v[124:125], v67 offset:24576\n"
    "v_exp_f32 v86, v86\n"
    "v_exp_f32 v87, v87\n"
    "v_add_f32 v80, 1.0, v80\n"
    "v_mfma_f32_16x16x32_bf16 v[200:203], v[100:103], a[148:151], v[200:203]\n"
    "v_add_f32 v81, 1.0, v81\n"
    "v_add_f32 v82, 1.0, v82\n"
    "v_add_f32 v83, 1.0, v83\n"
    "v_mfma_f32_16x16x32_bf16 v[232:235], v[100:103], a[156:159], v[232:235]\n"
    "ds_read_b64_tr_b16 v[126:127], v67 offset:25600\n"
    "v_add_f32 v84, 1.0, v84\n"
    "v_add_f32 v85, 1.0, v85\n"
    "v_add_f32 v86, 1.0, v86\n"
    "v_mfma_f32_16x16x32_bf16 v[232:235], v[96:99], a[152:155], v[232:235]\n"
    "v_add_f32 v87, 1.0, v87\n"
    "v_rcp_f32 v80, v80\n"
    "v_rcp_f32 v81, v81\n"
    "s_waitcnt lgkmcnt(8)\n"
    "v_mfma_f32_16x16x32_bf16 v[236:239], v[104:107], a[152:155], v[236:239]\n"
    "ds_read_b64_tr_b16 v[96:97], v66 offset:32768\n"
    "v_rcp_f32 v82, v82\n"
    "v_rcp_f32 v83, v83\n"
    "v_rcp_f32 v84, v84\n"
    "v_mfma_f32_16x16x32_bf16 v[236:239], v[108:111], a[156:159], v[236:239]\n"
    "v_rcp_f32 v85, v85\n"
    "v_rcp_f32 v86, v86\n"
    "v_rcp_f32 v87, v87\n"
    "v_mfma_f32_16x16x32_bf16 v[204:207], v[108:111], a[148:151], v[204:207]\n"
    "ds_read_b64_tr_b16 v[98:99], v66 offset:33792\n"
    "v_mul_f32 v80, v80, v160\n"
    "v_mul_f32 v81, v81, v161\n"
    "v_mul_f32 v82, v82, v162\n"
    "v_mfma_f32_16x16x32_bf16 v[204:207], v[104:107], a[144:147], v[204:207]\n"
    "v_mul_f32 v83, v83, v163\n"
    "v_mul_f32 v84, v84, v164\n"
    "v_mul_f32 v85, v85, v165\n"
    "v_mfma_f32_16x16x32_bf16 v[172:175], v[104:107], a[136:139], v[172:175]\n"
    "ds_read_b64_tr_b16 v[100:101], v66 offset:40960\n"
    "v_mul_f32 v86, v86, v166\n"
    "v_mul_f32 v87, v87, v167\n"
    "v_cvt_pk_bf16_f32 v160, v80, v81\n"
    "v_mfma_f32_16x16x32_bf16 v[172:175], v[108:111], a[140:143], v[172:175]\n"
    "v_cvt_pk_bf16_f32 v161, v82, v83\n"
    "v_cvt_pk_bf16_f32 v162, v84, v85\n"
    "v_cvt_pk_bf16_f32 v163, v86, v87\n"
    "v_mfma_f32_16x16x32_bf16 v[140:143], v[108:111], a[132:135], v[140:143]\n"
    "ds_read_b64_tr_b16 v[102:103], v66 offset:41984\n"
    "v_cndmask_b32 v164, v160, v162, s[92:93]\n"
    "v_cndmask_b32 v165, v161, v163, s[92:93]\n"
    "v_cndmask_b32 v166, v162, v160, s[92:93]\n"
    "v_mfma_f32_16x16x32_bf16 v[140:143], v[104:107], a[128:131], v[140:143]\n"
    "v_cndmask_b32 v167, v163, v161, s[92:93]\n"
    "s_waitcnt lgkmcnt(8)\n"
    "v_mfma_f32_16x16x32_bf16 v[136:139], v[112:115], a[160:163], v[136:139]\n"
    "ds_read_b64_tr_b16 v[104:105], v67 offset:32768\n"
    "v_mfma_f32_16x16x32_bf16 v[136:139], v[116:119], a[164:167], v[136:139]\n"
    "buffer_store_dwordx4 v[164:167], v46, s[60:63], s89 offen offset:0 sc1\n"
    "v_mfma_f32_16x16x32_bf16 v[168:171], v[116:119], a[172:175], v[168:171]\n"
    "ds_read_b64_tr_b16 v[106:107], v67 offset:33792\n"
    "v_mul_f32 v72, v192, v192\n"
    "v_mul_f32 v73, v193, v193\n"
    "v_mul_f32 v74, v194, v194\n"
    "v_mfma_f32_16x16x32_bf16 v[168:171], v[112:115], a[168:171], v[168:171]\n"
    "v_mul_f32 v75, v195, v195\n"
    "v_mul_f32 v76, v196, v196\n"
    "v_mul_f32 v77, v197, v197\n"
    "v_mfma_f32_16x16x32_bf16 v[200:203], v[112:115], a[176:179], v[200:203]\n"
    "ds_read_b64_tr_b16 v[108:109], v67 offset:40960\n"
    "v_mul_f32 v78, v198, v198\n"
    "v_mul_f32 v79, v199, v199\n"
    "v_fma_f32 v72, v72, v44, v45\n"
    "v_mfma_f32_16x16x32_bf16 v[200:203], v[116:119], a[180:183], v[200:203]\n"
    "v_fma_f32 v73, v73, v44, v45\n"
    "v_fma_f32 v74, v74, v44, v45\n"
    "v_fma_f32 v75, v75, v44, v45\n"
    "v_mfma_f32_16x16x32_bf16 v[232:235], v[116:119], a[188:191], v[232:235]\n"
    "ds_read_b64_tr_b16 v[110:111], v67 offset:41984\n"
    "v_fma_f32 v76, v76, v44, v45\n"
    "v_fma_f32 v77, v77, v44, v45\n"
    "v_fma_f32 v78, v78, v44, v45\n"
    "v_mfma_f32_16x16x32_bf16 v[232:235], v[112:115], a[184:187], v[232:235]\n"
    "v_fma_f32 v79, v79, v44, v45\n"
    "v_mul_f32 v72, v72, v192\n"
    "v_mul_f32 v73, v73, v193\n"
    "s_waitcnt lgkmcnt(8)\n"
    "v_mfma_f32_16x16x32_bf16 v[236:239], v[120:123], a[184:187], v[236:239]\n"
    "ds_read_b64_tr_b16 v[112:113], v70 offset:0\n"
    "v_mul_f32 v74, v74, v194\n"
    "v_mul_f32 v75, v75, v195\n"
    "v_mul_f32 v76, v76, v196\n"
    "v_mfma_f32_16x16x32_bf16 v[236:239], v[124:127], a[188:191], v[236:239]\n"
    "v_mul_f32 v77, v77, v197\n"
    "v_mul_f32 v78, v78, v198\n"
    "v_mul_f32 v79, v79, v199\n"
    "v_mfma_f32_16x16x32_bf16 v[204:207], v[124:127], a[180:183], v[204:207]\n"
    "ds_read_b64_tr_b16 v[114:115], v70 offset:1024\n"
    "v_exp_f32 v72, v72\n"
    "v_exp_f32 v73, v73\n"
    "v_exp_f32 v74, v74\n"
    "v_mfma_f32_16x16x32_bf16 v[204:207], v[120:123], a[176:179], v[204:207]\n"
    "v_exp_f32 v75, v75\n"
    "v_exp_f32 v76, v76\n"
    "v_exp_f32 v77, v77\n"
    "v_mfma_f32_16x16x32_bf16 v[172:175], v[120:123], a[168:171], v[172:175]\n"
    "ds_read_b64_tr_b16 v[116:117], v70 offset:8192\n"
    "v_exp_f32 v78, v78\n"
    "v_exp_f32 v79, v79\n"
    "v_add_f32 v72, 1.0, v72\n"
    "v_mfma_f32_16x16x32_bf16 v[172:175], v[124:127], a[172:175], v[172:175]\n"
    "v_add_f32 v73, 1.0, v73\n"
    "v_add_f32 v74, 1.0, v74\n"
    "v_add_f32 v75, 1.0, v75\n"
    "v_mfma_f32_16x16x32_bf16 v[140:143], v[124:127], a[164:167], v[140:143]\n"
    "ds_read_b64_tr_b16 v[118:119], v70 offset:9216\n"
    "v_add_f32 v76, 1.0, v76\n"
    "v_add_f32 v77, 1.0, v77\n"
    "v_add_f32 v78, 1.0, v78\n"
    "v_mfma_f32_16x16x32_bf16 v[140:143], v[120:123], a[160:163], v[140:143]\n"
    "v_add_f32 v79, 1.0, v79\n"
    "v_rcp_f32 v72, v72\n"
    "v_rcp_f32 v73, v73\n"
    "s_waitcnt lgkmcnt(8)\n"
    "v_mfma_f32_16x16x32_bf16 v[136:139], v[96:99], a[192:195], v[136:139]\n"
    "ds_read_b64_tr_b16 v[120:121], v71 offset:0\n"
    "v_rcp_f32 v74, v74\n"
    "v_rcp_f32 v75, v75\n"
    "v_rcp_f32 v76, v76\n"
    "v_mfma_f32_16x16x32_bf16 v[136:139], v[100:103], a[196:199], v[136:139]\n"
    "v_rcp_f32 v77, v77\n"
    "v_rcp_f32 v78, v78\n"
    "v_rcp_f32 v79, v79\n"
    "v_mfma_f32_16x16x32_bf16 v[168:171], v[100:103], a[204:207], v[168:171]\n"
    "ds_read_b64_tr_b16 v[122:123], v71 offset:1024\n"
    "v_mul_f32 v72, v72, v192\n"
    "v_mul_f32 v73, v73, v193\n"
    "v_mul_f32 v74, v74, v194\n"
    "v_mfma_f32_16x16x32_bf16 v[168:171], v[96:99], a[200:203], v[168:171]\n"
    "v_mul_f32 v75, v75, v195\n"
    "v_mul_f32 v76, v76, v196\n"
    "v_mul_f32 v77, v77, v197\n"
    "v_mfma_f32_16x16x32_bf16 v[200:203], v[96:99], a[208:211], v[200:203]\n"
    "ds_read_b64_tr_b16 v[124:125], v71 offset:8192\n"
    "v_mul_f32 v78, v78, v198\n"
    "v_mul_f32 v79, v79, v199\n"
    "v_cvt_pk_bf16_f32 v192, v72, v73\n"
    "v_mfma_f32_16x16x32_bf16 v[200:203], v[100:103], a[212:215], v[200:203]\n"
    "v_cvt_pk_bf16_f32 v193, v74, v75\n"
    "v_cvt_pk_bf16_f32 v194, v76, v77\n"
    "v_cvt_pk_bf16_f32 v195, v78, v79\n"
    "v_mfma_f32_16x16x32_bf16 v[232:235], v[100:103], a[220:223], v[232:235]\n"
    "ds_read_b64_tr_b16 v[126:127], v71 offset:9216\n"
    "v_cndmask_b32 v196, v192, v194, s[92:93]\n"
    "v_cndmask_b32 v197, v193, v195, s[92:93]\n"
    "v_cndmask_b32 v198, v194, v192, s[92:93]\n"
    "v_mfma_f32_16x16x32_bf16 v[232:235], v[96:99], a[216:219], v[232:235]\n"
    "v_cndmask_b32 v199, v195, v193, s[92:93]\n"
    "s_waitcnt lgkmcnt(8)\n"
    "v_mfma_f32_16x16x32_bf16 v[236:239], v[104:107], a[216:219], v[236:239]\n"
    "ds_read_b64_tr_b16 v[96:97], v60 offset:33280\n"
    "v_mfma_f32_16x16x32_bf16 v[236:239], v[108:111], a[220:223], v[236:239]\n"
    "buffer_store_dwordx4 v[196:199], v46, s[60:63], s90 offen offset:0 sc1\n"
    "v_mfma_f32_16x16x32_bf16 v[204:207], v[108:111], a[212:215], v[204:207]\n"
    "ds_read_b64_tr_b16 v[98:99], v60 offset:34304\n"
    "v_mul_f32 v80, v224, v224\n"
    "v_mul_f32 v81, v225, v225\n"
    "v_mul_f32 v82, v226, v226\n"
    "v_mfma_f32_16x16x32_bf16 v[204:207], v[104:107], a[208:211], v[204:207]\n"
    "v_mul_f32 v83, v227, v227\n"
    "v_mul_f32 v84, v228, v228\n"
    "v_mul_f32 v85, v229, v229\n"
    "v_mfma_f32_16x16x32_bf16 v[172:175], v[104:107], a[200:203], v[172:175]\n"
    "ds_read_b64_tr_b16 v[100:101], v60 offset:41472\n"
    "v_mul_f32 v86, v230, v230\n"
    "v_mul_f32 v87, v231, v231\n"
    "v_fma_f32 v80, v80, v44, v45\n"
    "v_mfma_f32_16x16x32_bf16 v[172:175], v[108:111], a[204:207], v[172:175]\n"
    "v_fma_f32 v81, v81, v44, v45\n"
    "v_fma_f32 v82, v82, v44, v45\n"
    "v_fma_f32 v83, v83, v44, v45\n"
    "v_mfma_f32_16x16x32_bf16 v[140:143], v[108:111], a[196:199], v[140:143]\n"
    "ds_read_b64_tr_b16 v[102:103], v60 offset:42496\n"
    "v_fma_f32 v84, v84, v44, v45\n"
    "v_fma_f32 v85, v85, v44, v45\n"
    "v_fma_f32 v86, v86, v44, v45\n"
    "v_mfma_f32_16x16x32_bf16 v[140:143], v[104:107], a[192:195], v[140:143]\n"
    "v_fma_f32 v87, v87, v44, v45\n"
    "v_mul_f32 v80, v80, v224\n"
    "v_mul_f32 v81, v81, v225\n"
    "s_waitcnt lgkmcnt(8)\n"
    "v_mfma_f32_16x16x32_bf16 v[136:139], v[112:115], a[224:227], v[136:139]\n"
    "ds_read_b64_tr_b16 v[104:105], v61 offset:33280\n"
    "v_mul_f32 v82, v82, v226\n"
    "v_mul_f32 v83, v83, v227\n"
    "v_mul_f32 v84, v84, v228\n"
    "v_mfma_f32_16x16x32_bf16 v[136:139], v[116:119], a[228:231], v[136:139]\n"
    "v_mul_f32 v85, v85, v229\n"
    "v_mul_f32 v86, v86, v230\n"
    "v_mul_f32 v87, v87, v231\n"
    "v_mfma_f32_16x16x32_bf16 v[168:171], v[116:119], a[236:239], v[168:171]\n"
    "ds_read_b64_tr_b16 v[106:107], v61 offset:34304\n"
    "v_exp_f32 v80, v80\n"
    "v_exp_f32 v81, v81\n"
    "v_exp_f32 v82, v82\n"
    "v_mfma_f32_16x16x32_bf16 v[168:171], v[112:115], a[232:235], v[168:171]\n"
    "v_exp_f32 v83, v83\n"
    "v_exp_f32 v84, v84\n"
    "v_exp_f32 v85, v85\n"
    "v_mfma_f32_16x16x32_bf16 v[200:203], v[112:115], a[240:243], v[200:203]\n"
    "ds_read_b64_tr_b16 v[108:109], v61 offset:41472\n"
    "v_exp_f32 v86, v86\n"
    "v_exp_f32 v87, v87\n"
    "v_add_f32 v80, 1.0, v80\n"
    "v_mfma_f32_16x16x32_bf16 v[200:203], v[116:119], a[244:247], v[200:203]\n"
    "v_add_f32 v81, 1.0, v81\n"
    "v_add_f32 v82, 1.0, v82\n"
    "v_add_f32 v83, 1.0, v83\n"
    "v_mfma_f32_16x16x32_bf16 v[232:235], v[116:119], a[252:255], v[232:235]\n"
    "ds_read_b64_tr_b16 v[110:111], v61 offset:42496\n"
    "v_add_f32 v84, 1.0, v84\n"
    "v_add_f32 v85, 1.0, v85\n"
    "v_add_f32 v86, 1.0, v86\n"
    "v_mfma_f32_16x16x32_bf16 v[232:235], v[112:115], a[248:251], v[232:235]\n"
    "v_add_f32 v87, 1.0, v87\n"
    "v_rcp_f32 v80, v80\n"
    "v_rcp_f32 v81, v81\n"
    "s_waitcnt lgkmcnt(8)\n"
    "v_mfma_f32_16x16x32_bf16 v[236:239], v[120:123], a[248:251], v[236:239]\n"
    "ds_read_b64_tr_b16 v[112:113], v60 offset:49664\n"
    "v_rcp_f32 v82, v82\n"
    "v_rcp_f32 v83, v83\n"
    "v_rcp_f32 v84, v84\n"
    "v_mfma_f32_16x16x32_bf16 v[236:239], v[124:127], a[252:255], v[236:239]\n"
    "v_rcp_f32 v85, v85\n"
    "v_rcp_f32 v86, v86\n"
    "v_rcp_f32 v87, v87\n"
    "v_mfma_f32_16x16x32_bf16 v[204:207], v[124:127], a[244:247], v[204:207]\n"
    "ds_read_b64_tr_b16 v[114:115], v60 offset:50688\n"
    "v_mul_f32 v80, v80, v224\n"
    "v_mul_f32 v81, v81, v225\n"
    "v_mul_f32 v82, v82, v226\n"
    "v_mfma_f32_16x16x32_bf16 v[204:207], v[120:123], a[240:243], v[204:207]\n"
    "v_mul_f32 v83, v83, v227\n"
    "v_mul_f32 v84, v84, v228\n"
    "v_mul_f32 v85, v85, v229\n"
    "v_mfma_f32_16x16x32_bf16 v[172:175], v[120:123], a[232:235], v[172:175]\n"
    "ds_read_b64_tr_b16 v[116:117], v60 offset:57856\n"
    "v_mul_f32 v86, v86, v230\n"
    "v_mul_f32 v87, v87, v231\n"
    "v_cvt_pk_bf16_f32 v224, v80, v81\n"
    "v_mfma_f32_16x16x32_bf16 v[172:175], v[124:127], a[236:239], v[172:175]\n"
    "v_cvt_pk_bf16_f32 v225, v82, v83\n"
    "v_cvt_pk_bf16_f32 v226, v84, v85\n"
    "v_cvt_pk_bf16_f32 v227, v86, v87\n"
    "v_mfma_f32_16x16x32_bf16 v[140:143], v[124:127], a[228:231], v[140:143]\n"
    "ds_read_b64_tr_b16 v[118:119], v60 offset:58880\n"
    "v_cndmask_b32 v228, v224, v226, s[92:93]\n"
    "v_cndmask_b32 v229, v225, v227, s[92:93]\n"
    "v_cndmask_b32 v230, v226, v224, s[92:93]\n"
    "v_mfma_f32_16x16x32_bf16 v[140:143], v[120:123], a[224:227], v[140:143]\n"
    "v_cndmask_b32 v231, v227, v225, s[92:93]\n"
    "buffer_store_dwordx4 v[228:231], v46, s[60:63], s91 offen offset:0 sc1\n"
    "s_waitcnt lgkmcnt(8)\n"
    "v_mfma_f32_16x16x32_bf16 v[144:147], v[96:99], a[64:67], v[144:147]\n"
    "ds_read_b64_tr_b16 v[120:121], v61 offset:49664\n"
    "s_nop 0\n"
    "s_nop 0\n"
    "s_nop 0\n"
    "v_mfma_f32_16x16x32_bf16 v[144:147], v[100:103], a[68:71], v[144:147]\n"
    "s_nop 0\n"
    "s_nop 0\n"
    "s_nop 0\n"
    "v_mfma_f32_16x16x32_bf16 v[176:179], v[100:103], a[76:79], v[176:179]\n"
    "ds_read_b64_tr_b16 v[122:123], v61 offset:50688\n"
    "v_mul_f32 v72, v136, v136\n"
    "v_mul_f32 v73, v137, v137\n"
    "v_mul_f32 v74, v138, v138\n"
    "v_mfma_f32_16x16x32_bf16 v[176:179], v[96:99], a[72:75], v[176:179]\n"
    "v_mul_f32 v75, v139, v139\n"
    "v_mul_f32 v76, v140, v140\n"
    "v_mul_f32 v77, v141, v141\n"
    "v_mfma_f32_16x16x32_bf16 v[208:211], v[96:99], a[80:83], v[208:211]\n"
    "ds_read_b64_tr_b16 v[124:125], v61 offset:57856\n"
    "v_mul_f32 v78, v142, v142\n"
    "v_mul_f32 v79, v143, v143\n"
    "v_fma_f32 v72, v72, v44, v45\n"
    "v_mfma_f32_16x16x32_bf16 v[208:211], v[100:103], a[84:87], v[208:211]\n"
    "v_fma_f32 v73, v73, v44, v45\n"
    "v_fma_f32 v74, v74, v44, v45\n"
    "v_fma_f32 v75, v75, v44, v45\n"
    "v_mfma_f32_16x16x32_bf16 v[240:243], v[100:103], a[92:95], v[240:243]\n"
    "ds_read_b64_tr_b16 v[126:127], v61 offset:58880\n"
    "v_fma_f32 v76, v76, v44, v45\n"
    "v_fma_f32 v77, v77, v44, v45\n"
    "v_fma_f32 v78, v78, v44, v45\n"
    "v_mfma_f32_16x16x32_bf16 v[240:243], v[96:99], a[88:91], v[240:243]\n"
    "v_fma_f32 v79, v79, v44, v45\n"
    "v_mul_f32 v72, v72, v136\n"
    "v_mul_f32 v73, v73, v137\n"
    "s_waitcnt lgkmcnt(8)\n"
    "v_mfma_f32_16x16x32_bf16 v[244:247], v[104:107], a[88:91], v[244:247]\n"
    "ds_read_b64_tr_b16 v[96:97], v64 offset:512\n"
    "v_mul_f32 v74, v74, v138\n"
    "v_mul_f32 v75, v75, v139\n"
    "v_mul_f32 v76, v76, v140\n"
    "v_mfma_f32_16x16x32_bf16 v[244:247], v[108:111], a[92:95], v[244:247]\n"
    "v_mul_f32 v77, v77, v141\n"
    "v_mul_f32 v78, v78, v142\n"
    "v_mul_f32 v79, v79, v143\n"
    "v_mfma_f32_16x16x32_bf16 v[212:215], v[108:111], a[84:87], v[212:215]\n"
    "ds_read_b64_tr_b16 v[98:99], v64 offset:1536\n"
    "v_exp_f32 v72, v72\n"
    "v_exp_f32 v73, v73\n"
    "v_exp_f32 v74, v74\n"
    "v_mfma_f32_16x16x32_bf16 v[212:215], v[104:107], a[80:83], v[212:215]\n"
    "v_exp_f32 v75, v75\n"
    "v_exp_f32 v76, v76\n"
    "v_exp_f32 v77, v77\n"
    "v_mfma_f32_16x16x32_bf16 v[180:183], v[104:107], a[72:75], v[180:183]\n"
    "ds_read_b64_tr_b16 v[100:101], v64 offset:8704\n"
    "v_exp_f32 v78, v78\n"
    "v_exp_f32 v79, v79\n"
    "v_add_f32 v72, 1.0, v72\n"
    "v_mfma_f32_16x16x32_bf16 v[180:183], v[108:111], a[76:79], v[180:183]\n"
    "v_add_f32 v73, 1.0, v73\n"
    "v_add_f32 v74, 1.0, v74\n"
    "v_add_f32 v75, 1.0, v75\n"
    "v_mfma_f32_16x16x32_bf16 v[148:151], v[108:111], a[68:71], v[148:151]\n"
    "ds_read_b64_tr_b16 v[102:103], v64 offset:9728\n"
    "v_add_f32 v76, 1.0, v76\n"
    "v_add_f32 v77, 1.0, v77\n"
    "v_add_f32 v78, 1.0, v78\n"
    "v_mfma_f32_16x16x32_bf16 v[148:151], v[104:107], a[64:67], v[148:151]\n"
    "v_add_f32 v79, 1.0, v79\n"
    "v_rcp_f32 v72, v72\n"
    "v_rcp_f32 v73, v73\n"
    "s_waitcnt lgkmcnt(8)\n"
    "v_mfma_f32_16x16x32_bf16 v[144:147], v[112:115], a[96:99], v[144:147]\n"
    "ds_read_b64_tr_b16 v[104:105], v65 offset:512\n"
    "v_rcp_f32 v74, v74\n"
    "v_rcp_f32 v75, v75\n"
    "v_rcp_f32 v76, v76\n"
    "v_mfma_f32_16x16x32_bf16 v[144:147], v[116:119], a[100:103], v[144:147]\n"
    "v_rcp_f32 v77, v77\n"
    "v_rcp_f32 v78, v78\n"
    "v_rcp_f32 v79, v79\n"
    "v_mfma_f32_16x16x32_bf16 v[176:179], v[116:119], a[108:111], v[176:179]\n"
    "ds_read_b64_tr_b16 v[106:107], v65 offset:1536\n"
    "v_mul_f32 v72, v72, v136\n"
    "v_mul_f32 v73, v73, v137\n"
    "v_mul_f32 v74, v74, v138\n"
    "v_mfma_f32_16x16x32_bf16 v[176:179], v[112:115], a[104:107], v[176:179]\n"
    "v_mul_f32 v75, v75, v139\n"
    "v_mul_f32 v76, v76, v140\n"
    "v_mul_f32 v77, v77, v141\n"
    "v_mfma_f32_16x16x32_bf16 v[208:211], v[112:115], a[112:115], v[208:211]\n"
    "ds_read_b64_tr_b16 v[108:109], v65 offset:8704\n"
    "v_mul_f32 v78, v78, v142\n"
    "v_mul_f32 v79, v79, v143\n"
    "v_cvt_pk_bf16_f32 v136, v72, v73\n"
    "v_mfma_f32_16x16x32_bf16 v[208:211], v[116:119], a[116:119], v[208:211]\n"
    "v_cvt_pk_bf16_f32 v137, v74, v75\n"
    "v_cvt_pk_bf16_f32 v138, v76, v77\n"
    "v_cvt_pk_bf16_f32 v139, v78, v79\n"
    "v_mfma_f32_16x16x32_bf16 v[240:243], v[116:119], a[124:127], v[240:243]\n"
    "ds_read_b64_tr_b16 v[110:111], v65 offset:9728\n"
    "v_cndmask_b32 v140, v136, v138, s[92:93]\n"
    "v_cndmask_b32 v141, v137, v139, s[92:93]\n"
    "v_cndmask_b32 v142, v138, v136, s[92:93]\n"
    "v_mfma_f32_16x16x32_bf16 v[240:243], v[112:115], a[120:123], v[240:243]\n"
    "v_cndmask_b32 v143, v139, v137, s[92:93]\n"
    "s_waitcnt lgkmcnt(8)\n"
    "v_mfma_f32_16x16x32_bf16 v[244:247], v[120:123], a[120:123], v[244:247]\n"
    "ds_read_b64_tr_b16 v[112:113], v64 offset:16896\n"
    "v_mfma_f32_16x16x32_bf16 v[244:247], v[124:127], a[124:127], v[244:247]\n"
    "buffer_store_dwordx4 v[140:143], v46, s[60:63], s88 offen offset:64 sc1\n"
    "v_mfma_f32_16x16x32_bf16 v[212:215], v[124:127], a[116:119], v[212:215]\n"
    "ds_read_b64_tr_b16 v[114:115], v64 offset:17920\n"
    "v_mul_f32 v80, v168, v168\n"
    "v_mul_f32 v81, v169, v169\n"
    "v_mul_f32 v82, v170, v170\n"
    "v_mfma_f32_16x16x32_bf16 v[212:215], v[120:123], a[112:115], v[212:215]\n"
    "v_mul_f32 v83, v171, v171\n"
    "v_mul_f32 v84, v172, v172\n"
    "v_mul_f32 v85, v173, v173\n"
    "v_mfma_f32_16x16x32_bf16 v[180:183], v[120:123], a[104:107], v[180:183]\n"
    "ds_read_b64_tr_b16 v[116:117], v64 offset:25088\n"
    "v_mul_f32 v86, v174, v174\n"
    "v_mul_f32 v87, v175, v175\n"
    "v_fma_f32 v80, v80, v44, v45\n"
    "v_mfma_f32_16x16x32_bf16 v[180:183], v[124:127], a[108:111], v[180:183]\n"
    "v_fma_f32 v81, v81, v44, v45\n"
    "v_fma_f32 v82, v82, v44, v45\n"
    "v_fma_f32 v83, v83, v44, v45\n"
    "v_mfma_f32_16x16x32_bf16 v[148:151], v[124:127], a[100:103], v[148:151]\n"
    "ds_read_b64_tr_b16 v[118:119], v64 offset:26112\n"
    "v_fma_f32 v84, v84, v44, v45\n"
    "v_fma_f32 v85, v85, v44, v45\n"
    "v_fma_f32 v86, v86, v44, v45\n"
    "v_mfma_f32_16x16x32_bf16 v[148:151], v[120:123], a[96:99], v[148:151]\n"
    "v_fma_f32 v87, v87, v44, v45\n"
    "v_mul_f32 v80, v80, v168\n"
    "v_mul_f32 v81, v81, v169\n"
    "s_waitcnt lgkmcnt(8)\n"
    "v_mfma_f32_16x16x32_bf16 v[144:147], v[96:99], a[128:131], v[144:147]\n"
    "ds_read_b64_tr_b16 v[120:121], v65 offset:16896\n"
    "v_mul_f32 v82, v82, v170\n"
    "v_mul_f32 v83, v83, v171\n"
    "v_mul_f32 v84, v84, v172\n"
    "v_mfma_f32_16x16x32_bf16 v[144:147], v[100:103], a[132:135], v[144:147]\n"
    "v_mul_f32 v85, v85, v173\n"
    "v_mul_f32 v86, v86, v174\n"
    "v_mul_f32 v87, v87, v175\n"
    "v_mfma_f32_16x16x32_bf16 v[176:179], v[100:103], a[140:143], v[176:179]\n"
    "ds_read_b64_tr_b16 v[122:123], v65 offset:17920\n"
    "v_exp_f32 v80, v80\n"
    "v_exp_f32 v81, v81\n"
    "v_exp_f32 v82, v82\n"
    "v_mfma_f32_16x16x32_bf16 v[176:179], v[96:99], a[136:139], v[176:179]\n"
    "v_exp_f32 v83, v83\n"
    "v_exp_f32 v84, v84\n"
    "v_exp_f32 v85, v85\n"
    "v_mfma_f32_16x16x32_bf16 v[208:211], v[96:99], a[144:147], v[208:211]\n"
    "ds_read_b64_tr_b16 v[124:125], v65 offset:25088\n"
    "v_exp_f32 v86, v86\n"
    "v_exp_f32 v87, v87\n"
    "v_add_f32 v80, 1.0, v80\n"
    "v_mfma_f32_16x16x32_bf16 v[208:211], v[100:103], a[148:151], v[208:211]\n"
    "v_add_f32 v81, 1.0, v81\n"
    "v_add_f32 v82, 1.0, v82\n"
    "v_add_f32 v83, 1.0, v83\n"
    "v_mfma_f32_16x16x32_bf16 v[240:243], v[100:103], a[156:159], v[240:243]\n"
    "ds_read_b64_tr_b16 v[126:127], v65 offset:26112\n"
    "v_add_f32 v84, 1.0, v84\n"
    "v_add_f32 v85, 1.0, v85\n"
    "v_add_f32 v86, 1.0, v86\n"
    "v_mfma_f32_16x16x32_bf16 v[240:243], v[96:99], a[152:155], v[240:243]\n"
    "v_add_f32 v87, 1.0, v87\n"
    "v_rcp_f32 v80, v80\n"
    "v_rcp_f32 v81, v81\n"
    "s_waitcnt lgkmcnt(8)\n"
    "v_mfma_f32_16x16x32_bf16 v[244:247], v[104:107], a[152:155], v[244:247]\n"
    "ds_read_b64_tr_b16 v[96:97], v64 offset:33280\n"
    "v_rcp_f32 v82, v82\n"
    "v_rcp_f32 v83, v83\n"
    "v_rcp_f32 v84, v84\n"
    "v_mfma_f32_16x16x32_bf16 v[244:247], v[108:111], a[156:159], v[244:247]\n"
    "v_rcp_f32 v85, v85\n"
    "v_rcp_f32 v86, v86\n"
    "v_rcp_f32 v87, v87\n"
    "v_mfma_f32_16x16x32_bf16 v[212:215], v[108:111], a[148:151], v[212:215]\n"
    "ds_read_b64_tr_b16 v[98:99], v64 offset:34304\n"
    "v_mul_f32 v80, v80, v168\n"
    "v_mul_f32 v81, v81, v169\n"
    "v_mul_f32 v82, v82, v170\n"
    "v_mfma_f32_16x16x32_bf16 v[212:215], v[104:107], a[144:147], v[212:215]\n"
    "v_mul_f32 v83, v83, v171\n"
    "v_mul_f32 v84, v84, v172\n"
    "v_mul_f32 v85, v85, v173\n"
    "v_mfma_f32_16x16x32_bf16 v[180:183], v[104:107], a[136:139], v[180:183]\n"
    "ds_read_b64_tr_b16 v[100:101], v64 offset:41472\n"
    "v_mul_f32 v86, v86, v174\n"
    "v_mul_f32 v87, v87, v175\n"
    "v_cvt_pk_bf16_f32 v168, v80, v81\n"
    "v_mfma_f32_16x16x32_bf16 v[180:183], v[108:111], a[140:143], v[180:183]\n"
    "v_cvt_pk_bf16_f32 v169, v82, v83\n"
    "v_cvt_pk_bf16_f32 v170, v84, v85\n"
    "v_cvt_pk_bf16_f32 v171, v86, v87\n"
    "v_mfma_f32_16x16x32_bf16 v[148:151], v[108:111], a[132:135], v[148:151]\n"
    "ds_read_b64_tr_b16 v[102:103], v64 offset:42496\n"
    "v_cndmask_b32 v172, v168, v170, s[92:93]\n"
    "v_cndmask_b32 v173, v169, v171, s[92:93]\n"
    "v_cndmask_b32 v174, v170, v168, s[92:93]\n"
    "v_mfma_f32_16x16x32_bf16 v[148:151], v[104:107], a[128:131], v[148:151]\n"
    "v_cndmask_b32 v175, v171, v169, s[92:93]\n"
    "s_waitcnt lgkmcnt(8)\n"
    "v_mfma_f32_16x16x32_bf16 v[144:147], v[112:115], a[160:163], v[144:147]\n"
    "ds_read_b64_tr_b16 v[104:105], v65 offset:33280\n"
    "v_mfma_f32_16x16x32_bf16 v[144:147], v[116:119], a[164:167], v[144:147]\n"
    "buffer_store_dwordx4 v[172:175], v46, s[60:63], s89 offen offset:64 sc1\n"
    "v_mfma_f32_16x16x32_bf16 v[176:179], v[116:119], a[172:175], v[176:179]\n"
    "ds_read_b64_tr_b16 v[106:107], v65 offset:34304\n"
    "v_mul_f32 v72, v200, v200\n"
    "v_mul_f32 v73, v201, v201\n"
    "v_mul_f32 v74, v202, v202\n"
    "v_mfma_f32_16x16x32_bf16 v[176:179], v[112:115], a[168:171], v[176:179]\n"
    "v_mul_f32 v75, v203, v203\n"
    "v_mul_f32 v76, v204, v204\n"
    "v_mul_f32 v77, v205, v205\n"
    "v_mfma_f32_16x16x32_bf16 v[208:211], v[112:115], a[176:179], v[208:211]\n"
    "ds_read_b64_tr_b16 v[108:109], v65 offset:41472\n"
    "v_mul_f32 v78, v206, v206\n"
    "v_mul_f32 v79, v207, v207\n"
    "v_fma_f32 v72, v72, v44, v45\n"
    "v_mfma_f32_16x16x32_bf16 v[208:211], v[116:119], a[180:183], v[208:211]\n"
    "v_fma_f32 v73, v73, v44, v45\n"
    "v_fma_f32 v74, v74, v44, v45\n"
    "v_fma_f32 v75, v75, v44, v45\n"
    "v_mfma_f32_16x16x32_bf16 v[240:243], v[116:119], a[188:191], v[240:243]\n"
    "ds_read_b64_tr_b16 v[110:111], v65 offset:42496\n"
    "v_fma_f32 v76, v76, v44, v45\n"
    "v_fma_f32 v77, v77, v44, v45\n"
    "v_fma_f32 v78, v78, v44, v45\n"
    "v_mfma_f32_16x16x32_bf16 v[240:243], v[112:115], a[184:187], v[240:243]\n"
    "v_fma_f32 v79, v79, v44, v45\n"
    "v_mul_f32 v72, v72, v200\n"
    "v_mul_f32 v73, v73, v201\n"
    "s_waitcnt lgkmcnt(8)\n"
    "v_mfma_f32_16x16x32_bf16 v[244:247], v[120:123], a[184:187], v[244:247]\n"
    "ds_read_b64_tr_b16 v[112:113], v68 offset:512\n"
    "v_mul_f32 v74, v74, v202\n"
    "v_mul_f32 v75, v75, v203\n"
    "v_mul_f32 v76, v76, v204\n"
    "v_mfma_f32_16x16x32_bf16 v[244:247], v[124:127], a[188:191], v[244:247]\n"
    "v_mul_f32 v77, v77, v205\n"
    "v_mul_f32 v78, v78, v206\n"
    "v_mul_f32 v79, v79, v207\n"
    "v_mfma_f32_16x16x32_bf16 v[212:215], v[124:127], a[180:183], v[212:215]\n"
    "ds_read_b64_tr_b16 v[114:115], v68 offset:1536\n"
    "v_exp_f32 v72, v72\n"
    "v_exp_f32 v73, v73\n"
    "v_exp_f32 v74, v74\n"
    "v_mfma_f32_16x16x32_bf16 v[212:215], v[120:123], a[176:179], v[212:215]\n"
    "v_exp_f32 v75, v75\n"
    "v_exp_f32 v76, v76\n"
    "v_exp_f32 v77, v77\n"
    "v_mfma_f32_16x16x32_bf16 v[180:183], v[120:123], a[168:171], v[180:183]\n"
    "ds_read_b64_tr_b16 v[116:117], v68 offset:8704\n"
    "v_exp_f32 v78, v78\n"
    "v_exp_f32 v79, v79\n"
    "v_add_f32 v72, 1.0, v72\n"
    "v_mfma_f32_16x16x32_bf16 v[180:183], v[124:127], a[172:175], v[180:183]\n"
    "v_add_f32 v73, 1.0, v73\n"
    "v_add_f32 v74, 1.0, v74\n"
    "v_add_f32 v75, 1.0, v75\n"
    "v_mfma_f32_16x16x32_bf16 v[148:151], v[124:127], a[164:167], v[148:151]\n"
    "ds_read_b64_tr_b16 v[118:119], v68 offset:9728\n"
    "v_add_f32 v76, 1.0, v76\n"
    "v_add_f32 v77, 1.0, v77\n"
    "v_add_f32 v78, 1.0, v78\n"
    "v_mfma_f32_16x16x32_bf16 v[148:151], v[120:123], a[160:163], v[148:151]\n"
    "v_add_f32 v79, 1.0, v79\n"
    "v_rcp_f32 v72, v72\n"
    "v_rcp_f32 v73, v73\n"
    "s_waitcnt lgkmcnt(8)\n"
    "v_mfma_f32_16x16x32_bf16 v[144:147], v[96:99], a[192:195], v[144:147]\n"
    "ds_read_b64_tr_b16 v[120:121], v69 offset:512\n"
    "v_rcp_f32 v74, v74\n"
    "v_rcp_f32 v75, v75\n"
    "v_rcp_f32 v76, v76\n"
    "v_mfma_f32_16x16x32_bf16 v[144:147], v[100:103], a[196:199], v[144:147]\n"
    "v_rcp_f32 v77, v77\n"
    "v_rcp_f32 v78, v78\n"
    "v_rcp_f32 v79, v79\n"
    "v_mfma_f32_16x16x32_bf16 v[176:179], v[100:103], a[204:207], v[176:179]\n"
    "ds_read_b64_tr_b16 v[122:123], v69 offset:1536\n"
    "v_mul_f32 v72, v72, v200\n"
    "v_mul_f32 v73, v73, v201\n"
    "v_mul_f32 v74, v74, v202\n"
    "v_mfma_f32_16x16x32_bf16 v[176:179], v[96:99], a[200:203], v[176:179]\n"
    "v_mul_f32 v75, v75, v203\n"
    "v_mul_f32 v76, v76, v204\n"
    "v_mul_f32 v77, v77, v205\n"
    "v_mfma_f32_16x16x32_bf16 v[208:211], v[96:99], a[208:211], v[208:211]\n"
    "ds_read_b64_tr_b16 v[124:125], v69 offset:8704\n"
    "v_mul_f32 v78, v78, v206\n"
    "v_mul_f32 v79, v79, v207\n"
    "v_cvt_pk_bf16_f32 v200, v72, v73\n"
    "v_mfma_f32_16x16x32_bf16 v[208:211], v[100:103], a[212:215], v[208:211]\n"
    "v_cvt_pk_bf16_f32 v201, v74, v75\n"
    "v_cvt_pk_bf16_f32 v202, v76, v77\n"
    "v_cvt_pk_bf16_f32 v203, v78, v79\n"
    "v_mfma_f32_16x16x32_bf16 v[240:243], v[100:103], a[220:223], v[240:243]\n"
    "ds_read_b64_tr_b16 v[126:127], v69 offset:9728\n"
    "v_cndmask_b32 v204, v200, v202, s[92:93]\n"
    "v_cndmask_b32 v205, v201, v203, s[92:93]\n"
    "v_cndmask_b32 v206, v202, v200, s[92:93]\n"
    "v_mfma_f32_16x16x32_bf16 v[240:243], v[96:99], a[216:219], v[240:243]\n"
    "v_cndmask_b32 v207, v203, v201, s[92:93]\n"
    "s_waitcnt lgkmcnt(8)\n"
    "v_mfma_f32_16x16x32_bf16 v[244:247], v[104:107], a[216:219], v[244:247]\n"
    "ds_read_b64_tr_b16 v[96:97], v62 offset:33280\n"
    "v_mfma_f32_16x16x32_bf16 v[244:247], v[108:111], a[220:223], v[244:247]\n"
    "buffer_store_dwordx4 v[204:207], v46, s[60:63], s90 offen offset:64 sc1\n"
    "v_mfma_f32_16x16x32_bf16 v[212:215], v[108:111], a[212:215], v[212:215]\n"
    "ds_read_b64_tr_b16 v[98:99], v62 offset:34304\n"
    "v_mul_f32 v80, v232, v232\n"
    "v_mul_f32 v81, v233, v233\n"
    "v_mul_f32 v82, v234, v234\n"
    "v_mfma_f32_16x16x32_bf16 v[212:215], v[104:107], a[208:211], v[212:215]\n"
    "v_mul_f32 v83, v235, v235\n"
    "v_mul_f32 v84, v236, v236\n"
    "v_mul_f32 v85, v237, v237\n"
    "v_mfma_f32_16x16x32_bf16 v[180:183], v[104:107], a[200:203], v[180:183]\n"
    "ds_read_b64_tr_b16 v[100:101], v62 offset:41472\n"
    "v_mul_f32 v86, v238, v238\n"
    "v_mul_f32 v87, v239, v239\n"
    "v_fma_f32 v80, v80, v44, v45\n"
    "v_mfma_f32_16x16x32_bf16 v[180:183], v[108:111], a[204:207], v[180:183]\n"
    "v_fma_f32 v81, v81, v44, v45\n"
    "v_fma_f32 v82, v82, v44, v45\n"
    "v_fma_f32 v83, v83, v44, v45\n"
    "v_mfma_f32_16x16x32_bf16 v[148:151], v[108:111], a[196:199], v[148:151]\n"
    "ds_read_b64_tr_b16 v[102:103], v62 offset:42496\n"
    "v_fma_f32 v84, v84, v44, v45\n"
    "v_fma_f32 v85, v85, v44, v45\n"
    "v_fma_f32 v86, v86, v44, v45\n"
    "v_mfma_f32_16x16x32_bf16 v[148:151], v[104:107], a[192:195], v[148:151]\n"
    "v_fma_f32 v87, v87, v44, v45\n"
    "v_mul_f32 v80, v80, v232\n"
    "v_mul_f32 v81, v81, v233\n"
    "s_waitcnt lgkmcnt(8)\n"
    "v_mfma_f32_16x16x32_bf16 v[144:147], v[112:115], a[224:227], v[144:147]\n"
    "ds_read_b64_tr_b16 v[104:105], v63 offset:33280\n"
    "v_mul_f32 v82, v82, v234\n"
    "v_mul_f32 v83, v83, v235\n"
    "v_mul_f32 v84, v84, v236\n"
    "v_mfma_f32_16x16x32_bf16 v[144:147], v[116:119], a[228:231], v[144:147]\n"
    "v_mul_f32 v85, v85, v237\n"
    "v_mul_f32 v86, v86, v238\n"
    "v_mul_f32 v87, v87, v239\n"
    "v_mfma_f32_16x16x32_bf16 v[176:179], v[116:119], a[236:239], v[176:179]\n"
    "ds_read_b64_tr_b16 v[106:107], v63 offset:34304\n"
    "v_exp_f32 v80, v80\n"
    "v_exp_f32 v81, v81\n"
    "v_exp_f32 v82, v82\n"
    "v_mfma_f32_16x16x32_bf16 v[176:179], v[112:115], a[232:235], v[176:179]\n"
    "v_exp_f32 v83, v83\n"
    "v_exp_f32 v84, v84\n"
    "v_exp_f32 v85, v85\n"
    "v_mfma_f32_16x16x32_bf16 v[208:211], v[112:115], a[240:243], v[208:211]\n"
    "ds_read_b64_tr_b16 v[108:109], v63 offset:41472\n"
    "v_exp_f32 v86, v86\n"
    "v_exp_f32 v87, v87\n"
    "v_add_f32 v80, 1.0, v80\n"
    "v_mfma_f32_16x16x32_bf16 v[208:211], v[116:119], a[244:247], v[208:211]\n"
    "v_add_f32 v81, 1.0, v81\n"
    "v_add_f32 v82, 1.0, v82\n"
    "v_add_f32 v83, 1.0, v83\n"
    "v_mfma_f32_16x16x32_bf16 v[240:243], v[116:119], a[252:255], v[240:243]\n"
    "ds_read_b64_tr_b16 v[110:111], v63 offset:42496\n"
    "v_add_f32 v84, 1.0, v84\n"
    "v_add_f32 v85, 1.0, v85\n"
    "v_add_f32 v86, 1.0, v86\n"
    "v_mfma_f32_16x16x32_bf16 v[240:243], v[112:115], a[248:251], v[240:243]\n"
    "v_add_f32 v87, 1.0, v87\n"
    "v_rcp_f32 v80, v80\n"
    "v_rcp_f32 v81, v81\n"
    "s_waitcnt lgkmcnt(8)\n"
    "v_mfma_f32_16x16x32_bf16 v[244:247], v[120:123], a[248:251], v[244:247]\n"
    "ds_read_b64_tr_b16 v[112:113], v62 offset:49664\n"
    "v_rcp_f32 v82, v82\n"
    "v_rcp_f32 v83, v83\n"
    "v_rcp_f32 v84, v84\n"
    "v_mfma_f32_16x16x32_bf16 v[244:247], v[124:127], a[252:255], v[244:247]\n"
    "v_rcp_f32 v85, v85\n"
    "v_rcp_f32 v86, v86\n"
    "v_rcp_f32 v87, v87\n"
    "v_mfma_f32_16x16x32_bf16 v[212:215], v[124:127], a[244:247], v[212:215]\n"
    "ds_read_b64_tr_b16 v[114:115], v62 offset:50688\n"
    "v_mul_f32 v80, v80, v232\n"
    "v_mul_f32 v81, v81, v233\n"
    "v_mul_f32 v82, v82, v234\n"
    "v_mfma_f32_16x16x32_bf16 v[212:215], v[120:123], a[240:243], v[212:215]\n"
    "v_mul_f32 v83, v83, v235\n"
    "v_mul_f32 v84, v84, v236\n"
    "v_mul_f32 v85, v85, v237\n"
    "v_mfma_f32_16x16x32_bf16 v[180:183], v[120:123], a[232:235], v[180:183]\n"
    "ds_read_b64_tr_b16 v[116:117], v62 offset:57856\n"
    "v_mul_f32 v86, v86, v238\n"
    "v_mul_f32 v87, v87, v239\n"
    "v_cvt_pk_bf16_f32 v232, v80, v81\n"
    "v_mfma_f32_16x16x32_bf16 v[180:183], v[124:127], a[236:239], v[180:183]\n"
    "v_cvt_pk_bf16_f32 v233, v82, v83\n"
    "v_cvt_pk_bf16_f32 v234, v84, v85\n"
    "v_cvt_pk_bf16_f32 v235, v86, v87\n"
    "v_mfma_f32_16x16x32_bf16 v[148:151], v[124:127], a[228:231], v[148:151]\n"
    "ds_read_b64_tr_b16 v[118:119], v62 offset:58880\n"
    "v_cndmask_b32 v236, v232, v234, s[92:93]\n"
    "v_cndmask_b32 v237, v233, v235, s[92:93]\n"
    "v_cndmask_b32 v238, v234, v232, s[92:93]\n"
    "v_mfma_f32_16x16x32_bf16 v[148:151], v[120:123], a[224:227], v[148:151]\n"
    "v_cndmask_b32 v239, v235, v233, s[92:93]\n"
    "buffer_store_dwordx4 v[236:239], v46, s[60:63], s91 offen offset:64 sc1\n"
    "s_waitcnt lgkmcnt(8)\n"
    "v_mfma_f32_16x16x32_bf16 v[152:155], v[96:99], a[64:67], v[152:155]\n"
    "ds_read_b64_tr_b16 v[120:121], v63 offset:49664\n"
    "s_nop 0\n"
    "s_nop 0\n"
    "s_nop 0\n"
    "v_mfma_f32_16x16x32_bf16 v[152:155], v[100:103], a[68:71], v[152:155]\n"
    "s_nop 0\n"
    "s_nop 0\n"
    "s_nop 0\n"
    "v_mfma_f32_16x16x32_bf16 v[184:187], v[100:103], a[76:79], v[184:187]\n"
    "ds_read_b64_tr_b16 v[122:123], v63 offset:50688\n"
    "v_mul_f32 v72, v144, v144\n"
    "v_mul_f32 v73, v145, v145\n"
    "v_mul_f32 v74, v146, v146\n"
    "v_mfma_f32_16x16x32_bf16 v[184:187], v[96:99], a[72:75], v[184:187]\n"
    "v_mul_f32 v75, v147, v147\n"
    "v_mul_f32 v76, v148, v148\n"
    "v_mul_f32 v77, v149, v149\n"
    "v_mfma_f32_16x16x32_bf16 v[216:219], v[96:99], a[80:83], v[216:219]\n"
    "ds_read_b64_tr_b16 v[124:125], v63 offset:57856\n"
    "v_mul_f32 v78, v150, v150\n"
    "v_mul_f32 v79, v151, v151\n"
    "v_fma_f32 v72, v72, v44, v45\n"
    "v_mfma_f32_16x16x32_bf16 v[216:219], v[100:103], a[84:87], v[216:219]\n"
    "v_fma_f32 v73, v73, v44, v45\n"
    "v_fma_f32 v74, v74, v44, v45\n"
    "v_fma_f32 v75, v75, v44, v45\n"
    "v_mfma_f32_16x16x32_bf16 v[248:251], v[100:103], a[92:95], v[248:251]\n"
    "ds_read_b64_tr_b16 v[126:127], v63 offset:58880\n"
    "v_fma_f32 v76, v76, v44, v45\n"
    "v_fma_f32 v77, v77, v44, v45\n"
    "v_fma_f32 v78, v78, v44, v45\n"
    "v_mfma_f32_16x16x32_bf16 v[248:251], v[96:99], a[88:91], v[248:251]\n"
    "v_fma_f32 v79, v79, v44, v45\n"
    "v_mul_f32 v72, v72, v144\n"
    "v_mul_f32 v73, v73, v145\n"
    "s_waitcnt lgkmcnt(8)\n"
    "v_mfma_f32_16x16x32_bf16 v[252:255], v[104:107], a[88:91], v[252:255]\n"
    "ds_read_b64_tr_b16 v[96:97], v66 offset:512\n"
    "v_mul_f32 v74, v74, v146\n"
    "v_mul_f32 v75, v75, v147\n"
    "v_mul_f32 v76, v76, v148\n"
    "v_mfma_f32_16x16x32_bf16 v[252:255], v[108:111], a[92:95], v[252:255]\n"
    "v_mul_f32 v77, v77, v149\n"
    "v_mul_f32 v78, v78, v150\n"
    "v_mul_f32 v79, v79, v151\n"
    "v_mfma_f32_16x16x32_bf16 v[220:223], v[108:111], a[84:87], v[220:223]\n"
    "ds_read_b64_tr_b16 v[98:99], v66 offset:1536\n"
    "v_exp_f32 v72, v72\n"
    "v_exp_f32 v73, v73\n"
    "v_exp_f32 v74, v74\n"
    "v_mfma_f32_16x16x32_bf16 v[220:223], v[104:107], a[80:83], v[220:223]\n"
    "v_exp_f32 v75, v75\n"
    "v_exp_f32 v76, v76\n"
    "v_exp_f32 v77, v77\n"
    "v_mfma_f32_16x16x32_bf16 v[188:191], v[104:107], a[72:75], v[188:191]\n"
    "ds_read_b64_tr_b16 v[100:101], v66 offset:8704\n"
    "v_exp_f32 v78, v78\n"
    "v_exp_f32 v79, v79\n"
    "v_add_f32 v72, 1.0, v72\n"
    "v_mfma_f32_16x16x32_bf16 v[188:191], v[108:111], a[76:79], v[188:191]\n"
    "v_add_f32 v73, 1.0, v73\n"
    "v_add_f32 v74, 1.0, v74\n"
    "v_add_f32 v75, 1.0, v75\n"
    "v_mfma_f32_16x16x32_bf16 v[156:159], v[108:111], a[68:71], v[156:159]\n"
    "ds_read_b64_tr_b16 v[102:103], v66 offset:9728\n"
    "v_add_f32 v76, 1.0, v76\n"
    "v_add_f32 v77, 1.0, v77\n"
    "v_add_f32 v78, 1.0, v78\n"
    "v_mfma_f32_16x16x32_bf16 v[156:159], v[104:107], a[64:67], v[156:159]\n"
    "v_add_f32 v79, 1.0, v79\n"
    "v_rcp_f32 v72, v72\n"
    "v_rcp_f32 v73, v73\n"
    "s_waitcnt lgkmcnt(8)\n"
    "v_mfma_f32_16x16x32_bf16 v[152:155], v[112:115], a[96:99], v[152:155]\n"
    "ds_read_b64_tr_b16 v[104:105], v67 offset:512\n"
    "v_rcp_f32 v74, v74\n"
    "v_rcp_f32 v75, v75\n"
    "v_rcp_f32 v76, v76\n"
    "v_mfma_f32_16x16x32_bf16 v[152:155], v[116:119], a[100:103], v[152:155]\n"
    "v_rcp_f32 v77, v77\n"
    "v_rcp_f32 v78, v78\n"
    "v_rcp_f32 v79, v79\n"
    "v_mfma_f32_16x16x32_bf16 v[184:187], v[116:119], a[108:111], v[184:187]\n"
    "ds_read_b64_tr_b16 v[106:107], v67 offset:1536\n"
    "v_mul_f32 v72, v72, v144\n"
    "v_mul_f32 v73, v73, v145\n"
    "v_mul_f32 v74, v74, v146\n"
    "v_mfma_f32_16x16x32_bf16 v[184:187], v[112:115], a[104:107], v[184:187]\n"
    "v_mul_f32 v75, v75, v147\n"
    "v_mul_f32 v76, v76, v148\n"
    "v_mul_f32 v77, v77, v149\n"
    "v_mfma_f32_16x16x32_bf16 v[216:219], v[112:115], a[112:115], v[216:219]\n"
    "ds_read_b64_tr_b16 v[108:109], v67 offset:8704\n"
    "v_mul_f32 v78, v78, v150\n"
    "v_mul_f32 v79, v79, v151\n"
    "v_cvt_pk_bf16_f32 v144, v72, v73\n"
    "v_mfma_f32_16x16x32_bf16 v[216:219], v[116:119], a[116:119], v[216:219]\n"
    "v_cvt_pk_bf16_f32 v145, v74, v75\n"
    "v_cvt_pk_bf16_f32 v146, v76, v77\n"
    "v_cvt_pk_bf16_f32 v147, v78, v79\n"
    "v_mfma_f32_16x16x32_bf16 v[248:251], v[116:119], a[124:127], v[248:251]\n"
    "ds_read_b64_tr_b16 v[110:111], v67 offset:9728\n"
    "v_cndmask_b32 v148, v144, v146, s[92:93]\n"
    "v_cndmask_b32 v149, v145, v147, s[92:93]\n"
    "v_cndmask_b32 v150, v146, v144, s[92:93]\n"
    "v_mfma_f32_16x16x32_bf16 v[248:251], v[112:115], a[120:123], v[248:251]\n"
    "v_cndmask_b32 v151, v147, v145, s[92:93]\n"
    "s_waitcnt lgkmcnt(8)\n"
    "v_mfma_f32_16x16x32_bf16 v[252:255], v[120:123], a[120:123], v[252:255]\n"
    "ds_read_b64_tr_b16 v[112:113], v66 offset:16896\n"
    "v_mfma_f32_16x16x32_bf16 v[252:255], v[124:127], a[124:127], v[252:255]\n"
    "buffer_store_dwordx4 v[148:151], v46, s[60:63], s88 offen offset:128 sc1\n"
    "v_mfma_f32_16x16x32_bf16 v[220:223], v[124:127], a[116:119], v[220:223]\n"
    "ds_read_b64_tr_b16 v[114:115], v66 offset:17920\n"
    "v_mul_f32 v80, v176, v176\n"
    "v_mul_f32 v81, v177, v177\n"
    "v_mul_f32 v82, v178, v178\n"
    "v_mfma_f32_16x16x32_bf16 v[220:223], v[120:123], a[112:115], v[220:223]\n"
    "v_mul_f32 v83, v179, v179\n"
    "v_mul_f32 v84, v180, v180\n"
    "v_mul_f32 v85, v181, v181\n"
    "v_mfma_f32_16x16x32_bf16 v[188:191], v[120:123], a[104:107], v[188:191]\n"
    "ds_read_b64_tr_b16 v[116:117], v66 offset:25088\n"
    "v_mul_f32 v86, v182, v182\n"
    "v_mul_f32 v87, v183, v183\n"
    "v_fma_f32 v80, v80, v44, v45\n"
    "v_mfma_f32_16x16x32_bf16 v[188:191], v[124:127], a[108:111], v[188:191]\n"
    "v_fma_f32 v81, v81, v44, v45\n"
    "v_fma_f32 v82, v82, v44, v45\n"
    "v_fma_f32 v83, v83, v44, v45\n"
    "v_mfma_f32_16x16x32_bf16 v[156:159], v[124:127], a[100:103], v[156:159]\n"
    "ds_read_b64_tr_b16 v[118:119], v66 offset:26112\n"
    "v_fma_f32 v84, v84, v44, v45\n"
    "v_fma_f32 v85, v85, v44, v45\n"
    "v_fma_f32 v86, v86, v44, v45\n"
    "v_mfma_f32_16x16x32_bf16 v[156:159], v[120:123], a[96:99], v[156:159]\n"
    "v_fma_f32 v87, v87, v44, v45\n"
    "v_mul_f32 v80, v80, v176\n"
    "v_mul_f32 v81, v81, v177\n"
    "s_waitcnt lgkmcnt(8)\n"
    "v_mfma_f32_16x16x32_bf16 v[152:155], v[96:99], a[128:131], v[152:155]\n"
    "ds_read_b64_tr_b16 v[120:121], v67 offset:16896\n"
    "v_mul_f32 v82, v82, v178\n"
    "v_mul_f32 v83, v83, v179\n"
    "v_mul_f32 v84, v84, v180\n"
    "v_mfma_f32_16x16x32_bf16 v[152:155], v[100:103], a[132:135], v[152:155]\n"
    "v_mul_f32 v85, v85, v181\n"
    "v_mul_f32 v86, v86, v182\n"
    "v_mul_f32 v87, v87, v183\n"
    "v_mfma_f32_16x16x32_bf16 v[184:187], v[100:103], a[140:143], v[184:187]\n"
    "ds_read_b64_tr_b16 v[122:123], v67 offset:17920\n"
    "v_exp_f32 v80, v80\n"
    "v_exp_f32 v81, v81\n"
    "v_exp_f32 v82, v82\n"
    "v_mfma_f32_16x16x32_bf16 v[184:187], v[96:99], a[136:139], v[184:187]\n"
    "v_exp_f32 v83, v83\n"
    "v_exp_f32 v84, v84\n"
    "v_exp_f32 v85, v85\n"
    "v_mfma_f32_16x16x32_bf16 v[216:219], v[96:99], a[144:147], v[216:219]\n"
    "ds_read_b64_tr_b16 v[124:125], v67 offset:25088\n"
    "v_exp_f32 v86, v86\n"
    "v_exp_f32 v87, v87\n"
    "v_add_f32 v80, 1.0, v80\n"
    "v_mfma_f32_16x16x32_bf16 v[216:219], v[100:103], a[148:151], v[216:219]\n"
    "v_add_f32 v81, 1.0, v81\n"
    "v_add_f32 v82, 1.0, v82\n"
    "v_add_f32 v83, 1.0, v83\n"
    "v_mfma_f32_16x16x32_bf16 v[248:251], v[100:103], a[156:159], v[248:251]\n"
    "ds_read_b64_tr_b16 v[126:127], v67 offset:26112\n"
    "v_add_f32 v84, 1.0, v84\n"
    "v_add_f32 v85, 1.0, v85\n"
    "v_add_f32 v86, 1.0, v86\n"
    "v_mfma_f32_16x16x32_bf16 v[248:251], v[96:99], a[152:155], v[248:251]\n"
    "v_add_f32 v87, 1.0, v87\n"
    "v_rcp_f32 v80, v80\n"
    "v_rcp_f32 v81, v81\n"
    "s_waitcnt lgkmcnt(8)\n"
    "v_mfma_f32_16x16x32_bf16 v[252:255], v[104:107], a[152:155], v[252:255]\n"
    "ds_read_b64_tr_b16 v[96:97], v66 offset:33280\n"
    "v_rcp_f32 v82, v82\n"
    "v_rcp_f32 v83, v83\n"
    "v_rcp_f32 v84, v84\n"
    "v_mfma_f32_16x16x32_bf16 v[252:255], v[108:111], a[156:159], v[252:255]\n"
    "v_rcp_f32 v85, v85\n"
    "v_rcp_f32 v86, v86\n"
    "v_rcp_f32 v87, v87\n"
    "v_mfma_f32_16x16x32_bf16 v[220:223], v[108:111], a[148:151], v[220:223]\n"
    "ds_read_b64_tr_b16 v[98:99], v66 offset:34304\n"
    "v_mul_f32 v80, v80, v176\n"
    "v_mul_f32 v81, v81, v177\n"
    "v_mul_f32 v82, v82, v178\n"
    "v_mfma_f32_16x16x32_bf16 v[220:223], v[104:107], a[144:147], v[220:223]\n"
    "v_mul_f32 v83, v83, v179\n"
    "v_mul_f32 v84, v84, v180\n"
    "v_mul_f32 v85, v85, v181\n"
    "v_mfma_f32_16x16x32_bf16 v[188:191], v[104:107], a[136:139], v[188:191]\n"
    "ds_read_b64_tr_b16 v[100:101], v66 offset:41472\n"
    "v_mul_f32 v86, v86, v182\n"
    "v_mul_f32 v87, v87, v183\n"
    "v_cvt_pk_bf16_f32 v176, v80, v81\n"
    "v_mfma_f32_16x16x32_bf16 v[188:191], v[108:111], a[140:143], v[188:191]\n"
    "v_cvt_pk_bf16_f32 v177, v82, v83\n"
    "v_cvt_pk_bf16_f32 v178, v84, v85\n"
    "v_cvt_pk_bf16_f32 v179, v86, v87\n"
    "v_mfma_f32_16x16x32_bf16 v[156:159], v[108:111], a[132:135], v[156:159]\n"
    "ds_read_b64_tr_b16 v[102:103], v66 offset:42496\n"
    "v_cndmask_b32 v180, v176, v178, s[92:93]\n"
    "v_cndmask_b32 v181, v177, v179, s[92:93]\n"
    "v_cndmask_b32 v182, v178, v176, s[92:93]\n"
    "v_mfma_f32_16x16x32_bf16 v[156:159], v[104:107], a[128:131], v[156:159]\n"
    "v_cndmask_b32 v183, v179, v177, s[92:93]\n"
    "s_waitcnt lgkmcnt(8)\n"
    "v_mfma_f32_16x16x32_bf16 v[152:155], v[112:115], a[160:163], v[152:155]\n"
    "ds_read_b64_tr_b16 v[104:105], v67 offset:33280\n"
    "v_mfma_f32_16x16x32_bf16 v[152:155], v[116:119], a[164:167], v[152:155]\n"
    "buffer_store_dwordx4 v[180:183], v46, s[60:63], s89 offen offset:128 sc1\n"
    "v_mfma_f32_16x16x32_bf16 v[184:187], v[116:119], a[172:175], v[184:187]\n"
    "ds_read_b64_tr_b16 v[106:107], v67 offset:34304\n"
    "v_mul_f32 v72, v208, v208\n"
    "v_mul_f32 v73, v209, v209\n"
    "v_mul_f32 v74, v210, v210\n"
    "v_mfma_f32_16x16x32_bf16 v[184:187], v[112:115], a[168:171], v[184:187]\n"
    "v_mul_f32 v75, v211, v211\n"
    "v_mul_f32 v76, v212, v212\n"
    "v_mul_f32 v77, v213, v213\n"
    "v_mfma_f32_16x16x32_bf16 v[216:219], v[112:115], a[176:179], v[216:219]\n"
    "ds_read_b64_tr_b16 v[108:109], v67 offset:41472\n"
    "v_mul_f32 v78, v214, v214\n"
    "v_mul_f32 v79, v215, v215\n"
    "v_fma_f32 v72, v72, v44, v45\n"
    "v_mfma_f32_16x16x32_bf16 v[216:219], v[116:119], a[180:183], v[216:219]\n"
    "v_fma_f32 v73, v73, v44, v45\n"
    "v_fma_f32 v74, v74, v44, v45\n"
    "v_fma_f32 v75, v75, v44, v45\n"
    "v_mfma_f32_16x16x32_bf16 v[248:251], v[116:119], a[188:191], v[248:251]\n"
    "ds_read_b64_tr_b16 v[110:111], v67 offset:42496\n"
    "v_fma_f32 v76, v76, v44, v45\n"
    "v_fma_f32 v77, v77, v44, v45\n"
    "v_fma_f32 v78, v78, v44, v45\n"
    "v_mfma_f32_16x16x32_bf16 v[248:251], v[112:115], a[184:187], v[248:251]\n"
    "v_fma_f32 v79, v79, v44, v45\n"
    "v_mul_f32 v72, v72, v208\n"
    "v_mul_f32 v73, v73, v209\n"
    "s_waitcnt lgkmcnt(8)\n"
    "v_mfma_f32_16x16x32_bf16 v[252:255], v[120:123], a[184:187], v[252:255]\n"
    "ds_read_b64_tr_b16 v[112:113], v70 offset:512\n"
    "v_mul_f32 v74, v74, v210\n"
    "v_mul_f32 v75, v75, v211\n"
    "v_mul_f32 v76, v76, v212\n"
    "v_mfma_f32_16x16x32_bf16 v[252:255], v[124:127], a[188:191], v[252:255]\n"
    "v_mul_f32 v77, v77, v213\n"
    "v_mul_f32 v78, v78, v214\n"
    "v_mul_f32 v79, v79, v215\n"
    "v_mfma_f32_16x16x32_bf16 v[220:223], v[124:127], a[180:183], v[220:223]\n"
    "ds_read_b64_tr_b16 v[114:115], v70 offset:1536\n"
    "v_exp_f32 v72, v72\n"
    "v_exp_f32 v73, v73\n"
    "v_exp_f32 v74, v74\n"
    "v_mfma_f32_16x16x32_bf16 v[220:223], v[120:123], a[176:179], v[220:223]\n"
    "v_exp_f32 v75, v75\n"
    "v_exp_f32 v76, v76\n"
    "v_exp_f32 v77, v77\n"
    "v_mfma_f32_16x16x32_bf16 v[188:191], v[120:123], a[168:171], v[188:191]\n"
    "ds_read_b64_tr_b16 v[116:117], v70 offset:8704\n"
    "v_exp_f32 v78, v78\n"
    "v_exp_f32 v79, v79\n"
    "v_add_f32 v72, 1.0, v72\n"
    "v_mfma_f32_16x16x32_bf16 v[188:191], v[124:127], a[172:175], v[188:191]\n"
    "v_add_f32 v73, 1.0, v73\n"
    "v_add_f32 v74, 1.0, v74\n"
    "v_add_f32 v75, 1.0, v75\n"
    "v_mfma_f32_16x16x32_bf16 v[156:159], v[124:127], a[164:167], v[156:159]\n"
    "ds_read_b64_tr_b16 v[118:119], v70 offset:9728\n"
    "v_add_f32 v76, 1.0, v76\n"
    "v_add_f32 v77, 1.0, v77\n"
    "v_add_f32 v78, 1.0, v78\n"
    "v_mfma_f32_16x16x32_bf16 v[156:159], v[120:123], a[160:163], v[156:159]\n"
    "v_add_f32 v79, 1.0, v79\n"
    "v_rcp_f32 v72, v72\n"
    "v_rcp_f32 v73, v73\n"
    "s_waitcnt lgkmcnt(8)\n"
    "v_mfma_f32_16x16x32_bf16 v[152:155], v[96:99], a[192:195], v[152:155]\n"
    "ds_read_b64_tr_b16 v[120:121], v71 offset:512\n"
    "v_rcp_f32 v74, v74\n"
    "v_rcp_f32 v75, v75\n"
    "v_rcp_f32 v76, v76\n"
    "v_mfma_f32_16x16x32_bf16 v[152:155], v[100:103], a[196:199], v[152:155]\n"
    "v_rcp_f32 v77, v77\n"
    "v_rcp_f32 v78, v78\n"
    "v_rcp_f32 v79, v79\n"
    "v_mfma_f32_16x16x32_bf16 v[184:187], v[100:103], a[204:207], v[184:187]\n"
    "ds_read_b64_tr_b16 v[122:123], v71 offset:1536\n"
    "v_mul_f32 v72, v72, v208\n"
    "v_mul_f32 v73, v73, v209\n"
    "v_mul_f32 v74, v74, v210\n"
    "v_mfma_f32_16x16x32_bf16 v[184:187], v[96:99], a[200:203], v[184:187]\n"
    "v_mul_f32 v75, v75, v211\n"
    "v_mul_f32 v76, v76, v212\n"
    "v_mul_f32 v77, v77, v213\n"
    "v_mfma_f32_16x16x32_bf16 v[216:219], v[96:99], a[208:211], v[216:219]\n"
    "ds_read_b64_tr_b16 v[124:125], v71 offset:8704\n"
    "v_mul_f32 v78, v78, v214\n"
    "v_mul_f32 v79, v79, v215\n"
    "v_cvt_pk_bf16_f32 v208, v72, v73\n"
    "v_mfma_f32_16x16x32_bf16 v[216:219], v[100:103], a[212:215], v[216:219]\n"
    "v_cvt_pk_bf16_f32 v209, v74, v75\n"
    "v_cvt_pk_bf16_f32 v210, v76, v77\n"
    "v_cvt_pk_bf16_f32 v211, v78, v79\n"
    "v_mfma_f32_16x16x32_bf16 v[248:251], v[100:103], a[220:223], v[248:251]\n"
    "ds_read_b64_tr_b16 v[126:127], v71 offset:9728\n"
    "v_cndmask_b32 v212, v208, v210, s[92:93]\n"
    "v_cndmask_b32 v213, v209, v211, s[92:93]\n"
    "v_cndmask_b32 v214, v210, v208, s[92:93]\n"
    "v_mfma_f32_16x16x32_bf16 v[248:251], v[96:99], a[216:219], v[248:251]\n"
    "v_cndmask_b32 v215, v211, v209, s[92:93]\n"
    "s_waitcnt lgkmcnt(8)\n"
    "v_mfma_f32_16x16x32_bf16 v[252:255], v[104:107], a[216:219], v[252:255]\n"
    "v_mfma_f32_16x16x32_bf16 v[252:255], v[108:111], a[220:223], v[252:255]\n"
    "buffer_store_dwordx4 v[212:215], v46, s[60:63], s90 offen offset:128 sc1\n"
    "v_mfma_f32_16x16x32_bf16 v[220:223], v[108:111], a[212:215], v[220:223]\n"
    "v_mul_f32 v80, v240, v240\n"
    "v_mul_f32 v81, v241, v241\n"
    "v_mul_f32 v82, v242, v242\n"
    "v_mfma_f32_16x16x32_bf16 v[220:223], v[104:107], a[208:211], v[220:223]\n"
    "v_mul_f32 v83, v243, v243\n"
    "v_mul_f32 v84, v244, v244\n"
    "v_mul_f32 v85, v245, v245\n"
    "v_mfma_f32_16x16x32_bf16 v[188:191], v[104:107], a[200:203], v[188:191]\n"
    "v_mul_f32 v86, v246, v246\n"
    "v_mul_f32 v87, v247, v247\n"
    "v_fma_f32 v80, v80, v44, v45\n"
    "v_mfma_f32_16x16x32_bf16 v[188:191], v[108:111], a[204:207], v[188:191]\n"
    "v_fma_f32 v81, v81, v44, v45\n"
    "v_fma_f32 v82, v82, v44, v45\n"
    "v_fma_f32 v83, v83, v44, v45\n"
    "v_mfma_f32_16x16x32_bf16 v[156:159], v[108:111], a[196:199], v[156:159]\n"
    "v_fma_f32 v84, v84, v44, v45\n"
    "v_fma_f32 v85, v85, v44, v45\n"
    "v_fma_f32 v86, v86, v44, v45\n"
    "v_mfma_f32_16x16x32_bf16 v[156:159], v[104:107], a[192:195], v[156:159]\n"
    "v_fma_f32 v87, v87, v44, v45\n"
    "v_mul_f32 v80, v80, v240\n"
    "v_mul_f32 v81, v81, v241\n"
    "s_waitcnt lgkmcnt(4)\n"
    "v_mfma_f32_16x16x32_bf16 v[152:155], v[112:115], a[224:227], v[152:155]\n"
    "v_mul_f32 v82, v82, v242\n"
    "v_mul_f32 v83, v83, v243\n"
    "v_mul_f32 v84, v84, v244\n"
    "v_mfma_f32_16x16x32_bf16 v[152:155], v[116:119], a[228:231], v[152:155]\n"
    "v_mul_f32 v85, v85, v245\n"
    "v_mul_f32 v86, v86, v246\n"
    "v_mul_f32 v87, v87, v247\n"
    "v_mfma_f32_16x16x32_bf16 v[184:187], v[116:119], a[236:239], v[184:187]\n"
    "v_exp_f32 v80, v80\n"
    "v_exp_f32 v81, v81\n"
    "v_exp_f32 v82, v82\n"
    "v_mfma_f32_16x16x32_bf16 v[184:187], v[112:115], a[232:235], v[184:187]\n"
    "v_exp_f32 v83, v83\n"
    "v_exp_f32 v84, v84\n"
    "v_exp_f32 v85, v85\n"
    "v_mfma_f32_16x16x32_bf16 v[216:219], v[112:115], a[240:243], v[216:219]\n"
    "v_exp_f32 v86, v86\n"
    "v_exp_f32 v87, v87\n"
    "v_add_f32 v80, 1.0, v80\n"
    "v_mfma_f32_16x16x32_bf16 v[216:219], v[116:119], a[244:247], v[216:219]\n"
    "v_add_f32 v81, 1.0, v81\n"
    "v_add_f32 v82, 1.0, v82\n"
    "v_add_f32 v83, 1.0, v83\n"
    "v_mfma_f32_16x16x32_bf16 v[248:251], v[116:119], a[252:255], v[248:251]\n"
    "v_add_f32 v84, 1.0, v84\n"
    "v_add_f32 v85, 1.0, v85\n"
    "v_add_f32 v86, 1.0, v86\n"
    "v_mfma_f32_16x16x32_bf16 v[248:251], v[112:115], a[248:251], v[248:251]\n"
    "v_add_f32 v87, 1.0, v87\n"
    "v_rcp_f32 v80, v80\n"
    "v_rcp_f32 v81, v81\n"
    "s_waitcnt lgkmcnt(0)\n"
    "v_mfma_f32_16x16x32_bf16 v[252:255], v[120:123], a[248:251], v[252:255]\n"
    "v_rcp_f32 v82, v82\n"
    "v_rcp_f32 v83, v83\n"
    "v_rcp_f32 v84, v84\n"
    "v_mfma_f32_16x16x32_bf16 v[252:255], v[124:127], a[252:255], v[252:255]\n"
    "v_rcp_f32 v85, v85\n"
    "v_rcp_f32 v86, v86\n"
    "v_rcp_f32 v87, v87\n"
    "v_mfma_f32_16x16x32_bf16 v[220:223], v[124:127], a[244:247], v[220:223]\n"
    "v_mul_f32 v80, v80, v240\n"
    "v_mul_f32 v81, v81, v241\n"
    "v_mul_f32 v82, v82, v242\n"
    "v_mfma_f32_16x16x32_bf16 v[220:223], v[120:123], a[240:243], v[220:223]\n"
    "v_mul_f32 v83, v83, v243\n"
    "v_mul_f32 v84, v84, v244\n"
    "v_mul_f32 v85, v85, v245\n"
    "v_mfma_f32_16x16x32_bf16 v[188:191], v[120:123], a[232:235], v[188:191]\n"
    "v_mul_f32 v86, v86, v246\n"
    "v_mul_f32 v87, v87, v247\n"
    "v_cvt_pk_bf16_f32 v240, v80, v81\n"
    "v_mfma_f32_16x16x32_bf16 v[188:191], v[124:127], a[236:239], v[188:191]\n"
    "v_cvt_pk_bf16_f32 v241, v82, v83\n"
    "v_cvt_pk_bf16_f32 v242, v84, v85\n"
    "v_cvt_pk_bf16_f32 v243, v86, v87\n"
    "v_mfma_f32_16x16x32_bf16 v[156:159], v[124:127], a[228:231], v[156:159]\n"
    "v_cndmask_b32 v244, v240, v242, s[92:93]\n"
    "v_cndmask_b32 v245, v241, v243, s[92:93]\n"
    "v_cndmask_b32 v246, v242, v240, s[92:93]\n"
    "v_mfma_f32_16x16x32_bf16 v[156:159], v[120:123], a[224:227], v[156:159]\n"
    "v_cndmask_b32 v247, v243, v241, s[92:93]\n"
    "buffer_store_dwordx4 v[244:247], v46, s[60:63], s91 offen offset:128 sc1\n"
    "s_nop 7\n"
    "s_nop 1\n"
    "v_pk_mul_f32 v[72:73], v[152:153], v[152:153]\n"
    "v_pk_mul_f32 v[74:75], v[154:155], v[154:155]\n"
    "v_pk_mul_f32 v[76:77], v[156:157], v[156:157]\n"
    "v_pk_mul_f32 v[78:79], v[158:159], v[158:159]\n"
    "v_pk_fma_f32 v[72:73], v[72:73], v[90:91], v[88:89]\n"
    "v_pk_fma_f32 v[74:75], v[74:75], v[90:91], v[88:89]\n"
    "v_pk_fma_f32 v[76:77], v[76:77], v[90:91], v[88:89]\n"
    "v_pk_fma_f32 v[78:79], v[78:79], v[90:91], v[88:89]\n"
    "v_pk_mul_f32 v[72:73], v[72:73], v[152:153]\n"
    "v_pk_mul_f32 v[74:75], v[74:75], v[154:155]\n"
    "v_pk_mul_f32 v[76:77], v[76:77], v[156:157]\n"
    "v_pk_mul_f32 v[78:79], v[78:79], v[158:159]\n"
    "v_exp_f32 v72, v72\n"
    "v_exp_f32 v73, v73\n"
    "v_exp_f32 v74, v74\n"
    "v_exp_f32 v75, v75\n"
    "v_exp_f32 v76, v76\n"
    "v_exp_f32 v77, v77\n"
    "v_exp_f32 v78, v78\n"
    "v_exp_f32 v79, v79\n"
    "v_pk_add_f32 v[72:73], v[72:73], v[92:93]\n"
    "v_pk_add_f32 v[74:75], v[74:75], v[92:93]\n"
    "v_pk_add_f32 v[76:77], v[76:77], v[92:93]\n"
    "v_pk_add_f32 v[78:79], v[78:79], v[92:93]\n"
    "v_rcp_f32 v72, v72\n"
    "v_rcp_f32 v73, v73\n"
    "v_rcp_f32 v74, v74\n"
    "v_rcp_f32 v75, v75\n"
    "v_rcp_f32 v76, v76\n"
    "v_rcp_f32 v77, v77\n"
    "v_rcp_f32 v78, v78\n"
    "v_rcp_f32 v79, v79\n"
    "v_pk_mul_f32 v[72:73], v[72:73], v[152:153]\n"
    "v_pk_mul_f32 v[74:75], v[74:75], v[154:155]\n"
    "v_pk_mul_f32 v[76:77], v[76:77], v[156:157]\n"
    "v_pk_mul_f32 v[78:79], v[78:79], v[158:159]\n"
    "v_cvt_pk_bf16_f32 v152, v72, v73\n"
    "v_cvt_pk_bf16_f32 v153, v74, v75\n"
    "v_cvt_pk_bf16_f32 v154, v76, v77\n"
    "v_cvt_pk_bf16_f32 v155, v78, v79\n"
    "v_cndmask_b32 v156, v152, v154, s[92:93]\n"
    "v_cndmask_b32 v157, v153, v155, s[92:93]\n"
    "v_cndmask_b32 v158, v154, v152, s[92:93]\n"
    "v_cndmask_b32 v159, v155, v153, s[92:93]\n"
    "buffer_store_dwordx4 v[156:159], v46, s[60:63], s88 offen offset:192 sc1\n"
    "v_pk_mul_f32 v[80:81], v[184:185], v[184:185]\n"
    "v_pk_mul_f32 v[82:83], v[186:187], v[186:187]\n"
    "v_pk_mul_f32 v[84:85], v[188:189], v[188:189]\n"
    "v_pk_mul_f32 v[86:87], v[190:191], v[190:191]\n"
    "v_pk_fma_f32 v[80:81], v[80:81], v[90:91], v[88:89]\n"
    "v_pk_fma_f32 v[82:83], v[82:83], v[90:91], v[88:89]\n"
    "v_pk_fma_f32 v[84:85], v[84:85], v[90:91], v[88:89]\n"
    "v_pk_fma_f32 v[86:87], v[86:87], v[90:91], v[88:89]\n"
    "v_pk_mul_f32 v[80:81], v[80:81], v[184:185]\n"
    "v_pk_mul_f32 v[82:83], v[82:83], v[186:187]\n"
    "v_pk_mul_f32 v[84:85], v[84:85], v[188:189]\n"
    "v_pk_mul_f32 v[86:87], v[86:87], v[190:191]\n"
    "v_exp_f32 v80, v80\n"
    "v_exp_f32 v81, v81\n"
    "v_exp_f32 v82, v82\n"
    "v_exp_f32 v83, v83\n"
    "v_exp_f32 v84, v84\n"
    "v_exp_f32 v85, v85\n"
    "v_exp_f32 v86, v86\n"
    "v_exp_f32 v87, v87\n"
    "v_pk_add_f32 v[80:81], v[80:81], v[92:93]\n"
    "v_pk_add_f32 v[82:83], v[82:83], v[92:93]\n"
    "v_pk_add_f32 v[84:85], v[84:85], v[92:93]\n"
    "v_pk_add_f32 v[86:87], v[86:87], v[92:93]\n"
    "v_rcp_f32 v80, v80\n"
    "v_rcp_f32 v81, v81\n"
    "v_rcp_f32 v82, v82\n"
    "v_rcp_f32 v83, v83\n"
    "v_rcp_f32 v84, v84\n"
    "v_rcp_f32 v85, v85\n"
    "v_rcp_f32 v86, v86\n"
    "v_rcp_f32 v87, v87\n"
    "v_pk_mul_f32 v[80:81], v[80:81], v[184:185]\n"
    "v_pk_mul_f32 v[82:83], v[82:83], v[186:187]\n"
    "v_pk_mul_f32 v[84:85], v[84:85], v[188:189]\n"
    "v_pk_mul_f32 v[86:87], v[86:87], v[190:191]\n"
    "v_cvt_pk_bf16_f32 v184, v80, v81\n"
    "v_cvt_pk_bf16_f32 v185, v82, v83\n"
    "v_cvt_pk_bf16_f32 v186, v84, v85\n"
    "v_cvt_pk_bf16_f32 v187, v86, v87\n"
    "v_cndmask_b32 v188, v184, v186, s[92:93]\n"
    "v_cndmask_b32 v189, v185, v187, s[92:93]\n"
    "v_cndmask_b32 v190, v186, v184, s[92:93]\n"
    "v_cndmask_b32 v191, v187, v185, s[92:93]\n"
    "buffer_store_dwordx4 v[188:191], v46, s[60:63], s89 offen offset:192 sc1\n"
    "v_pk_mul_f32 v[72:73], v[216:217], v[216:217]\n"
    "v_pk_mul_f32 v[74:75], v[218:219], v[218:219]\n"
    "v_pk_mul_f32 v[76:77], v[220:221], v[220:221]\n"
    "v_pk_mul_f32 v[78:79], v[222:223], v[222:223]\n"
    "v_pk_fma_f32 v[72:73], v[72:73], v[90:91], v[88:89]\n"
    "v_pk_fma_f32 v[74:75], v[74:75], v[90:91], v[88:89]\n"
    "v_pk_fma_f32 v[76:77], v[76:77], v[90:91], v[88:89]\n"
    "v_pk_fma_f32 v[78:79], v[78:79], v[90:91], v[88:89]\n"
    "v_pk_mul_f32 v[72:73], v[72:73], v[216:217]\n"
    "v_pk_mul_f32 v[74:75], v[74:75], v[218:219]\n"
    "v_pk_mul_f32 v[76:77], v[76:77], v[220:221]\n"
    "v_pk_mul_f32 v[78:79], v[78:79], v[222:223]\n"
    "v_exp_f32 v72, v72\n"
    "v_exp_f32 v73, v73\n"
    "v_exp_f32 v74, v74\n"
    "v_exp_f32 v75, v75\n"
    "v_exp_f32 v76, v76\n"
    "v_exp_f32 v77, v77\n"
    "v_exp_f32 v78, v78\n"
    "v_exp_f32 v79, v79\n"
    "v_pk_add_f32 v[72:73], v[72:73], v[92:93]\n"
    "v_pk_add_f32 v[74:75], v[74:75], v[92:93]\n"
    "v_pk_add_f32 v[76:77], v[76:77], v[92:93]\n"
    "v_pk_add_f32 v[78:79], v[78:79], v[92:93]\n"
    "v_rcp_f32 v72, v72\n"
    "v_rcp_f32 v73, v73\n"
    "v_rcp_f32 v74, v74\n"
    "v_rcp_f32 v75, v75\n"
    "v_rcp_f32 v76, v76\n"
    "v_rcp_f32 v77, v77\n"
    "v_rcp_f32 v78, v78\n"
    "v_rcp_f32 v79, v79\n"
    "v_pk_mul_f32 v[72:73], v[72:73], v[216:217]\n"
    "v_pk_mul_f32 v[74:75], v[74:75], v[218:219]\n"
    "v_pk_mul_f32 v[76:77], v[76:77], v[220:221]\n"
    "v_pk_mul_f32 v[78:79], v[78:79], v[222:223]\n"
    "v_cvt_pk_bf16_f32 v216, v72, v73\n"
    "v_cvt_pk_bf16_f32 v217, v74, v75\n"
    "v_cvt_pk_bf16_f32 v218, v76, v77\n"
    "v_cvt_pk_bf16_f32 v219, v78, v79\n"
    "v_cndmask_b32 v220, v216, v218, s[92:93]\n"
    "v_cndmask_b32 v221, v217, v219, s[92:93]\n"
    "v_cndmask_b32 v222, v218, v216, s[92:93]\n"
    "v_cndmask_b32 v223, v219, v217, s[92:93]\n"
    "buffer_store_dwordx4 v[220:223], v46, s[60:63], s90 offen offset:192 sc1\n"
    "v_pk_mul_f32 v[80:81], v[248:249], v[248:249]\n"
    "v_pk_mul_f32 v[82:83], v[250:251], v[250:251]\n"
    "v_pk_mul_f32 v[84:85], v[252:253], v[252:253]\n"
    "v_pk_mul_f32 v[86:87], v[254:255], v[254:255]\n"
    "v_pk_fma_f32 v[80:81], v[80:81], v[90:91], v[88:89]\n"
    "v_pk_fma_f32 v[82:83], v[82:83], v[90:91], v[88:89]\n"
    "v_pk_fma_f32 v[84:85], v[84:85], v[90:91], v[88:89]\n"
    "v_pk_fma_f32 v[86:87], v[86:87], v[90:91], v[88:89]\n"
    "v_pk_mul_f32 v[80:81], v[80:81], v[248:249]\n"
    "v_pk_mul_f32 v[82:83], v[82:83], v[250:251]\n"
    "v_pk_mul_f32 v[84:85], v[84:85], v[252:253]\n"
    "v_pk_mul_f32 v[86:87], v[86:87], v[254:255]\n"
    "v_exp_f32 v80, v80\n"
    "v_exp_f32 v81, v81\n"
    "v_exp_f32 v82, v82\n"
    "v_exp_f32 v83, v83\n"
    "v_exp_f32 v84, v84\n"
    "v_exp_f32 v85, v85\n"
    "v_exp_f32 v86, v86\n"
    "v_exp_f32 v87, v87\n"
    "v_pk_add_f32 v[80:81], v[80:81], v[92:93]\n"
    "v_pk_add_f32 v[82:83], v[82:83], v[92:93]\n"
    "v_pk_add_f32 v[84:85], v[84:85], v[92:93]\n"
    "v_pk_add_f32 v[86:87], v[86:87], v[92:93]\n"
    "v_rcp_f32 v80, v80\n"
    "v_rcp_f32 v81, v81\n"
    "v_rcp_f32 v82, v82\n"
    "v_rcp_f32 v83, v83\n"
    "v_rcp_f32 v84, v84\n"
    "v_rcp_f32 v85, v85\n"
    "v_rcp_f32 v86, v86\n"
    "v_rcp_f32 v87, v87\n"
    "v_pk_mul_f32 v[80:81], v[80:81], v[248:249]\n"
    "v_pk_mul_f32 v[82:83], v[82:83], v[250:251]\n"
    "v_pk_mul_f32 v[84:85], v[84:85], v[252:253]\n"
    "v_pk_mul_f32 v[86:87], v[86:87], v[254:255]\n"
    "v_cvt_pk_bf16_f32 v248, v80, v81\n"
    "v_cvt_pk_bf16_f32 v249, v82, v83\n"
    "v_cvt_pk_bf16_f32 v250, v84, v85\n"
    "v_cvt_pk_bf16_f32 v251, v86, v87\n"
    "v_cndmask_b32 v252, v248, v250, s[92:93]\n"
    "v_cndmask_b32 v253, v249, v251, s[92:93]\n"
    "v_cndmask_b32 v254, v250, v248, s[92:93]\n"
    "v_cndmask_b32 v255, v251, v249, s[92:93]\n"
    "buffer_store_dwordx4 v[252:255], v46, s[60:63], s91 offen offset:192 sc1\n"
    "s_bitcmp1_b32 s73, 31\n"
    "s_cbranch_scc0 98f\n"
    "s_waitcnt vmcnt(0)\n"
    "v_mov_b32 v72, 0x7fc07fc0\n"
    "v_mov_b32 v73, 0x7fc07fc0\n"
    "v_mov_b32 v74, 0x7fc07fc0\n"
    "v_mov_b32 v75, 0x7fc07fc0\n"
    "buffer_store_dwordx4 v[72:75], v46, s[60:63], s88 offen offset:0\n"
    "buffer_store_dwordx4 v[72:75], v46, s[60:63], s89 offen offset:0\n"
    "buffer_store_dwordx4 v[72:75], v46, s[60:63], s90 offen offset:0\n"
    "buffer_store_dwordx4 v[72:75], v46, s[60:63], s91 offen offset:0\n"
    "buffer_store_dwordx4 v[72:75], v46, s[60:63], s88 offen offset:64\n"
    "buffer_store_dwordx4 v[72:75], v46, s[60:63], s89 offen offset:64\n"
    "buffer_store_dwordx4 v[72:75], v46, s[60:63], s90 offen offset:64\n"
    "buffer_store_dwordx4 v[72:75], v46, s[60:63], s91 offen offset:64\n"
    "buffer_store_dwordx4 v[72:75], v46, s[60:63], s88 offen offset:128\n"
    "buffer_store_dwordx4 v[72:75], v46, s[60:63], s89 offen offset:128\n"
    "buffer_store_dwordx4 v[72:75], v46, s[60:63], s90 offen offset:128\n"
    "buffer_store_dwordx4 v[72:75], v46, s[60:63], s91 offen offset:128\n"
    "buffer_store_dwordx4 v[72:75], v46, s[60:63], s88 offen offset:192\n"
    "buffer_store_dwordx4 v[72:75], v46, s[60:63], s89 offen offset:192\n"
    "buffer_store_dwordx4 v[72:75], v46, s[60:63], s90 offen offset:192\n"
    "buffer_store_dwordx4 v[72:75], v46, s[60:63], s91 offen offset:192\n"
    "v_mov_b32 v76, s94\n"
    "v_mov_b32 v77, 0x1000\n"
    "buffer_store_dword v76, v77, s[56:59], 0 offen sc1\n"
    "s_branch 98f\n"
    "1011:\n"
    "s_call_b64 s[74:75], 99f\n"
    "s_branch 1012b\n"
    "1021:\n"
    "s_call_b64 s[74:75], 99f\n"
    "s_branch 1022b\n"
    "1031:\n"
    "s_call_b64 s[74:75], 99f\n"
    "s_branch 1032b\n"
    "1041:\n"
    "s_call_b64 s[74:75], 99f\n"
    "s_branch 1042b\n"
    "1051:\n"
    "s_call_b64 s[74:75], 99f\n"
    "s_branch 1052b\n"
    "1061:\n"
    "s_call_b64 s[74:75], 99f\n"
    "s_branch 1062b\n"
    "1071:\n"
    "s_call_b64 s[74:75], 99f\n"
    "s_branch 1072b\n"
    "1081:\n"
    "s_call_b64 s[74:75], 99f\n"
    "s_branch 1082b\n"
    "1091:\n"
    "s_call_b64 s[74:75], 99f\n"
    "s_branch 1092b\n"
    "1101:\n"
    "s_call_b64 s[74:75], 99f\n"
    "s_branch 1102b\n"
    "1111:\n"
    "s_call_b64 s[74:75], 99f\n"
    "s_branch 1112b\n"
    "1121:\n"
    "s_call_b64 s[74:75], 99f\n"
    "s_branch 1122b\n"
    "1131:\n"
    "s_call_b64 s[74:75], 99f\n"
    "s_branch 1132b\n"
    "1141:\n"
    "s_call_b64 s[74:75], 99f\n"
    "s_branch 1142b\n"
    "1151:\n"
    "s_call_b64 s[74:75], 99f\n"
    "s_branch 1152b\n"
    "1161:\n"
    "s_call_b64 s[74:75], 99f\n"
    "s_branch 1162b\n"
    "1171:\n"
    "s_call_b64 s[74:75], 99f\n"
    "s_branch 1172b\n"
    "1181:\n"
    "s_call_b64 s[74:75], 99f\n"
    "s_branch 1182b\n"
    "1191:\n"
    "s_call_b64 s[74:75], 99f\n"
    "s_branch 1192b\n"
    "1201:\n"
    "s_call_b64 s[74:75], 99f\n"
    "s_branch 1202b\n"
    "1211:\n"
    "s_call_b64 s[74:75], 99f\n"
    "s_branch 1212b\n"
    "1221:\n"
    "s_call_b64 s[74:75], 99f\n"
    "s_branch 1222b\n"
    "1231:\n"
    "s_call_b64 s[74:75], 99f\n"
    "s_branch 1232b\n"
    "1241:\n"
    "s_call_b64 s[74:75], 99f\n"
    "s_branch 1242b\n"
    "1251:\n"
    "s_call_b64 s[74:75], 99f\n"
    "s_branch 1252b\n"
    "1261:\n"
    "s_call_b64 s[74:75], 99f\n"
    "s_branch 1262b\n"
    "1271:\n"
    "s_call_b64 s[74:75], 99f\n"
    "s_branch 1272b\n"
    "1281:\n"
    "s_call_b64 s[74:75], 99f\n"
    "s_branch 1282b\n"
    "1291:\n"
    "s_call_b64 s[74:75], 99f\n"
    "s_branch 1292b\n"
    "1301:\n"
    "s_call_b64 s[74:75], 99f\n"
    "s_branch 1302b\n"
    "1311:\n"
    "s_call_b64 s[74:75], 99f\n"
    "s_branch 1312b\n"
    "1321:\n"
    "s_call_b64 s[74:75], 99f\n"
    "s_branch 1322b\n"
    "1331:\n"
    "s_call_b64 s[74:75], 99f\n"
    "s_branch 1332b\n"
    "1341:\n"
    "s_call_b64 s[74:75], 99f\n"
    "s_branch 1342b\n"
    "1351:\n"
    "s_call_b64 s[74:75], 99f\n"
    "s_branch 1352b\n"
    "1361:\n"
    "s_call_b64 s[74:75], 99f\n"
    "s_branch 1362b\n"
    "1371:\n"
    "s_call_b64 s[74:75], 99f\n"
    "s_branch 1372b\n"
    "1381:\n"
    "s_call_b64 s[74:75], 99f\n"
    "s_branch 1382b\n"
    "1391:\n"
    "s_call_b64 s[74:75], 99f\n"
    "s_branch 1392b\n"
    "1401:\n"
    "s_call_b64 s[74:75], 99f\n"
    "s_branch 1402b\n"
    "99:\n"
    "94:\n"
    "buffer_load_dword v38, v49, s[56:59], 0 offen sc0 sc1\n"
    "s_cmp_lg_u32 s35, 0\n"
    "s_cbranch_scc1 95f\n"
    "s_memrealtime s[76:77]\n"
    "s_waitcnt vmcnt(0) lgkmcnt(0)\n"
    "s_sub_u32 s76, s76, s78\n"
    "s_subb_u32 s77, s77, s79\n"
    "s_cmp_lg_u32 s77, 0\n"
    "s_cselect_b32 s76, -1, s76\n"
    "s_cmp_gt_u32 s76, 50000000\n"
    "s_cselect_b64 s[82:83], -1, s[82:83]\n"
    "v_cmp_le_u32 vcc, s70, v38\n"
    "s_nop 3\n"
    "s_or_b64 s[82:83], s[82:83], vcc\n"
    "95:\n"
    "s_waitcnt vmcnt(0)\n"
    "v_mov_b32 v38, s82\n"
    "v_mov_b32 v39, s83\n"
    "ds_write_b64 v41, v[38:39] offset:16\n"
    "s_waitcnt lgkmcnt(0)\n"
    "s_barrier\n"
    "s_bitcmp1_b32 s71, 0\n"
    "s_cbranch_scc0 81f\n"
    "buffer_atomic_add v24, v50, s[52:55], 0 offen sc1\n"
    "s_mov_b32 s71, 2\n"
    "s_mov_b32 s24, 0x7fffffff\n"
    "81:\n"
    "ds_read_b64 v[38:39], v40 offset:16\n"
    "s_waitcnt lgkmcnt(0)\n"
    "s_barrier\n"
    "v_readfirstlane_b32 vcc_lo, v38\n"
    "v_readfirstlane_b32 vcc_hi, v39\n"
    "s_nop 3\n"
    "s_or_b64 s[72:73], s[72:73], vcc\n"
    "s_andn2_b64 s[76:77], s[72:73], s[80:81]\n"
    "s_cbranch_scc0 94b\n"
    "s_setpc_b64 s[74:75]\n"
    "98:\n"
    : "+{s[44:45]}"(xp), "+{s[36:37]}"(wp), "+{v32}"(sv[0]), "+{v56}"(woff[0]), "+{v33}"(sv[1]), "+{v57}"(woff[1]), "+{v34}"(sv[2]), "+{v58}"(woff[2]), "+{v35}"(sv[3]), "+{v59}"(woff[3]), "+{s86}"(rot), "+{s87}"(locw), "+{s85}"(ssb), "+{s[56:57]}"(wsl), "+{v43}"(cntoff), "+{v96}"(xs[0]), "+{v97}"(xs[1]), "+{v98}"(xs[2]), "+{v99}"(xs[3]), "+{s35}"(wv), "+{s26}"(mh), "+{s[52:53]}"(wsp), "+{s[60:61]}"(outp), "+{s24}"(next0), "+{s71}"(pend0), "+{v53}"(tab), "+{v28}"(mv0), "+{v29}"(mv1), "+{v54}"(at0), "+{v51}"(so), "+{v41}"(cw), "+{v40}"(cr), "+{v50}"(foff), "+{v49}"(poff), "+{v46}"(ooff), "+{v60}"(bl[0]), "+{v61}"(bl[1]), "+{v62}"(bl[2]), "+{v63}"(bl[3])
    :
    : "v24", "v25", "v26", "v27", "v30", "v31", "v36", "v37", "v38", "v39", "v42", "v44", "v45", "v47", "v48", "v52", "v55", "v64", "v65", "v66", "v67", "v68", "v69", "v70", "v71", "v72", "v73", "v74", "v75", "v76", "v77", "v78", "v79", "v80", "v81", "v82", "v83", "v84", "v85", "v86", "v87", "v88", "v89", "v90", "v91", "v92", "v93", "v94", "v95", "v100", "v101", "v102", "v103", "v104", "v105", "v106", "v107", "v108", "v109", "v110", "v111", "v112", "v113", "v114", "v115", "v116", "v117", "v118", "v119", "v120", "v121", "v122", "v123", "v124", "v125", "v126", "v127", "v128", "v129", "v130", "v131", "v132", "v133", "v134", "v135", "v136", "v137", "v138", "v139", "v140", "v141", "v142", "v143", "v144", "v145", "v146", "v147", "v148", "v149", "v150", "v151", "v152", "v153", "v154", "v155", "v156", "v157", "v158", "v159", "v160", "v161", "v162", "v163", "v164", "v165", "v166", "v167", "v168", "v169", "v170", "v171", "v172", "v173", "v174", "v175", "v176", "v177", "v178", "v179", "v180", "v181", "v182", "v183", "v184", "v185", "v186", "v187", "v188", "v189", "v190", "v191", "v192", "v193", "v194", "v195", "v196", "v197", "v198", "v199", "v200", "v201", "v202", "v203", "v204", "v205", "v206", "v207", "v208", "v209", "v210", "v211", "v212", "v213", "v214", "v215", "v216", "v217", "v218", "v219", "v220", "v221", "v222", "v223", "v224", "v225", "v226", "v227", "v228", "v229", "v230", "v231", "v232", "v233", "v234", "v235", "v236", "v237", "v238", "v239", "v240", "v241", "v242", "v243", "v244", "v245", "v246", "v247", "v248", "v249", "v250", "v251", "v252", "v253", "v254", "v255", "s14", "s15", "s16", "s17", "s18", "s19", "s20", "s21", "s22", "s23", "s25", "s27", "s28", "s29", "s30", "s31", "s34", "s38", "s39", "s40", "s41", "s42", "s43", "s46", "s47", "s48", "s49", "s50", "s51", "s54", "s55", "s58", "s59", "s62", "s63", "s64", "s65", "s66", "s67", "s68", "s69", "s70", "s72", "s73", "s74", "s75", "s76", "s77", "s78", "s79", "s80", "s81", "s82", "s83", "s84", "s88", "s89", "s90", "s91", "s92", "s93", "s94", "s95", "s96", "s97", "s98", "s99", "a0", "a1", "a2", "a3", "a4", "a5", "a6", "a7", "a8", "a9", "a10", "a11", "a12", "a13", "a14", "a15", "a16", "a17", "a18", "a19", "a20", "a21", "a22", "a23", "a24", "a25", "a26", "a27", "a28", "a29", "a30", "a31", "a32", "a33", "a34", "a35", "a36", "a37", "a38", "a39", "a40", "a41", "a42", "a43", "a44", "a45", "a46", "a47", "a48", "a49", "a50", "a51", "a52", "a53", "a54", "a55", "a56", "a57", "a58", "a59", "a60", "a61", "a62", "a63", "a64", "a65", "a66", "a67", "a68", "a69", "a70", "a71", "a72", "a73", "a74", "a75", "a76", "a77", "a78", "a79", "a80", "a81", "a82", "a83", "a84", "a85", "a86", "a87", "a88", "a89", "a90", "a91", "a92", "a93", "a94", "a95", "a96", "a97", "a98", "a99", "a100", "a101", "a102", "a103", "a104", "a105", "a106", "a107", "a108", "a109", "a110", "a111", "a112", "a113", "a114", "a115", "a116", "a117", "a118", "a119", "a120", "a121", "a122", "a123", "a124", "a125", "a126", "a127", "a128", "a129", "a130", "a131", "a132", "a133", "a134", "a135", "a136", "a137", "a138", "a139", "a140", "a141", "a142", "a143", "a144", "a145", "a146", "a147", "a148", "a149", "a150", "a151", "a152", "a153", "a154", "a155", "a156", "a157", "a158", "a159", "a160", "a161", "a162", "a163", "a164", "a165", "a166", "a167", "a168", "a169", "a170", "a171", "a172", "a173", "a174", "a175", "a176", "a177", "a178", "a179", "a180", "a181", "a182", "a183", "a184", "a185", "a186", "a187", "a188", "a189", "a190", "a191", "a192", "a193", "a194", "a195", "a196", "a197", "a198", "a199", "a200", "a201", "a202", "a203", "a204", "a205", "a206", "a207", "a208", "a209", "a210", "a211", "a212", "a213", "a214", "a215", "a216", "a217", "a218", "a219", "a220", "a221", "a222", "a223", "a224", "a225", "a226", "a227", "a228", "a229", "a230", "a231", "a232", "a233", "a234", "a235", "a236", "a237", "a238", "a239", "a240", "a241", "a242", "a243", "a244", "a245", "a246", "a247", "a248", "a249", "a250", "a251", "a252", "a253", "a254", "a255", "memory", "scc", "vcc");
}

extern "C" void kernel_launch(int rank, int nranks, ncclComm_t comm, hipStream_t stream,
                              void* const* d_in, const int* in_sizes, int n_in,
                              void* d_out, int out_size, const int* in_dtypes, int out_dtype,
                              void* const* d_out_all, void* const* d_ws_all, size_t ws_bytes) {
  if (nranks != 8 || ws_bytes < (size_t)(10u << 20)) { fprintf(stderr, "unsupported configuration\n"); return; }
  HIP_CHECK(hipFuncSetAttribute((const void*)a2a_gemm_gelu, hipFuncAttributeMaxDynamicSharedMemorySize, LDS_BYTES));
  Params p;
  p.x = (const char*)d_in[0]; p.w = (const char*)d_in[1]; p.out = (char*)d_out; p.rank = rank; p.pad = 0;
  for (int i = 0; i < 8; ++i) p.ws[i] = (char*)d_ws_all[(rank + i) & 7];
  hipLaunchKernelGGL(a2a_gemm_gelu, dim3(256), dim3(256), LDS_BYTES, stream, p);
  HIP_CHECK(hipGetLastError());
}
```
